```python
import numpy as np

kernel_name = "gpu_gemm_ar_m8192_k8192_n8192_bf16_silu"


ABSMAX_RTOL = 0.008

COMPUTE_DTYPE = "bfloat16"
EPILOGUE = "silu"
IN_QUANT = None
OUT_QUANT = None
IN_DTYPES = {"x": "bfloat16", "w_mat": "bfloat16"}
WORLD_SIZE = 8

def _to_bf16(x):
    u = np.ascontiguousarray(x, dtype=np.float32).view(np.uint32)
    return ((u + 0x7FFF + ((u >> 16) & 1)) & 0xFFFF0000).view(np.float32)

def setup_inputs():

    x = _to_bf16(np.random.randn(8192, 8192).astype(np.float32))
    w_mat = _to_bf16((np.random.randn(8192, 8192) / np.sqrt(8192)).astype(np.float32))
    return {"x": x, "w_mat": w_mat}

def _epilogue(y):
    return (y / (1.0 + np.exp(-np.clip(y, -60.0, 60.0)))).astype(np.float32)

def reference(x, w_mat):
    y = (x @ w_mat).astype(np.float32)
    return _epilogue(y)
```

```cpp
#include <hip/hip_runtime.h>
#include <rccl/rccl.h>
#include <cstdio>
#include <cstdint>

#define HIP_CHECK(expr) do { hipError_t _e = (expr); if (_e != hipSuccess) { \
    fprintf(stderr, "HIP error %s at %s:%d\n", hipGetErrorString(_e), __FILE__, __LINE__); \
    return; } } while (0)

namespace {

#define LAS __attribute__((address_space(3)))
typedef short bf16x8 __attribute__((ext_vector_type(8)));
typedef float f32x4 __attribute__((ext_vector_type(4)));
typedef unsigned u32x4 __attribute__((ext_vector_type(4)));

constexpr int NR = 8;
constexpr int KS = 1024;
constexpr int NTILE = 128;
#define NCW 16
#define WPW 1
constexpr int WPL = NCW * WPW;
constexpr int NGEMM = 128;
constexpr int GRID = 256;
constexpr int N_IN = 24 * 64;
constexpr int N_OUT = NTILE * 16;
static_assert(WPL <= 128, "flag slots");
static_assert(7 * NCW <= GRID - NGEMM, "too many comm workgroups");

constexpr unsigned OFF_SEQ = 0, OFF_DONE = 128, OFF_TMO = 256;
constexpr unsigned OFF_XFLAG = 4096;
constexpr unsigned OFF_WFLAG = OFF_XFLAG + 8 * 16 * 64 * 4;
constexpr unsigned OFF_TFLAG = OFF_WFLAG + 8 * 8 * 64 * 4;
constexpr unsigned OFF_OFLAG = OFF_TFLAG + 1024;
constexpr unsigned OFF_QCTR = OFF_OFLAG + 8 * 128 * 4;
constexpr unsigned OFF_DBG = 128 * 1024;
constexpr unsigned OFF_XS = 1u << 20;
constexpr unsigned OFF_WTS = OFF_XS + (64u << 20);
constexpr unsigned WS_NEED = OFF_WTS + (32u << 20);
static_assert(WS_NEED <= (128u << 20), "buffer window");
constexpr unsigned long long DEADLINE = 40000000ull;

constexpr int BM = 256, BK = 64, HALF = 128, HTB = HALF * BK * 2, STAGE_BYTES = 8 * HTB;
constexpr int LDS_BYTES = STAGE_BYTES;

struct Params {
    const char* x;
    const char* w;
    char* out[NR];
    char* ws[NR];
    int rank;
    unsigned order[NTILE / 4];
    unsigned pos[NTILE / 4];
};

typedef const __attribute__((address_space(4))) Params& KP;

__device__ __forceinline__ int lds_byte(int r, int c) { const int st = (r >> 4) * 2 + (c >> 5), rr = r & 15, cc = c & 31, ob = rr * 64 + cc * 2; return st * 1024 + (ob ^ (((ob >> 9) & 1) << 5)); }
__device__ __forceinline__ void stage_rc(int b, int& R, int& C) { const int st = b / 1024, sb = b % 1024, swz = sb ^ (((sb >> 9) & 1) << 5); R = (st >> 1) * 16 + swz / 64; C = (st & 1) * 32 + (swz % 64) / 2; }
__device__ __forceinline__ int perm32(int rho) { const int n = rho >> 4, i = rho & 15; return 8 * (i >> 2) + 4 * n + (i & 3); }

__device__ __forceinline__ unsigned ld_sys(const char* p) { return __hip_atomic_load((const unsigned*)p, __ATOMIC_RELAXED, __HIP_MEMORY_SCOPE_SYSTEM); }
__device__ __forceinline__ unsigned ld_agent(const char* p) { return __hip_atomic_load((const unsigned*)p, __ATOMIC_RELAXED, __HIP_MEMORY_SCOPE_AGENT); }
__device__ __forceinline__ void st_sys(char* p, unsigned v) { __hip_atomic_store((unsigned*)p, v, __ATOMIC_RELAXED, __HIP_MEMORY_SCOPE_SYSTEM); }
__device__ __forceinline__ void st_agent(char* p, unsigned v) { __hip_atomic_store((unsigned*)p, v, __ATOMIC_RELAXED, __HIP_MEMORY_SCOPE_AGENT); }
__device__ __forceinline__ unsigned cvt_pk_bf16(float lo, float hi) { unsigned r; asm("v_cvt_pk_bf16_f32 %0, %1, %2" : "=v"(r) : "v"(lo), "v"(hi)); return r; }
__device__ __forceinline__ float silu(float y) {
    const float t = fminf(fmaxf(y, -60.0f), 60.0f);
    const float e = __builtin_amdgcn_exp2f(t * -1.44269504088896f);
    return y * __builtin_amdgcn_rcpf(1.0f + e);
}
__device__ __forceinline__ __amdgpu_buffer_rsrc_t mk_rsrc(const void* p, unsigned bytes) {
    return __builtin_amdgcn_make_buffer_rsrc((void*)p, 0, bytes, 0x00020000);
}
#define WAIT_V(n) asm volatile("s_waitcnt vmcnt(" #n ")" ::: "memory")
#define WAIT_L(n) asm volatile("s_waitcnt lgkmcnt(" #n ")" ::: "memory")

#define STAMP(idx) do { } while (0)

__device__ __forceinline__ bool give_up(char* ws, unsigned call, unsigned long long t0) {
    const unsigned tm = __builtin_amdgcn_readfirstlane(ld_agent(ws + OFF_TMO));
    if (tm == call || __builtin_amdgcn_s_memrealtime() - t0 > DEADLINE) {
        if ((threadIdx.x & 63) == 0) st_agent(ws + OFF_TMO, call);
        return true;
    }
    return false;
}

struct Piece { int type; unsigned src, dst, flag; };

typedef unsigned long long u64;
__device__ __forceinline__ u64 uni64(const void* p) {
    const u64 a = (u64)p;
    return ((u64)(unsigned)__builtin_amdgcn_readfirstlane((unsigned)(a >> 32)) << 32) | (unsigned)__builtin_amdgcn_readfirstlane((unsigned)a);
}
__device__ __forceinline__ u32x4 mk_srd(u64 a) {
    u32x4 r;
    r.x = (unsigned)a;
    r.y = (unsigned)(a >> 32) & 0xffffu;
    r.z = 128u << 20;
    r.w = 0x00020000u;
    return r;
}
__device__ __forceinline__ unsigned selv(int t, unsigned a, unsigned b, unsigned c) {
    asm("" : "+v"(a)); asm("" : "+v"(b)); asm("" : "+v"(c));
    return t == 0 ? a : (t == 1 ? b : c);
}
__device__ __forceinline__ u64 sel3(int t, u64 a, u64 b, u64 c) {
    asm("" : "+s"(a)); asm("" : "+s"(b)); asm("" : "+s"(c));
    return t == 0 ? a : (t == 1 ? b : c);
}
#define CE_CLOB "memory", "scc", "vcc", "v157", "v158", "v159", "v160", "v161", "v162", "v163", "v164", "v165", "v166", "v167", "v168", "v169", "v170", "v171", "v172", "v173", "v174", "v175", "v176", "v177", "v178", "v179", "v180", "v181", "v182", "v183", "v184", "v185", "v186", "v187", "v188", "v189", "v190", "v191", "v192", "v193", "v194", "v195", "v196", "v197", "v198", "v199", "v200", "v201", "v202", "v203", "v204", "v205", "v206", "v207", "v208", "v209", "v210", "v211", "v212", "v213", "v214", "v215", "v216", "v217", "v218", "v219", "v220", "v221", "v222", "v223", "v224", "v225", "v226", "v227", "v228", "v229", "v230", "v231", "v232", "v233", "v234", "v235", "v236", "v237", "v238", "v239", "v240", "v241", "v242", "v243", "v244", "v245", "v246", "v247", "v248", "v249", "v250", "v251", "v252", "v253", "v254", "v255"
template <int R> __device__ __forceinline__ void ld8(unsigned voff, const u32x4& srd, unsigned soff, unsigned step) {
    unsigned t;
    asm volatile(
        "s_nop 4\n\t"
        "buffer_load_dwordx4 v[%c5:%c5+3], %1, %2, %3 offen sc1\n\t"
        "s_add_u32 %0, %3, %4\n\t"
        "buffer_load_dwordx4 v[%c5+4:%c5+7], %1, %2, %0 offen sc1\n\t"
        "s_add_u32 %0, %0, %4\n\t"
        "buffer_load_dwordx4 v[%c5+8:%c5+11], %1, %2, %0 offen sc1\n\t"
        "s_add_u32 %0, %0, %4\n\t"
        "buffer_load_dwordx4 v[%c5+12:%c5+15], %1, %2, %0 offen sc1\n\t"
        "s_add_u32 %0, %0, %4\n\t"
        "buffer_load_dwordx4 v[%c5+16:%c5+19], %1, %2, %0 offen sc1\n\t"
        "s_add_u32 %0, %0, %4\n\t"
        "buffer_load_dwordx4 v[%c5+20:%c5+23], %1, %2, %0 offen sc1\n\t"
        "s_add_u32 %0, %0, %4\n\t"
        "buffer_load_dwordx4 v[%c5+24:%c5+27], %1, %2, %0 offen sc1\n\t"
        "s_add_u32 %0, %0, %4\n\t"
        "buffer_load_dwordx4 v[%c5+28:%c5+31], %1, %2, %0 offen sc1"
        : "=&s"(t)
        : "v"(voff), "s"(srd), "s"(soff), "s"(step), "i"(R)
        : CE_CLOB);
}
template <int R> __device__ __forceinline__ void st8(unsigned voff, const u32x4& srd, unsigned soff, unsigned step) {
    unsigned t;
    asm volatile(
        "s_nop 4\n\t"
        "buffer_store_dwordx4 v[%c5:%c5+3], %1, %2, %3 offen sc0 sc1\n\t"
        "s_add_u32 %0, %3, %4\n\t"
        "buffer_store_dwordx4 v[%c5+4:%c5+7], %1, %2, %0 offen sc0 sc1\n\t"
        "s_add_u32 %0, %0, %4\n\t"
        "buffer_store_dwordx4 v[%c5+8:%c5+11], %1, %2, %0 offen sc0 sc1\n\t"
        "s_add_u32 %0, %0, %4\n\t"
        "buffer_store_dwordx4 v[%c5+12:%c5+15], %1, %2, %0 offen sc0 sc1\n\t"
        "s_add_u32 %0, %0, %4\n\t"
        "buffer_store_dwordx4 v[%c5+16:%c5+19], %1, %2, %0 offen sc0 sc1\n\t"
        "s_add_u32 %0, %0, %4\n\t"
        "buffer_store_dwordx4 v[%c5+20:%c5+23], %1, %2, %0 offen sc0 sc1\n\t"
        "s_add_u32 %0, %0, %4\n\t"
        "buffer_store_dwordx4 v[%c5+24:%c5+27], %1, %2, %0 offen sc0 sc1\n\t"
        "s_add_u32 %0, %0, %4\n\t"
        "buffer_store_dwordx4 v[%c5+28:%c5+31], %1, %2, %0 offen sc0 sc1\n\t"
        "s_nop 1"
        : "=&s"(t)
        : "v"(voff), "s"(srd), "s"(soff), "s"(step), "i"(R)
        : CE_CLOB);
}
template <int R> __device__ __forceinline__ void tr8() {
    asm volatile(
        "v_perm_b32 v224, v[%c2+4], v[%c2+0], %0\n\t"
        "v_perm_b32 v225, v[%c2+12], v[%c2+8], %0\n\t"
        "v_perm_b32 v226, v[%c2+20], v[%c2+16], %0\n\t"
        "v_perm_b32 v227, v[%c2+28], v[%c2+24], %0\n\t"
        "v_perm_b32 v228, v[%c2+4], v[%c2+0], %1\n\t"
        "v_perm_b32 v229, v[%c2+12], v[%c2+8], %1\n\t"
        "v_perm_b32 v230, v[%c2+20], v[%c2+16], %1\n\t"
        "v_perm_b32 v231, v[%c2+28], v[%c2+24], %1\n\t"
        "v_perm_b32 v232, v[%c2+5], v[%c2+1], %0\n\t"
        "v_perm_b32 v233, v[%c2+13], v[%c2+9], %0\n\t"
        "v_perm_b32 v234, v[%c2+21], v[%c2+17], %0\n\t"
        "v_perm_b32 v235, v[%c2+29], v[%c2+25], %0\n\t"
        "v_perm_b32 v236, v[%c2+5], v[%c2+1], %1\n\t"
        "v_perm_b32 v237, v[%c2+13], v[%c2+9], %1\n\t"
        "v_perm_b32 v238, v[%c2+21], v[%c2+17], %1\n\t"
        "v_perm_b32 v239, v[%c2+29], v[%c2+25], %1\n\t"
        "v_perm_b32 v240, v[%c2+6], v[%c2+2], %0\n\t"
        "v_perm_b32 v241, v[%c2+14], v[%c2+10], %0\n\t"
        "v_perm_b32 v242, v[%c2+22], v[%c2+18], %0\n\t"
        "v_perm_b32 v243, v[%c2+30], v[%c2+26], %0\n\t"
        "v_perm_b32 v244, v[%c2+6], v[%c2+2], %1\n\t"
        "v_perm_b32 v245, v[%c2+14], v[%c2+10], %1\n\t"
        "v_perm_b32 v246, v[%c2+22], v[%c2+18], %1\n\t"
        "v_perm_b32 v247, v[%c2+30], v[%c2+26], %1\n\t"
        "v_perm_b32 v248, v[%c2+7], v[%c2+3], %0\n\t"
        "v_perm_b32 v249, v[%c2+15], v[%c2+11], %0\n\t"
        "v_perm_b32 v250, v[%c2+23], v[%c2+19], %0\n\t"
        "v_perm_b32 v251, v[%c2+31], v[%c2+27], %0\n\t"
        "v_perm_b32 v252, v[%c2+7], v[%c2+3], %1\n\t"
        "v_perm_b32 v253, v[%c2+15], v[%c2+11], %1\n\t"
        "v_perm_b32 v254, v[%c2+23], v[%c2+19], %1\n\t"
        "v_perm_b32 v255, v[%c2+31], v[%c2+27], %1\n\t"
        "s_nop 1"
        :: "s"(0x05040100u), "s"(0x07060302u), "i"(R)
        : CE_CLOB);
}
__device__ __forceinline__ void ticket_issue(u64 ctr) {
    u64 sv;
    asm volatile(
        "s_mov_b64 %0, exec\n\t"
        "s_mov_b64 exec, 1\n\t"
        "v_mov_b32 v158, 0\n\t"
        "v_mov_b32 v159, 1\n\t"
        "s_nop 4\n\t"
        "global_atomic_add v159, v158, v159, %1 sc0\n\t"
        "s_mov_b64 exec, %0"
        : "=&s"(sv) : "s"(ctr) : CE_CLOB);
}
__device__ __forceinline__ int ticket_read() {
    int q;
    asm volatile("s_nop 1\n\tv_readfirstlane_b32 %0, v159\n\ts_nop 4" : "=s"(q) :: CE_CLOB);
    return q;
}
__device__ __forceinline__ void tflags_issue(u64 base, unsigned voff) {
    asm volatile(
        "s_nop 4\n\t"
        "global_load_dword v157, %0, %1 sc1\n\t"
        "global_load_dword v158, %0, %1 offset:256 sc1"
        :: "v"(voff), "s"(base) : CE_CLOB);
}
__device__ __forceinline__ int tflags_read(unsigned call) {
    u64 m0, m1;
    asm volatile("s_nop 1\n\tv_cmp_le_u32 %0, %2, v157\n\tv_cmp_le_u32 %1, %2, v158\n\ts_nop 4" : "=&s"(m0), "=&s"(m1) : "s"(call) : CE_CLOB);
    return (~m0 != 0ull) ? __builtin_ctzll(~m0) : 64 + ((~m1 != 0ull) ? __builtin_ctzll(~m1) : 64);
}
#define CE_WAIT(n) asm volatile("s_waitcnt vmcnt(" #n ")" ::: CE_CLOB)
constexpr int RA = 160, RB = 192, RT = 224;

struct Comm {
    KP p;
    int rank, dst, wv, lane;
    unsigned call;
    u64 rx, rw, ro_src, rws_dst, ro_dst;
    unsigned lin_off, w_src_off, w_dst_off, o_off;
    char* wsd;

    __device__ __forceinline__ Comm(KP p_, int dst_, int wv_, int lane_, unsigned call_) : p(p_), rank(p_.rank), dst(dst_), wv(wv_), lane(lane_), call(call_) {
        rx = uni64(p.x);
        rw = uni64(p.w);
        ro_src = uni64(p.out[rank]);
        wsd = p.ws[dst];
        rws_dst = uni64(wsd);
        ro_dst = uni64(p.out[dst]);
        lin_off = lane * 16;
        w_src_off = (lane >> 3) * 8 * 16384 + (lane & 7) * 16;
        w_dst_off = (lane & 7) * 8 * 2048 + (lane >> 3) * 16;
        o_off = (lane >> 5) * 16384 + (lane & 31) * 16;
    }
    __device__ __forceinline__ Piece desc(int q) const {
        Piece d;
        q = __builtin_amdgcn_readfirstlane(q);
        const int e = q >> 6, pc = q & 63;
        const bool isw = e < 8 ? (e & 1) : (e < 12);
        const int panel = e < 8 ? (e >> 1) : (e < 12 ? e - 4 : e - 8);
        const int kb = pc >> 2, nb = pc & 3;
        const int qq = q < N_IN ? 0 : q - N_IN, o = qq >> 4, po = qq & 15;
        const int t = (p.order[o >> 2] >> (8 * (o & 3))) & 255, pm = t >> 3, pn = t & 7;
        const unsigned xs = (unsigned)((dst >> 2) * 4096 + panel * 256) * 2048u + pc * 8192u;
        const unsigned xd = OFF_XS + rank * (8u << 20) + panel * (512u << 10) + pc * 8192u;
        const unsigned xf = OFF_XFLAG + ((rank * 16 + panel) * 64 + pc) * 4;
        const unsigned wsr = (unsigned)(kb * 64 * 8192 + (dst & 3) * 2048 + panel * 256 + nb * 64) * 2u;
        const unsigned wd = OFF_WTS + rank * (4u << 20) + (unsigned)((panel * 256 + nb * 64) * 1024 + kb * 64) * 2u;
        const unsigned wf = OFF_WFLAG + ((rank * 8 + panel) * 64 + pc) * 4;
        const unsigned os = (unsigned)((rank >> 2) * 4096 + pm * 256 + po * 16) * 16384u + (unsigned)((rank & 3) * 2048 + pn * 256) * 2u;
        d.type = q >= N_IN ? 2 : (isw ? 1 : 0);
        d.src = d.type == 0 ? xs : (d.type == 1 ? wsr : os);
        d.dst = d.type == 0 ? xd : (d.type == 1 ? wd : os);
        d.flag = d.type == 0 ? xf : (d.type == 1 ? wf : 0u);
        return d;
    }
    template <int R> __device__ __forceinline__ void load(const Piece& d) const {
        const unsigned voff = selv(d.type, lin_off, w_src_off, o_off);
        const unsigned step = d.type == 0 ? 1024u : (d.type == 1 ? 16384u : 32768u);
        ld8<R>(voff, mk_srd(sel3(d.type, rx, rw, ro_src)), d.src, step);
    }
    template <int R> __device__ __forceinline__ void store(const Piece& d) const {
        if (d.type == 1) {
            tr8<R>();
            st8<RT>(w_dst_off, mk_srd(rws_dst), d.dst, 2048u);
        } else {
            const unsigned voff = selv(d.type, lin_off, o_off, o_off);
            const unsigned step = d.type == 0 ? 1024u : 32768u;
            st8<R>(voff, mk_srd(sel3(d.type, rws_dst, rws_dst, ro_dst)), d.dst, step);
        }
    }
    __device__ __forceinline__ void flag(unsigned off) const {
        if (off != 0 && lane == 0) st_sys(wsd + off, call);
    }
};

__device__ __forceinline__ void comm_role(KP p, int cidx, unsigned call) {
    const int rank = p.rank;
    char* ws = p.ws[rank];
    const int lane = threadIdx.x & 63, wid = __builtin_amdgcn_readfirstlane(threadIdx.x >> 6);
    const int link = cidx % 7, wgl = cidx / 7;
    const int dst = (rank ^ (link + 1)) & (NR - 1);
    const int wv = wgl * WPW + wid;
    if (wid >= WPW) return;
    Comm c(p, dst, wv, lane, call);
    STAMP(1024 + (cidx * 8 + wid) * 4 + 0);
    constexpr int N_ALL = N_IN + N_OUT;
    const u64 ctr = uni64(ws + OFF_QCTR + link * 128), tfl = uni64(ws + OFF_TFLAG);
    const unsigned tvoff = lane * 4;
    const unsigned long long t0 = __builtin_amdgcn_s_memrealtime();
    int nready = 0;
    ticket_issue(ctr);
    CE_WAIT(0);
    int qn = ticket_read();
    while (qn < N_ALL) {
        while (qn >= N_IN && ((qn - N_IN) >> 4) >= nready) {
            tflags_issue(tfl, tvoff);
            CE_WAIT(0);
            nready = tflags_read(call);
            if (((qn - N_IN) >> 4) < nready) break;
            if (give_up(ws, call, t0)) { nready = NTILE; break; }
            __builtin_amdgcn_s_sleep(8);
        }
        int qc = qn, qp = N_IN;
        ticket_issue(ctr);
        c.load<RA>(c.desc(qc));
        CE_WAIT(8);
        qn = ticket_read();
        bool first = true;
#define CE_STEP(X, Y) { \
            if (!(qn < N_ALL && (qn < N_IN || ((qn - N_IN) >> 4) < nready))) { \
                CE_WAIT(0); c.flag(c.desc(qp).flag); c.store<X>(c.desc(qc)); CE_WAIT(0); c.flag(c.desc(qc).flag); break; } \
            const bool snap = nready < NTILE && qn >= N_IN - 2 * WPL; \
            ticket_issue(ctr); \
            if (snap) tflags_issue(tfl, tvoff); \
            c.load<Y>(c.desc(qn)); \
            if (first) CE_WAIT(9); else CE_WAIT(17); \
            first = false; \
            c.store<X>(c.desc(qc)); \
            CE_WAIT(16); \
            c.flag(c.desc(qp).flag); \
            qp = qc; qc = qn; \
            qn = ticket_read(); \
            if (snap) nready = max(nready, tflags_read(call)); }
        for (;;) {
            CE_STEP(RA, RB)
            CE_STEP(RB, RA)
        }
#undef CE_STEP
    }
    if (lane == 0) st_sys(c.wsd + OFF_OFLAG + (rank * 128 + wv) * 4, call);
    STAMP(1024 + (cidx * 8 + wid) * 4 + 2);
    if (wgl == 0 && wid == 0) {
        const unsigned long long t1 = __builtin_amdgcn_s_memrealtime();
        for (;;) {
            bool ok = true;
            if (lane < WPL) ok = ld_sys(ws + OFF_OFLAG + (dst * 128 + lane) * 4) >= call;
            if (WPL > 64 && lane + 64 < WPL) ok = ok && (ld_sys(ws + OFF_OFLAG + (dst * 128 + 64 + lane) * 4) >= call);
            if (__all(ok)) break;
            if (give_up(ws, call, t1)) break;
            __builtin_amdgcn_s_sleep(4);
        }
        STAMP(1024 + (cidx * 8 + wid) * 4 + 3);
    }
}

__device__ __forceinline__ void gemm_role(KP p, int g, int xcd, int slot, unsigned call, LAS unsigned char* lds) {
    const int rank = p.rank;
    char* ws = p.ws[rank];
    const int tid = threadIdx.x, wid = __builtin_amdgcn_readfirstlane(tid >> 6), lane = tid & 63, wr = wid >> 2, wc = wid & 3, fr = lane & 15, fq = lane >> 4;
    const int pm = (xcd >> 1) * 4 + (slot >> 2), pn = (xcd & 1) * 4 + (slot & 3);
    if (wid == 0) STAMP(g * 8 + 0);

    if (wid < 4) {
        Comm c(p, rank, 0, lane, call);
        const int id = g * 4 + wid, panel = id >> 6, pc = id & 63;
        const int e = panel < 4 ? 2 * panel + 1 : panel + 4;
        const Piece d = c.desc(e * 64 + pc);
        c.load<RA>(d);
        CE_WAIT(0);
        c.store<RA>(d);
        CE_WAIT(0);
        c.flag(d.flag);
    }

    unsigned voffA[2], voffB[2];
#pragma unroll
    for (int i = 0; i < 2; ++i) { int R, C; stage_rc(tid * 16 + i * 8192, R, C); const int Rb = (R & ~31) + perm32(R & 31);
        voffA[i] = (unsigned)(R * KS + C) * 2u; voffB[i] = (unsigned)(Rb * KS + C) * 2u; }
    constexpr size_t kstep = (size_t)(BK * 2);
    constexpr size_t hstep = (size_t)HALF * KS * 2;
    const unsigned ldsw = (unsigned)wid * 1024u;
    const int aoff = lds_byte(wr * 64 + fr, fq * 8), boff = lds_byte(wc * 32 + fr, fq * 8);
#define SA(b, h) (((b) * 2 + (h)) * HTB)
#define SB(b, h) ((4 + (b) * 2 + (h)) * HTB)
#define STAGE(bufoff, gbase, voff) do { _Pragma("unroll") for (int _i = 0; _i < 2; ++_i) \
        __builtin_amdgcn_global_load_lds((const unsigned*)((const char*)(gbase) + (voff)[_i]), (LAS unsigned*)(lds + (bufoff) + ldsw + _i * 8192), 16, 0, 0); } while (0)
#define LDA(dst, b, h) do { _Pragma("unroll") for (int m = 0; m < 4; ++m) _Pragma("unroll") for (int k = 0; k < 2; ++k) dst[m][k] = *(const LAS bf16x8*)(lds + SA(b, h) + aoff + m * 2048 + k * 1024); } while (0)
#define LDB(dst, b, h) do { _Pragma("unroll") for (int n = 0; n < 2; ++n) _Pragma("unroll") for (int k = 0; k < 2; ++k) dst[n][k] = *(const LAS bf16x8*)(lds + SB(b, h) + boff + n * 2048 + k * 1024); } while (0)
#define MMA(ai, bj, At, Bt) do { __builtin_amdgcn_s_setprio(1); _Pragma("unroll") for (int m = 0; m < 4; ++m) _Pragma("unroll") for (int n = 0; n < 2; ++n) _Pragma("unroll") for (int k = 0; k < 2; ++k) \
        acc[ai][bj][m][n] = __builtin_amdgcn_mfma_f32_16x16x32_bf16(Bt[n][k], At[m][k], acc[ai][bj][m][n], 0, 0, 0); __builtin_amdgcn_s_setprio(0); } while (0)
#define BAR __builtin_amdgcn_s_barrier()
#define SCHED __builtin_amdgcn_sched_barrier(0)

    auto hook = [&](bool remote) {
        if (wid == 0) {
            const unsigned long long t0 = __builtin_amdgcn_s_memrealtime();
            if (!remote) {
                for (;;) {
                    const bool ok = ld_sys(ws + OFF_WFLAG + ((rank * 8 + pn) * 64 + lane) * 4) >= call;
                    if (__all(ok)) break;
                    if (give_up(ws, call, t0)) break;
                    __builtin_amdgcn_s_sleep(4);
                }
            } else {
                const int s1 = rank ^ 1;
                for (;;) {
                    const bool ok = (ld_sys(ws + OFF_XFLAG + ((s1 * 16 + pm) * 64 + lane) * 4) >= call) & (ld_sys(ws + OFF_WFLAG + ((s1 * 8 + pn) * 64 + lane) * 4) >= call);
                    if (__all(ok)) break;
                    if (give_up(ws, call, t0)) break;
                    __builtin_amdgcn_s_sleep(16);
                }
                for (;;) {
                    bool ok = true;
#pragma unroll
                    for (int l = 2; l < 8; ++l) {
                        const int s = rank ^ l;
                        ok = ok & (ld_sys(ws + OFF_XFLAG + ((s * 16 + pm) * 64 + lane) * 4) >= call) & (ld_sys(ws + OFF_WFLAG + ((s * 8 + pn) * 64 + lane) * 4) >= call);
                    }
                    if (__all(ok)) break;
                    if (give_up(ws, call, t0)) break;
                    __builtin_amdgcn_s_sleep(8);
                }
            }
            __builtin_amdgcn_fence(__ATOMIC_ACQUIRE, "");
            asm volatile("s_waitcnt vmcnt(0)" ::: "memory");
        }
        asm volatile("" ::: "memory"); __builtin_amdgcn_s_barrier(); asm volatile("" ::: "memory");
    };

    f32x4 acc[2][2][4][2];
#pragma unroll
    for (int a = 0; a < 2; ++a)
#pragma unroll
        for (int b = 0; b < 2; ++b)
#pragma unroll
            for (int m = 0; m < 4; ++m)
#pragma unroll
                for (int n = 0; n < 2; ++n) acc[a][b][m][n] = (f32x4){0.f, 0.f, 0.f, 0.f};
    bf16x8 At[4][2], B0[2][2], B1[2][2];

    const char* xsA = ws + OFF_XS + (size_t)pm * (512u << 10);
    const char* wsB = ws + OFF_WTS + (size_t)pn * (512u << 10);
    const char* cA = p.x + (size_t)((rank >> 2) * 4096 + pm * 256) * 2048;
    const char* cB = wsB + (size_t)rank * (4u << 20);
    hook(false);
    if (wid == 0) STAMP(g * 8 + 1);
    STAGE(SB(0, 0), cB, voffB); STAGE(SA(0, 0), cA, voffA); STAGE(SB(0, 1), cB + hstep, voffB); STAGE(SA(0, 1), cA + hstep, voffA);
    if (wr == 1) BAR;
    WAIT_V(4); BAR;
    STAGE(SB(1, 0), cB + kstep, voffB); STAGE(SA(1, 0), cA + kstep, voffA); STAGE(SB(1, 1), cB + hstep + kstep, voffB);
    WAIT_V(6); BAR;
    constexpr int nt = KS / BK;
    for (int u = 0; u < NR; ++u) {
        const bool has_next = u + 1 < NR;
        const int sn = rank ^ (u + 1);
        const char* nA = has_next ? xsA + (size_t)sn * (8u << 20) : cA;
        const char* nB = has_next ? wsB + (size_t)sn * (4u << 20) : cB;
        for (int t = 0; t < nt; t += 2) {
            const bool last = (t == nt - 2);
            const char* a1 = cA + (size_t)(t + 1) * kstep;
            const char* a2 = last ? nA : cA + (size_t)(t + 2) * kstep; const char* b2 = last ? nB : cB + (size_t)(t + 2) * kstep;
            const char* a3 = a2 + kstep; const char* b3 = b2 + kstep;
            if (last && u == 0) {
                if (wid == 0) STAMP(g * 8 + 2);
                hook(true);
                if (wid == 0) STAMP(g * 8 + 3);
            }
            LDB(B0, 0, 0); SCHED; LDA(At, 0, 0); STAGE(SA(1, 1), a1 + hstep, voffA);
            WAIT_L(8); BAR; WAIT_L(0); MMA(0, 0, At, B0); BAR; SCHED;
            LDB(B1, 0, 1); STAGE(SB(0, 0), b2, voffB);
            BAR; WAIT_L(0); MMA(0, 1, At, B1); BAR;
            LDA(At, 0, 1); STAGE(SA(0, 0), a2, voffA);
            BAR; WAIT_L(0); MMA(1, 0, At, B0); BAR; SCHED;
            STAGE(SB(0, 1), b2 + hstep, voffB);
            WAIT_V(6); BAR; MMA(1, 1, At, B1); BAR;
            LDB(B0, 1, 0); SCHED; LDA(At, 1, 0); STAGE(SA(0, 1), a2 + hstep, voffA);
            WAIT_L(8); BAR; WAIT_L(0); MMA(0, 0, At, B0); BAR; SCHED;
            LDB(B1, 1, 1); STAGE(SB(1, 0), b3, voffB);
            BAR; WAIT_L(0); MMA(0, 1, At, B1); BAR;
            LDA(At, 1, 1); STAGE(SA(1, 0), a3, voffA);
            BAR; WAIT_L(0); MMA(1, 0, At, B0); BAR; SCHED;
            STAGE(SB(1, 1), b3 + hstep, voffB);
            WAIT_V(6); BAR; MMA(1, 1, At, B1); BAR;
        }
        cA = nA; cB = nB;
    }
    if (wid == 0) STAMP(g * 8 + 4);
    {
        const __amdgpu_buffer_rsrc_t ro = mk_rsrc(p.out[rank], 128u << 20);
        const unsigned base = (unsigned)((rank >> 2) * 4096 + pm * BM) * 16384u + (unsigned)((rank & 3) * 2048 + pn * BM) * 2u;
        const unsigned voff = (unsigned)(wr * 64 + fr) * 16384u + (unsigned)(wc * 32 + 8 * fq) * 2u;
#pragma unroll
        for (int ai = 0; ai < 2; ++ai)
#pragma unroll
            for (int m = 0; m < 4; ++m)
#pragma unroll
                for (int bj = 0; bj < 2; ++bj) {
                    const f32x4 v0 = acc[ai][bj][m][0], v1 = acc[ai][bj][m][1];
                    u32x4 o;
                    o.x = cvt_pk_bf16(silu(v0[0]), silu(v0[1])); o.y = cvt_pk_bf16(silu(v0[2]), silu(v0[3]));
                    o.z = cvt_pk_bf16(silu(v1[0]), silu(v1[1])); o.w = cvt_pk_bf16(silu(v1[2]), silu(v1[3]));
                    __builtin_amdgcn_raw_buffer_store_b128(o, ro, voff, base + (unsigned)(ai * HALF + m * 16) * 16384u + (unsigned)(bj * HALF) * 2u, 16);
                }
    }
    WAIT_V(0);
    if (wr == 0) BAR;
    BAR;
    if (tid == 0) st_agent(ws + OFF_TFLAG + ((p.pos[(pm * 8 + pn) >> 2] >> (8 * (pn & 3))) & 255) * 4, call);
    if (wid == 0) STAMP(g * 8 + 5);
#undef SA
#undef SB
#undef STAGE
#undef LDA
#undef LDB
#undef MMA
#undef BAR
#undef SCHED
}

__global__ __launch_bounds__(512, 2) void fused_gemm_allgather(const Params p_by_value) {
    KP p = *(const __attribute__((address_space(4))) Params*)__builtin_amdgcn_kernarg_segment_ptr();
    extern __shared__ __attribute__((aligned(16))) unsigned char lds_raw[];
    LAS unsigned char* lds = (LAS unsigned char*)lds_raw;
    const int b = blockIdx.x, xcd = b & 7, slot = b >> 3;
    char* ws = p.ws[p.rank];
    const unsigned call = __builtin_amdgcn_readfirstlane(ld_agent(ws + OFF_SEQ)) + 1u;
    if (slot >= 16) {
        gemm_role(p, (slot - 16) * 8 + xcd, xcd, slot - 16, call, lds);
    } else {
        const int cidx = slot * 8 + xcd;
        if (cidx < 7 * NCW) comm_role(p, cidx, call);
    }
    asm volatile("s_waitcnt vmcnt(0)" ::: "memory");
    __syncthreads();
    if (threadIdx.x == 0) {
        const unsigned t = __hip_atomic_fetch_add((unsigned*)(ws + OFF_DONE), 1u, __ATOMIC_RELAXED, __HIP_MEMORY_SCOPE_AGENT);
        if ((t + 1u) % (unsigned)GRID == 0u) {
            if (ld_agent(ws + OFF_TMO) == call) {
                const size_t off = (size_t)((p.rank >> 2) * 4096) * 16384 + (size_t)((p.rank & 3) * 2048) * 2;
                st_agent(p.out[p.rank] + off, 0x7fc07fc0u);
            }
            for (int l = 0; l < NR; ++l) st_agent(ws + OFF_QCTR + l * 128, 0u);
            st_agent(ws + OFF_SEQ, call);
        }
    }
}

}

extern "C" void kernel_launch(int rank, int nranks, ncclComm_t comm,
                              hipStream_t stream,
                              void* const* d_in, const int* in_sizes, int n_in,
                              void* d_out, int out_size,
                              const int* in_dtypes, int out_dtype,
                              void* const* d_out_all,
                              void* const* d_ws_all, size_t ws_bytes) {
    if (nranks != NR || (unsigned)rank >= (unsigned)NR || ws_bytes < WS_NEED || n_in < 2) {
        fprintf(stderr, "unsupported configuration\n");
        return;
    }
    static bool attr_set[NR] = {};
    if (!attr_set[rank]) {
        HIP_CHECK(hipFuncSetAttribute((const void*)fused_gemm_allgather, hipFuncAttributeMaxDynamicSharedMemorySize, LDS_BYTES));
        attr_set[rank] = true;
    }
    Params p;
    p.x = (const char*)d_in[0];
    p.w = (const char*)d_in[1];
    for (int r = 0; r < NR; ++r) { p.out[r] = (char*)d_out_all[r]; p.ws[r] = (char*)d_ws_all[r]; }
    p.rank = rank;
    {
        auto xpos = [](int pm) { return pm < 4 ? 2 * pm : pm + 8; };
        auto wpos = [](int pn) { return pn < 4 ? 2 * pn + 1 : pn + 4; };
        int n = 0;
        for (int e = 0; e < 24; ++e)
            for (int t = 0; t < NTILE; ++t) {
                const int a = xpos(t >> 3), bb = wpos(t & 7);
                if ((a > bb ? a : bb) == e) { ((unsigned char*)p.order)[n] = (unsigned char)t; ((unsigned char*)p.pos)[t] = (unsigned char)n; ++n; }
            }
    }
    hipLaunchKernelGGL(fused_gemm_allgather, dim3(GRID), dim3(512), LDS_BYTES, stream, p);
    HIP_CHECK(hipGetLastError());
}
```

```cpp
#include <hip/hip_runtime.h>
#include <rccl/rccl.h>
#include <cstdio>
#include <cstdint>

#define HIP_CHECK(expr) do { hipError_t _e = (expr); if (_e != hipSuccess) { \
    fprintf(stderr, "HIP error %s at %s:%d\n", hipGetErrorString(_e), __FILE__, __LINE__); \
    return; } } while (0)

namespace {

#define LAS __attribute__((address_space(3)))
typedef short bf16x8 __attribute__((ext_vector_type(8)));
typedef float f32x4 __attribute__((ext_vector_type(4)));
typedef unsigned u32x4 __attribute__((ext_vector_type(4)));
typedef unsigned long long u64;

constexpr int NR = 8;
#define PM_ 7
constexpr int PM = PM_;
constexpr int KS = 1024;
constexpr int NPOS = 128;
#define NCW 12
#define WPW 1
#define WPWP 1
#define NFW 90
#define FLAG_ 14
#define PSL_S 24
#define PSL_L 120
#define CSL 24
#define GPX 16
constexpr int WPL = NCW * WPW;
constexpr int GRID = 256;
constexpr int N_IN = 32 * 32;
constexpr int N_FIN = NPOS * 16;
constexpr int NF6 = NFW / 6;
constexpr int NSEQ = NPOS + NF6;
constexpr int N_ALL3 = N_IN + NSEQ * 16;
constexpr int N_ALLP = N_FIN + (NPOS - NFW) * 16;
static_assert(WPL <= 64 && NCW * WPWP <= 64 && NFW % 6 == 0 && NFW < NPOS && NF6 <= 32, "config");
static_assert(NCW <= 32 - GPX && GPX >= 16 && GPX <= 32, "too many workgroups");

constexpr unsigned OFF_SEQ = 0, OFF_DONE = 128, OFF_TMO = 256;
constexpr unsigned OFF_INF = 4096;
constexpr unsigned OFF_OWNW = OFF_INF + 6 * 32 * 32 * 4;
constexpr unsigned OFF_TFLAG = OFF_OWNW + 16 * 64 * 4;
constexpr unsigned OFF_SFLAG = OFF_TFLAG + 512;
constexpr unsigned OFF_PFLAG = OFF_SFLAG + 512;
constexpr unsigned OFF_FCNT = OFF_PFLAG + 128 * 16 * 4;
constexpr unsigned OFF_OCNT = OFF_FCNT + 6 * 64 * 4;
constexpr unsigned OFF_QCTR = OFF_OCNT + 8 * 128;
constexpr unsigned OFF_DBG = 128 * 1024;
constexpr unsigned OFF_XS = 1u << 20;
constexpr unsigned OFF_WTS = OFF_XS + (24u << 20);
constexpr unsigned OFF_PS = OFF_WTS + (32u << 20);
constexpr unsigned OFF_PR = OFF_PS + (16u << 20);
constexpr unsigned WS_NEED = OFF_PR + (16u << 20);
static_assert(OFF_QCTR + 16 * 128 <= OFF_DBG && WS_NEED <= (128u << 20), "layout");
constexpr unsigned long long DEADLINE = 40000000ull;

constexpr int BM = 256, BK = 64, HALF = 128, HTB = HALF * BK * 2, STAGE_BYTES = 8 * HTB;
constexpr int LDS_BYTES = STAGE_BYTES;

struct Params {
    const char* x;
    const char* w;
    char* out[NR];
    char* ws[NR];
    int rank;
    unsigned forder[2][NPOS / 4];
    unsigned pos[256 / 4];
    unsigned xq[8][32 / 4];
    unsigned seq[160 / 4];
    unsigned spf[132 / 4];
    unsigned xlink[2];
    unsigned spw[36 / 4];
};
typedef const __attribute__((address_space(4))) Params& KP;

__host__ __device__ __forceinline__ int role_of(int r) { return (r >> 2) & 1; }
__host__ __device__ __forceinline__ int quad_of(int r) { return role_of(r) ? (r ^ PM) : r; }
__host__ __device__ __forceinline__ int uidx(int m) { return m < PM ? m - 1 : m - 2; }

__device__ __forceinline__ int lds_byte(int r, int c) { const int st = (r >> 4) * 2 + (c >> 5), rr = r & 15, cc = c & 31, ob = rr * 64 + cc * 2; return st * 1024 + (ob ^ (((ob >> 9) & 1) << 5)); }
__device__ __forceinline__ void stage_rc(int b, int& R, int& C) { const int st = b / 1024, sb = b % 1024, swz = sb ^ (((sb >> 9) & 1) << 5); R = (st >> 1) * 16 + swz / 64; C = (st & 1) * 32 + (swz % 64) / 2; }
__device__ __forceinline__ int perm32(int rho) { const int n = rho >> 4, i = rho & 15; return 8 * (i >> 2) + 4 * n + (i & 3); }

__device__ __forceinline__ unsigned ld_sys(const char* p) { return __hip_atomic_load((const unsigned*)p, __ATOMIC_RELAXED, __HIP_MEMORY_SCOPE_SYSTEM); }
__device__ __forceinline__ unsigned ld_agent(const char* p) { return __hip_atomic_load((const unsigned*)p, __ATOMIC_RELAXED, __HIP_MEMORY_SCOPE_AGENT); }
__device__ __forceinline__ void st_sys(char* p, unsigned v) { __hip_atomic_store((unsigned*)p, v, __ATOMIC_RELAXED, __HIP_MEMORY_SCOPE_SYSTEM); }
__device__ __forceinline__ void st_agent(char* p, unsigned v) { __hip_atomic_store((unsigned*)p, v, __ATOMIC_RELAXED, __HIP_MEMORY_SCOPE_AGENT); }
__device__ __forceinline__ unsigned cvt_pk_bf16(float lo, float hi) { unsigned r; asm("v_cvt_pk_bf16_f32 %0, %1, %2" : "=v"(r) : "v"(lo), "v"(hi)); return r; }
__device__ __forceinline__ float bf_lo(unsigned u) { return __uint_as_float(u << 16); }
__device__ __forceinline__ float bf_hi(unsigned u) { return __uint_as_float(u & 0xffff0000u); }
__device__ __forceinline__ float silu(float y) {
    const float t = fminf(fmaxf(y, -60.0f), 60.0f);
    const float e = __builtin_amdgcn_exp2f(t * -1.44269504088896f);
    return y * __builtin_amdgcn_rcpf(1.0f + e);
}
__device__ __forceinline__ __amdgpu_buffer_rsrc_t mk_rsrc(const void* p, unsigned bytes) {
    return __builtin_amdgcn_make_buffer_rsrc((void*)p, 0, bytes, 0x00020000);
}
template <class T> __device__ __forceinline__ int tab8(const T& t, int i) { return (t[i >> 2] >> (8 * (i & 3))) & 255; }
#define WAIT_V(n) asm volatile("s_waitcnt vmcnt(" #n ")" ::: "memory")
#define WAIT_L(n) asm volatile("s_waitcnt lgkmcnt(" #n ")" ::: "memory")

#define STAMP(idx) do { } while (0)

__device__ __forceinline__ bool give_up(char* ws, unsigned call, unsigned long long t0) {
    const unsigned tm = __builtin_amdgcn_readfirstlane(ld_agent(ws + OFF_TMO));
    if (tm == call || __builtin_amdgcn_s_memrealtime() - t0 > DEADLINE) {
        if ((threadIdx.x & 63) == 0) st_agent(ws + OFF_TMO, call);
        return true;
    }
    return false;
}

__device__ __forceinline__ u64 uni64(const void* p) {
    const u64 a = (u64)p;
    return ((u64)(unsigned)__builtin_amdgcn_readfirstlane((unsigned)(a >> 32)) << 32) | (unsigned)__builtin_amdgcn_readfirstlane((unsigned)a);
}
__device__ __forceinline__ u32x4 mk_srd(u64 a) {
    u32x4 r;
    r.x = (unsigned)a;
    r.y = (unsigned)(a >> 32) & 0xffffu;
    r.z = 128u << 20;
    r.w = 0x00020000u;
    return r;
}
#define CE_CLOB "memory", "scc", "vcc", "v156", "v157", "v158", "v159", "v160", "v161", "v162", "v163", "v164", "v165", "v166", "v167", "v168", "v169", "v170", "v171", "v172", "v173", "v174", "v175", "v176", "v177", "v178", "v179", "v180", "v181", "v182", "v183", "v184", "v185", "v186", "v187", "v188", "v189", "v190", "v191", "v192", "v193", "v194", "v195", "v196", "v197", "v198", "v199", "v200", "v201", "v202", "v203", "v204", "v205", "v206", "v207", "v208", "v209", "v210", "v211", "v212", "v213", "v214", "v215", "v216", "v217", "v218", "v219", "v220", "v221", "v222", "v223", "v224", "v225", "v226", "v227", "v228", "v229", "v230", "v231", "v232", "v233", "v234", "v235", "v236", "v237", "v238", "v239", "v240", "v241", "v242", "v243", "v244", "v245", "v246", "v247", "v248", "v249", "v250", "v251", "v252", "v253", "v254", "v255"
#define LDF "sc0 sc1"
template <int R> __device__ __forceinline__ void ld8(unsigned voff, const u32x4& srd, unsigned soff, unsigned step) {
    unsigned t;
    asm volatile(
        "s_nop 4\n\t"
        "buffer_load_dwordx4 v[%c5+0:%c5+3], %1, %2, %3 offen " LDF "\n\t"
        "s_add_u32 %0, %3, %4\n\t"
        "buffer_load_dwordx4 v[%c5+4:%c5+7], %1, %2, %0 offen " LDF "\n\t"
        "s_add_u32 %0, %0, %4\n\t"
        "buffer_load_dwordx4 v[%c5+8:%c5+11], %1, %2, %0 offen " LDF "\n\t"
        "s_add_u32 %0, %0, %4\n\t"
        "buffer_load_dwordx4 v[%c5+12:%c5+15], %1, %2, %0 offen " LDF "\n\t"
        "s_add_u32 %0, %0, %4\n\t"
        "buffer_load_dwordx4 v[%c5+16:%c5+19], %1, %2, %0 offen " LDF "\n\t"
        "s_add_u32 %0, %0, %4\n\t"
        "buffer_load_dwordx4 v[%c5+20:%c5+23], %1, %2, %0 offen " LDF "\n\t"
        "s_add_u32 %0, %0, %4\n\t"
        "buffer_load_dwordx4 v[%c5+24:%c5+27], %1, %2, %0 offen " LDF "\n\t"
        "s_add_u32 %0, %0, %4\n\t"
        "buffer_load_dwordx4 v[%c5+28:%c5+31], %1, %2, %0 offen " LDF "\n\t"
        "s_nop 0"
        : "=&s"(t)
        : "v"(voff), "s"(srd), "s"(soff), "s"(step), "i"(R)
        : CE_CLOB);
}
#define SWIN 4
#define SWN(j) ((j) < SWIN ? SWIN + 8 : SWIN - 1)
template <int R> __device__ __forceinline__ void st8(unsigned voff, const u32x4& srd, unsigned soff, unsigned step) {
    unsigned t;
    asm volatile(
        "s_nop 4\n\t"
        "s_waitcnt vmcnt(%c6)\n\t"
        "buffer_store_dwordx4 v[%c5+0:%c5+3], %1, %2, %3 offen sc0 sc1\n\t"
        "s_add_u32 %0, %3, %4\n\t"
        "s_waitcnt vmcnt(%c7)\n\t"
        "buffer_store_dwordx4 v[%c5+4:%c5+7], %1, %2, %0 offen sc0 sc1\n\t"
        "s_add_u32 %0, %0, %4\n\t"
        "s_waitcnt vmcnt(%c8)\n\t"
        "buffer_store_dwordx4 v[%c5+8:%c5+11], %1, %2, %0 offen sc0 sc1\n\t"
        "s_add_u32 %0, %0, %4\n\t"
        "s_waitcnt vmcnt(%c9)\n\t"
        "buffer_store_dwordx4 v[%c5+12:%c5+15], %1, %2, %0 offen sc0 sc1\n\t"
        "s_add_u32 %0, %0, %4\n\t"
        "s_waitcnt vmcnt(%c10)\n\t"
        "buffer_store_dwordx4 v[%c5+16:%c5+19], %1, %2, %0 offen sc0 sc1\n\t"
        "s_add_u32 %0, %0, %4\n\t"
        "s_waitcnt vmcnt(%c11)\n\t"
        "buffer_store_dwordx4 v[%c5+20:%c5+23], %1, %2, %0 offen sc0 sc1\n\t"
        "s_add_u32 %0, %0, %4\n\t"
        "s_waitcnt vmcnt(%c12)\n\t"
        "buffer_store_dwordx4 v[%c5+24:%c5+27], %1, %2, %0 offen sc0 sc1\n\t"
        "s_add_u32 %0, %0, %4\n\t"
        "s_waitcnt vmcnt(%c13)\n\t"
        "buffer_store_dwordx4 v[%c5+28:%c5+31], %1, %2, %0 offen sc0 sc1\n\t"
        "s_nop 1"
        : "=&s"(t)
        : "v"(voff), "s"(srd), "s"(soff), "s"(step), "i"(R), "i"(SWN(0)), "i"(SWN(1)), "i"(SWN(2)), "i"(SWN(3)), "i"(SWN(4)), "i"(SWN(5)), "i"(SWN(6)), "i"(SWN(7))
        : CE_CLOB);
}
template <int R> __device__ __forceinline__ void tr8() {
    asm volatile(
        "v_perm_b32 v224, v[%c2+4], v[%c2+0], %0\n\t"
        "v_perm_b32 v225, v[%c2+12], v[%c2+8], %0\n\t"
        "v_perm_b32 v226, v[%c2+20], v[%c2+16], %0\n\t"
        "v_perm_b32 v227, v[%c2+28], v[%c2+24], %0\n\t"
        "v_perm_b32 v228, v[%c2+4], v[%c2+0], %1\n\t"
        "v_perm_b32 v229, v[%c2+12], v[%c2+8], %1\n\t"
        "v_perm_b32 v230, v[%c2+20], v[%c2+16], %1\n\t"
        "v_perm_b32 v231, v[%c2+28], v[%c2+24], %1\n\t"
        "v_perm_b32 v232, v[%c2+5], v[%c2+1], %0\n\t"
        "v_perm_b32 v233, v[%c2+13], v[%c2+9], %0\n\t"
        "v_perm_b32 v234, v[%c2+21], v[%c2+17], %0\n\t"
        "v_perm_b32 v235, v[%c2+29], v[%c2+25], %0\n\t"
        "v_perm_b32 v236, v[%c2+5], v[%c2+1], %1\n\t"
        "v_perm_b32 v237, v[%c2+13], v[%c2+9], %1\n\t"
        "v_perm_b32 v238, v[%c2+21], v[%c2+17], %1\n\t"
        "v_perm_b32 v239, v[%c2+29], v[%c2+25], %1\n\t"
        "v_perm_b32 v240, v[%c2+6], v[%c2+2], %0\n\t"
        "v_perm_b32 v241, v[%c2+14], v[%c2+10], %0\n\t"
        "v_perm_b32 v242, v[%c2+22], v[%c2+18], %0\n\t"
        "v_perm_b32 v243, v[%c2+30], v[%c2+26], %0\n\t"
        "v_perm_b32 v244, v[%c2+6], v[%c2+2], %1\n\t"
        "v_perm_b32 v245, v[%c2+14], v[%c2+10], %1\n\t"
        "v_perm_b32 v246, v[%c2+22], v[%c2+18], %1\n\t"
        "v_perm_b32 v247, v[%c2+30], v[%c2+26], %1\n\t"
        "v_perm_b32 v248, v[%c2+7], v[%c2+3], %0\n\t"
        "v_perm_b32 v249, v[%c2+15], v[%c2+11], %0\n\t"
        "v_perm_b32 v250, v[%c2+23], v[%c2+19], %0\n\t"
        "v_perm_b32 v251, v[%c2+31], v[%c2+27], %0\n\t"
        "v_perm_b32 v252, v[%c2+7], v[%c2+3], %1\n\t"
        "v_perm_b32 v253, v[%c2+15], v[%c2+11], %1\n\t"
        "v_perm_b32 v254, v[%c2+23], v[%c2+19], %1\n\t"
        "v_perm_b32 v255, v[%c2+31], v[%c2+27], %1\n\t"
        "s_nop 1"
        :: "s"(0x05040100u), "s"(0x07060302u), "i"(R)
        : CE_CLOB);
}
__device__ __forceinline__ void ticket_issue(u64 ctr) {
    u64 sv;
    asm volatile(
        "s_mov_b64 %0, exec\n\t"
        "s_mov_b64 exec, 1\n\t"
        "v_mov_b32 v158, 0\n\t"
        "v_mov_b32 v159, 1\n\t"
        "s_nop 4\n\t"
        "global_atomic_add v159, v158, v159, %1 sc0\n\t"
        "s_mov_b64 exec, %0"
        : "=&s"(sv) : "s"(ctr) : CE_CLOB);
}
__device__ __forceinline__ int ticket_read() {
    int q;
    asm volatile("s_nop 1\n\tv_readfirstlane_b32 %0, v159\n\ts_nop 4" : "=s"(q) :: CE_CLOB);
    return q;
}
__device__ __forceinline__ void tflags_issue(u64 base, unsigned voff) {
    asm volatile(
        "s_nop 4\n\t"
        "global_load_dword v157, %0, %1 sc1\n\t"
        "global_load_dword v158, %0, %1 offset:256 sc1"
        :: "v"(voff), "s"(base) : CE_CLOB);
}
__device__ __forceinline__ int tflags_read(unsigned call) {
    u64 m0, m1;
    asm volatile("s_nop 1\n\tv_cmp_le_u32 %0, %2, v157\n\tv_cmp_le_u32 %1, %2, v158\n\ts_nop 4" : "=&s"(m0), "=&s"(m1) : "s"(call) : CE_CLOB);
    return (~m0 != 0ull) ? __builtin_ctzll(~m0) : 64 + ((~m1 != 0ull) ? __builtin_ctzll(~m1) : 64);
}
__device__ __forceinline__ void fcnt_issue(u64 base, unsigned voff) {
    asm volatile("s_nop 4\n\tglobal_load_dword v156, %0, %1 sc0 sc1" :: "v"(voff), "s"(base) : CE_CLOB);
}
__device__ __forceinline__ int fcnt_read(unsigned need) {
    u64 m0;
    asm volatile("s_nop 1\n\tv_cmp_le_u32 %0, %1, v156\n\ts_nop 4" : "=&s"(m0) : "s"(need) : CE_CLOB);
    return (~m0 != 0ull) ? __builtin_ctzll(~m0) : 64;
}
#define CE_WAIT(n) asm volatile("s_waitcnt vmcnt(" #n ")" ::: CE_CLOB)
#define CE_WAITX(n) CE_WAIT(n)
#define SDEPTH 17
constexpr int RA = 160, RB = 192, RT = 224;

__device__ __forceinline__ unsigned selv(bool c0, bool c1, unsigned a, unsigned b, unsigned c) {
    asm("" : "+v"(a)); asm("" : "+v"(b)); asm("" : "+v"(c));
    return c0 ? a : (c1 ? b : c);
}
__device__ __forceinline__ u64 sels(bool c, u64 a, u64 b) {
    asm("" : "+s"(a)); asm("" : "+s"(b));
    return uni64((const void*)(c ? a : b));
}

struct Piece { int type; unsigned src, dst, flag; };

struct Comm {
    KP p;
    int rank, dst, lane;
    unsigned call;
    bool partner;
    u64 rx, rw, ro_src, rws_src, rws_dst, ro_dst;
    unsigned lin_off, w_src_off, w_dst_off, o_off;
    char* wsd;

    __device__ __forceinline__ Comm(KP p_, int dst_, int lane_, unsigned call_) : p(p_), rank(p_.rank), dst(dst_), lane(lane_), call(call_) {
        partner = (rank ^ dst) == PM;
        rx = uni64(p.x);
        rw = uni64(p.w);
        ro_src = uni64(p.out[rank]);
        rws_src = uni64(p.ws[rank]);
        wsd = p.ws[dst];
        rws_dst = uni64(wsd);
        ro_dst = uni64(p.out[dst]);
        lin_off = lane * 16;
        w_src_off = (lane >> 3) * 8 * 16384 + (lane & 7) * 16;
        w_dst_off = (lane & 7) * 8 * 2048 + (lane >> 3) * 16;
        o_off = (lane >> 5) * 16384 + (lane & 31) * 16;
    }
    __device__ __forceinline__ unsigned out_off(int a, int o, int po) const {
        const int t = tab8(p.forder[role_of(a)], o), pm = t >> 4, pn = t & 15, qd = quad_of(a);
        return (unsigned)((qd >> 1) * 4096 + pm * 256 + po * 16) * 16384u + (unsigned)((qd & 1) * 4096 + pn * 256) * 2u;
    }
    __device__ __forceinline__ Piece desc(int q) const {
        Piece d;
        q = __builtin_amdgcn_readfirstlane(q);
        if (partner) {
            if (q < N_FIN) {
                d.type = 3;
                d.src = OFF_PS + q * 8192u; d.dst = OFF_PR + q * 8192u; d.flag = OFF_PFLAG + q * 4;
            } else {
                const int qq = q - N_FIN;
                d.type = 2;
                d.src = d.dst = out_off(rank, NFW + (qq >> 4), qq & 15); d.flag = 0;
            }
        } else {
            const int u = uidx(rank ^ dst);
            if (q < N_IN) {
                const int e = q >> 5, pc = q & 31, panel = e >> 1, hd = role_of(dst), qd = quad_of(dst);
                d.flag = OFF_INF + (u * 1024 + q) * 4;
                if (!(e & 1)) {
                    d.type = 0;
                    d.src = (unsigned)((qd >> 1) * 4096 + panel * 256 + pc * 8) * 2048u + hd * 1024u;
                    d.dst = OFF_XS + (u >> 1) * (8u << 20) + (unsigned)(panel * 256 + pc * 8) * 2048u + (u & 1) * 1024u;
                } else {
                    const int kb = pc >> 2, nb = pc & 3;
                    d.type = 1;
                    d.src = (unsigned)((hd * 512 + kb * 64) * 8192 + (qd & 1) * 4096 + panel * 256 + nb * 64) * 2u;
                    d.dst = OFF_WTS + (u >> 1) * (8u << 20) + (unsigned)((panel * 256 + nb * 64) * 1024 + (u & 1) * 512 + kb * 64) * 2u;
                }
            } else {
                const int qq = q - N_IN, sq = tab8(p.seq, qq >> 4), po = qq & 15;
                d.type = 2;
                if (sq < NPOS) {
                    d.src = d.dst = out_off(rank, sq, po);
                    d.flag = (sq < NFW && sq % 6 == u) ? (OFF_FCNT + (u * 64 + sq / 6) * 4) | 1u : 0u;
                } else {
                    const int a = dst ^ PM;
                    d.src = d.dst = out_off(a, uidx(a ^ rank) + 6 * (sq - NPOS), po); d.flag = 0;
                }
            }
        }
        d.type = __builtin_amdgcn_readfirstlane(d.type);
        d.src = __builtin_amdgcn_readfirstlane(d.src);
        d.dst = __builtin_amdgcn_readfirstlane(d.dst);
        d.flag = __builtin_amdgcn_readfirstlane(d.flag);
        return d;
    }
    template <int R> __device__ __forceinline__ void load(const Piece& d) const {
        const unsigned voff = selv(d.type == 1, d.type == 2, w_src_off, o_off, lin_off);
        const unsigned step = d.type == 0 ? 2048u : (d.type == 1 ? 16384u : (d.type == 2 ? 32768u : 1024u));
        const u64 base = sels(d.type < 2, sels(d.type == 0, rx, rw), sels(d.type == 2, ro_src, rws_src));
        ld8<R>(voff, mk_srd(base), d.src, step);
    }
    template <int R> __device__ __forceinline__ void store(const Piece& d) const {
        if (d.type == 1) {
            tr8<R>();
            st8<RT>(w_dst_off, mk_srd(rws_dst), d.dst, 2048u);
        } else {
            const unsigned voff = selv(d.type == 2, false, o_off, lin_off, lin_off);
            const unsigned step = d.type == 0 ? 2048u : (d.type == 2 ? 32768u : 1024u);
            st8<R>(voff, mk_srd(sels(d.type == 2, ro_dst, rws_dst)), d.dst, step);
        }
    }
    __device__ __forceinline__ void flag(unsigned off) const {
        if (off != 0 && lane == 0) {
            if (off & 1u) __hip_atomic_fetch_add((unsigned*)(wsd + (off & ~1u)), 1u, __ATOMIC_RELAXED, __HIP_MEMORY_SCOPE_SYSTEM);
            else st_sys(wsd + off, call);
        }
    }
};

__device__ __forceinline__ void comm_role(KP p, int dst, int slot, int cidx, unsigned call) {
    const int rank = p.rank;
    char* ws = p.ws[rank];
    const int lane = threadIdx.x & 63, wid = __builtin_amdgcn_readfirstlane(threadIdx.x >> 6);
    if ((unsigned)dst >= (unsigned)NR || dst == rank) return;
    const int wpw = ((rank ^ dst) == PM) ? WPWP : WPW, wpl = wpw * NCW;
    if (wid >= wpw) return;
    Comm c(p, dst, lane, call);
    STAMP(4096 + (cidx * 8 + wid) * 4 + 0);
    const bool partner = c.partner;
    const int n_all = partner ? N_ALLP : N_ALL3;
    const int q_none = n_all - 1;
    const u64 ctr = uni64(ws + OFF_QCTR + dst * 128), tfl = uni64(ws + OFF_TFLAG), sfl = uni64(ws + OFF_SFLAG);
    const u64 fcn = uni64(ws + OFF_FCNT + (partner ? 0 : uidx((dst ^ PM) ^ rank)) * 256);
    const unsigned tvoff = lane * 4;
    const unsigned long long t0 = __builtin_amdgcn_s_memrealtime();
    unsigned ndone = 0;
    int qready = partner ? 0 : N_IN;
#define SNAP_ISSUE() do { tflags_issue(sels(partner && qready < N_FIN, sfl, tfl), tvoff); fcnt_issue(fcn, tvoff); } while (0)
#define SNAP_APPLY() do { const int n_ = tflags_read(call), w_ = min(fcnt_read(call * 16u), NF6); \
        const int r_ = partner ? (qready < N_FIN ? 16 * n_ : N_FIN + 16 * (n_ - NFW)) : N_IN + 16 * min(tab8(p.spf, n_), tab8(p.spw, w_)); \
        qready = __builtin_amdgcn_readfirstlane(max(qready, r_)); } while (0)
    ticket_issue(ctr);
    CE_WAIT(0);
    int qn = ticket_read();
    while (qn < n_all) {
        while (qn >= qready) {
            SNAP_ISSUE();
            CE_WAIT(0);
            const int before = qready;
            SNAP_APPLY();
            if (qready != before) continue;
            if (give_up(ws, call, t0)) { qready = n_all; break; }
            __builtin_amdgcn_s_sleep(CSL);
        }
        qready = __builtin_amdgcn_readfirstlane(qready);
        int qc = qn, qp = q_none;
        ticket_issue(ctr);
        c.load<RA>(c.desc(qc));
        CE_WAIT(8);
        qn = ticket_read();
        bool first = true;
#define CE_STEP(X, Y) { \
            if (!(qn < n_all && qn < qready)) { \
                CE_WAIT(0); c.flag(c.desc(qp).flag); c.store<X>(c.desc(qc)); CE_WAIT(0); c.flag(c.desc(qc).flag); ++ndone; break; } \
            const bool snap = qready < n_all && qn + 3 * wpl >= qready; \
            ticket_issue(ctr); \
            if (snap) SNAP_ISSUE(); \
            c.load<Y>(c.desc(qn)); \
            if (first) CE_WAIT(9); else CE_WAITX(SDEPTH); \
            first = false; \
            c.store<X>(c.desc(qc)); \
            CE_WAIT(16); \
            c.flag(c.desc(qp).flag); \
            qp = qc; qc = qn; ++ndone; \
            qn = ticket_read(); \
            if (snap) SNAP_APPLY(); }
        for (;;) {
            CE_STEP(RA, RB)
            CE_STEP(RB, RA)
        }
#undef CE_STEP
    }
    if (lane == 0 && ndone != 0) __hip_atomic_fetch_add((unsigned*)(c.wsd + OFF_OCNT + rank * 128), ndone, __ATOMIC_RELAXED, __HIP_MEMORY_SCOPE_SYSTEM);
    STAMP(4096 + (cidx * 8 + wid) * 4 + 2);
    if (slot == 0 && wid == 0) {
        const unsigned long long t1 = __builtin_amdgcn_s_memrealtime();
        for (;;) {
            if (__builtin_amdgcn_readfirstlane(ld_sys(ws + OFF_OCNT + dst * 128)) >= call * (unsigned)n_all) break;
            if (give_up(ws, call, t1)) break;
            __builtin_amdgcn_s_sleep(4);
        }
        STAMP(4096 + (cidx * 8 + wid) * 4 + 3);
    }
}

__device__ __forceinline__ void gemm_role(KP p, int xcd, int slot, unsigned call, LAS unsigned char* lds) {
    const int rank = p.rank;
    char* ws = p.ws[rank];
    const int tid = threadIdx.x, wid = __builtin_amdgcn_readfirstlane(tid >> 6), lane = tid & 63, wr = wid >> 2, wc = wid & 3, fr = lane & 15, fq = lane >> 4;
    const int qd = quad_of(rank), role = role_of(rank);

    if (slot < 16) {
        const int id = (slot * 8 + xcd) * 8 + wid, panel = id >> 6, pc = id & 63, kb = pc >> 2, nb = pc & 3;
        const unsigned w_src_off = (lane >> 3) * 8 * 16384 + (lane & 7) * 16, w_dst_off = (lane & 7) * 8 * 2048 + (lane >> 3) * 16;
        ld8<RA>(w_src_off, mk_srd(uni64(p.w)), (unsigned)(kb * 64 * 8192 + (qd & 1) * 4096 + panel * 256 + nb * 64) * 2u, 16384u);
        CE_WAIT(0);
        tr8<RA>();
        st8<RT>(w_dst_off, mk_srd(uni64(ws)), OFF_WTS + 3 * (8u << 20) + (unsigned)((panel * 256 + nb * 64) * 1024 + kb * 64) * 2u, 2048u);
        CE_WAIT(0);
        if (lane == 0) st_sys(ws + OFF_OWNW + id * 4, call);
    }

    unsigned voffA[2], voffB[2];
#pragma unroll
    for (int i = 0; i < 2; ++i) { int R, C; stage_rc(tid * 16 + i * 8192, R, C); const int Rb = (R & ~31) + perm32(R & 31);
        voffA[i] = (unsigned)(R * KS + C) * 2u; voffB[i] = (unsigned)(Rb * KS + C) * 2u; }
    constexpr size_t kstep = (size_t)(BK * 2);
    constexpr size_t hstep = (size_t)HALF * KS * 2;
    const unsigned ldsw = (unsigned)wid * 1024u;
    const int aoff = lds_byte(wr * 64 + fr, fq * 8), boff = lds_byte(wc * 32 + fr, fq * 8);
#define SA(b, h) (((b) * 2 + (h)) * HTB)
#define SB(b, h) ((4 + (b) * 2 + (h)) * HTB)
#define STAGE(bufoff, gbase, voff) do { _Pragma("unroll") for (int _i = 0; _i < 2; ++_i) \
        __builtin_amdgcn_global_load_lds((const unsigned*)((const char*)(gbase) + (voff)[_i]), (LAS unsigned*)(lds + (bufoff) + ldsw + _i * 8192), 16, 0, 0); } while (0)
#define LDA(dst, b, h) do { _Pragma("unroll") for (int m = 0; m < 4; ++m) _Pragma("unroll") for (int k = 0; k < 2; ++k) dst[m][k] = *(const LAS bf16x8*)(lds + SA(b, h) + aoff + m * 2048 + k * 1024); } while (0)
#define LDB(dst, b, h) do { _Pragma("unroll") for (int n = 0; n < 2; ++n) _Pragma("unroll") for (int k = 0; k < 2; ++k) dst[n][k] = *(const LAS bf16x8*)(lds + SB(b, h) + boff + n * 2048 + k * 1024); } while (0)
#define MMA(ai, bj, At, Bt) do { __builtin_amdgcn_s_setprio(1); _Pragma("unroll") for (int m = 0; m < 4; ++m) _Pragma("unroll") for (int n = 0; n < 2; ++n) _Pragma("unroll") for (int k = 0; k < 2; ++k) \
        acc[ai][bj][m][n] = __builtin_amdgcn_mfma_f32_16x16x32_bf16(Bt[n][k], At[m][k], acc[ai][bj][m][n], 0, 0, 0); __builtin_amdgcn_s_setprio(0); } while (0)
#define BAR __builtin_amdgcn_s_barrier()
#define SCHED __builtin_amdgcn_sched_barrier(0)

    int qi = slot;
    while (qi < 32) {
        const int tile = tab8(p.xq[xcd], qi), pm = tile >> 4, pn = tile & 15, pos = tab8(p.pos, tile);
        const bool mine = ((pm + pn) & 1) == role;
        if (wid == 0) STAMP(tile * 8 + 0);

        auto hook = [&](bool remote) {
            if (wid == 0) {
                const unsigned long long t0 = __builtin_amdgcn_s_memrealtime();
                if (!remote) {
                    for (;;) {
                        const bool ok = ld_sys(ws + OFF_OWNW + (pn * 64 + lane) * 4) >= call;
                        if (__all(ok)) break;
                        if (give_up(ws, call, t0)) break;
                        __builtin_amdgcn_s_sleep(4);
                    }
                } else {
                    const char* fb = ws + OFF_INF + ((lane < 32 ? 2 * pm : 2 * pn + 1) * 32 + (lane & 31)) * 4;
                    for (;;) {
                        const bool ok = ld_sys(fb) >= call;
                        if (__all(ok)) break;
                        if (give_up(ws, call, t0)) break;
                        __builtin_amdgcn_s_sleep(16);
                    }
                    for (;;) {
                        bool ok = true;
#pragma unroll
                        for (int u = 1; u < 6; ++u) ok = ok & (ld_sys(fb + u * 4096) >= call);
                        if (__all(ok)) break;
                        if (give_up(ws, call, t0)) break;
                        __builtin_amdgcn_s_sleep(8);
                    }
                }
                __builtin_amdgcn_fence(__ATOMIC_ACQUIRE, "");
                asm volatile("s_waitcnt vmcnt(0)" ::: "memory");
            }
            asm volatile("" ::: "memory"); __builtin_amdgcn_s_barrier(); asm volatile("" ::: "memory");
        };

        f32x4 acc[2][2][4][2];
#pragma unroll
        for (int a = 0; a < 2; ++a)
#pragma unroll
            for (int b = 0; b < 2; ++b)
#pragma unroll
                for (int m = 0; m < 4; ++m)
#pragma unroll
                    for (int n = 0; n < 2; ++n) acc[a][b][m][n] = (f32x4){0.f, 0.f, 0.f, 0.f};
        bf16x8 At[4][2], B0[2][2], B1[2][2];

        const char* xsA = ws + OFF_XS + (size_t)pm * (512u << 10);
        const char* wsB = ws + OFF_WTS + (size_t)pn * (512u << 10);
        const char* cA = p.x + (size_t)((qd >> 1) * 4096 + pm * 256) * 2048;
        const char* cB = wsB + (size_t)3 * (8u << 20);
        hook(false);
        if (wid == 0) STAMP(tile * 8 + 1);
        STAGE(SB(0, 0), cB, voffB); STAGE(SA(0, 0), cA, voffA); STAGE(SB(0, 1), cB + hstep, voffB); STAGE(SA(0, 1), cA + hstep, voffA);
        if (wr == 1) BAR;
        WAIT_V(4); BAR;
        STAGE(SB(1, 0), cB + kstep, voffB); STAGE(SA(1, 0), cA + kstep, voffA); STAGE(SB(1, 1), cB + hstep + kstep, voffB);
        WAIT_V(6); BAR;
        constexpr int nt = KS / BK;
        for (int u = 0; u < 4; ++u) {
            const bool has_next = u + 1 < 4;
            const char* nA = has_next ? xsA + (size_t)u * (8u << 20) : cA;
            const char* nB = has_next ? wsB + (size_t)u * (8u << 20) : cB;
            for (int t = 0; t < nt; t += 2) {
                const bool last = (t == nt - 2);
                const char* a1 = cA + (size_t)(t + 1) * kstep;
                const char* a2 = last ? nA : cA + (size_t)(t + 2) * kstep; const char* b2 = last ? nB : cB + (size_t)(t + 2) * kstep;
                const char* a3 = a2 + kstep; const char* b3 = b2 + kstep;
                if (last && u == 0) {
                    if (wid == 0) STAMP(tile * 8 + 2);
                    hook(true);
                    if (wid == 0) STAMP(tile * 8 + 3);
                }
                LDB(B0, 0, 0); SCHED; LDA(At, 0, 0); STAGE(SA(1, 1), a1 + hstep, voffA);
                WAIT_L(8); BAR; WAIT_L(0); MMA(0, 0, At, B0); BAR; SCHED;
                LDB(B1, 0, 1); STAGE(SB(0, 0), b2, voffB);
                BAR; WAIT_L(0); MMA(0, 1, At, B1); BAR;
                LDA(At, 0, 1); STAGE(SA(0, 0), a2, voffA);
                BAR; WAIT_L(0); MMA(1, 0, At, B0); BAR; SCHED;
                STAGE(SB(0, 1), b2 + hstep, voffB);
                WAIT_V(6); BAR; MMA(1, 1, At, B1); BAR;
                LDB(B0, 1, 0); SCHED; LDA(At, 1, 0); STAGE(SA(0, 1), a2 + hstep, voffA);
                WAIT_L(8); BAR; WAIT_L(0); MMA(0, 0, At, B0); BAR; SCHED;
                LDB(B1, 1, 1); STAGE(SB(1, 0), b3, voffB);
                BAR; WAIT_L(0); MMA(0, 1, At, B1); BAR;
                LDA(At, 1, 1); STAGE(SA(1, 0), a3, voffA);
                BAR; WAIT_L(0); MMA(1, 0, At, B0); BAR; SCHED;
                STAGE(SB(1, 1), b3 + hstep, voffB);
                WAIT_V(6); BAR; MMA(1, 1, At, B1); BAR;
            }
            cA = nA; cB = nB;
        }
        if (wid == 0) STAMP(tile * 8 + 4);
        if (wr == 0) BAR;
        const __amdgpu_buffer_rsrc_t rw = mk_rsrc(ws, WS_NEED);
        const unsigned pbase = (unsigned)pos * (128u << 10) + (unsigned)wid * (16u << 10);
        if (!mine) {
#pragma unroll
            for (int ai = 0; ai < 2; ++ai)
#pragma unroll
                for (int m = 0; m < 4; ++m)
#pragma unroll
                    for (int bj = 0; bj < 2; ++bj) {
                        const f32x4 v0 = acc[ai][bj][m][0], v1 = acc[ai][bj][m][1];
                        u32x4 o;
                        o.x = cvt_pk_bf16(v0[0], v0[1]); o.y = cvt_pk_bf16(v0[2], v0[3]);
                        o.z = cvt_pk_bf16(v1[0], v1[1]); o.w = cvt_pk_bf16(v1[2], v1[3]);
                        __builtin_amdgcn_raw_buffer_store_b128(o, rw, lane * 16, OFF_PS + pbase + ((ai * 4 + m) * 2 + bj) * 1024u, 16);
                    }
        } else {
            if (wid == 0) {
                const unsigned long long t0 = __builtin_amdgcn_s_memrealtime();
                for (;;) {
                    const int fi = lane < 16 ? pos * 16 + lane : max(pos - 4, 0) * 16 + 15;
                    const unsigned long long m = __ballot(ld_sys(ws + OFF_PFLAG + fi * 4) >= call);
                    if ((m & 0xffffull) == 0xffffull) break;
                    if (give_up(ws, call, t0)) break;
                    if (m >> 16) __builtin_amdgcn_s_sleep(PSL_S); else __builtin_amdgcn_s_sleep(PSL_L);
                }
            }
            asm volatile("" ::: "memory"); __builtin_amdgcn_s_barrier(); asm volatile("" ::: "memory");
            if (wid == 0) STAMP(tile * 8 + 5);
            const __amdgpu_buffer_rsrc_t ro = mk_rsrc(p.out[rank], 128u << 20);
            const unsigned base = (unsigned)((qd >> 1) * 4096 + pm * BM) * 16384u + (unsigned)((qd & 1) * 4096 + pn * BM) * 2u;
            const unsigned voff = (unsigned)(wr * 64 + fr) * 16384u + (unsigned)(wc * 32 + 8 * fq) * 2u;
            u32x4 pr[2][4][2];
#pragma unroll
            for (int ai = 0; ai < 2; ++ai)
#pragma unroll
                for (int m = 0; m < 4; ++m)
#pragma unroll
                    for (int bj = 0; bj < 2; ++bj)
                        pr[ai][m][bj] = __builtin_amdgcn_raw_buffer_load_b128(rw, lane * 16, OFF_PR + pbase + ((ai * 4 + m) * 2 + bj) * 1024u, 17);
            __builtin_amdgcn_sched_barrier(0);
#pragma unroll
            for (int ai = 0; ai < 2; ++ai)
#pragma unroll
                for (int m = 0; m < 4; ++m)
#pragma unroll
                    for (int bj = 0; bj < 2; ++bj) {
                        const f32x4 v0 = acc[ai][bj][m][0], v1 = acc[ai][bj][m][1];
                        const u32x4 q = pr[ai][m][bj];
                        u32x4 o;
                        o.x = cvt_pk_bf16(silu(v0[0] + bf_lo(q.x)), silu(v0[1] + bf_hi(q.x))); o.y = cvt_pk_bf16(silu(v0[2] + bf_lo(q.y)), silu(v0[3] + bf_hi(q.y)));
                        o.z = cvt_pk_bf16(silu(v1[0] + bf_lo(q.z)), silu(v1[1] + bf_hi(q.z))); o.w = cvt_pk_bf16(silu(v1[2] + bf_lo(q.w)), silu(v1[3] + bf_hi(q.w)));
                        __builtin_amdgcn_raw_buffer_store_b128(o, ro, voff, base + (unsigned)(ai * HALF + m * 16) * 16384u + (unsigned)(bj * HALF) * 2u, 16);
                    }
        }
        WAIT_V(0);
        BAR;
        if (tid == 0) {
            st_agent(ws + (mine ? OFF_TFLAG : OFF_SFLAG) + pos * 4, call);
            const unsigned t = __hip_atomic_fetch_add((unsigned*)(ws + OFF_QCTR + (8 + xcd) * 128), 1u, __ATOMIC_RELAXED, __HIP_MEMORY_SCOPE_AGENT);
            *(volatile LAS unsigned*)lds = GPX + t;
        }
        if (wid == 0) STAMP(tile * 8 + 6);
        __syncthreads();
        qi = __builtin_amdgcn_readfirstlane(*(volatile LAS unsigned*)lds);
        __syncthreads();
    }
#undef SA
#undef SB
#undef STAGE
#undef LDA
#undef LDB
#undef MMA
#undef BAR
#undef SCHED
}

__global__ __launch_bounds__(512, 2) void fused_gemm_allreduce(const Params p_by_value) {
    KP p = *(const __attribute__((address_space(4))) Params*)__builtin_amdgcn_kernarg_segment_ptr();
    extern __shared__ __attribute__((aligned(16))) unsigned char lds_raw[];
    LAS unsigned char* lds = (LAS unsigned char*)lds_raw;
    const int b = blockIdx.x, xcd = b & 7, slot = b >> 3;
    char* ws = p.ws[p.rank];
    const unsigned call = __builtin_amdgcn_readfirstlane(ld_agent(ws + OFF_SEQ)) + 1u;
    if (slot >= 32 - GPX) {
        gemm_role(p, xcd, slot - (32 - GPX), call, lds);
    } else {
        unsigned xcc;
        asm volatile("s_getreg_b32 %0, hwreg(HW_REG_XCC_ID)" : "=s"(xcc));
        const int xsel = slot == 0 ? xcd : (int)(xcc & 7u);
        if (slot < NCW) comm_role(p, tab8(p.xlink, xsel), slot, slot * 8 + xcd, call);
    }
    asm volatile("s_waitcnt vmcnt(0)" ::: "memory");
    __syncthreads();
    if (threadIdx.x == 0) {
        const unsigned t = __hip_atomic_fetch_add((unsigned*)(ws + OFF_DONE), 1u, __ATOMIC_RELAXED, __HIP_MEMORY_SCOPE_AGENT);
        if ((t + 1u) % (unsigned)GRID == 0u) {
            if (ld_agent(ws + OFF_TMO) == call) {
                const int t0 = tab8(p.forder[role_of(p.rank)], 0), qd = quad_of(p.rank);
                const size_t off = (size_t)((qd >> 1) * 4096 + (t0 >> 4) * 256) * 16384 + (size_t)((qd & 1) * 4096 + (t0 & 15) * 256) * 2;
                st_agent(p.out[p.rank] + off, 0x7fc07fc0u);
            }
            for (int l = 0; l < 16; ++l) st_agent(ws + OFF_QCTR + l * 128, 0u);
            st_agent(ws + OFF_SEQ, call);
        }
    }
}

}

extern "C" void kernel_launch(int rank, int nranks, ncclComm_t comm,
                              hipStream_t stream,
                              void* const* d_in, const int* in_sizes, int n_in,
                              void* d_out, int out_size,
                              const int* in_dtypes, int out_dtype,
                              void* const* d_out_all,
                              void* const* d_ws_all, size_t ws_bytes) {
    if (nranks != NR || (unsigned)rank >= (unsigned)NR || ws_bytes < WS_NEED || n_in < 2) {
        fprintf(stderr, "unsupported configuration\n");
        return;
    }
    static bool attr_set[NR] = {};
    if (!attr_set[rank]) {
        HIP_CHECK(hipFuncSetAttribute((const void*)fused_gemm_allreduce, hipFuncAttributeMaxDynamicSharedMemorySize, LDS_BYTES));
        attr_set[rank] = true;
    }
    Params p;
    p.x = (const char*)d_in[0];
    p.w = (const char*)d_in[1];
    for (int r = 0; r < NR; ++r) { p.out[r] = (char*)d_out_all[r]; p.ws[r] = (char*)d_ws_all[r]; }
    p.rank = rank;
    {
        static const unsigned char xl[NR][8] = {
            {6, 255, 2, 3, 1, 5, 4, 7}, {7, 255, 4, 5, 0, 3, 2, 6}, {3, 255, 0, 6, 4, 5, 1, 7}, {2, 255, 0, 6, 1, 5, 4, 7},
            {5, 255, 1, 7, 0, 2, 3, 6}, {4, 255, 1, 7, 2, 3, 0, 6}, {0, 255, 2, 3, 4, 7, 1, 5}, {1, 255, 4, 5, 3, 6, 0, 2}};
        for (int i = 0; i < 8; ++i) ((unsigned char*)p.xlink)[i] = xl[rank][i];
    }
    {
        int nf[2] = {0, 0}, nx[8] = {0, 0, 0, 0, 0, 0, 0, 0};
        for (int e = 0; e < 32; ++e)
            for (int t = 0; t < 256; ++t) {
                const int pm = t >> 4, pn = t & 15, a = 2 * pm, bb = 2 * pn + 1;
                if ((a > bb ? a : bb) != e) continue;
                const int h = (pm + pn) & 1, g = (pm & 3) * 2 + ((pn >> 1) & 1);
                ((unsigned char*)p.forder[h])[nf[h]] = (unsigned char)t;
                ((unsigned char*)p.pos)[t] = (unsigned char)nf[h]++;
                ((unsigned char*)p.xq[g])[nx[g]++] = (unsigned char)t;
            }
    }
    {
        unsigned char* seq = (unsigned char*)p.seq; unsigned char* spf = (unsigned char*)p.spf; unsigned char* spw = (unsigned char*)p.spw;
        for (int i = 0; i < 160; ++i) seq[i] = 0;
        for (int i = 0; i < 132; ++i) spf[i] = (unsigned char)NSEQ;
        for (int i = 0; i < 36; ++i) spw[i] = (unsigned char)NSEQ;
        int n = 0, wi = 0;
        for (int o = 0; o < NPOS; ++o) {
            while (wi < NF6 && (o == NPOS - 1 || 6 * wi + FLAG_ < o)) { spw[wi] = (unsigned char)n; seq[n++] = (unsigned char)(NPOS + wi); ++wi; }
            spf[o] = (unsigned char)n; seq[n++] = (unsigned char)o;
        }
    }
    hipLaunchKernelGGL(fused_gemm_allreduce, dim3(GRID), dim3(512), LDS_BYTES, stream, p);
    HIP_CHECK(hipGetLastError());
}
```

```cpp
#include <hip/hip_runtime.h>
#include <rccl/rccl.h>
#include <cstdio>
#include <cstdint>

#define HIP_CHECK(expr) do { hipError_t _e = (expr); if (_e != hipSuccess) { \
    fprintf(stderr, "HIP error %s at %s:%d\n", hipGetErrorString(_e), __FILE__, __LINE__); \
    return; } } while (0)

namespace {

#define LAS __attribute__((address_space(3)))
typedef short bf16x8 __attribute__((ext_vector_type(8)));
typedef float f32x4 __attribute__((ext_vector_type(4)));
typedef unsigned u32x4 __attribute__((ext_vector_type(4)));
typedef unsigned long long u64;

constexpr int NR = 8;
#define PM_ 7
constexpr int PM = PM_;
constexpr int KS = 1024;
constexpr int NPOS = 128;
#define NCW 12
#define WPW 1
#define WPWP 1
#define PSL_S 24
#define PSL_L 120
#define CSL 24
#define GPX 16
constexpr int WPL = NCW * WPW;
constexpr int GRID = 256;
constexpr int WAITER = 1;
constexpr int N_IN = 32 * 32;
constexpr int N_FIN = NPOS * 16;
constexpr int N_OWN3 = N_IN + N_FIN;
constexpr int N_OWNP = N_FIN;
constexpr int SEGSH = 7;
constexpr int NSEG = N_FIN >> SEGSH;
constexpr unsigned N_TOTAL = 6 * N_OWN3 + N_OWNP + N_FIN;
static_assert(WPL <= 64 && NCW * WPWP <= 64 && NSEG <= 64, "config");
static_assert(NCW <= 32 - GPX && GPX >= 16 && GPX <= 32, "too many workgroups");

constexpr unsigned OFF_SEQ = 0, OFF_DONE = 128, OFF_TMO = 256;
constexpr unsigned OFF_INF = 4096;
constexpr unsigned OFF_OWNW = OFF_INF + 6 * 32 * 32 * 4;
constexpr unsigned OFF_TFLAG = OFF_OWNW + 16 * 64 * 4;
constexpr unsigned OFF_SFLAG = OFF_TFLAG + 512;
constexpr unsigned OFF_PFLAG = OFF_SFLAG + 512;
constexpr unsigned OFF_FCNT = OFF_PFLAG + 128 * 16 * 4;
constexpr unsigned OFF_OCNT = OFF_FCNT + 6 * 64 * 4;
constexpr unsigned OFF_QQ = OFF_OCNT + 128;
constexpr unsigned OFF_QCTR = OFF_QQ + 256;
constexpr unsigned OFF_DBG = 128 * 1024;
constexpr unsigned OFF_XS = 1u << 20;
constexpr unsigned OFF_WTS = OFF_XS + (24u << 20);
constexpr unsigned OFF_PS = OFF_WTS + (32u << 20);
constexpr unsigned OFF_PR = OFF_PS + (16u << 20);
constexpr unsigned WS_NEED = OFF_PR + (16u << 20);
static_assert(OFF_QCTR + 16 * 128 <= OFF_DBG && WS_NEED <= (128u << 20), "layout");
constexpr unsigned long long DEADLINE = 40000000ull;

constexpr int BM = 256, BK = 64, HALF = 128, HTB = HALF * BK * 2, STAGE_BYTES = 8 * HTB;
constexpr int LDS_BYTES = STAGE_BYTES;

struct Params {
    const char* x;
    const char* w;
    char* out[NR];
    char* ws[NR];
    int rank;
    unsigned forder[2][NPOS / 4];
    unsigned pos[256 / 4];
    unsigned xq[8][32 / 4];
    unsigned xlink[2];
};
typedef const __attribute__((address_space(4))) Params& KP;

__host__ __device__ __forceinline__ int role_of(int r) { return (r >> 2) & 1; }
__host__ __device__ __forceinline__ int quad_of(int r) { return role_of(r) ? (r ^ PM) : r; }
__host__ __device__ __forceinline__ int uidx(int m) { return m < PM ? m - 1 : m - 2; }

__device__ __forceinline__ int lds_byte(int r, int c) { const int st = (r >> 4) * 2 + (c >> 5), rr = r & 15, cc = c & 31, ob = rr * 64 + cc * 2; return st * 1024 + (ob ^ (((ob >> 9) & 1) << 5)); }
__device__ __forceinline__ void stage_rc(int b, int& R, int& C) { const int st = b / 1024, sb = b % 1024, swz = sb ^ (((sb >> 9) & 1) << 5); R = (st >> 1) * 16 + swz / 64; C = (st & 1) * 32 + (swz % 64) / 2; }
__device__ __forceinline__ int perm32(int rho) { const int n = rho >> 4, i = rho & 15; return 8 * (i >> 2) + 4 * n + (i & 3); }

__device__ __forceinline__ unsigned ld_sys(const char* p) { return __hip_atomic_load((const unsigned*)p, __ATOMIC_RELAXED, __HIP_MEMORY_SCOPE_SYSTEM); }
__device__ __forceinline__ unsigned ld_agent(const char* p) { return __hip_atomic_load((const unsigned*)p, __ATOMIC_RELAXED, __HIP_MEMORY_SCOPE_AGENT); }
__device__ __forceinline__ void st_sys(char* p, unsigned v) { __hip_atomic_store((unsigned*)p, v, __ATOMIC_RELAXED, __HIP_MEMORY_SCOPE_SYSTEM); }
__device__ __forceinline__ void st_agent(char* p, unsigned v) { __hip_atomic_store((unsigned*)p, v, __ATOMIC_RELAXED, __HIP_MEMORY_SCOPE_AGENT); }
__device__ __forceinline__ unsigned cvt_pk_bf16(float lo, float hi) { unsigned r; asm("v_cvt_pk_bf16_f32 %0, %1, %2" : "=v"(r) : "v"(lo), "v"(hi)); return r; }
__device__ __forceinline__ float bf_lo(unsigned u) { return __uint_as_float(u << 16); }
__device__ __forceinline__ float bf_hi(unsigned u) { return __uint_as_float(u & 0xffff0000u); }
__device__ __forceinline__ float silu(float y) {
    const float t = fminf(fmaxf(y, -60.0f), 60.0f);
    const float e = __builtin_amdgcn_exp2f(t * -1.44269504088896f);
    return y * __builtin_amdgcn_rcpf(1.0f + e);
}
__device__ __forceinline__ __amdgpu_buffer_rsrc_t mk_rsrc(const void* p, unsigned bytes) {
    return __builtin_amdgcn_make_buffer_rsrc((void*)p, 0, bytes, 0x00020000);
}
template <class T> __device__ __forceinline__ int tab8(const T& t, int i) { return (t[i >> 2] >> (8 * (i & 3))) & 255; }
#define WAIT_V(n) asm volatile("s_waitcnt vmcnt(" #n ")" ::: "memory")
#define WAIT_L(n) asm volatile("s_waitcnt lgkmcnt(" #n ")" ::: "memory")

#define STAMP(idx) do { } while (0)

__device__ __forceinline__ bool give_up(char* ws, unsigned call, unsigned long long t0) {
    const unsigned tm = __builtin_amdgcn_readfirstlane(ld_agent(ws + OFF_TMO));
    if (tm == call || __builtin_amdgcn_s_memrealtime() - t0 > DEADLINE) {
        if ((threadIdx.x & 63) == 0) st_agent(ws + OFF_TMO, call);
        return true;
    }
    return false;
}

__device__ __forceinline__ u64 uni64(const void* p) {
    const u64 a = (u64)p;
    return ((u64)(unsigned)__builtin_amdgcn_readfirstlane((unsigned)(a >> 32)) << 32) | (unsigned)__builtin_amdgcn_readfirstlane((unsigned)a);
}
__device__ __forceinline__ u32x4 mk_srd(u64 a) {
    u32x4 r;
    r.x = (unsigned)a;
    r.y = (unsigned)(a >> 32) & 0xffffu;
    r.z = 128u << 20;
    r.w = 0x00020000u;
    return r;
}
#define CE_CLOB "memory", "scc", "vcc", "v156", "v157", "v158", "v159", "v160", "v161", "v162", "v163", "v164", "v165", "v166", "v167", "v168", "v169", "v170", "v171", "v172", "v173", "v174", "v175", "v176", "v177", "v178", "v179", "v180", "v181", "v182", "v183", "v184", "v185", "v186", "v187", "v188", "v189", "v190", "v191", "v192", "v193", "v194", "v195", "v196", "v197", "v198", "v199", "v200", "v201", "v202", "v203", "v204", "v205", "v206", "v207", "v208", "v209", "v210", "v211", "v212", "v213", "v214", "v215", "v216", "v217", "v218", "v219", "v220", "v221", "v222", "v223", "v224", "v225", "v226", "v227", "v228", "v229", "v230", "v231", "v232", "v233", "v234", "v235", "v236", "v237", "v238", "v239", "v240", "v241", "v242", "v243", "v244", "v245", "v246", "v247", "v248", "v249", "v250", "v251", "v252", "v253", "v254", "v255"
#define LDF "sc0 sc1"
template <int R> __device__ __forceinline__ void ld8(unsigned voff, const u32x4& srd, unsigned soff, unsigned step) {
    unsigned t;
    asm volatile(
        "s_nop 4\n\t"
        "buffer_load_dwordx4 v[%c5+0:%c5+3], %1, %2, %3 offen " LDF "\n\t"
        "s_add_u32 %0, %3, %4\n\t"
        "buffer_load_dwordx4 v[%c5+4:%c5+7], %1, %2, %0 offen " LDF "\n\t"
        "s_add_u32 %0, %0, %4\n\t"
        "buffer_load_dwordx4 v[%c5+8:%c5+11], %1, %2, %0 offen " LDF "\n\t"
        "s_add_u32 %0, %0, %4\n\t"
        "buffer_load_dwordx4 v[%c5+12:%c5+15], %1, %2, %0 offen " LDF "\n\t"
        "s_add_u32 %0, %0, %4\n\t"
        "buffer_load_dwordx4 v[%c5+16:%c5+19], %1, %2, %0 offen " LDF "\n\t"
        "s_add_u32 %0, %0, %4\n\t"
        "buffer_load_dwordx4 v[%c5+20:%c5+23], %1, %2, %0 offen " LDF "\n\t"
        "s_add_u32 %0, %0, %4\n\t"
        "buffer_load_dwordx4 v[%c5+24:%c5+27], %1, %2, %0 offen " LDF "\n\t"
        "s_add_u32 %0, %0, %4\n\t"
        "buffer_load_dwordx4 v[%c5+28:%c5+31], %1, %2, %0 offen " LDF "\n\t"
        "s_nop 0"
        : "=&s"(t)
        : "v"(voff), "s"(srd), "s"(soff), "s"(step), "i"(R)
        : CE_CLOB);
}
#define SWIN 4
#define SWN(j) ((j) < SWIN ? SWIN + 8 : SWIN - 1)
template <int R> __device__ __forceinline__ void st8(unsigned voff, const u32x4& srd, unsigned soff, unsigned step) {
    unsigned t;
    asm volatile(
        "s_nop 4\n\t"
        "s_waitcnt vmcnt(%c6)\n\t"
        "buffer_store_dwordx4 v[%c5+0:%c5+3], %1, %2, %3 offen sc0 sc1\n\t"
        "s_add_u32 %0, %3, %4\n\t"
        "s_waitcnt vmcnt(%c7)\n\t"
        "buffer_store_dwordx4 v[%c5+4:%c5+7], %1, %2, %0 offen sc0 sc1\n\t"
        "s_add_u32 %0, %0, %4\n\t"
        "s_waitcnt vmcnt(%c8)\n\t"
        "buffer_store_dwordx4 v[%c5+8:%c5+11], %1, %2, %0 offen sc0 sc1\n\t"
        "s_add_u32 %0, %0, %4\n\t"
        "s_waitcnt vmcnt(%c9)\n\t"
        "buffer_store_dwordx4 v[%c5+12:%c5+15], %1, %2, %0 offen sc0 sc1\n\t"
        "s_add_u32 %0, %0, %4\n\t"
        "s_waitcnt vmcnt(%c10)\n\t"
        "buffer_store_dwordx4 v[%c5+16:%c5+19], %1, %2, %0 offen sc0 sc1\n\t"
        "s_add_u32 %0, %0, %4\n\t"
        "s_waitcnt vmcnt(%c11)\n\t"
        "buffer_store_dwordx4 v[%c5+20:%c5+23], %1, %2, %0 offen sc0 sc1\n\t"
        "s_add_u32 %0, %0, %4\n\t"
        "s_waitcnt vmcnt(%c12)\n\t"
        "buffer_store_dwordx4 v[%c5+24:%c5+27], %1, %2, %0 offen sc0 sc1\n\t"
        "s_add_u32 %0, %0, %4\n\t"
        "s_waitcnt vmcnt(%c13)\n\t"
        "buffer_store_dwordx4 v[%c5+28:%c5+31], %1, %2, %0 offen sc0 sc1\n\t"
        "s_nop 1"
        : "=&s"(t)
        : "v"(voff), "s"(srd), "s"(soff), "s"(step), "i"(R), "i"(SWN(0)), "i"(SWN(1)), "i"(SWN(2)), "i"(SWN(3)), "i"(SWN(4)), "i"(SWN(5)), "i"(SWN(6)), "i"(SWN(7))
        : CE_CLOB);
}
template <int R> __device__ __forceinline__ void tr8() {
    asm volatile(
        "v_perm_b32 v224, v[%c2+4], v[%c2+0], %0\n\t"
        "v_perm_b32 v225, v[%c2+12], v[%c2+8], %0\n\t"
        "v_perm_b32 v226, v[%c2+20], v[%c2+16], %0\n\t"
        "v_perm_b32 v227, v[%c2+28], v[%c2+24], %0\n\t"
        "v_perm_b32 v228, v[%c2+4], v[%c2+0], %1\n\t"
        "v_perm_b32 v229, v[%c2+12], v[%c2+8], %1\n\t"
        "v_perm_b32 v230, v[%c2+20], v[%c2+16], %1\n\t"
        "v_perm_b32 v231, v[%c2+28], v[%c2+24], %1\n\t"
        "v_perm_b32 v232, v[%c2+5], v[%c2+1], %0\n\t"
        "v_perm_b32 v233, v[%c2+13], v[%c2+9], %0\n\t"
        "v_perm_b32 v234, v[%c2+21], v[%c2+17], %0\n\t"
        "v_perm_b32 v235, v[%c2+29], v[%c2+25], %0\n\t"
        "v_perm_b32 v236, v[%c2+5], v[%c2+1], %1\n\t"
        "v_perm_b32 v237, v[%c2+13], v[%c2+9], %1\n\t"
        "v_perm_b32 v238, v[%c2+21], v[%c2+17], %1\n\t"
        "v_perm_b32 v239, v[%c2+29], v[%c2+25], %1\n\t"
        "v_perm_b32 v240, v[%c2+6], v[%c2+2], %0\n\t"
        "v_perm_b32 v241, v[%c2+14], v[%c2+10], %0\n\t"
        "v_perm_b32 v242, v[%c2+22], v[%c2+18], %0\n\t"
        "v_perm_b32 v243, v[%c2+30], v[%c2+26], %0\n\t"
        "v_perm_b32 v244, v[%c2+6], v[%c2+2], %1\n\t"
        "v_perm_b32 v245, v[%c2+14], v[%c2+10], %1\n\t"
        "v_perm_b32 v246, v[%c2+22], v[%c2+18], %1\n\t"
        "v_perm_b32 v247, v[%c2+30], v[%c2+26], %1\n\t"
        "v_perm_b32 v248, v[%c2+7], v[%c2+3], %0\n\t"
        "v_perm_b32 v249, v[%c2+15], v[%c2+11], %0\n\t"
        "v_perm_b32 v250, v[%c2+23], v[%c2+19], %0\n\t"
        "v_perm_b32 v251, v[%c2+31], v[%c2+27], %0\n\t"
        "v_perm_b32 v252, v[%c2+7], v[%c2+3], %1\n\t"
        "v_perm_b32 v253, v[%c2+15], v[%c2+11], %1\n\t"
        "v_perm_b32 v254, v[%c2+23], v[%c2+19], %1\n\t"
        "v_perm_b32 v255, v[%c2+31], v[%c2+27], %1\n\t"
        "s_nop 1"
        :: "s"(0x05040100u), "s"(0x07060302u), "i"(R)
        : CE_CLOB);
}
__device__ __forceinline__ void ticket_issue(u64 ctr) {
    u64 sv;
    asm volatile(
        "s_mov_b64 %0, exec\n\t"
        "s_mov_b64 exec, 1\n\t"
        "v_mov_b32 v158, 0\n\t"
        "v_mov_b32 v159, 1\n\t"
        "s_nop 4\n\t"
        "global_atomic_add v159, v158, v159, %1 sc0 sc1\n\t"
        "s_mov_b64 exec, %0"
        : "=&s"(sv) : "s"(ctr) : CE_CLOB);
}
__device__ __forceinline__ int ticket_read() {
    int q;
    asm volatile("s_nop 1\n\tv_readfirstlane_b32 %0, v159\n\ts_nop 4" : "=s"(q) :: CE_CLOB);
    return q;
}
__device__ __forceinline__ void tflags_issue(u64 base, unsigned voff) {
    asm volatile(
        "s_nop 4\n\t"
        "global_load_dword v157, %0, %1 sc1\n\t"
        "global_load_dword v158, %0, %1 offset:256 sc1"
        :: "v"(voff), "s"(base) : CE_CLOB);
}
__device__ __forceinline__ int tflags_read(unsigned call) {
    u64 m0, m1;
    asm volatile("s_nop 1\n\tv_cmp_le_u32 %0, %2, v157\n\tv_cmp_le_u32 %1, %2, v158\n\ts_nop 4" : "=&s"(m0), "=&s"(m1) : "s"(call) : CE_CLOB);
    return (~m0 != 0ull) ? __builtin_ctzll(~m0) : 64 + ((~m1 != 0ull) ? __builtin_ctzll(~m1) : 64);
}
__device__ __forceinline__ void fcnt_issue(u64 base, unsigned voff) {
    asm volatile("s_nop 4\n\tglobal_load_dword v156, %0, %1 sc0 sc1" :: "v"(voff), "s"(base) : CE_CLOB);
}
__device__ __forceinline__ int fcnt_read(unsigned need) {
    u64 m0;
    asm volatile("s_nop 1\n\tv_cmp_le_u32 %0, %1, v156\n\ts_nop 4" : "=&s"(m0) : "s"(need) : CE_CLOB);
    return (~m0 != 0ull) ? __builtin_ctzll(~m0) : 64;
}
#define CE_WAIT(n) asm volatile("s_waitcnt vmcnt(" #n ")" ::: CE_CLOB)
constexpr int RA = 160, RB = 192, RT = 224;

__device__ __forceinline__ unsigned selv(bool c0, bool c1, unsigned a, unsigned b, unsigned c) {
    asm("" : "+v"(a)); asm("" : "+v"(b)); asm("" : "+v"(c));
    return c0 ? a : (c1 ? b : c);
}
__device__ __forceinline__ u64 sels(bool c, u64 a, u64 b) {
    asm("" : "+s"(a)); asm("" : "+s"(b));
    return uni64((const void*)(c ? a : b));
}

struct Piece { int type; unsigned src, dst, flag; };

struct Comm {
    KP p;
    int rank, dst, lane;
    unsigned call;
    bool partner;
    u64 rx, rw, ro_src, rws_src, rws_dst, ro_dst;
    unsigned lin_off, w_src_off, w_dst_off, o_off;
    char* wsd;

    __device__ __forceinline__ Comm(KP p_, int dst_, int lane_, unsigned call_) : p(p_), rank(p_.rank), dst(dst_), lane(lane_), call(call_) {
        partner = (rank ^ dst) == PM;
        rx = uni64(p.x);
        rw = uni64(p.w);
        ro_src = uni64(p.out[rank]);
        rws_src = uni64(p.ws[rank]);
        wsd = p.ws[dst];
        rws_dst = uni64(wsd);
        ro_dst = uni64(p.out[dst]);
        lin_off = lane * 16;
        w_src_off = (lane >> 3) * 8 * 16384 + (lane & 7) * 16;
        w_dst_off = (lane & 7) * 8 * 2048 + (lane >> 3) * 16;
        o_off = (lane >> 5) * 16384 + (lane & 31) * 16;
    }
    __device__ __forceinline__ unsigned out_off(int a, int o, int po) const {
        const int t = tab8(p.forder[role_of(a)], o), pm = t >> 4, pn = t & 15, qd = quad_of(a);
        return (unsigned)((qd >> 1) * 4096 + pm * 256 + po * 16) * 16384u + (unsigned)((qd & 1) * 4096 + pn * 256) * 2u;
    }
    __device__ __forceinline__ Piece desc(int q) const {
        Piece d;
        q = __builtin_amdgcn_readfirstlane(q);
        const int n_own = partner ? N_OWNP : N_OWN3;
        if (q >= n_own) {
            const int qq = q - n_own;
            d.type = 2;
            d.src = d.dst = out_off(dst ^ PM, qq >> 4, qq & 15); d.flag = 0;
        } else if (partner) {
            d.type = 3;
            d.src = OFF_PS + q * 8192u; d.dst = OFF_PR + q * 8192u; d.flag = OFF_PFLAG + q * 4;
        } else {
            const int u = uidx(rank ^ dst);
            if (q < N_IN) {
                const int e = q >> 5, pc = q & 31, panel = e >> 1, hd = role_of(dst), qd = quad_of(dst);
                d.flag = OFF_INF + (u * 1024 + q) * 4;
                if (!(e & 1)) {
                    d.type = 0;
                    d.src = (unsigned)((qd >> 1) * 4096 + panel * 256 + pc * 8) * 2048u + hd * 1024u;
                    d.dst = OFF_XS + (u >> 1) * (8u << 20) + (unsigned)(panel * 256 + pc * 8) * 2048u + (u & 1) * 1024u;
                } else {
                    const int kb = pc >> 2, nb = pc & 3;
                    d.type = 1;
                    d.src = (unsigned)((hd * 512 + kb * 64) * 8192 + (qd & 1) * 4096 + panel * 256 + nb * 64) * 2u;
                    d.dst = OFF_WTS + (u >> 1) * (8u << 20) + (unsigned)((panel * 256 + nb * 64) * 1024 + (u & 1) * 512 + kb * 64) * 2u;
                }
            } else {
                const int qq = q - N_IN;
                d.type = 2;
                d.src = d.dst = out_off(rank, qq >> 4, qq & 15);
                d.flag = (OFF_FCNT + (u * 64 + (qq >> SEGSH)) * 4) | 1u;
            }
        }
        d.type = __builtin_amdgcn_readfirstlane(d.type);
        d.src = __builtin_amdgcn_readfirstlane(d.src);
        d.dst = __builtin_amdgcn_readfirstlane(d.dst);
        d.flag = __builtin_amdgcn_readfirstlane(d.flag);
        return d;
    }
    template <int R> __device__ __forceinline__ void load(const Piece& d) const {
        const unsigned voff = selv(d.type == 1, d.type == 2, w_src_off, o_off, lin_off);
        const unsigned step = d.type == 0 ? 2048u : (d.type == 1 ? 16384u : (d.type == 2 ? 32768u : 1024u));
        const u64 base = sels(d.type < 2, sels(d.type == 0, rx, rw), sels(d.type == 2, ro_src, rws_src));
        ld8<R>(voff, mk_srd(base), d.src, step);
    }
    template <int R> __device__ __forceinline__ void store(const Piece& d) const {
        if (d.type == 1) {
            tr8<R>();
            st8<RT>(w_dst_off, mk_srd(rws_dst), d.dst, 2048u);
        } else {
            const unsigned voff = selv(d.type == 2, false, o_off, lin_off, lin_off);
            const unsigned step = d.type == 0 ? 2048u : (d.type == 2 ? 32768u : 1024u);
            st8<R>(voff, mk_srd(sels(d.type == 2, ro_dst, rws_dst)), d.dst, step);
        }
    }
    unsigned seg_off = 0, seg_cnt = 0;
    __device__ __forceinline__ void flush() {
        if (seg_cnt != 0 && lane == 0) __hip_atomic_fetch_add((unsigned*)(wsd + seg_off), seg_cnt, __ATOMIC_RELAXED, __HIP_MEMORY_SCOPE_SYSTEM);
        seg_cnt = 0;
    }
    __device__ __forceinline__ void flag(unsigned off) {
        if (off & 1u) {
            if ((off & ~1u) != seg_off) { flush(); seg_off = off & ~1u; }
            ++seg_cnt;
        } else if (off != 0 && lane == 0) st_sys(wsd + off, call);
    }
};

__device__ __forceinline__ void comm_role(KP p, int dst, int slot, int cidx) {
    const int rank = p.rank;
    char* ws = p.ws[rank];
    const int lane = threadIdx.x & 63, wid = __builtin_amdgcn_readfirstlane(threadIdx.x >> 6);
    if ((unsigned)dst >= (unsigned)NR || dst == rank) return;
    const int wpw = ((rank ^ dst) == PM) ? WPWP : WPW, wpl = wpw * NCW;
    if (wid >= wpw) return;
    const u64 ctr_own = uni64(ws + OFF_QCTR + dst * 128);
    ticket_issue(ctr_own);
    const unsigned call = __builtin_amdgcn_readfirstlane(ld_agent(ws + OFF_SEQ)) + 1u;
    Comm c(p, dst, lane, call);
    STAMP(4096 + (cidx * 8 + wid) * 4 + 0);
    const bool partner = c.partner;
    const int n_own = partner ? N_OWNP : N_OWN3, n_all = n_own + N_FIN;
    const int q_none = n_all - 1;
    const u64 ctr_q = uni64(c.wsd + OFF_QQ + (call & 1u) * 128);
    const u64 tfl = uni64(ws + OFF_TFLAG), sfl = uni64(ws + OFF_SFLAG);
    const u64 fcn = uni64(ws + OFF_FCNT + (partner ? 0 : uidx((dst ^ PM) ^ rank)) * 256);
    const unsigned tvoff = lane * 4;
    const unsigned long long t0 = __builtin_amdgcn_s_memrealtime();
    unsigned ndone = 0;
    int qready = partner ? 0 : N_IN;
    int qbase = 0, qlim = n_own;
    u64 ctr = ctr_own;
#define SNAP_ISSUE() do { tflags_issue(sels(partner && qbase == 0, sfl, tfl), tvoff); if (qbase != 0) fcnt_issue(fcn, tvoff); } while (0)
#define SNAP_APPLY() do { const int n_ = tflags_read(call); \
        int r_; \
        if (qbase == 0) r_ = (partner ? 0 : N_IN) + 16 * n_; \
        else r_ = n_own + (partner ? 16 * n_ : (min(fcnt_read(call << SEGSH), NSEG) << SEGSH)); \
        qready = __builtin_amdgcn_readfirstlane(max(qready, r_)); } while (0)
    CE_WAIT(0);
    int qn = ticket_read();
    for (;;) {
        if (qn >= qlim) {
            if (qbase != 0) break;
            if (partner) {
                while (__builtin_amdgcn_readfirstlane(ld_agent(ws + OFF_TFLAG)) < call) {
                    if (give_up(ws, call, t0)) break;
                    __builtin_amdgcn_s_sleep(CSL);
                }
            }
            STAMP(4096 + (cidx * 8 + wid) * 4 + 3);
            qbase = n_own; qlim = n_all; ctr = ctr_q; qready = n_own;
            ticket_issue(ctr);
            SNAP_ISSUE();
            CE_WAIT(0);
            qn = qbase + ticket_read();
            SNAP_APPLY();
            continue;
        }
        while (qn >= qready) {
            SNAP_ISSUE();
            CE_WAIT(0);
            const int before = qready;
            SNAP_APPLY();
            if (qready != before) continue;
            if (give_up(ws, call, t0)) { qready = qlim; break; }
            __builtin_amdgcn_s_sleep(CSL);
        }
        qready = __builtin_amdgcn_readfirstlane(qready);
        ctr = uni64((const void*)ctr);
        int qc = qn, qp = q_none;
        ticket_issue(ctr);
        c.load<RA>(c.desc(qc));
        CE_WAIT(8);
        qn = qbase + ticket_read();
        bool first = true;
#define CE_STEP(X, Y) { \
            if (!(qn < qlim && qn < qready)) { \
                const bool sw = !partner && qbase == 0 && qn >= qlim;     \
                if (sw) { STAMP(4096 + (cidx * 8 + wid) * 4 + 3); qbase = n_own; qlim = n_all; ctr = ctr_q; qready = n_own; ticket_issue(ctr_q); SNAP_ISSUE(); CE_WAIT(4); } \
                else CE_WAIT(0); \
                c.flag(c.desc(qp).flag); c.store<X>(c.desc(qc)); CE_WAIT(0); c.flag(c.desc(qc).flag); c.flush(); ++ndone; \
                if (sw) { qn = qbase + ticket_read(); SNAP_APPLY(); } \
                break; } \
            const bool snap = qready < qlim && qn + (qbase != 0 ? 16 : 3) * wpl >= qready; \
            ticket_issue(ctr); \
            if (snap) SNAP_ISSUE(); \
            c.load<Y>(c.desc(qn)); \
            if (first) CE_WAIT(9); else CE_WAIT(17); \
            first = false; \
            c.store<X>(c.desc(qc)); \
            CE_WAIT(16); \
            c.flag(c.desc(qp).flag); \
            qp = qc; qc = qn; ++ndone; \
            qn = qbase + ticket_read(); \
            if (snap) SNAP_APPLY(); }
        for (;;) {
            CE_STEP(RA, RB)
            CE_STEP(RB, RA)
        }
#undef CE_STEP
    }
    if (lane == 0 && ndone != 0) __hip_atomic_fetch_add((unsigned*)(c.wsd + OFF_OCNT), ndone, __ATOMIC_RELAXED, __HIP_MEMORY_SCOPE_SYSTEM);
    STAMP(4096 + (cidx * 8 + wid) * 4 + 2);
}

__device__ __forceinline__ void gemm_role(KP p, int xcd, int slot, unsigned call, LAS unsigned char* lds) {
    const int rank = p.rank;
    char* ws = p.ws[rank];
    const int tid = threadIdx.x, wid = __builtin_amdgcn_readfirstlane(tid >> 6), lane = tid & 63, wr = wid >> 2, wc = wid & 3, fr = lane & 15, fq = lane >> 4;
    const int qd = quad_of(rank), role = role_of(rank);

    if (slot < 16) {
        const int id = (slot * 8 + xcd) * 8 + wid, panel = id >> 6, pc = id & 63, kb = pc >> 2, nb = pc & 3;
        const unsigned w_src_off = (lane >> 3) * 8 * 16384 + (lane & 7) * 16, w_dst_off = (lane & 7) * 8 * 2048 + (lane >> 3) * 16;
        ld8<RA>(w_src_off, mk_srd(uni64(p.w)), (unsigned)(kb * 64 * 8192 + (qd & 1) * 4096 + panel * 256 + nb * 64) * 2u, 16384u);
        CE_WAIT(0);
        tr8<RA>();
        st8<RT>(w_dst_off, mk_srd(uni64(ws)), OFF_WTS + 3 * (8u << 20) + (unsigned)((panel * 256 + nb * 64) * 1024 + kb * 64) * 2u, 2048u);
        CE_WAIT(0);
        if (lane == 0) st_sys(ws + OFF_OWNW + id * 4, call);
    }

    unsigned voffA[2], voffB[2];
#pragma unroll
    for (int i = 0; i < 2; ++i) { int R, C; stage_rc(tid * 16 + i * 8192, R, C); const int Rb = (R & ~31) + perm32(R & 31);
        voffA[i] = (unsigned)(R * KS + C) * 2u; voffB[i] = (unsigned)(Rb * KS + C) * 2u; }
    constexpr size_t kstep = (size_t)(BK * 2);
    constexpr size_t hstep = (size_t)HALF * KS * 2;
    const unsigned ldsw = (unsigned)wid * 1024u;
    const int aoff = lds_byte(wr * 64 + fr, fq * 8), boff = lds_byte(wc * 32 + fr, fq * 8);
#define SA(b, h) (((b) * 2 + (h)) * HTB)
#define SB(b, h) ((4 + (b) * 2 + (h)) * HTB)
#define STAGE(bufoff, gbase, voff) do { _Pragma("unroll") for (int _i = 0; _i < 2; ++_i) \
        __builtin_amdgcn_global_load_lds((const unsigned*)((const char*)(gbase) + (voff)[_i]), (LAS unsigned*)(lds + (bufoff) + ldsw + _i * 8192), 16, 0, 0); } while (0)
#define LDA(dst, b, h) do { _Pragma("unroll") for (int m = 0; m < 4; ++m) _Pragma("unroll") for (int k = 0; k < 2; ++k) dst[m][k] = *(const LAS bf16x8*)(lds + SA(b, h) + aoff + m * 2048 + k * 1024); } while (0)
#define LDB(dst, b, h) do { _Pragma("unroll") for (int n = 0; n < 2; ++n) _Pragma("unroll") for (int k = 0; k < 2; ++k) dst[n][k] = *(const LAS bf16x8*)(lds + SB(b, h) + boff + n * 2048 + k * 1024); } while (0)
#define MMA(ai, bj, At, Bt) do { __builtin_amdgcn_s_setprio(1); _Pragma("unroll") for (int m = 0; m < 4; ++m) _Pragma("unroll") for (int n = 0; n < 2; ++n) _Pragma("unroll") for (int k = 0; k < 2; ++k) \
        acc[ai][bj][m][n] = __builtin_amdgcn_mfma_f32_16x16x32_bf16(Bt[n][k], At[m][k], acc[ai][bj][m][n], 0, 0, 0); __builtin_amdgcn_s_setprio(0); } while (0)
#define BAR __builtin_amdgcn_s_barrier()
#define SCHED __builtin_amdgcn_sched_barrier(0)

    int qi = slot;
    while (qi < 32) {
        const int tile = tab8(p.xq[xcd], qi), pm = tile >> 4, pn = tile & 15, pos = tab8(p.pos, tile);
        const bool mine = ((pm + pn) & 1) == role;
        if (wid == 0) STAMP(tile * 8 + 0);

        auto hook = [&](bool remote) {
            if (wid == 0) {
                const unsigned long long t0 = __builtin_amdgcn_s_memrealtime();
                if (!remote) {
                    for (;;) {
                        const bool ok = ld_sys(ws + OFF_OWNW + (pn * 64 + lane) * 4) >= call;
                        if (__all(ok)) break;
                        if (give_up(ws, call, t0)) break;
                        __builtin_amdgcn_s_sleep(4);
                    }
                } else {
                    const char* fb = ws + OFF_INF + ((lane < 32 ? 2 * pm : 2 * pn + 1) * 32 + (lane & 31)) * 4;
                    for (;;) {
                        const bool ok = ld_sys(fb) >= call;
                        if (__all(ok)) break;
                        if (give_up(ws, call, t0)) break;
                        __builtin_amdgcn_s_sleep(16);
                    }
                    for (;;) {
                        bool ok = true;
#pragma unroll
                        for (int u = 1; u < 6; ++u) ok = ok & (ld_sys(fb + u * 4096) >= call);
                        if (__all(ok)) break;
                        if (give_up(ws, call, t0)) break;
                        __builtin_amdgcn_s_sleep(8);
                    }
                }
                __builtin_amdgcn_fence(__ATOMIC_ACQUIRE, "");
                asm volatile("s_waitcnt vmcnt(0)" ::: "memory");
            }
            asm volatile("" ::: "memory"); __builtin_amdgcn_s_barrier(); asm volatile("" ::: "memory");
        };

        f32x4 acc[2][2][4][2];
#pragma unroll
        for (int a = 0; a < 2; ++a)
#pragma unroll
            for (int b = 0; b < 2; ++b)
#pragma unroll
                for (int m = 0; m < 4; ++m)
#pragma unroll
                    for (int n = 0; n < 2; ++n) acc[a][b][m][n] = (f32x4){0.f, 0.f, 0.f, 0.f};
        bf16x8 At[4][2], B0[2][2], B1[2][2];

        const char* xsA = ws + OFF_XS + (size_t)pm * (512u << 10);
        const char* wsB = ws + OFF_WTS + (size_t)pn * (512u << 10);
        const char* cA = p.x + (size_t)((qd >> 1) * 4096 + pm * 256) * 2048;
        const char* cB = wsB + (size_t)3 * (8u << 20);
        hook(false);
        if (wid == 0) STAMP(tile * 8 + 1);
        STAGE(SB(0, 0), cB, voffB); STAGE(SA(0, 0), cA, voffA); STAGE(SB(0, 1), cB + hstep, voffB); STAGE(SA(0, 1), cA + hstep, voffA);
        if (wr == 1) BAR;
        WAIT_V(4); BAR;
        STAGE(SB(1, 0), cB + kstep, voffB); STAGE(SA(1, 0), cA + kstep, voffA); STAGE(SB(1, 1), cB + hstep + kstep, voffB);
        WAIT_V(6); BAR;
        constexpr int nt = KS / BK;
        for (int u = 0; u < 4; ++u) {
            const bool has_next = u + 1 < 4;
            const char* nA = has_next ? xsA + (size_t)u * (8u << 20) : cA;
            const char* nB = has_next ? wsB + (size_t)u * (8u << 20) : cB;
            for (int t = 0; t < nt; t += 2) {
                const bool last = (t == nt - 2);
                const char* a1 = cA + (size_t)(t + 1) * kstep;
                const char* a2 = last ? nA : cA + (size_t)(t + 2) * kstep; const char* b2 = last ? nB : cB + (size_t)(t + 2) * kstep;
                const char* a3 = a2 + kstep; const char* b3 = b2 + kstep;
                if (last && u == 0) {
                    if (wid == 0) STAMP(tile * 8 + 2);
                    hook(true);
                    if (wid == 0) STAMP(tile * 8 + 3);
                }
                LDB(B0, 0, 0); SCHED; LDA(At, 0, 0); STAGE(SA(1, 1), a1 + hstep, voffA);
                WAIT_L(8); BAR; WAIT_L(0); MMA(0, 0, At, B0); BAR; SCHED;
                LDB(B1, 0, 1); STAGE(SB(0, 0), b2, voffB);
                BAR; WAIT_L(0); MMA(0, 1, At, B1); BAR;
                LDA(At, 0, 1); STAGE(SA(0, 0), a2, voffA);
                BAR; WAIT_L(0); MMA(1, 0, At, B0); BAR; SCHED;
                STAGE(SB(0, 1), b2 + hstep, voffB);
                WAIT_V(6); BAR; MMA(1, 1, At, B1); BAR;
                LDB(B0, 1, 0); SCHED; LDA(At, 1, 0); STAGE(SA(0, 1), a2 + hstep, voffA);
                WAIT_L(8); BAR; WAIT_L(0); MMA(0, 0, At, B0); BAR; SCHED;
                LDB(B1, 1, 1); STAGE(SB(1, 0), b3, voffB);
                BAR; WAIT_L(0); MMA(0, 1, At, B1); BAR;
                LDA(At, 1, 1); STAGE(SA(1, 0), a3, voffA);
                BAR; WAIT_L(0); MMA(1, 0, At, B0); BAR; SCHED;
                STAGE(SB(1, 1), b3 + hstep, voffB);
                WAIT_V(6); BAR; MMA(1, 1, At, B1); BAR;
            }
            cA = nA; cB = nB;
        }
        if (wid == 0) STAMP(tile * 8 + 4);
        if (wr == 0) BAR;
        const __amdgpu_buffer_rsrc_t rw = mk_rsrc(ws, WS_NEED);
        const unsigned pbase = (unsigned)pos * (128u << 10) + (unsigned)wid * (16u << 10);
        if (!mine) {
#pragma unroll
            for (int ai = 0; ai < 2; ++ai)
#pragma unroll
                for (int m = 0; m < 4; ++m)
#pragma unroll
                    for (int bj = 0; bj < 2; ++bj) {
                        const f32x4 v0 = acc[ai][bj][m][0], v1 = acc[ai][bj][m][1];
                        u32x4 o;
                        o.x = cvt_pk_bf16(v0[0], v0[1]); o.y = cvt_pk_bf16(v0[2], v0[3]);
                        o.z = cvt_pk_bf16(v1[0], v1[1]); o.w = cvt_pk_bf16(v1[2], v1[3]);
                        __builtin_amdgcn_raw_buffer_store_b128(o, rw, lane * 16, OFF_PS + pbase + ((ai * 4 + m) * 2 + bj) * 1024u, 16);
                    }
        } else {
            if (wid == 0) {
                const unsigned long long t0 = __builtin_amdgcn_s_memrealtime();
                for (;;) {
                    const int fi = lane < 16 ? pos * 16 + lane : max(pos - 4, 0) * 16 + 15;
                    const unsigned long long m = __ballot(ld_sys(ws + OFF_PFLAG + fi * 4) >= call);
                    if ((m & 0xffffull) == 0xffffull) break;
                    if (give_up(ws, call, t0)) break;
                    if (m >> 16) __builtin_amdgcn_s_sleep(PSL_S); else __builtin_amdgcn_s_sleep(PSL_L);
                }
            }
            asm volatile("" ::: "memory"); __builtin_amdgcn_s_barrier(); asm volatile("" ::: "memory");
            if (wid == 0) STAMP(tile * 8 + 5);
            const __amdgpu_buffer_rsrc_t ro = mk_rsrc(p.out[rank], 128u << 20);
            const unsigned base = (unsigned)((qd >> 1) * 4096 + pm * BM) * 16384u + (unsigned)((qd & 1) * 4096 + pn * BM) * 2u;
            const unsigned voff = (unsigned)(wr * 64 + fr) * 16384u + (unsigned)(wc * 32 + 8 * fq) * 2u;
            u32x4 pr[2][4][2];
#pragma unroll
            for (int ai = 0; ai < 2; ++ai)
#pragma unroll
                for (int m = 0; m < 4; ++m)
#pragma unroll
                    for (int bj = 0; bj < 2; ++bj)
                        pr[ai][m][bj] = __builtin_amdgcn_raw_buffer_load_b128(rw, lane * 16, OFF_PR + pbase + ((ai * 4 + m) * 2 + bj) * 1024u, 17);
            __builtin_amdgcn_sched_barrier(0);
#pragma unroll
            for (int ai = 0; ai < 2; ++ai)
#pragma unroll
                for (int m = 0; m < 4; ++m)
#pragma unroll
                    for (int bj = 0; bj < 2; ++bj) {
                        const f32x4 v0 = acc[ai][bj][m][0], v1 = acc[ai][bj][m][1];
                        const u32x4 q = pr[ai][m][bj];
                        u32x4 o;
                        o.x = cvt_pk_bf16(silu(v0[0] + bf_lo(q.x)), silu(v0[1] + bf_hi(q.x))); o.y = cvt_pk_bf16(silu(v0[2] + bf_lo(q.y)), silu(v0[3] + bf_hi(q.y)));
                        o.z = cvt_pk_bf16(silu(v1[0] + bf_lo(q.z)), silu(v1[1] + bf_hi(q.z))); o.w = cvt_pk_bf16(silu(v1[2] + bf_lo(q.w)), silu(v1[3] + bf_hi(q.w)));
                        __builtin_amdgcn_raw_buffer_store_b128(o, ro, voff, base + (unsigned)(ai * HALF + m * 16) * 16384u + (unsigned)(bj * HALF) * 2u, 16);
                    }
        }
        WAIT_V(0);
        BAR;
        if (tid == 0) {
            st_agent(ws + (mine ? OFF_TFLAG : OFF_SFLAG) + pos * 4, call);
            const unsigned t = __hip_atomic_fetch_add((unsigned*)(ws + OFF_QCTR + (8 + xcd) * 128), 1u, __ATOMIC_RELAXED, __HIP_MEMORY_SCOPE_AGENT);
            *(volatile LAS unsigned*)lds = GPX + t;
        }
        if (wid == 0) STAMP(tile * 8 + 6);
        __syncthreads();
        qi = __builtin_amdgcn_readfirstlane(*(volatile LAS unsigned*)lds);
        __syncthreads();
    }
#undef SA
#undef SB
#undef STAGE
#undef LDA
#undef LDB
#undef MMA
#undef BAR
#undef SCHED
}

__global__ __launch_bounds__(512, 2) void fused_gemm_allreduce(const Params p_by_value) {
    KP p = *(const __attribute__((address_space(4))) Params*)__builtin_amdgcn_kernarg_segment_ptr();
    extern __shared__ __attribute__((aligned(16))) unsigned char lds_raw[];
    LAS unsigned char* lds = (LAS unsigned char*)lds_raw;
    const int b = blockIdx.x, xcd = b & 7, slot = b >> 3;
    char* ws = p.ws[p.rank];
    if (slot < NCW) {
        unsigned xcc;
        asm volatile("s_getreg_b32 %0, hwreg(HW_REG_XCC_ID)" : "=s"(xcc));
        const int xsel = slot == 0 ? xcd : (int)(xcc & 7u);
        comm_role(p, tab8(p.xlink, xsel), slot, slot * 8 + xcd);
    }
    const unsigned call = __builtin_amdgcn_readfirstlane(ld_agent(ws + OFF_SEQ)) + 1u;
    if (slot >= 32 - GPX) gemm_role(p, xcd, slot - (32 - GPX), call, lds);
    if (b == WAITER && threadIdx.x < 64) {
        const unsigned long long t1 = __builtin_amdgcn_s_memrealtime();
        for (;;) {
            const unsigned got = __builtin_amdgcn_readfirstlane(ld_sys(ws + OFF_OCNT));
            if (got >= call * N_TOTAL) break;
            if (got == (call - 1u) * N_TOTAL) {
                if (give_up(ws, call, t1)) break;
                __builtin_amdgcn_s_sleep(100);
            } else if (__builtin_amdgcn_s_memrealtime() - t1 > DEADLINE) {
                if (give_up(ws, call, t1)) break;
            }
        }
        STAMP(4095);
    }
    asm volatile("s_waitcnt vmcnt(0)" ::: "memory");
    __syncthreads();
    if (threadIdx.x == 0) {
        const unsigned t = __hip_atomic_fetch_add((unsigned*)(ws + OFF_DONE), 1u, __ATOMIC_RELAXED, __HIP_MEMORY_SCOPE_AGENT);
        if ((t + 1u) % (unsigned)GRID == 0u) {
            for (int l = 0; l < 16; ++l) st_agent(ws + OFF_QCTR + l * 128, 0u);
            st_sys(ws + OFF_QQ + ((call + 1u) & 1u) * 128, 0u);
            st_agent(ws + OFF_SEQ, call);
            if (ld_agent(ws + OFF_TMO) == call) {
                const int t0 = tab8(p.forder[role_of(p.rank)], 0), qd = quad_of(p.rank);
                const size_t off = (size_t)((qd >> 1) * 4096 + (t0 >> 4) * 256) * 16384 + (size_t)((qd & 1) * 4096 + (t0 & 15) * 256) * 2;
                st_agent(p.out[p.rank] + off, 0x7fc07fc0u);
            }
        }
    }
}

}

extern "C" void kernel_launch(int rank, int nranks, ncclComm_t comm,
                              hipStream_t stream,
                              void* const* d_in, const int* in_sizes, int n_in,
                              void* d_out, int out_size,
                              const int* in_dtypes, int out_dtype,
                              void* const* d_out_all,
                              void* const* d_ws_all, size_t ws_bytes) {
    if (nranks != NR || (unsigned)rank >= (unsigned)NR || ws_bytes < WS_NEED || n_in < 2) {
        fprintf(stderr, "unsupported configuration\n");
        return;
    }
    static bool attr_set[NR] = {};
    if (!attr_set[rank]) {
        HIP_CHECK(hipFuncSetAttribute((const void*)fused_gemm_allreduce, hipFuncAttributeMaxDynamicSharedMemorySize, LDS_BYTES));
        attr_set[rank] = true;
    }
    Params p;
    p.x = (const char*)d_in[0];
    p.w = (const char*)d_in[1];
    for (int r = 0; r < NR; ++r) { p.out[r] = (char*)d_out_all[r]; p.ws[r] = (char*)d_ws_all[r]; }
    p.rank = rank;
    {
        static const unsigned char xl[NR][8] = {
            {6, 255, 2, 3, 1, 5, 4, 7}, {7, 255, 4, 5, 0, 3, 2, 6}, {3, 255, 0, 6, 4, 5, 1, 7}, {2, 255, 0, 6, 1, 5, 4, 7},
            {5, 255, 1, 7, 0, 2, 3, 6}, {4, 255, 1, 7, 2, 3, 0, 6}, {0, 255, 2, 3, 4, 7, 1, 5}, {1, 255, 4, 5, 3, 6, 0, 2}};
        for (int i = 0; i < 8; ++i) ((unsigned char*)p.xlink)[i] = xl[rank][i];
    }
    {
        int nf[2] = {0, 0}, nx[8] = {0, 0, 0, 0, 0, 0, 0, 0};
        for (int e = 0; e < 32; ++e)
            for (int t = 0; t < 256; ++t) {
                const int pm = t >> 4, pn = t & 15, a = 2 * pm, bb = 2 * pn + 1;
                if ((a > bb ? a : bb) != e) continue;
                const int h = (pm + pn) & 1, g = (pm & 3) * 2 + ((pn >> 1) & 1);
                ((unsigned char*)p.forder[h])[nf[h]] = (unsigned char)t;
                ((unsigned char*)p.pos)[t] = (unsigned char)nf[h]++;
                ((unsigned char*)p.xq[g])[nx[g]++] = (unsigned char)t;
            }
    }
    hipLaunchKernelGGL(fused_gemm_allreduce, dim3(GRID), dim3(512), LDS_BYTES, stream, p);
    HIP_CHECK(hipGetLastError());
}
```

```cpp
#include <hip/hip_runtime.h>
#include <rccl/rccl.h>
#include <cstdio>
#include <cstdint>

#define HIP_CHECK(expr) do { hipError_t _e = (expr); if (_e != hipSuccess) { \
    fprintf(stderr, "HIP error %s at %s:%d\n", hipGetErrorString(_e), __FILE__, __LINE__); \
    return; } } while (0)

namespace {

#define LAS __attribute__((address_space(3)))
typedef short bf16x8 __attribute__((ext_vector_type(8)));
typedef float f32x4 __attribute__((ext_vector_type(4)));
typedef unsigned u32x4 __attribute__((ext_vector_type(4)));
typedef unsigned long long u64;

constexpr int NR = 8;
#define PM_ 7
constexpr int PM = PM_;
constexpr int KS = 1024;
constexpr int NPOS = 128;
#define NCW 12
#define WPW 1
#define NCP 12
#define EPUSH 8
#define STG_SLOT 4
#define PGATE (N_PUSH - 48)
#define GSL 32
#define PSL_S 24
#define PSL_L 120
#define CSL 24
#define GPX 16
constexpr int WPL = NCW * WPW;
constexpr int GRID = 256;
constexpr int N_IN = 32 * 32;
constexpr int N_FIN = NPOS * 16;
constexpr int N_PUSH = EPUSH * 32;
constexpr int N_PIN = N_IN - N_PUSH;
constexpr int N_STG = 2 * 2 * (16 - EPUSH / 2) * 64;
static_assert(EPUSH % 2 == 0 && EPUSH >= 2 && EPUSH <= 32, "EPUSH");
constexpr int NSPARE = 32 - GPX - (NCW > NCP ? NCW : NCP);
constexpr int NSTG_WG = (NSPARE + GPX - STG_SLOT) * 8;
static_assert(NSTG_WG > 0, "stagers");
constexpr int J_NONE = -N_IN - 1;
constexpr int SEGSH = 7;
constexpr int NSEG = N_FIN >> SEGSH;
static_assert(WPL <= 64 && NSEG <= 64, "config");
static_assert(NCW <= 32 - GPX && NCP <= 32 - GPX && GPX >= 16 && GPX <= 32, "too many workgroups");

constexpr unsigned OFF_SEQ = 0, OFF_DONE = 128, OFF_TMO = 256;
constexpr unsigned OFF_INF = 4096;
constexpr unsigned OFF_OWNW = OFF_INF + 6 * 32 * 32 * 4;
constexpr unsigned OFF_RT = OFF_OWNW + 16 * 64 * 4;
constexpr unsigned OFF_RS = OFF_RT + 8 * 512;
constexpr unsigned OFF_PFLAG = OFF_RS + 512;
constexpr unsigned OFF_HAVE = OFF_PFLAG + 128 * 16 * 4;
constexpr unsigned OFF_SR = OFF_HAVE + 6 * 64 * 4;
constexpr unsigned OFF_QCTR = OFF_SR + 128;
constexpr unsigned OFF_DBG = 128 * 1024;
constexpr unsigned OFF_XS = 1u << 20;
constexpr unsigned OFF_WTS = OFF_XS + (24u << 20);
constexpr unsigned OFF_PS = OFF_WTS + (32u << 20);
constexpr unsigned OFF_PR = OFF_PS + (16u << 20);
constexpr unsigned OFF_XO = OFF_PR + (16u << 20);
constexpr unsigned OFF_WO = OFF_XO + (16u << 20);
constexpr unsigned WS_NEED = OFF_WO + (16u << 20);
static_assert(OFF_QCTR + 27 * 128 <= OFF_DBG && WS_NEED <= (128u << 20), "layout");
constexpr unsigned long long DEADLINE = 40000000ull;

constexpr int BM = 256, BK = 64, HALF = 128, HTB = HALF * BK * 2, STAGE_BYTES = 8 * HTB;
constexpr int LDS_BYTES = STAGE_BYTES;

struct Params {
    const char* x;
    const char* w;
    char* out[NR];
    char* ws[NR];
    int rank;
    unsigned forder[2][NPOS / 4];
    unsigned pos[256 / 4];
    unsigned xq[8][32 / 4];
    unsigned xlink[2];
};
typedef const __attribute__((address_space(4))) Params& KP;

__host__ __device__ __forceinline__ int role_of(int r) { return (r >> 2) & 1; }
__host__ __device__ __forceinline__ int quad_of(int r) { return role_of(r) ? (r ^ PM) : r; }
__host__ __device__ __forceinline__ int uidx(int m) { return m < PM ? m - 1 : m - 2; }

__device__ __forceinline__ int lds_byte(int r, int c) { const int st = (r >> 4) * 2 + (c >> 5), rr = r & 15, cc = c & 31, ob = rr * 64 + cc * 2; return st * 1024 + (ob ^ (((ob >> 9) & 1) << 5)); }
__device__ __forceinline__ void stage_rc(int b, int& R, int& C) { const int st = b / 1024, sb = b % 1024, swz = sb ^ (((sb >> 9) & 1) << 5); R = (st >> 1) * 16 + swz / 64; C = (st & 1) * 32 + (swz % 64) / 2; }
__device__ __forceinline__ int perm32(int rho) { const int n = rho >> 4, i = rho & 15; return 8 * (i >> 2) + 4 * n + (i & 3); }

__device__ __forceinline__ unsigned ld_sys(const char* p) { return __hip_atomic_load((const unsigned*)p, __ATOMIC_RELAXED, __HIP_MEMORY_SCOPE_SYSTEM); }
__device__ __forceinline__ unsigned ld_agent(const char* p) { return __hip_atomic_load((const unsigned*)p, __ATOMIC_RELAXED, __HIP_MEMORY_SCOPE_AGENT); }
__device__ __forceinline__ void st_sys(char* p, unsigned v) { __hip_atomic_store((unsigned*)p, v, __ATOMIC_RELAXED, __HIP_MEMORY_SCOPE_SYSTEM); }
__device__ __forceinline__ void st_agent(char* p, unsigned v) { __hip_atomic_store((unsigned*)p, v, __ATOMIC_RELAXED, __HIP_MEMORY_SCOPE_AGENT); }
__device__ __forceinline__ unsigned cvt_pk_bf16(float lo, float hi) { unsigned r; asm("v_cvt_pk_bf16_f32 %0, %1, %2" : "=v"(r) : "v"(lo), "v"(hi)); return r; }
__device__ __forceinline__ float bf_lo(unsigned u) { return __uint_as_float(u << 16); }
__device__ __forceinline__ float bf_hi(unsigned u) { return __uint_as_float(u & 0xffff0000u); }
__device__ __forceinline__ float silu(float y) {
    const float t = fminf(fmaxf(y, -60.0f), 60.0f);
    const float e = __builtin_amdgcn_exp2f(t * -1.44269504088896f);
    return y * __builtin_amdgcn_rcpf(1.0f + e);
}
__device__ __forceinline__ __amdgpu_buffer_rsrc_t mk_rsrc(const void* p, unsigned bytes) {
    return __builtin_amdgcn_make_buffer_rsrc((void*)p, 0, bytes, 0x00020000);
}
template <class T> __device__ __forceinline__ int tab8(const T& t, int i) { return (t[i >> 2] >> (8 * (i & 3))) & 255; }
#define WAIT_V(n) asm volatile("s_waitcnt vmcnt(" #n ")" ::: "memory")
#define WAIT_L(n) asm volatile("s_waitcnt lgkmcnt(" #n ")" ::: "memory")

#define STAMP(idx) do { } while (0)

__device__ __forceinline__ bool give_up(char* ws, unsigned call, unsigned long long t0) {
    const unsigned tm = __builtin_amdgcn_readfirstlane(ld_agent(ws + OFF_TMO));
    if (tm == call || __builtin_amdgcn_s_memrealtime() - t0 > DEADLINE) {
        if ((threadIdx.x & 63) == 0) st_agent(ws + OFF_TMO, call);
        return true;
    }
    return false;
}

__device__ __forceinline__ u64 uni64(const void* p) {
    const u64 a = (u64)p;
    return ((u64)(unsigned)__builtin_amdgcn_readfirstlane((unsigned)(a >> 32)) << 32) | (unsigned)__builtin_amdgcn_readfirstlane((unsigned)a);
}
__device__ __forceinline__ u32x4 mk_srd(u64 a) {
    u32x4 r;
    r.x = (unsigned)a;
    r.y = (unsigned)(a >> 32) & 0xffffu;
    r.z = 128u << 20;
    r.w = 0x00020000u;
    return r;
}
#define CE_CLOB "memory", "scc", "vcc", "v156", "v157", "v158", "v159", "v160", "v161", "v162", "v163", "v164", "v165", "v166", "v167", "v168", "v169", "v170", "v171", "v172", "v173", "v174", "v175", "v176", "v177", "v178", "v179", "v180", "v181", "v182", "v183", "v184", "v185", "v186", "v187", "v188", "v189", "v190", "v191", "v192", "v193", "v194", "v195", "v196", "v197", "v198", "v199", "v200", "v201", "v202", "v203", "v204", "v205", "v206", "v207", "v208", "v209", "v210", "v211", "v212", "v213", "v214", "v215", "v216", "v217", "v218", "v219", "v220", "v221", "v222", "v223", "v224", "v225", "v226", "v227", "v228", "v229", "v230", "v231", "v232", "v233", "v234", "v235", "v236", "v237", "v238", "v239", "v240", "v241", "v242", "v243", "v244", "v245", "v246", "v247", "v248", "v249", "v250", "v251", "v252", "v253", "v254", "v255"
#define LDF "sc0 sc1"
template <int R> __device__ __forceinline__ void ld8(unsigned voff, const u32x4& srd, unsigned soff, unsigned step) {
    unsigned t;
    asm volatile(
        "s_nop 4\n\t"
        "buffer_load_dwordx4 v[%c5+0:%c5+3], %1, %2, %3 offen " LDF "\n\t"
        "s_add_u32 %0, %3, %4\n\t"
        "buffer_load_dwordx4 v[%c5+4:%c5+7], %1, %2, %0 offen " LDF "\n\t"
        "s_add_u32 %0, %0, %4\n\t"
        "buffer_load_dwordx4 v[%c5+8:%c5+11], %1, %2, %0 offen " LDF "\n\t"
        "s_add_u32 %0, %0, %4\n\t"
        "buffer_load_dwordx4 v[%c5+12:%c5+15], %1, %2, %0 offen " LDF "\n\t"
        "s_add_u32 %0, %0, %4\n\t"
        "buffer_load_dwordx4 v[%c5+16:%c5+19], %1, %2, %0 offen " LDF "\n\t"
        "s_add_u32 %0, %0, %4\n\t"
        "buffer_load_dwordx4 v[%c5+20:%c5+23], %1, %2, %0 offen " LDF "\n\t"
        "s_add_u32 %0, %0, %4\n\t"
        "buffer_load_dwordx4 v[%c5+24:%c5+27], %1, %2, %0 offen " LDF "\n\t"
        "s_add_u32 %0, %0, %4\n\t"
        "buffer_load_dwordx4 v[%c5+28:%c5+31], %1, %2, %0 offen " LDF "\n\t"
        "s_nop 0"
        : "=&s"(t)
        : "v"(voff), "s"(srd), "s"(soff), "s"(step), "i"(R)
        : CE_CLOB);
}
#define SWIN 4
#define SWN(j) ((j) < SWIN ? SWIN + 8 : SWIN - 1)
template <int R> __device__ __forceinline__ void st8(unsigned voff, const u32x4& srd, unsigned soff, unsigned step) {
    unsigned t;
    asm volatile(
        "s_nop 4\n\t"
        "s_waitcnt vmcnt(%c6)\n\t"
        "buffer_store_dwordx4 v[%c5+0:%c5+3], %1, %2, %3 offen sc0 sc1\n\t"
        "s_add_u32 %0, %3, %4\n\t"
        "s_waitcnt vmcnt(%c7)\n\t"
        "buffer_store_dwordx4 v[%c5+4:%c5+7], %1, %2, %0 offen sc0 sc1\n\t"
        "s_add_u32 %0, %0, %4\n\t"
        "s_waitcnt vmcnt(%c8)\n\t"
        "buffer_store_dwordx4 v[%c5+8:%c5+11], %1, %2, %0 offen sc0 sc1\n\t"
        "s_add_u32 %0, %0, %4\n\t"
        "s_waitcnt vmcnt(%c9)\n\t"
        "buffer_store_dwordx4 v[%c5+12:%c5+15], %1, %2, %0 offen sc0 sc1\n\t"
        "s_add_u32 %0, %0, %4\n\t"
        "s_waitcnt vmcnt(%c10)\n\t"
        "buffer_store_dwordx4 v[%c5+16:%c5+19], %1, %2, %0 offen sc0 sc1\n\t"
        "s_add_u32 %0, %0, %4\n\t"
        "s_waitcnt vmcnt(%c11)\n\t"
        "buffer_store_dwordx4 v[%c5+20:%c5+23], %1, %2, %0 offen sc0 sc1\n\t"
        "s_add_u32 %0, %0, %4\n\t"
        "s_waitcnt vmcnt(%c12)\n\t"
        "buffer_store_dwordx4 v[%c5+24:%c5+27], %1, %2, %0 offen sc0 sc1\n\t"
        "s_add_u32 %0, %0, %4\n\t"
        "s_waitcnt vmcnt(%c13)\n\t"
        "buffer_store_dwordx4 v[%c5+28:%c5+31], %1, %2, %0 offen sc0 sc1\n\t"
        "s_nop 1"
        : "=&s"(t)
        : "v"(voff), "s"(srd), "s"(soff), "s"(step), "i"(R), "i"(SWN(0)), "i"(SWN(1)), "i"(SWN(2)), "i"(SWN(3)), "i"(SWN(4)), "i"(SWN(5)), "i"(SWN(6)), "i"(SWN(7))
        : CE_CLOB);
}
template <int R> __device__ __forceinline__ void tr8() {
    asm volatile(
        "v_perm_b32 v224, v[%c2+4], v[%c2+0], %0\n\t"
        "v_perm_b32 v225, v[%c2+12], v[%c2+8], %0\n\t"
        "v_perm_b32 v226, v[%c2+20], v[%c2+16], %0\n\t"
        "v_perm_b32 v227, v[%c2+28], v[%c2+24], %0\n\t"
        "v_perm_b32 v228, v[%c2+4], v[%c2+0], %1\n\t"
        "v_perm_b32 v229, v[%c2+12], v[%c2+8], %1\n\t"
        "v_perm_b32 v230, v[%c2+20], v[%c2+16], %1\n\t"
        "v_perm_b32 v231, v[%c2+28], v[%c2+24], %1\n\t"
        "v_perm_b32 v232, v[%c2+5], v[%c2+1], %0\n\t"
        "v_perm_b32 v233, v[%c2+13], v[%c2+9], %0\n\t"
        "v_perm_b32 v234, v[%c2+21], v[%c2+17], %0\n\t"
        "v_perm_b32 v235, v[%c2+29], v[%c2+25], %0\n\t"
        "v_perm_b32 v236, v[%c2+5], v[%c2+1], %1\n\t"
        "v_perm_b32 v237, v[%c2+13], v[%c2+9], %1\n\t"
        "v_perm_b32 v238, v[%c2+21], v[%c2+17], %1\n\t"
        "v_perm_b32 v239, v[%c2+29], v[%c2+25], %1\n\t"
        "v_perm_b32 v240, v[%c2+6], v[%c2+2], %0\n\t"
        "v_perm_b32 v241, v[%c2+14], v[%c2+10], %0\n\t"
        "v_perm_b32 v242, v[%c2+22], v[%c2+18], %0\n\t"
        "v_perm_b32 v243, v[%c2+30], v[%c2+26], %0\n\t"
        "v_perm_b32 v244, v[%c2+6], v[%c2+2], %1\n\t"
        "v_perm_b32 v245, v[%c2+14], v[%c2+10], %1\n\t"
        "v_perm_b32 v246, v[%c2+22], v[%c2+18], %1\n\t"
        "v_perm_b32 v247, v[%c2+30], v[%c2+26], %1\n\t"
        "v_perm_b32 v248, v[%c2+7], v[%c2+3], %0\n\t"
        "v_perm_b32 v249, v[%c2+15], v[%c2+11], %0\n\t"
        "v_perm_b32 v250, v[%c2+23], v[%c2+19], %0\n\t"
        "v_perm_b32 v251, v[%c2+31], v[%c2+27], %0\n\t"
        "v_perm_b32 v252, v[%c2+7], v[%c2+3], %1\n\t"
        "v_perm_b32 v253, v[%c2+15], v[%c2+11], %1\n\t"
        "v_perm_b32 v254, v[%c2+23], v[%c2+19], %1\n\t"
        "v_perm_b32 v255, v[%c2+31], v[%c2+27], %1\n\t"
        "s_nop 1"
        :: "s"(0x05040100u), "s"(0x07060302u), "i"(R)
        : CE_CLOB);
}
template <int R> __device__ __forceinline__ void ld4(unsigned voff, const u32x4& srd, unsigned soff, unsigned step) {
    asm volatile(
        "s_nop 4\n\t"
        "buffer_load_dwordx4 v[%c4+0:%c4+3], %1, %2, %0 offen sc0 sc1\n\t"
        "s_add_u32 %0, %0, %3\n\t"
        "buffer_load_dwordx4 v[%c4+4:%c4+7], %1, %2, %0 offen sc0 sc1\n\t"
        "s_add_u32 %0, %0, %3\n\t"
        "buffer_load_dwordx4 v[%c4+8:%c4+11], %1, %2, %0 offen sc0 sc1\n\t"
        "s_add_u32 %0, %0, %3\n\t"
        "buffer_load_dwordx4 v[%c4+12:%c4+15], %1, %2, %0 offen sc0 sc1\n\t"
        "s_nop 0"
        : "+s"(soff)
        : "v"(voff), "s"(srd), "s"(step), "i"(R)
        : CE_CLOB);
}
template <int X, int Y> __device__ __forceinline__ void xfer(unsigned dst, unsigned src4, unsigned nsrc, unsigned voff, const u32x4& dsrd, unsigned step,
                                                         const u32x4& ssrd, unsigned nvoff, const u32x4& nsrd, unsigned nstep) {
    asm volatile(
        "s_nop 4\n\t"
        "s_waitcnt vmcnt(6)\n\t"
        "buffer_store_dwordx4 v[%c12+0:%c12+3], %3, %4, %0 offen sc0 sc1\n\t"
        "s_add_u32 %0, %0, %5\n\t"
        "buffer_load_dwordx4 v[%c12+16:%c12+19], %3, %6, %1 offen sc0 sc1\n\t"
        "s_add_u32 %1, %1, %5\n\t"
        "s_waitcnt vmcnt(6)\n\t"
        "buffer_store_dwordx4 v[%c12+4:%c12+7], %3, %4, %0 offen sc0 sc1\n\t"
        "s_add_u32 %0, %0, %5\n\t"
        "buffer_load_dwordx4 v[%c12+20:%c12+23], %3, %6, %1 offen sc0 sc1\n\t"
        "s_add_u32 %1, %1, %5\n\t"
        "s_waitcnt vmcnt(6)\n\t"
        "buffer_store_dwordx4 v[%c12+8:%c12+11], %3, %4, %0 offen sc0 sc1\n\t"
        "s_add_u32 %0, %0, %5\n\t"
        "buffer_load_dwordx4 v[%c12+24:%c12+27], %3, %6, %1 offen sc0 sc1\n\t"
        "s_add_u32 %1, %1, %5\n\t"
        "s_waitcnt vmcnt(6)\n\t"
        "buffer_store_dwordx4 v[%c12+12:%c12+15], %3, %4, %0 offen sc0 sc1\n\t"
        "s_add_u32 %0, %0, %5\n\t"
        "buffer_load_dwordx4 v[%c12+28:%c12+31], %3, %6, %1 offen sc0 sc1\n\t"
        "s_waitcnt vmcnt(6)\n\t"
        "buffer_store_dwordx4 v[%c12+16:%c12+19], %3, %4, %0 offen sc0 sc1\n\t"
        "s_add_u32 %0, %0, %5\n\t"
        "buffer_load_dwordx4 v[%c13+0:%c13+3], %7, %8, %2 offen sc0 sc1\n\t"
        "s_add_u32 %2, %2, %9\n\t"
        "s_waitcnt vmcnt(6)\n\t"
        "buffer_store_dwordx4 v[%c12+20:%c12+23], %3, %4, %0 offen sc0 sc1\n\t"
        "s_add_u32 %0, %0, %5\n\t"
        "buffer_load_dwordx4 v[%c13+4:%c13+7], %7, %8, %2 offen sc0 sc1\n\t"
        "s_add_u32 %2, %2, %9\n\t"
        "s_waitcnt vmcnt(6)\n\t"
        "buffer_store_dwordx4 v[%c12+24:%c12+27], %3, %4, %0 offen sc0 sc1\n\t"
        "s_add_u32 %0, %0, %5\n\t"
        "buffer_load_dwordx4 v[%c13+8:%c13+11], %7, %8, %2 offen sc0 sc1\n\t"
        "s_add_u32 %2, %2, %9\n\t"
        "s_waitcnt vmcnt(6)\n\t"
        "buffer_store_dwordx4 v[%c12+28:%c12+31], %3, %4, %0 offen sc0 sc1\n\t"
        "buffer_load_dwordx4 v[%c13+12:%c13+15], %7, %8, %2 offen sc0 sc1\n\t"
        "s_nop 1"
        : "+s"(dst), "+s"(src4), "+s"(nsrc)
        : "v"(voff), "s"(dsrd), "s"(step), "s"(ssrd), "v"(nvoff), "s"(nsrd), "s"(nstep), "i"(0), "i"(0), "i"(X), "i"(Y)
        : CE_CLOB);
}
__device__ __forceinline__ void ticket_issue(u64 ctr) {
    u64 sv;
    asm volatile(
        "s_mov_b64 %0, exec\n\t"
        "s_mov_b64 exec, 1\n\t"
        "v_mov_b32 v158, 0\n\t"
        "v_mov_b32 v159, 1\n\t"
        "s_nop 4\n\t"
        "global_atomic_add v159, v158, v159, %1 sc0\n\t"
        "s_mov_b64 exec, %0"
        : "=&s"(sv) : "s"(ctr) : CE_CLOB);
}
__device__ __forceinline__ int ticket_read() {
    int q;
    asm volatile("s_nop 1\n\tv_readfirstlane_b32 %0, v159\n\ts_nop 4" : "=s"(q) :: CE_CLOB);
    return q;
}
__device__ __forceinline__ void tflags_issue(u64 base, unsigned voff) {
    asm volatile(
        "s_nop 4\n\t"
        "global_load_dword v157, %0, %1 sc0 sc1\n\t"
        "global_load_dword v158, %0, %1 offset:256 sc0 sc1"
        :: "v"(voff), "s"(base) : CE_CLOB);
}
__device__ __forceinline__ int tflags_read(unsigned call) {
    u64 m0, m1;
    asm volatile("s_nop 1\n\tv_cmp_le_u32 %0, %2, v157\n\tv_cmp_le_u32 %1, %2, v158\n\ts_nop 4" : "=&s"(m0), "=&s"(m1) : "s"(call) : CE_CLOB);
    return (~m0 != 0ull) ? __builtin_ctzll(~m0) : 64 + ((~m1 != 0ull) ? __builtin_ctzll(~m1) : 64);
}
__device__ __forceinline__ void fcnt_issue(u64 base, unsigned voff) {
    asm volatile("s_nop 4\n\tglobal_load_dword v156, %0, %1 sc0 sc1" :: "v"(voff), "s"(base) : CE_CLOB);
}
__device__ __forceinline__ int fcnt_read(unsigned need) {
    u64 m0;
    asm volatile("s_nop 1\n\tv_cmp_le_u32 %0, %1, v156\n\ts_nop 4" : "=&s"(m0) : "s"(need) : CE_CLOB);
    return (~m0 != 0ull) ? __builtin_ctzll(~m0) : 64;
}
#define CE_WAIT(n) asm volatile("s_waitcnt vmcnt(" #n ")" ::: CE_CLOB)
constexpr int RA = 160, RB = 192, RT = 224;

__device__ __forceinline__ unsigned selv(bool c0, bool c1, unsigned a, unsigned b, unsigned c) {
    asm("" : "+v"(a)); asm("" : "+v"(b)); asm("" : "+v"(c));
    return c0 ? a : (c1 ? b : c);
}
__device__ __forceinline__ u64 sels(bool c, u64 a, u64 b) {
    asm("" : "+s"(a)); asm("" : "+s"(b));
    return uni64((const void*)(c ? a : b));
}

struct Piece { int type; unsigned src, dst, flag; };

struct Comm {
    KP p;
    int rank, peer, lane;
    unsigned call;
    bool partner;
    u64 rx, rw, ro_me, rws_me, rws_peer, ro_peer;
    unsigned lin_off, w_src_off, w_dst_off, o_off;
    char* wsp;
    char* wsh;
    char* wsm;

    __device__ __forceinline__ Comm(KP p_, int peer_, int lane_, unsigned call_) : p(p_), rank(p_.rank), peer(peer_), lane(lane_), call(call_) {
        partner = (rank ^ peer) == PM;
        rx = uni64(p.x);
        rw = uni64(p.w);
        ro_me = uni64(p.out[rank]);
        wsm = p.ws[rank];
        rws_me = uni64(wsm);
        wsp = p.ws[peer];
        wsh = p.ws[peer ^ PM];
        rws_peer = uni64(wsp);
        ro_peer = uni64(p.out[peer]);
        lin_off = lane * 16;
        w_src_off = (lane >> 3) * 8 * 16384 + (lane & 7) * 16;
        w_dst_off = (lane & 7) * 8 * 2048 + (lane >> 3) * 16;
        o_off = (lane >> 5) * 16384 + (lane & 31) * 16;
    }
    __device__ __forceinline__ unsigned out_off(int a, int o, int po) const {
        const int t = tab8(p.forder[role_of(a)], o), pm = t >> 4, pn = t & 15, qd = quad_of(a);
        return (unsigned)((qd >> 1) * 4096 + pm * 256 + po * 16) * 16384u + (unsigned)((qd & 1) * 4096 + pn * 256) * 2u;
    }
    __device__ __forceinline__ Piece desc_in(int q) const {
        Piece d;
        q = __builtin_amdgcn_readfirstlane(q);
        const int u = uidx(rank ^ peer);
        const int e = q >> 5, pc = q & 31, panel = e >> 1, hd = role_of(peer), qd = quad_of(peer);
        d.flag = OFF_INF + (u * 1024 + q) * 4;
        if (!(e & 1)) {
            d.type = 0;
            d.src = (unsigned)((qd >> 1) * 4096 + panel * 256 + pc * 8) * 2048u + hd * 1024u;
            d.dst = OFF_XS + (u >> 1) * (8u << 20) + (unsigned)(panel * 256 + pc * 8) * 2048u + (u & 1) * 1024u;
        } else {
            const int kb = pc >> 2, nb = pc & 3;
            d.type = 1;
            d.src = (unsigned)((hd * 512 + kb * 64) * 8192 + (qd & 1) * 4096 + panel * 256 + nb * 64) * 2u;
            d.dst = OFF_WTS + (u >> 1) * (8u << 20) + (unsigned)((panel * 256 + nb * 64) * 1024 + (u & 1) * 512 + kb * 64) * 2u;
        }
        d.type = __builtin_amdgcn_readfirstlane(d.type);
        d.src = __builtin_amdgcn_readfirstlane(d.src);
        d.dst = __builtin_amdgcn_readfirstlane(d.dst);
        d.flag = __builtin_amdgcn_readfirstlane(d.flag);
        return d;
    }
    template <int R> __device__ __forceinline__ void load(const Piece& d) const {
        ld8<R>(selv(d.type == 1, false, w_src_off, lin_off, lin_off), mk_srd(sels(d.type == 0, rx, rw)), d.src, d.type == 0 ? 2048u : 16384u);
    }
    template <int R> __device__ __forceinline__ void store(const Piece& d) const {
        if (d.type == 1) {
            tr8<R>();
            st8<RT>(w_dst_off, mk_srd(rws_peer), d.dst, 2048u);
        } else {
            st8<R>(lin_off, mk_srd(rws_peer), d.dst, 2048u);
        }
    }
    __device__ __forceinline__ void flag_in(int q) const {
        if (q >= 0 && lane == 0) st_sys(wsp + OFF_INF + (uidx(rank ^ peer) * 1024 + q) * 4, call);
    }
    __device__ __forceinline__ Piece desc_pull(int j) const {
        Piece d;
        j = __builtin_amdgcn_readfirstlane(j);
        if (j < -N_PIN) {
            d.type = 4; d.src = d.dst = OFF_PS; d.flag = 0;
        } else if (j < 0) {
            const int q = j + N_IN, e = q >> 5, pc = q & 31, panel = e >> 1, u = uidx(rank ^ peer), qd = quad_of(rank);
            const unsigned row = (unsigned)(panel * 256 + pc * 8) * 2048u;
            d.type = 5;
            d.src = ((e & 1) ? OFF_WO + (qd & 1) * (8u << 20) : OFF_XO + (qd >> 1) * (8u << 20)) + row + role_of(rank) * 1024u;
            d.dst = ((e & 1) ? OFF_WTS : OFF_XS) + (u >> 1) * (8u << 20) + row + (u & 1) * 1024u;
            d.flag = OFF_INF + (u * 1024 + q) * 4;
        } else if (j >= N_FIN) {
            const int qq = j - N_FIN;
            d.type = 2;
            d.src = d.dst = out_off(rank ^ PM, qq >> 4, qq & 15); d.flag = 0;
        } else if (partner) {
            d.type = 3;
            d.src = OFF_PS + j * 8192u; d.dst = OFF_PR + j * 8192u; d.flag = OFF_PFLAG + j * 4;
        } else {
            d.type = 2;
            d.src = d.dst = out_off(peer, j >> 4, j & 15);
            d.flag = (OFF_HAVE + (uidx(rank ^ peer ^ PM) * 64 + (j >> SEGSH)) * 4) | 1u;
        }
        d.type = __builtin_amdgcn_readfirstlane(d.type);
        d.src = __builtin_amdgcn_readfirstlane(d.src);
        d.dst = __builtin_amdgcn_readfirstlane(d.dst);
        d.flag = __builtin_amdgcn_readfirstlane(d.flag);
        return d;
    }
    __device__ __forceinline__ unsigned pvoff(const Piece& d) const { return selv(d.type == 2, false, o_off, lin_off, lin_off); }
    __device__ __forceinline__ unsigned pstep(const Piece& d) const { return d.type == 2 ? 32768u : (d.type == 5 ? 2048u : 1024u); }
    __device__ __forceinline__ u32x4 psrc(const Piece& d) const { return mk_srd(sels(d.type == 2, ro_peer, sels(d.type != 4, rws_peer, rws_me))); }
    __device__ __forceinline__ u32x4 pdst(const Piece& d) const { return mk_srd(sels(d.type == 2, ro_me, rws_me)); }
    template <int R> __device__ __forceinline__ void pull_first(const Piece& d) const { ld4<R>(pvoff(d), psrc(d), d.src, pstep(d)); }
    template <int X, int Y> __device__ __forceinline__ void pull_step(const Piece& c, const Piece& n) const {
        xfer<X, Y>(c.dst, c.src + 4 * pstep(c), n.src, pvoff(c), pdst(c), pstep(c), psrc(c), pvoff(n), psrc(n), pstep(n));
    }
    unsigned seg_off = 0, seg_cnt = 0;
    __device__ __forceinline__ void flush() {
        if (seg_cnt != 0 && lane == 0) __hip_atomic_fetch_add((unsigned*)(wsh + seg_off), seg_cnt, __ATOMIC_RELAXED, __HIP_MEMORY_SCOPE_SYSTEM);
        seg_cnt = 0;
    }
    __device__ __forceinline__ void flag_pull(unsigned off) {
        if (off & 1u) {
            if ((off & ~1u) != seg_off) { flush(); seg_off = off & ~1u; }
            ++seg_cnt;
        } else if (off != 0) { if (lane == 0) st_sys(wsm + off, call); }
        else flush();
    }
};

__device__ __forceinline__ void comm_role(KP p, int peer, int slot, int cidx) {
    const int rank = p.rank;
    char* ws = p.ws[rank];
    const int lane = threadIdx.x & 63, wid = __builtin_amdgcn_readfirstlane(threadIdx.x >> 6);
    if ((unsigned)peer >= (unsigned)NR || peer == rank) return;
    if (wid >= 2 || (wid == 0 && ((rank ^ peer) == PM || slot >= NCW)) || (wid == 1 && slot >= NCP)) return;
    const u64 ctr_own = uni64(ws + OFF_QCTR + (wid * 17 + peer) * 128);
    ticket_issue(ctr_own);
    const unsigned call = __builtin_amdgcn_readfirstlane(ld_agent(ws + OFF_SEQ)) + 1u;
    Comm c(p, peer, lane, call);
    STAMP(4096 + (cidx * 8 + wid) * 4 + 0);
    const bool partner = c.partner;
    CE_WAIT(0);
    int qn = ticket_read();

    if (wid == 0) {
        if (qn >= N_PUSH) return;
        int qc = qn, qp = -1;
        ticket_issue(ctr_own);
        c.load<RA>(c.desc_in(qc));
        CE_WAIT(8);
        qn = ticket_read();
        bool first = true;
#define CE_STEP(X, Y) { \
            if (qn >= N_PUSH) { CE_WAIT(0); c.flag_in(qp); c.store<X>(c.desc_in(qc)); CE_WAIT(0); c.flag_in(qc); break; } \
            ticket_issue(ctr_own); \
            c.load<Y>(c.desc_in(qn)); \
            if (first) CE_WAIT(9); else CE_WAIT(17); \
            first = false; \
            c.store<X>(c.desc_in(qc)); \
            CE_WAIT(16); \
            c.flag_in(qp); \
            qp = qc; qc = qn; \
            qn = ticket_read(); }
        for (;;) {
            CE_STEP(RA, RB)
            CE_STEP(RB, RA)
        }
#undef CE_STEP
        STAMP(4096 + (cidx * 8 + wid) * 4 + 1);
        return;
    }

    const u64 ctr_q = uni64(ws + OFF_QCTR + 16 * 128);
    const u64 rtf = uni64(ws + OFF_RT + peer * 512), rsf = uni64(ws + OFF_RS);
    const u64 hav = uni64(ws + OFF_HAVE + (partner ? 0 : uidx(rank ^ peer)) * 256);
    const unsigned tvoff = lane * 4;
    const unsigned long long t0 = __builtin_amdgcn_s_memrealtime();
    int qoff = partner ? 0 : -N_PIN;
    int qready = 0, qlim = N_FIN;
    bool inq = false;
    u64 ctr = ctr_own;
    if (!partner) {
        const char* gate = lane < 32 ? ws + OFF_INF + (uidx(rank ^ peer) * 1024 + PGATE + lane) * 4 : ws + OFF_SR + peer * 4;
        for (;;) {
            if (__all(ld_sys(gate) >= call)) break;
            if (give_up(ws, call, t0)) break;
            __builtin_amdgcn_s_sleep(GSL);
        }
    }
    STAMP(4096 + (cidx * 8 + wid) * 4 + 1);
    qn += qoff;
#define SNAP_ISSUE() do { tflags_issue(sels(partner && !inq, rsf, rtf), tvoff); if (inq) fcnt_issue(hav, tvoff); } while (0)
#define SNAP_APPLY() do { const int n_ = tflags_read(call); \
        int r_; \
        if (!inq) r_ = 16 * n_; \
        else r_ = N_FIN + (partner ? 16 * n_ : (min(fcnt_read(call << SEGSH), NSEG) << SEGSH)); \
        qready = __builtin_amdgcn_readfirstlane(max(qready, r_)); } while (0)
#define TO_QUEUE() do { STAMP(4096 + (cidx * 8 + wid) * 4 + 3); inq = true; qoff = N_FIN; qlim = 2 * N_FIN; ctr = ctr_q; qready = N_FIN; } while (0)
    for (;;) {
        if (qn >= qlim) {
            if (inq) break;
            TO_QUEUE();
            ticket_issue(ctr_q);
            SNAP_ISSUE();
            CE_WAIT(0);
            qn = qoff + ticket_read();
            SNAP_APPLY();
            continue;
        }
        while (qn >= qready) {
            SNAP_ISSUE();
            CE_WAIT(0);
            const int before = qready;
            SNAP_APPLY();
            if (qready != before) continue;
            if (give_up(ws, call, t0)) { qready = qlim; break; }
            __builtin_amdgcn_s_sleep(CSL);
        }
        qready = __builtin_amdgcn_readfirstlane(qready);
        ctr = uni64((const void*)ctr);
        int qc = qn, qp = J_NONE;
        ticket_issue(ctr);
        c.pull_first<RA>(c.desc_pull(qc));
        CE_WAIT(0);
        qn = qoff + ticket_read();
#define CE_STEP(X, Y) { \
            if (!inq && qn >= qlim) {         \
                TO_QUEUE(); ticket_issue(ctr_q); SNAP_ISSUE(); CE_WAIT(0); qn = qoff + ticket_read(); SNAP_APPLY(); \
                ctr = uni64((const void*)ctr); } \
            if (!(qn < qlim && qn < qready)) { \
                c.pull_step<X, Y>(c.desc_pull(qc), c.desc_pull(J_NONE)); \
                CE_WAIT(0); c.flag_pull(c.desc_pull(qp).flag); c.flag_pull(c.desc_pull(qc).flag); c.flush(); \
                break; } \
            const bool snap = qready < qlim && qn + (inq ? 16 : 3) * NCP >= qready; \
            ticket_issue(ctr); \
            if (snap) SNAP_ISSUE(); \
            c.pull_step<X, Y>(c.desc_pull(qc), c.desc_pull(qn)); \
            c.flag_pull(c.desc_pull(qp).flag); \
            qp = qc; qc = qn; \
            qn = qoff + ticket_read(); \
            if (snap) SNAP_APPLY(); }
        for (;;) {
            CE_STEP(RA, RB)
            CE_STEP(RB, RA)
        }
#undef CE_STEP
    }
    STAMP(4096 + (cidx * 8 + wid) * 4 + 2);
}

__device__ __forceinline__ void stage_role(KP p, int sidx) {
    const int rank = p.rank;
    char* ws = p.ws[rank];
    const int lane = threadIdx.x & 63, wid = __builtin_amdgcn_readfirstlane(threadIdx.x >> 6);
    const u32x4 sx = mk_srd(uni64(p.x)), sw = mk_srd(uni64(p.w)), sd = mk_srd(uni64(ws));
    const unsigned w_src_off = (lane >> 3) * 8 * 16384 + (lane & 7) * 16, w_dst_off = (lane & 7) * 8 * 2048 + (lane >> 3) * 16;
    for (int t = sidx * 8 + wid; t < N_STG / 2; t += NSTG_WG * 8) {
        const int pc = t & 63, hf = (t >> 6) & 1, panel = EPUSH / 2 + (t >> 7), kb = pc >> 2, nb = pc & 3;
        const unsigned xo = (unsigned)(hf * 4096 + panel * 256 + pc * 4) * 2048u;
        const int n0 = hf * 4096 + panel * 256 + nb * 64;
        ld8<RA>(lane * 16, sx, xo, 1024u);
        ld8<RB>(w_src_off, sw, (unsigned)(kb * 64 * 8192 + n0) * 2u, 16384u);
        CE_WAIT(8);
        st8<RA>(lane * 16, sd, OFF_XO + xo, 1024u);
        CE_WAIT(8);
        tr8<RB>();
        st8<RT>(w_dst_off, sd, OFF_WO + (unsigned)(n0 * 1024 + kb * 64) * 2u, 2048u);
    }
    CE_WAIT(0);
    __syncthreads();
    if (wid == 0) {
        unsigned tot = 0;
        if (lane == 0) tot = __hip_atomic_fetch_add((unsigned*)(ws + OFF_QCTR + 26 * 128), 1u, __ATOMIC_RELAXED, __HIP_MEMORY_SCOPE_AGENT) + 1u;
        if (__builtin_amdgcn_readfirstlane(tot) == (unsigned)NSTG_WG) {
            const unsigned call = ld_agent(ws + OFF_SEQ) + 1u;
            if (lane < NR && lane != rank) st_sys(p.ws[lane] + OFF_SR + rank * 4, call);
            STAMP(4094);
        }
    }
}

__device__ __forceinline__ void gemm_role(KP p, int xcd, int slot, unsigned call, LAS unsigned char* lds) {
    const int rank = p.rank;
    char* ws = p.ws[rank];
    const int tid = threadIdx.x, wid = __builtin_amdgcn_readfirstlane(tid >> 6), lane = tid & 63, wr = wid >> 2, wc = wid & 3, fr = lane & 15, fq = lane >> 4;
    const int qd = quad_of(rank), role = role_of(rank);

    if (slot < 16) {
        const int id = (slot * 8 + xcd) * 8 + wid, panel = id >> 6, pc = id & 63, kb = pc >> 2, nb = pc & 3;
        const unsigned w_src_off = (lane >> 3) * 8 * 16384 + (lane & 7) * 16, w_dst_off = (lane & 7) * 8 * 2048 + (lane >> 3) * 16;
        ld8<RA>(w_src_off, mk_srd(uni64(p.w)), (unsigned)(kb * 64 * 8192 + (qd & 1) * 4096 + panel * 256 + nb * 64) * 2u, 16384u);
        CE_WAIT(0);
        tr8<RA>();
        st8<RT>(w_dst_off, mk_srd(uni64(ws)), OFF_WTS + 3 * (8u << 20) + (unsigned)((panel * 256 + nb * 64) * 1024 + kb * 64) * 2u, 2048u);
        CE_WAIT(0);
        if (lane == 0) st_sys(ws + OFF_OWNW + id * 4, call);
    }

    if (slot >= STG_SLOT) stage_role(p, (NSPARE + slot - STG_SLOT) * 8 + xcd);
    unsigned voffA[2], voffB[2];
#pragma unroll
    for (int i = 0; i < 2; ++i) { int R, C; stage_rc(tid * 16 + i * 8192, R, C); const int Rb = (R & ~31) + perm32(R & 31);
        voffA[i] = (unsigned)(R * KS + C) * 2u; voffB[i] = (unsigned)(Rb * KS + C) * 2u; }
    constexpr size_t kstep = (size_t)(BK * 2);
    constexpr size_t hstep = (size_t)HALF * KS * 2;
    const unsigned ldsw = (unsigned)wid * 1024u;
    const int aoff = lds_byte(wr * 64 + fr, fq * 8), boff = lds_byte(wc * 32 + fr, fq * 8);
#define SA(b, h) (((b) * 2 + (h)) * HTB)
#define SB(b, h) ((4 + (b) * 2 + (h)) * HTB)
#define STAGE(bufoff, gbase, voff) do { _Pragma("unroll") for (int _i = 0; _i < 2; ++_i) \
        __builtin_amdgcn_global_load_lds((const unsigned*)((const char*)(gbase) + (voff)[_i]), (LAS unsigned*)(lds + (bufoff) + ldsw + _i * 8192), 16, 0, 0); } while (0)
#define LDA(dst, b, h) do { _Pragma("unroll") for (int m = 0; m < 4; ++m) _Pragma("unroll") for (int k = 0; k < 2; ++k) dst[m][k] = *(const LAS bf16x8*)(lds + SA(b, h) + aoff + m * 2048 + k * 1024); } while (0)
#define LDB(dst, b, h) do { _Pragma("unroll") for (int n = 0; n < 2; ++n) _Pragma("unroll") for (int k = 0; k < 2; ++k) dst[n][k] = *(const LAS bf16x8*)(lds + SB(b, h) + boff + n * 2048 + k * 1024); } while (0)
#define MMA(ai, bj, At, Bt) do { __builtin_amdgcn_s_setprio(1); _Pragma("unroll") for (int m = 0; m < 4; ++m) _Pragma("unroll") for (int n = 0; n < 2; ++n) _Pragma("unroll") for (int k = 0; k < 2; ++k) \
        acc[ai][bj][m][n] = __builtin_amdgcn_mfma_f32_16x16x32_bf16(Bt[n][k], At[m][k], acc[ai][bj][m][n], 0, 0, 0); __builtin_amdgcn_s_setprio(0); } while (0)
#define BAR __builtin_amdgcn_s_barrier()
#define SCHED __builtin_amdgcn_sched_barrier(0)

    int qi = slot;
    while (qi < 32) {
        const int tile = tab8(p.xq[xcd], qi), pm = tile >> 4, pn = tile & 15, pos = tab8(p.pos, tile);
        const bool mine = ((pm + pn) & 1) == role;
        if (wid == 0) STAMP(tile * 8 + 0);

        auto hook = [&](bool remote) {
            if (wid == 0) {
                const unsigned long long t0 = __builtin_amdgcn_s_memrealtime();
                if (!remote) {
                    for (;;) {
                        const bool ok = ld_sys(ws + OFF_OWNW + (pn * 64 + lane) * 4) >= call;
                        if (__all(ok)) break;
                        if (give_up(ws, call, t0)) break;
                        __builtin_amdgcn_s_sleep(4);
                    }
                } else {
                    const char* fb = ws + OFF_INF + ((lane < 32 ? 2 * pm : 2 * pn + 1) * 32 + (lane & 31)) * 4;
                    for (;;) {
                        const bool ok = ld_sys(fb) >= call;
                        if (__all(ok)) break;
                        if (give_up(ws, call, t0)) break;
                        __builtin_amdgcn_s_sleep(16);
                    }
                    for (;;) {
                        bool ok = true;
#pragma unroll
                        for (int u = 1; u < 6; ++u) ok = ok & (ld_sys(fb + u * 4096) >= call);
                        if (__all(ok)) break;
                        if (give_up(ws, call, t0)) break;
                        __builtin_amdgcn_s_sleep(8);
                    }
                }
                __builtin_amdgcn_fence(__ATOMIC_ACQUIRE, "");
                asm volatile("s_waitcnt vmcnt(0)" ::: "memory");
            }
            asm volatile("" ::: "memory"); __builtin_amdgcn_s_barrier(); asm volatile("" ::: "memory");
        };

        f32x4 acc[2][2][4][2];
#pragma unroll
        for (int a = 0; a < 2; ++a)
#pragma unroll
            for (int b = 0; b < 2; ++b)
#pragma unroll
                for (int m = 0; m < 4; ++m)
#pragma unroll
                    for (int n = 0; n < 2; ++n) acc[a][b][m][n] = (f32x4){0.f, 0.f, 0.f, 0.f};
        bf16x8 At[4][2], B0[2][2], B1[2][2];

        const char* xsA = ws + OFF_XS + (size_t)pm * (512u << 10);
        const char* wsB = ws + OFF_WTS + (size_t)pn * (512u << 10);
        const char* cA = p.x + (size_t)((qd >> 1) * 4096 + pm * 256) * 2048;
        const char* cB = wsB + (size_t)3 * (8u << 20);
        hook(false);
        if (wid == 0) STAMP(tile * 8 + 1);
        STAGE(SB(0, 0), cB, voffB); STAGE(SA(0, 0), cA, voffA); STAGE(SB(0, 1), cB + hstep, voffB); STAGE(SA(0, 1), cA + hstep, voffA);
        if (wr == 1) BAR;
        WAIT_V(4); BAR;
        STAGE(SB(1, 0), cB + kstep, voffB); STAGE(SA(1, 0), cA + kstep, voffA); STAGE(SB(1, 1), cB + hstep + kstep, voffB);
        WAIT_V(6); BAR;
        constexpr int nt = KS / BK;
        for (int u = 0; u < 4; ++u) {
            const bool has_next = u + 1 < 4;
            const char* nA = has_next ? xsA + (size_t)u * (8u << 20) : cA;
            const char* nB = has_next ? wsB + (size_t)u * (8u << 20) : cB;
            for (int t = 0; t < nt; t += 2) {
                const bool last = (t == nt - 2);
                const char* a1 = cA + (size_t)(t + 1) * kstep;
                const char* a2 = last ? nA : cA + (size_t)(t + 2) * kstep; const char* b2 = last ? nB : cB + (size_t)(t + 2) * kstep;
                const char* a3 = a2 + kstep; const char* b3 = b2 + kstep;
                if (last && u == 0) {
                    if (wid == 0) STAMP(tile * 8 + 2);
                    hook(true);
                    if (wid == 0) STAMP(tile * 8 + 3);
                }
                LDB(B0, 0, 0); SCHED; LDA(At, 0, 0); STAGE(SA(1, 1), a1 + hstep, voffA);
                WAIT_L(8); BAR; WAIT_L(0); MMA(0, 0, At, B0); BAR; SCHED;
                LDB(B1, 0, 1); STAGE(SB(0, 0), b2, voffB);
                BAR; WAIT_L(0); MMA(0, 1, At, B1); BAR;
                LDA(At, 0, 1); STAGE(SA(0, 0), a2, voffA);
                BAR; WAIT_L(0); MMA(1, 0, At, B0); BAR; SCHED;
                STAGE(SB(0, 1), b2 + hstep, voffB);
                WAIT_V(6); BAR; MMA(1, 1, At, B1); BAR;
                LDB(B0, 1, 0); SCHED; LDA(At, 1, 0); STAGE(SA(0, 1), a2 + hstep, voffA);
                WAIT_L(8); BAR; WAIT_L(0); MMA(0, 0, At, B0); BAR; SCHED;
                LDB(B1, 1, 1); STAGE(SB(1, 0), b3, voffB);
                BAR; WAIT_L(0); MMA(0, 1, At, B1); BAR;
                LDA(At, 1, 1); STAGE(SA(1, 0), a3, voffA);
                BAR; WAIT_L(0); MMA(1, 0, At, B0); BAR; SCHED;
                STAGE(SB(1, 1), b3 + hstep, voffB);
                WAIT_V(6); BAR; MMA(1, 1, At, B1); BAR;
            }
            cA = nA; cB = nB;
        }
        if (wid == 0) STAMP(tile * 8 + 4);
        if (wr == 0) BAR;
        const __amdgpu_buffer_rsrc_t rw = mk_rsrc(ws, WS_NEED);
        const unsigned pbase = (unsigned)pos * (128u << 10) + (unsigned)wid * (16u << 10);
        if (!mine) {
#pragma unroll
            for (int ai = 0; ai < 2; ++ai)
#pragma unroll
                for (int m = 0; m < 4; ++m)
#pragma unroll
                    for (int bj = 0; bj < 2; ++bj) {
                        const f32x4 v0 = acc[ai][bj][m][0], v1 = acc[ai][bj][m][1];
                        u32x4 o;
                        o.x = cvt_pk_bf16(v0[0], v0[1]); o.y = cvt_pk_bf16(v0[2], v0[3]);
                        o.z = cvt_pk_bf16(v1[0], v1[1]); o.w = cvt_pk_bf16(v1[2], v1[3]);
                        __builtin_amdgcn_raw_buffer_store_b128(o, rw, lane * 16, OFF_PS + pbase + ((ai * 4 + m) * 2 + bj) * 1024u, 17);
                    }
        } else {
            if (wid == 0) {
                const unsigned long long t0 = __builtin_amdgcn_s_memrealtime();
                for (;;) {
                    const int fi = lane < 16 ? pos * 16 + lane : max(pos - 4, 0) * 16 + 15;
                    const unsigned long long m = __ballot(ld_sys(ws + OFF_PFLAG + fi * 4) >= call);
                    if ((m & 0xffffull) == 0xffffull) break;
                    if (give_up(ws, call, t0)) break;
                    if (m >> 16) __builtin_amdgcn_s_sleep(PSL_S); else __builtin_amdgcn_s_sleep(PSL_L);
                }
            }
            asm volatile("" ::: "memory"); __builtin_amdgcn_s_barrier(); asm volatile("" ::: "memory");
            if (wid == 0) STAMP(tile * 8 + 5);
            const __amdgpu_buffer_rsrc_t ro = mk_rsrc(p.out[rank], 128u << 20);
            const unsigned base = (unsigned)((qd >> 1) * 4096 + pm * BM) * 16384u + (unsigned)((qd & 1) * 4096 + pn * BM) * 2u;
            const unsigned voff = (unsigned)(wr * 64 + fr) * 16384u + (unsigned)(wc * 32 + 8 * fq) * 2u;
            u32x4 pr[2][4][2];
#pragma unroll
            for (int ai = 0; ai < 2; ++ai)
#pragma unroll
                for (int m = 0; m < 4; ++m)
#pragma unroll
                    for (int bj = 0; bj < 2; ++bj)
                        pr[ai][m][bj] = __builtin_amdgcn_raw_buffer_load_b128(rw, lane * 16, OFF_PR + pbase + ((ai * 4 + m) * 2 + bj) * 1024u, 17);
            __builtin_amdgcn_sched_barrier(0);
#pragma unroll
            for (int ai = 0; ai < 2; ++ai)
#pragma unroll
                for (int m = 0; m < 4; ++m)
#pragma unroll
                    for (int bj = 0; bj < 2; ++bj) {
                        const f32x4 v0 = acc[ai][bj][m][0], v1 = acc[ai][bj][m][1];
                        const u32x4 q = pr[ai][m][bj];
                        u32x4 o;
                        o.x = cvt_pk_bf16(silu(v0[0] + bf_lo(q.x)), silu(v0[1] + bf_hi(q.x))); o.y = cvt_pk_bf16(silu(v0[2] + bf_lo(q.y)), silu(v0[3] + bf_hi(q.y)));
                        o.z = cvt_pk_bf16(silu(v1[0] + bf_lo(q.z)), silu(v1[1] + bf_hi(q.z))); o.w = cvt_pk_bf16(silu(v1[2] + bf_lo(q.w)), silu(v1[3] + bf_hi(q.w)));
                        __builtin_amdgcn_raw_buffer_store_b128(o, ro, voff, base + (unsigned)(ai * HALF + m * 16) * 16384u + (unsigned)(bj * HALF) * 2u, 17);
                    }
        }
        WAIT_V(0);
        BAR;
        if (tid < NR) {
            if (mine) { if (tid != rank) st_sys(p.ws[tid] + OFF_RT + rank * 512 + pos * 4, call); }
            else if (tid == (rank ^ PM)) st_sys(p.ws[tid] + OFF_RS + pos * 4, call);
        }
        if (tid == 0) {
            const unsigned t = __hip_atomic_fetch_add((unsigned*)(ws + OFF_QCTR + (8 + xcd) * 128), 1u, __ATOMIC_RELAXED, __HIP_MEMORY_SCOPE_AGENT);
            *(volatile LAS unsigned*)lds = GPX + t;
        }
        if (wid == 0) STAMP(tile * 8 + 6);
        __syncthreads();
        qi = __builtin_amdgcn_readfirstlane(*(volatile LAS unsigned*)lds);
        __syncthreads();
    }
#undef SA
#undef SB
#undef STAGE
#undef LDA
#undef LDB
#undef MMA
#undef BAR
#undef SCHED
}

__global__ __launch_bounds__(512, 2) void fused_gemm_allreduce(const Params p_by_value) {
    KP p = *(const __attribute__((address_space(4))) Params*)__builtin_amdgcn_kernarg_segment_ptr();
    extern __shared__ __attribute__((aligned(16))) unsigned char lds_raw[];
    LAS unsigned char* lds = (LAS unsigned char*)lds_raw;
    const int b = blockIdx.x, xcd = b & 7, slot = b >> 3;
    char* ws = p.ws[p.rank];
    if (slot < 32 - GPX) {
        unsigned xcc;
        asm volatile("s_getreg_b32 %0, hwreg(HW_REG_XCC_ID)" : "=s"(xcc));
        const int xsel = slot == 0 ? xcd : (int)(xcc & 7u);
        const int peer = (slot < NCW || slot < NCP) ? tab8(p.xlink, xsel) : 255;
        if (peer < NR) comm_role(p, peer, slot, slot * 8 + xcd);
        else if (slot >= 32 - GPX - NSPARE) stage_role(p, (slot - (32 - GPX - NSPARE)) * 8 + xcd);
    }
    const unsigned call = __builtin_amdgcn_readfirstlane(ld_agent(ws + OFF_SEQ)) + 1u;
    if (slot >= 32 - GPX) gemm_role(p, xcd, slot - (32 - GPX), call, lds);
    asm volatile("s_waitcnt vmcnt(0)" ::: "memory");
    __syncthreads();
    if (threadIdx.x == 0) {
        const unsigned t = __hip_atomic_fetch_add((unsigned*)(ws + OFF_DONE), 1u, __ATOMIC_RELAXED, __HIP_MEMORY_SCOPE_AGENT);
        if ((t + 1u) % (unsigned)GRID == 0u) {
            for (int l = 0; l < 27; ++l) st_agent(ws + OFF_QCTR + l * 128, 0u);
            st_agent(ws + OFF_SEQ, call);
            if (ld_agent(ws + OFF_TMO) == call) {
                const int t0 = tab8(p.forder[role_of(p.rank)], 0), qd = quad_of(p.rank);
                const size_t off = (size_t)((qd >> 1) * 4096 + (t0 >> 4) * 256) * 16384 + (size_t)((qd & 1) * 4096 + (t0 & 15) * 256) * 2;
                st_agent(p.out[p.rank] + off, 0x7fc07fc0u);
            }
        }
    }
}

}

extern "C" void kernel_launch(int rank, int nranks, ncclComm_t comm,
                              hipStream_t stream,
                              void* const* d_in, const int* in_sizes, int n_in,
                              void* d_out, int out_size,
                              const int* in_dtypes, int out_dtype,
                              void* const* d_out_all,
                              void* const* d_ws_all, size_t ws_bytes) {
    if (nranks != NR || (unsigned)rank >= (unsigned)NR || ws_bytes < WS_NEED || n_in < 2) {
        fprintf(stderr, "unsupported configuration\n");
        return;
    }
    static bool attr_set[NR] = {};
    if (!attr_set[rank]) {
        HIP_CHECK(hipFuncSetAttribute((const void*)fused_gemm_allreduce, hipFuncAttributeMaxDynamicSharedMemorySize, LDS_BYTES));
        attr_set[rank] = true;
    }
    Params p;
    p.x = (const char*)d_in[0];
    p.w = (const char*)d_in[1];
    for (int r = 0; r < NR; ++r) { p.out[r] = (char*)d_out_all[r]; p.ws[r] = (char*)d_ws_all[r]; }
    p.rank = rank;
    {
        static const unsigned char xl[NR][8] = {
            {6, 255, 2, 3, 1, 5, 4, 7}, {7, 255, 4, 5, 0, 3, 2, 6}, {3, 255, 0, 6, 4, 5, 1, 7}, {2, 255, 0, 6, 1, 5, 4, 7},
            {5, 255, 1, 7, 0, 2, 3, 6}, {4, 255, 1, 7, 2, 3, 0, 6}, {0, 255, 2, 3, 4, 7, 1, 5}, {1, 255, 4, 5, 3, 6, 0, 2}};
        for (int i = 0; i < 8; ++i) ((unsigned char*)p.xlink)[i] = xl[rank][i];
    }
    {
        int nf[2] = {0, 0}, nx[8] = {0, 0, 0, 0, 0, 0, 0, 0};
        for (int e = 0; e < 32; ++e)
            for (int t = 0; t < 256; ++t) {
                const int pm = t >> 4, pn = t & 15, a = 2 * pm, bb = 2 * pn + 1;
                if ((a > bb ? a : bb) != e) continue;
                const int h = (pm + pn) & 1, g = (pm & 3) * 2 + ((pn >> 1) & 1);
                ((unsigned char*)p.forder[h])[nf[h]] = (unsigned char)t;
                ((unsigned char*)p.pos)[t] = (unsigned char)nf[h]++;
                ((unsigned char*)p.xq[g])[nx[g]++] = (unsigned char)t;
            }
    }
    hipLaunchKernelGGL(fused_gemm_allreduce, dim3(GRID), dim3(512), LDS_BYTES, stream, p);
    HIP_CHECK(hipGetLastError());
}
```

```cpp
#include <hip/hip_runtime.h>
#include <rccl/rccl.h>
#include <cstdio>
#include <cstdint>

#define HIP_CHECK(expr) do { hipError_t _e = (expr); if (_e != hipSuccess) { \
    fprintf(stderr, "HIP error %s at %s:%d\n", hipGetErrorString(_e), __FILE__, __LINE__); \
    return; } } while (0)

namespace {

#define LAS __attribute__((address_space(3)))
typedef short bf16x8 __attribute__((ext_vector_type(8)));
typedef float f32x4 __attribute__((ext_vector_type(4)));
typedef unsigned u32x4 __attribute__((ext_vector_type(4)));
typedef unsigned long long u64;

constexpr int NR = 8;
#define PM_ 7
constexpr int PM = PM_;
constexpr int KS = 1024;
constexpr int NPOS = 128;
#define NCW 12
#define NCP 12
#define EPUSH 8
#define NHALF 50
#define STG_SLOT 4
#define PGATE (N_PUSH - 48)
#define GSL 32
#define PSL_S 24
#define PSL_L 120
#define CSL 24
#define GPX 16
constexpr int GRID = 256;
constexpr int N_IN = 32 * 32;
constexpr int N_FIN = NPOS * 16;
constexpr int N_PUSH = EPUSH * 32;
constexpr int N_PIN = N_IN - N_PUSH;
constexpr int N_STG = 2 * 2 * (16 - EPUSH / 2) * 64;
static_assert(EPUSH % 2 == 0 && EPUSH >= 2 && EPUSH <= 32, "EPUSH");
constexpr int NSPARE = 32 - GPX - (NCW > NCP ? NCW : NCP);
constexpr int NSTG_WG = (NSPARE + GPX - STG_SLOT) * 8;
static_assert(NSTG_WG > 0, "stagers");
constexpr int XQN = 48;
constexpr int J_NONE = -N_IN - 1;
constexpr int SEGSH = 7;
constexpr int NSEG = N_FIN >> SEGSH;
static_assert(NSEG <= 64, "config");
static_assert(NCW <= 32 - GPX && NCP <= 32 - GPX && GPX >= 16 && GPX <= 32, "too many workgroups");

constexpr unsigned OFF_SEQ = 0, OFF_DONE = 128, OFF_TMO = 256;
constexpr unsigned OFF_INF = 4096;
constexpr unsigned OFF_OWNW = OFF_INF + 6 * 32 * 32 * 4;
constexpr unsigned OFF_RT = OFF_OWNW + 16 * 64 * 4;
constexpr unsigned OFF_RS = OFF_RT + 8 * 512;
constexpr unsigned OFF_PFLAG = OFF_RS + 512;
constexpr unsigned OFF_HAVE = OFF_PFLAG + 128 * 16 * 4;
constexpr unsigned OFF_SR = OFF_HAVE + 6 * 64 * 4;
constexpr unsigned OFF_HCNT = OFF_SR + 128;
constexpr unsigned OFF_QCTR = OFF_HCNT + 512;
constexpr unsigned OFF_DBG = 128 * 1024;
constexpr unsigned OFF_XS = 1u << 20;
constexpr unsigned OFF_WTS = OFF_XS + (24u << 20);
constexpr unsigned OFF_PS = OFF_WTS + (32u << 20);
constexpr unsigned OFF_PR = OFF_PS + (16u << 20);
constexpr unsigned OFF_XO = OFF_PR + (16u << 20);
constexpr unsigned OFF_WO = OFF_XO + (16u << 20);
constexpr unsigned WS_NEED = OFF_WO + (16u << 20);
static_assert(OFF_QCTR + 27 * 128 <= OFF_DBG && WS_NEED <= (128u << 20), "layout");
constexpr unsigned long long DEADLINE = 40000000ull;

constexpr int BM = 256, BK = 64, HALF = 128, HTB = HALF * BK * 2, STAGE_BYTES = 8 * HTB;
constexpr int LDS_BYTES = STAGE_BYTES;

struct Params {
    const char* x;
    const char* w;
    char* out[NR];
    char* ws[NR];
    int rank;
    unsigned forder[2][NPOS / 4];
    unsigned pos[256 / 4];
    unsigned xq[8][XQN / 4];
    unsigned xm[8][XQN / 4];
    unsigned xlink[2];
};
typedef const __attribute__((address_space(4))) Params& KP;

__host__ __device__ __forceinline__ int role_of(int r) { return (r >> 2) & 1; }
__host__ __device__ __forceinline__ int quad_of(int r) { return role_of(r) ? (r ^ PM) : r; }
__host__ __device__ __forceinline__ int uidx(int m) { return m < PM ? m - 1 : m - 2; }

__device__ __forceinline__ int lds_byte(int r, int c) { const int st = (r >> 4) * 2 + (c >> 5), rr = r & 15, cc = c & 31, ob = rr * 64 + cc * 2; return st * 1024 + (ob ^ (((ob >> 9) & 1) << 5)); }
__device__ __forceinline__ void stage_rc(int b, int& R, int& C) { const int st = b / 1024, sb = b % 1024, swz = sb ^ (((sb >> 9) & 1) << 5); R = (st >> 1) * 16 + swz / 64; C = (st & 1) * 32 + (swz % 64) / 2; }
__device__ __forceinline__ int perm32(int rho) { const int n = rho >> 4, i = rho & 15; return 8 * (i >> 2) + 4 * n + (i & 3); }

__device__ __forceinline__ unsigned ld_sys(const char* p) { return __hip_atomic_load((const unsigned*)p, __ATOMIC_RELAXED, __HIP_MEMORY_SCOPE_SYSTEM); }
__device__ __forceinline__ unsigned ld_agent(const char* p) { return __hip_atomic_load((const unsigned*)p, __ATOMIC_RELAXED, __HIP_MEMORY_SCOPE_AGENT); }
__device__ __forceinline__ void st_sys(char* p, unsigned v) { __hip_atomic_store((unsigned*)p, v, __ATOMIC_RELAXED, __HIP_MEMORY_SCOPE_SYSTEM); }
__device__ __forceinline__ void st_agent(char* p, unsigned v) { __hip_atomic_store((unsigned*)p, v, __ATOMIC_RELAXED, __HIP_MEMORY_SCOPE_AGENT); }
__device__ __forceinline__ unsigned cvt_pk_bf16(float lo, float hi) { unsigned r; asm("v_cvt_pk_bf16_f32 %0, %1, %2" : "=v"(r) : "v"(lo), "v"(hi)); return r; }
__device__ __forceinline__ float bf_lo(unsigned u) { return __uint_as_float(u << 16); }
__device__ __forceinline__ float bf_hi(unsigned u) { return __uint_as_float(u & 0xffff0000u); }
__device__ __forceinline__ float silu(float y) {
    const float t = fminf(fmaxf(y, -60.0f), 60.0f);
    const float e = __builtin_amdgcn_exp2f(t * -1.44269504088896f);
    return y * __builtin_amdgcn_rcpf(1.0f + e);
}
__device__ __forceinline__ __amdgpu_buffer_rsrc_t mk_rsrc(const void* p, unsigned bytes) {
    return __builtin_amdgcn_make_buffer_rsrc((void*)p, 0, bytes, 0x00020000);
}
template <class T> __device__ __forceinline__ int tab8(const T& t, int i) { return (t[i >> 2] >> (8 * (i & 3))) & 255; }
#define WAIT_V(n) asm volatile("s_waitcnt vmcnt(" #n ")" ::: "memory")
#define WAIT_L(n) asm volatile("s_waitcnt lgkmcnt(" #n ")" ::: "memory")

#define STAMP(idx) do { } while (0)

__device__ __forceinline__ bool give_up(char* ws, unsigned call, unsigned long long t0) {
    const unsigned tm = __builtin_amdgcn_readfirstlane(ld_agent(ws + OFF_TMO));
    if (tm == call || __builtin_amdgcn_s_memrealtime() - t0 > DEADLINE) {
        if ((threadIdx.x & 63) == 0) st_agent(ws + OFF_TMO, call);
        return true;
    }
    return false;
}

__device__ __forceinline__ u64 uni64(const void* p) {
    const u64 a = (u64)p;
    return ((u64)(unsigned)__builtin_amdgcn_readfirstlane((unsigned)(a >> 32)) << 32) | (unsigned)__builtin_amdgcn_readfirstlane((unsigned)a);
}
__device__ __forceinline__ u32x4 mk_srd(u64 a) {
    u32x4 r;
    r.x = (unsigned)a;
    r.y = (unsigned)(a >> 32) & 0xffffu;
    r.z = 128u << 20;
    r.w = 0x00020000u;
    return r;
}
#define CE_CLOB "memory", "scc", "vcc", "v156", "v157", "v158", "v159", "v160", "v161", "v162", "v163", "v164", "v165", "v166", "v167", "v168", "v169", "v170", "v171", "v172", "v173", "v174", "v175", "v176", "v177", "v178", "v179", "v180", "v181", "v182", "v183", "v184", "v185", "v186", "v187", "v188", "v189", "v190", "v191", "v192", "v193", "v194", "v195", "v196", "v197", "v198", "v199", "v200", "v201", "v202", "v203", "v204", "v205", "v206", "v207", "v208", "v209", "v210", "v211", "v212", "v213", "v214", "v215", "v216", "v217", "v218", "v219", "v220", "v221", "v222", "v223", "v224", "v225", "v226", "v227", "v228", "v229", "v230", "v231", "v232", "v233", "v234", "v235", "v236", "v237", "v238", "v239", "v240", "v241", "v242", "v243", "v244", "v245", "v246", "v247", "v248", "v249", "v250", "v251", "v252", "v253", "v254", "v255"
#define LDF "sc0 sc1"
template <int R> __device__ __forceinline__ void ld8(unsigned voff, const u32x4& srd, unsigned soff, unsigned step) {
    unsigned t;
    asm volatile(
        "s_nop 4\n\t"
        "buffer_load_dwordx4 v[%c5+0:%c5+3], %1, %2, %3 offen " LDF "\n\t"
        "s_add_u32 %0, %3, %4\n\t"
        "buffer_load_dwordx4 v[%c5+4:%c5+7], %1, %2, %0 offen " LDF "\n\t"
        "s_add_u32 %0, %0, %4\n\t"
        "buffer_load_dwordx4 v[%c5+8:%c5+11], %1, %2, %0 offen " LDF "\n\t"
        "s_add_u32 %0, %0, %4\n\t"
        "buffer_load_dwordx4 v[%c5+12:%c5+15], %1, %2, %0 offen " LDF "\n\t"
        "s_add_u32 %0, %0, %4\n\t"
        "buffer_load_dwordx4 v[%c5+16:%c5+19], %1, %2, %0 offen " LDF "\n\t"
        "s_add_u32 %0, %0, %4\n\t"
        "buffer_load_dwordx4 v[%c5+20:%c5+23], %1, %2, %0 offen " LDF "\n\t"
        "s_add_u32 %0, %0, %4\n\t"
        "buffer_load_dwordx4 v[%c5+24:%c5+27], %1, %2, %0 offen " LDF "\n\t"
        "s_add_u32 %0, %0, %4\n\t"
        "buffer_load_dwordx4 v[%c5+28:%c5+31], %1, %2, %0 offen " LDF "\n\t"
        "s_nop 0"
        : "=&s"(t)
        : "v"(voff), "s"(srd), "s"(soff), "s"(step), "i"(R)
        : CE_CLOB);
}
#define SWIN 4
#define SWN(j) ((j) < SWIN ? SWIN + 8 : SWIN - 1)
template <int R> __device__ __forceinline__ void st8(unsigned voff, const u32x4& srd, unsigned soff, unsigned step) {
    unsigned t;
    asm volatile(
        "s_nop 4\n\t"
        "s_waitcnt vmcnt(%c6)\n\t"
        "buffer_store_dwordx4 v[%c5+0:%c5+3], %1, %2, %3 offen sc0 sc1\n\t"
        "s_add_u32 %0, %3, %4\n\t"
        "s_waitcnt vmcnt(%c7)\n\t"
        "buffer_store_dwordx4 v[%c5+4:%c5+7], %1, %2, %0 offen sc0 sc1\n\t"
        "s_add_u32 %0, %0, %4\n\t"
        "s_waitcnt vmcnt(%c8)\n\t"
        "buffer_store_dwordx4 v[%c5+8:%c5+11], %1, %2, %0 offen sc0 sc1\n\t"
        "s_add_u32 %0, %0, %4\n\t"
        "s_waitcnt vmcnt(%c9)\n\t"
        "buffer_store_dwordx4 v[%c5+12:%c5+15], %1, %2, %0 offen sc0 sc1\n\t"
        "s_add_u32 %0, %0, %4\n\t"
        "s_waitcnt vmcnt(%c10)\n\t"
        "buffer_store_dwordx4 v[%c5+16:%c5+19], %1, %2, %0 offen sc0 sc1\n\t"
        "s_add_u32 %0, %0, %4\n\t"
        "s_waitcnt vmcnt(%c11)\n\t"
        "buffer_store_dwordx4 v[%c5+20:%c5+23], %1, %2, %0 offen sc0 sc1\n\t"
        "s_add_u32 %0, %0, %4\n\t"
        "s_waitcnt vmcnt(%c12)\n\t"
        "buffer_store_dwordx4 v[%c5+24:%c5+27], %1, %2, %0 offen sc0 sc1\n\t"
        "s_add_u32 %0, %0, %4\n\t"
        "s_waitcnt vmcnt(%c13)\n\t"
        "buffer_store_dwordx4 v[%c5+28:%c5+31], %1, %2, %0 offen sc0 sc1\n\t"
        "s_nop 1"
        : "=&s"(t)
        : "v"(voff), "s"(srd), "s"(soff), "s"(step), "i"(R), "i"(SWN(0)), "i"(SWN(1)), "i"(SWN(2)), "i"(SWN(3)), "i"(SWN(4)), "i"(SWN(5)), "i"(SWN(6)), "i"(SWN(7))
        : CE_CLOB);
}
template <int R> __device__ __forceinline__ void tr8() {
    asm volatile(
        "v_perm_b32 v224, v[%c2+4], v[%c2+0], %0\n\t"
        "v_perm_b32 v225, v[%c2+12], v[%c2+8], %0\n\t"
        "v_perm_b32 v226, v[%c2+20], v[%c2+16], %0\n\t"
        "v_perm_b32 v227, v[%c2+28], v[%c2+24], %0\n\t"
        "v_perm_b32 v228, v[%c2+4], v[%c2+0], %1\n\t"
        "v_perm_b32 v229, v[%c2+12], v[%c2+8], %1\n\t"
        "v_perm_b32 v230, v[%c2+20], v[%c2+16], %1\n\t"
        "v_perm_b32 v231, v[%c2+28], v[%c2+24], %1\n\t"
        "v_perm_b32 v232, v[%c2+5], v[%c2+1], %0\n\t"
        "v_perm_b32 v233, v[%c2+13], v[%c2+9], %0\n\t"
        "v_perm_b32 v234, v[%c2+21], v[%c2+17], %0\n\t"
        "v_perm_b32 v235, v[%c2+29], v[%c2+25], %0\n\t"
        "v_perm_b32 v236, v[%c2+5], v[%c2+1], %1\n\t"
        "v_perm_b32 v237, v[%c2+13], v[%c2+9], %1\n\t"
        "v_perm_b32 v238, v[%c2+21], v[%c2+17], %1\n\t"
        "v_perm_b32 v239, v[%c2+29], v[%c2+25], %1\n\t"
        "v_perm_b32 v240, v[%c2+6], v[%c2+2], %0\n\t"
        "v_perm_b32 v241, v[%c2+14], v[%c2+10], %0\n\t"
        "v_perm_b32 v242, v[%c2+22], v[%c2+18], %0\n\t"
        "v_perm_b32 v243, v[%c2+30], v[%c2+26], %0\n\t"
        "v_perm_b32 v244, v[%c2+6], v[%c2+2], %1\n\t"
        "v_perm_b32 v245, v[%c2+14], v[%c2+10], %1\n\t"
        "v_perm_b32 v246, v[%c2+22], v[%c2+18], %1\n\t"
        "v_perm_b32 v247, v[%c2+30], v[%c2+26], %1\n\t"
        "v_perm_b32 v248, v[%c2+7], v[%c2+3], %0\n\t"
        "v_perm_b32 v249, v[%c2+15], v[%c2+11], %0\n\t"
        "v_perm_b32 v250, v[%c2+23], v[%c2+19], %0\n\t"
        "v_perm_b32 v251, v[%c2+31], v[%c2+27], %0\n\t"
        "v_perm_b32 v252, v[%c2+7], v[%c2+3], %1\n\t"
        "v_perm_b32 v253, v[%c2+15], v[%c2+11], %1\n\t"
        "v_perm_b32 v254, v[%c2+23], v[%c2+19], %1\n\t"
        "v_perm_b32 v255, v[%c2+31], v[%c2+27], %1\n\t"
        "s_nop 1"
        :: "s"(0x05040100u), "s"(0x07060302u), "i"(R)
        : CE_CLOB);
}
template <int R> __device__ __forceinline__ void ld4(unsigned voff, const u32x4& srd, unsigned soff, unsigned step) {
    asm volatile(
        "s_nop 4\n\t"
        "buffer_load_dwordx4 v[%c4+0:%c4+3], %1, %2, %0 offen sc0 sc1\n\t"
        "s_add_u32 %0, %0, %3\n\t"
        "buffer_load_dwordx4 v[%c4+4:%c4+7], %1, %2, %0 offen sc0 sc1\n\t"
        "s_add_u32 %0, %0, %3\n\t"
        "buffer_load_dwordx4 v[%c4+8:%c4+11], %1, %2, %0 offen sc0 sc1\n\t"
        "s_add_u32 %0, %0, %3\n\t"
        "buffer_load_dwordx4 v[%c4+12:%c4+15], %1, %2, %0 offen sc0 sc1\n\t"
        "s_nop 0"
        : "+s"(soff)
        : "v"(voff), "s"(srd), "s"(step), "i"(R)
        : CE_CLOB);
}
template <int X, int Y> __device__ __forceinline__ void xfer(unsigned dst, unsigned src4, unsigned nsrc, unsigned voff, const u32x4& dsrd, unsigned step,
                                                         const u32x4& ssrd, unsigned nvoff, const u32x4& nsrd, unsigned nstep) {
    asm volatile(
        "s_nop 4\n\t"
        "s_waitcnt vmcnt(6)\n\t"
        "buffer_store_dwordx4 v[%c12+0:%c12+3], %3, %4, %0 offen sc0 sc1\n\t"
        "s_add_u32 %0, %0, %5\n\t"
        "buffer_load_dwordx4 v[%c12+16:%c12+19], %3, %6, %1 offen sc0 sc1\n\t"
        "s_add_u32 %1, %1, %5\n\t"
        "s_waitcnt vmcnt(6)\n\t"
        "buffer_store_dwordx4 v[%c12+4:%c12+7], %3, %4, %0 offen sc0 sc1\n\t"
        "s_add_u32 %0, %0, %5\n\t"
        "buffer_load_dwordx4 v[%c12+20:%c12+23], %3, %6, %1 offen sc0 sc1\n\t"
        "s_add_u32 %1, %1, %5\n\t"
        "s_waitcnt vmcnt(6)\n\t"
        "buffer_store_dwordx4 v[%c12+8:%c12+11], %3, %4, %0 offen sc0 sc1\n\t"
        "s_add_u32 %0, %0, %5\n\t"
        "buffer_load_dwordx4 v[%c12+24:%c12+27], %3, %6, %1 offen sc0 sc1\n\t"
        "s_add_u32 %1, %1, %5\n\t"
        "s_waitcnt vmcnt(6)\n\t"
        "buffer_store_dwordx4 v[%c12+12:%c12+15], %3, %4, %0 offen sc0 sc1\n\t"
        "s_add_u32 %0, %0, %5\n\t"
        "buffer_load_dwordx4 v[%c12+28:%c12+31], %3, %6, %1 offen sc0 sc1\n\t"
        "s_waitcnt vmcnt(6)\n\t"
        "buffer_store_dwordx4 v[%c12+16:%c12+19], %3, %4, %0 offen sc0 sc1\n\t"
        "s_add_u32 %0, %0, %5\n\t"
        "buffer_load_dwordx4 v[%c13+0:%c13+3], %7, %8, %2 offen sc0 sc1\n\t"
        "s_add_u32 %2, %2, %9\n\t"
        "s_waitcnt vmcnt(6)\n\t"
        "buffer_store_dwordx4 v[%c12+20:%c12+23], %3, %4, %0 offen sc0 sc1\n\t"
        "s_add_u32 %0, %0, %5\n\t"
        "buffer_load_dwordx4 v[%c13+4:%c13+7], %7, %8, %2 offen sc0 sc1\n\t"
        "s_add_u32 %2, %2, %9\n\t"
        "s_waitcnt vmcnt(6)\n\t"
        "buffer_store_dwordx4 v[%c12+24:%c12+27], %3, %4, %0 offen sc0 sc1\n\t"
        "s_add_u32 %0, %0, %5\n\t"
        "buffer_load_dwordx4 v[%c13+8:%c13+11], %7, %8, %2 offen sc0 sc1\n\t"
        "s_add_u32 %2, %2, %9\n\t"
        "s_waitcnt vmcnt(6)\n\t"
        "buffer_store_dwordx4 v[%c12+28:%c12+31], %3, %4, %0 offen sc0 sc1\n\t"
        "buffer_load_dwordx4 v[%c13+12:%c13+15], %7, %8, %2 offen sc0 sc1\n\t"
        "s_nop 1"
        : "+s"(dst), "+s"(src4), "+s"(nsrc)
        : "v"(voff), "s"(dsrd), "s"(step), "s"(ssrd), "v"(nvoff), "s"(nsrd), "s"(nstep), "i"(0), "i"(0), "i"(X), "i"(Y)
        : CE_CLOB);
}
__device__ __forceinline__ void ticket_issue(u64 ctr) {
    u64 sv;
    asm volatile(
        "s_mov_b64 %0, exec\n\t"
        "s_mov_b64 exec, 1\n\t"
        "v_mov_b32 v158, 0\n\t"
        "v_mov_b32 v159, 1\n\t"
        "s_nop 4\n\t"
        "global_atomic_add v159, v158, v159, %1 sc0\n\t"
        "s_mov_b64 exec, %0"
        : "=&s"(sv) : "s"(ctr) : CE_CLOB);
}
__device__ __forceinline__ int ticket_read() {
    int q;
    asm volatile("s_nop 1\n\tv_readfirstlane_b32 %0, v159\n\ts_nop 4" : "=s"(q) :: CE_CLOB);
    return q;
}
__device__ __forceinline__ void tflags_issue(u64 base, unsigned voff) {
    asm volatile(
        "s_nop 4\n\t"
        "global_load_dword v157, %0, %1 sc0 sc1\n\t"
        "global_load_dword v158, %0, %1 offset:256 sc0 sc1"
        :: "v"(voff), "s"(base) : CE_CLOB);
}
__device__ __forceinline__ int tflags_read(unsigned call) {
    u64 m0, m1;
    asm volatile("s_nop 1\n\tv_cmp_le_u32 %0, %2, v157\n\tv_cmp_le_u32 %1, %2, v158\n\ts_nop 4" : "=&s"(m0), "=&s"(m1) : "s"(call) : CE_CLOB);
    return (~m0 != 0ull) ? __builtin_ctzll(~m0) : 64 + ((~m1 != 0ull) ? __builtin_ctzll(~m1) : 64);
}
__device__ __forceinline__ void fcnt_issue(u64 base, unsigned voff) {
    asm volatile("s_nop 4\n\tglobal_load_dword v156, %0, %1 sc0 sc1" :: "v"(voff), "s"(base) : CE_CLOB);
}
__device__ __forceinline__ int fcnt_read(unsigned need) {
    u64 m0;
    asm volatile("s_nop 1\n\tv_cmp_le_u32 %0, %1, v156\n\ts_nop 4" : "=&s"(m0) : "s"(need) : CE_CLOB);
    return (~m0 != 0ull) ? __builtin_ctzll(~m0) : 64;
}
#define CE_WAIT(n) asm volatile("s_waitcnt vmcnt(" #n ")" ::: CE_CLOB)
constexpr int RA = 160, RB = 192, RT = 224;

__device__ __forceinline__ unsigned selv(bool c0, bool c1, unsigned a, unsigned b, unsigned c) {
    asm("" : "+v"(a)); asm("" : "+v"(b)); asm("" : "+v"(c));
    return c0 ? a : (c1 ? b : c);
}
__device__ __forceinline__ u64 sels(bool c, u64 a, u64 b) {
    asm("" : "+s"(a)); asm("" : "+s"(b));
    return uni64((const void*)(c ? a : b));
}

struct Piece { int type; unsigned src, dst, flag; };

struct Comm {
    KP p;
    int rank, peer, lane;
    unsigned call;
    bool partner;
    u64 rx, rw, ro_me, rws_me, rws_peer, ro_peer;
    unsigned lin_off, w_src_off, w_dst_off, o_off;
    char* wsp;
    char* wsh;
    char* wsm;

    __device__ __forceinline__ Comm(KP p_, int peer_, int lane_, unsigned call_) : p(p_), rank(p_.rank), peer(peer_), lane(lane_), call(call_) {
        partner = (rank ^ peer) == PM;
        rx = uni64(p.x);
        rw = uni64(p.w);
        ro_me = uni64(p.out[rank]);
        wsm = p.ws[rank];
        rws_me = uni64(wsm);
        wsp = p.ws[peer];
        wsh = p.ws[peer ^ PM];
        rws_peer = uni64(wsp);
        ro_peer = uni64(p.out[peer]);
        lin_off = lane * 16;
        w_src_off = (lane >> 3) * 8 * 16384 + (lane & 7) * 16;
        w_dst_off = (lane & 7) * 8 * 2048 + (lane >> 3) * 16;
        o_off = (lane >> 5) * 16384 + (lane & 31) * 16;
    }
    __device__ __forceinline__ unsigned out_off(int a, int o, int po) const {
        const int t = tab8(p.forder[role_of(a)], o), pm = t >> 4, pn = t & 15, qd = quad_of(a);
        return (unsigned)((qd >> 1) * 4096 + pm * 256 + po * 16) * 16384u + (unsigned)((qd & 1) * 4096 + pn * 256) * 2u;
    }
    __device__ __forceinline__ Piece desc_in(int q) const {
        Piece d;
        q = __builtin_amdgcn_readfirstlane(q);
        const int u = uidx(rank ^ peer);
        const int e = q >> 5, pc = q & 31, panel = e >> 1, hd = role_of(peer), qd = quad_of(peer);
        d.flag = OFF_INF + (u * 1024 + q) * 4;
        if (!(e & 1)) {
            d.type = 0;
            d.src = (unsigned)((qd >> 1) * 4096 + panel * 256 + pc * 8) * 2048u + hd * 1024u;
            d.dst = OFF_XS + (u >> 1) * (8u << 20) + (unsigned)(panel * 256 + pc * 8) * 2048u + (u & 1) * 1024u;
        } else {
            const int kb = pc >> 2, nb = pc & 3;
            d.type = 1;
            d.src = (unsigned)((hd * 512 + kb * 64) * 8192 + (qd & 1) * 4096 + panel * 256 + nb * 64) * 2u;
            d.dst = OFF_WTS + (u >> 1) * (8u << 20) + (unsigned)((panel * 256 + nb * 64) * 1024 + (u & 1) * 512 + kb * 64) * 2u;
        }
        d.type = __builtin_amdgcn_readfirstlane(d.type);
        d.src = __builtin_amdgcn_readfirstlane(d.src);
        d.dst = __builtin_amdgcn_readfirstlane(d.dst);
        d.flag = __builtin_amdgcn_readfirstlane(d.flag);
        return d;
    }
    template <int R> __device__ __forceinline__ void load(const Piece& d) const {
        ld8<R>(selv(d.type == 1, false, w_src_off, lin_off, lin_off), mk_srd(sels(d.type == 0, rx, rw)), d.src, d.type == 0 ? 2048u : 16384u);
    }
    template <int R> __device__ __forceinline__ void store(const Piece& d) const {
        if (d.type == 1) {
            tr8<R>();
            st8<RT>(w_dst_off, mk_srd(rws_peer), d.dst, 2048u);
        } else {
            st8<R>(lin_off, mk_srd(rws_peer), d.dst, 2048u);
        }
    }
    __device__ __forceinline__ void flag_in(int q) const {
        if (q >= 0 && lane == 0) st_sys(wsp + OFF_INF + (uidx(rank ^ peer) * 1024 + q) * 4, call);
    }
    __device__ __forceinline__ Piece desc_pull(int j) const {
        Piece d;
        j = __builtin_amdgcn_readfirstlane(j);
        if (j < -N_PIN) {
            d.type = 4; d.src = d.dst = OFF_PS; d.flag = 0;
        } else if (j < 0) {
            const int q = j + N_IN, e = q >> 5, pc = q & 31, panel = e >> 1, u = uidx(rank ^ peer), qd = quad_of(rank);
            const unsigned row = (unsigned)(panel * 256 + pc * 8) * 2048u;
            d.type = 5;
            d.src = ((e & 1) ? OFF_WO + (qd & 1) * (8u << 20) : OFF_XO + (qd >> 1) * (8u << 20)) + row + role_of(rank) * 1024u;
            d.dst = ((e & 1) ? OFF_WTS : OFF_XS) + (u >> 1) * (8u << 20) + row + (u & 1) * 1024u;
            d.flag = OFF_INF + (u * 1024 + q) * 4;
        } else if (j >= N_FIN) {
            const int qq = j - N_FIN;
            d.type = 2;
            d.src = d.dst = out_off(rank ^ PM, qq >> 4, qq & 15); d.flag = 0;
        } else if (partner) {
            d.type = 3;
            d.src = OFF_PS + j * 8192u; d.dst = OFF_PR + j * 8192u; d.flag = OFF_PFLAG + j * 4;
        } else {
            d.type = 2;
            d.src = d.dst = out_off(peer, j >> 4, j & 15);
            d.flag = (OFF_HAVE + (uidx(rank ^ peer ^ PM) * 64 + (j >> SEGSH)) * 4) | 1u;
        }
        d.type = __builtin_amdgcn_readfirstlane(d.type);
        d.src = __builtin_amdgcn_readfirstlane(d.src);
        d.dst = __builtin_amdgcn_readfirstlane(d.dst);
        d.flag = __builtin_amdgcn_readfirstlane(d.flag);
        return d;
    }
    __device__ __forceinline__ unsigned pvoff(const Piece& d) const { return selv(d.type == 2, false, o_off, lin_off, lin_off); }
    __device__ __forceinline__ unsigned pstep(const Piece& d) const { return d.type == 2 ? 32768u : (d.type == 5 ? 2048u : 1024u); }
    __device__ __forceinline__ u32x4 psrc(const Piece& d) const { return mk_srd(sels(d.type == 2, ro_peer, sels(d.type != 4, rws_peer, rws_me))); }
    __device__ __forceinline__ u32x4 pdst(const Piece& d) const { return mk_srd(sels(d.type == 2, ro_me, rws_me)); }
    template <int R> __device__ __forceinline__ void pull_first(const Piece& d) const { ld4<R>(pvoff(d), psrc(d), d.src, pstep(d)); }
    template <int X, int Y> __device__ __forceinline__ void pull_step(const Piece& c, const Piece& n) const {
        xfer<X, Y>(c.dst, c.src + 4 * pstep(c), n.src, pvoff(c), pdst(c), pstep(c), psrc(c), pvoff(n), psrc(n), pstep(n));
    }
    unsigned seg_off = 0, seg_cnt = 0;
    __device__ __forceinline__ void flush() {
        if (seg_cnt != 0 && lane == 0) __hip_atomic_fetch_add((unsigned*)(wsh + seg_off), seg_cnt, __ATOMIC_RELAXED, __HIP_MEMORY_SCOPE_SYSTEM);
        seg_cnt = 0;
    }
    __device__ __forceinline__ void flag_pull(unsigned off) {
        if (off & 1u) {
            if ((off & ~1u) != seg_off) { flush(); seg_off = off & ~1u; }
            ++seg_cnt;
        } else if (off != 0) { if (lane == 0) st_sys(wsm + off, call); }
        else flush();
    }
};

__device__ __forceinline__ void comm_role(KP p, int peer, int slot, int cidx) {
    const int rank = p.rank;
    char* ws = p.ws[rank];
    const int lane = threadIdx.x & 63, wid = __builtin_amdgcn_readfirstlane(threadIdx.x >> 6);
    if ((unsigned)peer >= (unsigned)NR || peer == rank) return;
    if (wid >= 2 || (wid == 0 && ((rank ^ peer) == PM || slot >= NCW)) || (wid == 1 && slot >= NCP)) return;
    const u64 ctr_own = uni64(ws + OFF_QCTR + (wid * 17 + peer) * 128);
    ticket_issue(ctr_own);
    const unsigned call = __builtin_amdgcn_readfirstlane(ld_agent(ws + OFF_SEQ)) + 1u;
    Comm c(p, peer, lane, call);
    STAMP(4096 + (cidx * 8 + wid) * 4 + 0);
    const bool partner = c.partner;
    CE_WAIT(0);
    int qn = ticket_read();

    if (wid == 0) {
        if (qn >= N_PUSH) return;
        int qc = qn, qp = -1;
        ticket_issue(ctr_own);
        c.load<RA>(c.desc_in(qc));
        CE_WAIT(8);
        qn = ticket_read();
        bool first = true;
#define CE_STEP(X, Y) { \
            if (qn >= N_PUSH) { CE_WAIT(0); c.flag_in(qp); c.store<X>(c.desc_in(qc)); CE_WAIT(0); c.flag_in(qc); break; } \
            ticket_issue(ctr_own); \
            c.load<Y>(c.desc_in(qn)); \
            if (first) CE_WAIT(9); else CE_WAIT(17); \
            first = false; \
            c.store<X>(c.desc_in(qc)); \
            CE_WAIT(16); \
            c.flag_in(qp); \
            qp = qc; qc = qn; \
            qn = ticket_read(); }
        for (;;) {
            CE_STEP(RA, RB)
            CE_STEP(RB, RA)
        }
#undef CE_STEP
        STAMP(4096 + (cidx * 8 + wid) * 4 + 1);
        return;
    }

    const u64 ctr_q = uni64(ws + OFF_QCTR + 16 * 128);
    const u64 rtf = uni64(ws + OFF_RT + peer * 512), rsf = uni64(ws + OFF_RS);
    const u64 hav = uni64(ws + OFF_HAVE + (partner ? 0 : uidx(rank ^ peer)) * 256);
    const unsigned tvoff = lane * 4;
    const unsigned long long t0 = __builtin_amdgcn_s_memrealtime();
    int qoff = partner ? 0 : -N_PIN;
    int qready = 0, qlim = N_FIN;
    bool inq = false;
    u64 ctr = ctr_own;
    if (!partner) {
        const char* gate = lane < 32 ? ws + OFF_INF + (uidx(rank ^ peer) * 1024 + PGATE + lane) * 4 : ws + OFF_SR + peer * 4;
        for (;;) {
            if (__all(ld_sys(gate) >= call)) break;
            if (give_up(ws, call, t0)) break;
            __builtin_amdgcn_s_sleep(GSL);
        }
    }
    STAMP(4096 + (cidx * 8 + wid) * 4 + 1);
    qn += qoff;
#define SNAP_ISSUE() do { tflags_issue(sels(partner && !inq, rsf, rtf), tvoff); if (inq) fcnt_issue(hav, tvoff); } while (0)
#define SNAP_APPLY() do { const int n_ = tflags_read(call); \
        int r_; \
        if (!inq) r_ = 16 * n_; \
        else r_ = N_FIN + (partner ? 16 * n_ : (min(fcnt_read(call << SEGSH), NSEG) << SEGSH)); \
        qready = __builtin_amdgcn_readfirstlane(max(qready, r_)); } while (0)
#define TO_QUEUE() do { STAMP(4096 + (cidx * 8 + wid) * 4 + 3); inq = true; qoff = N_FIN; qlim = 2 * N_FIN; ctr = ctr_q; qready = N_FIN; } while (0)
    for (;;) {
        if (qn >= qlim) {
            if (inq) break;
            TO_QUEUE();
            ticket_issue(ctr_q);
            SNAP_ISSUE();
            CE_WAIT(0);
            qn = qoff + ticket_read();
            SNAP_APPLY();
            continue;
        }
        while (qn >= qready) {
            SNAP_ISSUE();
            CE_WAIT(0);
            const int before = qready;
            SNAP_APPLY();
            if (qready != before) continue;
            if (give_up(ws, call, t0)) { qready = qlim; break; }
            __builtin_amdgcn_s_sleep(CSL);
        }
        qready = __builtin_amdgcn_readfirstlane(qready);
        ctr = uni64((const void*)ctr);
        int qc = qn, qp = J_NONE;
        ticket_issue(ctr);
        c.pull_first<RA>(c.desc_pull(qc));
        CE_WAIT(0);
        qn = qoff + ticket_read();
#define CE_STEP(X, Y) { \
            if (!inq && qn >= qlim) {         \
                TO_QUEUE(); ticket_issue(ctr_q); SNAP_ISSUE(); CE_WAIT(0); qn = qoff + ticket_read(); SNAP_APPLY(); \
                ctr = uni64((const void*)ctr); } \
            if (!(qn < qlim && qn < qready)) { \
                c.pull_step<X, Y>(c.desc_pull(qc), c.desc_pull(J_NONE)); \
                CE_WAIT(0); c.flag_pull(c.desc_pull(qp).flag); c.flag_pull(c.desc_pull(qc).flag); c.flush(); \
                break; } \
            const bool snap = qready < qlim && qn + (inq ? 16 : 3) * NCP >= qready; \
            ticket_issue(ctr); \
            if (snap) SNAP_ISSUE(); \
            c.pull_step<X, Y>(c.desc_pull(qc), c.desc_pull(qn)); \
            c.flag_pull(c.desc_pull(qp).flag); \
            qp = qc; qc = qn; \
            qn = qoff + ticket_read(); \
            if (qc < 0 && qn >= 0) STAMP(4096 + (cidx * 8 + 2) * 4); \
            if (snap) SNAP_APPLY(); }
        for (;;) {
            CE_STEP(RA, RB)
            CE_STEP(RB, RA)
        }
#undef CE_STEP
    }
    STAMP(4096 + (cidx * 8 + wid) * 4 + 2);
}

__device__ __forceinline__ void stage_role(KP p, int sidx) {
    const int rank = p.rank;
    char* ws = p.ws[rank];
    const int lane = threadIdx.x & 63, wid = __builtin_amdgcn_readfirstlane(threadIdx.x >> 6);
    const u32x4 sx = mk_srd(uni64(p.x)), sw = mk_srd(uni64(p.w)), sd = mk_srd(uni64(ws));
    const unsigned w_src_off = (lane >> 3) * 8 * 16384 + (lane & 7) * 16, w_dst_off = (lane & 7) * 8 * 2048 + (lane >> 3) * 16;
    for (int t = sidx * 8 + wid; t < N_STG / 2; t += NSTG_WG * 8) {
        const int pc = t & 63, hf = (t >> 6) & 1, panel = EPUSH / 2 + (t >> 7), kb = pc >> 2, nb = pc & 3;
        const unsigned xo = (unsigned)(hf * 4096 + panel * 256 + pc * 4) * 2048u;
        const int n0 = hf * 4096 + panel * 256 + nb * 64;
        ld8<RA>(lane * 16, sx, xo, 1024u);
        ld8<RB>(w_src_off, sw, (unsigned)(kb * 64 * 8192 + n0) * 2u, 16384u);
        CE_WAIT(8);
        st8<RA>(lane * 16, sd, OFF_XO + xo, 1024u);
        CE_WAIT(8);
        tr8<RB>();
        st8<RT>(w_dst_off, sd, OFF_WO + (unsigned)(n0 * 1024 + kb * 64) * 2u, 2048u);
    }
    CE_WAIT(0);
    __syncthreads();
    if (wid == 0) {
        unsigned tot = 0;
        if (lane == 0) tot = __hip_atomic_fetch_add((unsigned*)(ws + OFF_QCTR + 26 * 128), 1u, __ATOMIC_RELAXED, __HIP_MEMORY_SCOPE_AGENT) + 1u;
        if (__builtin_amdgcn_readfirstlane(tot) == (unsigned)NSTG_WG) {
            const unsigned call = ld_agent(ws + OFF_SEQ) + 1u;
            if (lane < NR && lane != rank) st_sys(p.ws[lane] + OFF_SR + rank * 4, call);
            STAMP(4094);
        }
    }
}

__device__ __forceinline__ void gemm_role(KP p, int xcd, int slot, unsigned call, LAS unsigned char* lds) {
    const int rank = p.rank;
    char* ws = p.ws[rank];
    const int tid = threadIdx.x, wid = __builtin_amdgcn_readfirstlane(tid >> 6), lane = tid & 63, wr = wid >> 2, wc = wid & 3, fr = lane & 15, fq = lane >> 4;
    const int qd = quad_of(rank), role = role_of(rank);

    if (slot < 16) {
        const int id = (slot * 8 + xcd) * 8 + wid, panel = id >> 6, pc = id & 63, kb = pc >> 2, nb = pc & 3;
        const unsigned w_src_off = (lane >> 3) * 8 * 16384 + (lane & 7) * 16, w_dst_off = (lane & 7) * 8 * 2048 + (lane >> 3) * 16;
        ld8<RA>(w_src_off, mk_srd(uni64(p.w)), (unsigned)(kb * 64 * 8192 + (qd & 1) * 4096 + panel * 256 + nb * 64) * 2u, 16384u);
        CE_WAIT(0);
        tr8<RA>();
        st8<RT>(w_dst_off, mk_srd(uni64(ws)), OFF_WTS + 3 * (8u << 20) + (unsigned)((panel * 256 + nb * 64) * 1024 + kb * 64) * 2u, 2048u);
        CE_WAIT(0);
        if (lane == 0) st_sys(ws + OFF_OWNW + id * 4, call);
    }

    if (slot >= STG_SLOT) stage_role(p, (NSPARE + slot - STG_SLOT) * 8 + xcd);
    unsigned voffA[2], voffB[2];
#pragma unroll
    for (int i = 0; i < 2; ++i) { int R, C; stage_rc(tid * 16 + i * 8192, R, C); const int Rb = (R & ~31) + perm32(R & 31);
        voffA[i] = (unsigned)(R * KS + C) * 2u; voffB[i] = (unsigned)(Rb * KS + C) * 2u; }
    constexpr size_t kstep = (size_t)(BK * 2);
    constexpr size_t hstep = (size_t)HALF * KS * 2;
    const unsigned ldsw = (unsigned)wid * 1024u;
    const int aoff = lds_byte(wr * 64 + fr, fq * 8), boff = lds_byte(wc * 32 + fr, fq * 8);
#define SA(b, h) (((b) * 2 + (h)) * HTB)
#define SB(b, h) ((4 + (b) * 2 + (h)) * HTB)
#define STAGE(bufoff, gbase, voff) do { _Pragma("unroll") for (int _i = 0; _i < 2; ++_i) \
        __builtin_amdgcn_global_load_lds((const unsigned*)((const char*)(gbase) + (voff)[_i]), (LAS unsigned*)(lds + (bufoff) + ldsw + _i * 8192), 16, 0, 0); } while (0)
#define LDA(dst, b, h) do { _Pragma("unroll") for (int m = 0; m < 4; ++m) _Pragma("unroll") for (int k = 0; k < 2; ++k) dst[m][k] = *(const LAS bf16x8*)(lds + SA(b, h) + aoff + m * 2048 + k * 1024); } while (0)
#define LDB(dst, b, h) do { _Pragma("unroll") for (int n = 0; n < 2; ++n) _Pragma("unroll") for (int k = 0; k < 2; ++k) dst[n][k] = *(const LAS bf16x8*)(lds + SB(b, h) + boff + n * 2048 + k * 1024); } while (0)
#define MMA(ai, bj, At, Bt) do { __builtin_amdgcn_s_setprio(1); _Pragma("unroll") for (int m = 0; m < 4; ++m) _Pragma("unroll") for (int n = 0; n < 2; ++n) _Pragma("unroll") for (int k = 0; k < 2; ++k) \
        acc[ai][bj][m][n] = __builtin_amdgcn_mfma_f32_16x16x32_bf16(Bt[n][k], At[m][k], acc[ai][bj][m][n], 0, 0, 0); __builtin_amdgcn_s_setprio(0); } while (0)
#define BAR __builtin_amdgcn_s_barrier()
#define SCHED __builtin_amdgcn_sched_barrier(0)

    int qi = slot;
    while (qi < XQN) {
        const int mode = tab8(p.xm[xcd], qi);
        if (mode == 255) break;
        const int tile = tab8(p.xq[xcd], qi), pm = tile >> 4, pn = tile & 15, pos = tab8(p.pos, tile);
        const bool full = mode == 0;
        const size_t hstepA = full ? hstep : 0;
        const bool mine = ((pm + pn) & 1) == role;
        if (wid == 0) STAMP(tile * 8 + 0);

        auto hook = [&](bool remote) {
            if (wid == 0) {
                const unsigned long long t0 = __builtin_amdgcn_s_memrealtime();
                if (!remote) {
                    for (;;) {
                        const bool ok = ld_sys(ws + OFF_OWNW + (pn * 64 + lane) * 4) >= call;
                        if (__all(ok)) break;
                        if (give_up(ws, call, t0)) break;
                        __builtin_amdgcn_s_sleep(4);
                    }
                } else {
                    const char* fb = ws + OFF_INF + ((lane < 32 ? 2 * pm : 2 * pn + 1) * 32 + (lane & 31)) * 4;
                    for (;;) {
                        const bool ok = ld_sys(fb) >= call;
                        if (__all(ok)) break;
                        if (give_up(ws, call, t0)) break;
                        __builtin_amdgcn_s_sleep(16);
                    }
                    for (;;) {
                        bool ok = true;
#pragma unroll
                        for (int u = 1; u < 6; ++u) ok = ok & (ld_sys(fb + u * 4096) >= call);
                        if (__all(ok)) break;
                        if (give_up(ws, call, t0)) break;
                        __builtin_amdgcn_s_sleep(8);
                    }
                }
                __builtin_amdgcn_fence(__ATOMIC_ACQUIRE, "");
                asm volatile("s_waitcnt vmcnt(0)" ::: "memory");
            }
            asm volatile("" ::: "memory"); __builtin_amdgcn_s_barrier(); asm volatile("" ::: "memory");
        };

        f32x4 acc[2][2][4][2];
#pragma unroll
        for (int a = 0; a < 2; ++a)
#pragma unroll
            for (int b = 0; b < 2; ++b)
#pragma unroll
                for (int m = 0; m < 4; ++m)
#pragma unroll
                    for (int n = 0; n < 2; ++n) acc[a][b][m][n] = (f32x4){0.f, 0.f, 0.f, 0.f};
        bf16x8 At[4][2], B0[2][2], B1[2][2];

        const char* xsA = ws + OFF_XS + (size_t)pm * (512u << 10) + (mode == 2 ? hstep : 0);
        const char* wsB = ws + OFF_WTS + (size_t)pn * (512u << 10);
        const char* cA = p.x + (size_t)((qd >> 1) * 4096 + pm * 256) * 2048 + (mode == 2 ? hstep : 0);
        const char* cB = wsB + (size_t)3 * (8u << 20);
        hook(false);
        if (wid == 0) STAMP(tile * 8 + 1);
        STAGE(SB(0, 0), cB, voffB); STAGE(SA(0, 0), cA, voffA); STAGE(SB(0, 1), cB + hstep, voffB); STAGE(SA(0, 1), cA + hstepA, voffA);
        if (wr == 1) BAR;
        WAIT_V(4); BAR;
        STAGE(SB(1, 0), cB + kstep, voffB); STAGE(SA(1, 0), cA + kstep, voffA); STAGE(SB(1, 1), cB + hstep + kstep, voffB);
        WAIT_V(6); BAR;
        constexpr int nt = KS / BK;
        for (int u = 0; u < 4; ++u) {
            const bool has_next = u + 1 < 4;
            const char* nA = has_next ? xsA + (size_t)u * (8u << 20) : cA;
            const char* nB = has_next ? wsB + (size_t)u * (8u << 20) : cB;
            for (int t = 0; t < nt; t += 2) {
                const bool last = (t == nt - 2);
                const char* a1 = cA + (size_t)(t + 1) * kstep;
                const char* a2 = last ? nA : cA + (size_t)(t + 2) * kstep; const char* b2 = last ? nB : cB + (size_t)(t + 2) * kstep;
                const char* a3 = a2 + kstep; const char* b3 = b2 + kstep;
                if (last && u == 0) {
                    if (wid == 0) STAMP(tile * 8 + 2);
                    hook(true);
                    if (wid == 0) STAMP(tile * 8 + 3);
                }
                LDB(B0, 0, 0); SCHED; LDA(At, 0, 0); STAGE(SA(1, 1), a1 + hstepA, voffA);
                WAIT_L(8); BAR; WAIT_L(0); MMA(0, 0, At, B0); BAR; SCHED;
                LDB(B1, 0, 1); STAGE(SB(0, 0), b2, voffB);
                BAR; WAIT_L(0); MMA(0, 1, At, B1); BAR;
                if (full) LDA(At, 0, 1);
                STAGE(SA(0, 0), a2, voffA);
                BAR; WAIT_L(0); if (full) MMA(1, 0, At, B0); BAR; SCHED;
                STAGE(SB(0, 1), b2 + hstep, voffB);
                WAIT_V(6); BAR; if (full) MMA(1, 1, At, B1); BAR;
                LDB(B0, 1, 0); SCHED; LDA(At, 1, 0); STAGE(SA(0, 1), a2 + hstepA, voffA);
                WAIT_L(8); BAR; WAIT_L(0); MMA(0, 0, At, B0); BAR; SCHED;
                LDB(B1, 1, 1); STAGE(SB(1, 0), b3, voffB);
                BAR; WAIT_L(0); MMA(0, 1, At, B1); BAR;
                if (full) LDA(At, 1, 1);
                STAGE(SA(1, 0), a3, voffA);
                BAR; WAIT_L(0); if (full) MMA(1, 0, At, B0); BAR; SCHED;
                STAGE(SB(1, 1), b3 + hstep, voffB);
                WAIT_V(6); BAR; if (full) MMA(1, 1, At, B1); BAR;
            }
            cA = nA; cB = nB;
        }
        if (wid == 0) STAMP(tile * 8 + 4);
        if (wr == 0) BAR;
        const __amdgpu_buffer_rsrc_t rw = mk_rsrc(ws, WS_NEED);
        const unsigned pbase = (unsigned)pos * (128u << 10) + (unsigned)wid * (16u << 10);
        if (!mine) {
#pragma unroll
            for (int ai = 0; ai < 2; ++ai) {
                if (ai == 0 || full)
#pragma unroll
                for (int m = 0; m < 4; ++m)
#pragma unroll
                    for (int bj = 0; bj < 2; ++bj) {
                        const f32x4 v0 = acc[ai][bj][m][0], v1 = acc[ai][bj][m][1];
                        u32x4 o;
                        o.x = cvt_pk_bf16(v0[0], v0[1]); o.y = cvt_pk_bf16(v0[2], v0[3]);
                        o.z = cvt_pk_bf16(v1[0], v1[1]); o.w = cvt_pk_bf16(v1[2], v1[3]);
                        __builtin_amdgcn_raw_buffer_store_b128(o, rw, lane * 16, OFF_PS + pbase + (mode == 2 ? 8192u : 0u) + ((ai * 4 + m) * 2 + bj) * 1024u, 17);
                    }
            }
        } else {
            if (wid == 0) {
                const unsigned long long t0 = __builtin_amdgcn_s_memrealtime();
                for (;;) {
                    const int fi = lane < 16 ? pos * 16 + lane : max(pos - 4, 0) * 16 + 15;
                    const unsigned long long m = __ballot(ld_sys(ws + OFF_PFLAG + fi * 4) >= call);
                    if ((m & 0xffffull) == 0xffffull) break;
                    if (give_up(ws, call, t0)) break;
                    if (m >> 16) __builtin_amdgcn_s_sleep(PSL_S); else __builtin_amdgcn_s_sleep(PSL_L);
                }
            }
            asm volatile("" ::: "memory"); __builtin_amdgcn_s_barrier(); asm volatile("" ::: "memory");
            if (wid == 0) STAMP(tile * 8 + 5);
            const __amdgpu_buffer_rsrc_t ro = mk_rsrc(p.out[rank], 128u << 20);
            const unsigned base = (unsigned)((qd >> 1) * 4096 + pm * BM) * 16384u + (unsigned)((qd & 1) * 4096 + pn * BM) * 2u;
            const unsigned voff = (unsigned)(wr * 64 + fr) * 16384u + (unsigned)(wc * 32 + 8 * fq) * 2u;
            u32x4 pr[2][4][2];
#pragma unroll
            for (int ai = 0; ai < 2; ++ai)
#pragma unroll
                for (int m = 0; m < 4; ++m)
#pragma unroll
                    for (int bj = 0; bj < 2; ++bj)
                        pr[ai][m][bj] = __builtin_amdgcn_raw_buffer_load_b128(rw, lane * 16, OFF_PR + pbase + ((ai * 4 + m) * 2 + bj) * 1024u, 17);
            __builtin_amdgcn_sched_barrier(0);
#pragma unroll
            for (int ai = 0; ai < 2; ++ai)
#pragma unroll
                for (int m = 0; m < 4; ++m)
#pragma unroll
                    for (int bj = 0; bj < 2; ++bj) {
                        const f32x4 v0 = acc[ai][bj][m][0], v1 = acc[ai][bj][m][1];
                        const u32x4 q = pr[ai][m][bj];
                        u32x4 o;
                        o.x = cvt_pk_bf16(silu(v0[0] + bf_lo(q.x)), silu(v0[1] + bf_hi(q.x))); o.y = cvt_pk_bf16(silu(v0[2] + bf_lo(q.y)), silu(v0[3] + bf_hi(q.y)));
                        o.z = cvt_pk_bf16(silu(v1[0] + bf_lo(q.z)), silu(v1[1] + bf_hi(q.z))); o.w = cvt_pk_bf16(silu(v1[2] + bf_lo(q.w)), silu(v1[3] + bf_hi(q.w)));
                        __builtin_amdgcn_raw_buffer_store_b128(o, ro, voff, base + (unsigned)(ai * HALF + m * 16) * 16384u + (unsigned)(bj * HALF) * 2u, 17);
                    }
        }
        WAIT_V(0);
        BAR;
        if (tid < NR) {
            if (mine) { if (tid != rank) st_sys(p.ws[tid] + OFF_RT + rank * 512 + pos * 4, call); }
            else if (tid == (rank ^ PM)) {
                if (full || (__hip_atomic_fetch_add((unsigned*)(ws + OFF_HCNT + pos * 4), 1u, __ATOMIC_RELAXED, __HIP_MEMORY_SCOPE_AGENT) & 1u))
                    st_sys(p.ws[tid] + OFF_RS + pos * 4, call);
            }
        }
        if (tid == 0) {
            const unsigned t = __hip_atomic_fetch_add((unsigned*)(ws + OFF_QCTR + (8 + xcd) * 128), 1u, __ATOMIC_RELAXED, __HIP_MEMORY_SCOPE_AGENT);
            *(volatile LAS unsigned*)lds = GPX + t;
        }
        if (wid == 0) STAMP(tile * 8 + 6);
        __syncthreads();
        qi = __builtin_amdgcn_readfirstlane(*(volatile LAS unsigned*)lds);
        __syncthreads();
    }
#undef SA
#undef SB
#undef STAGE
#undef LDA
#undef LDB
#undef MMA
#undef BAR
#undef SCHED
}

__global__ __launch_bounds__(512, 2) void fused_gemm_allreduce(const Params p_by_value) {
    KP p = *(const __attribute__((address_space(4))) Params*)__builtin_amdgcn_kernarg_segment_ptr();
    extern __shared__ __attribute__((aligned(16))) unsigned char lds_raw[];
    LAS unsigned char* lds = (LAS unsigned char*)lds_raw;
    const int b = blockIdx.x, xcd = b & 7, slot = b >> 3;
    char* ws = p.ws[p.rank];
    if (slot < 32 - GPX) {
        unsigned xcc;
        asm volatile("s_getreg_b32 %0, hwreg(HW_REG_XCC_ID)" : "=s"(xcc));
        const int xsel = slot == 0 ? xcd : (int)(xcc & 7u);
        const int peer = (slot < NCW || slot < NCP) ? tab8(p.xlink, xsel) : 255;
        if (peer < NR) comm_role(p, peer, slot, slot * 8 + xcd);
        else if (slot >= 32 - GPX - NSPARE) stage_role(p, (slot - (32 - GPX - NSPARE)) * 8 + xcd);
    }
    const unsigned call = __builtin_amdgcn_readfirstlane(ld_agent(ws + OFF_SEQ)) + 1u;
    if (slot >= 32 - GPX) gemm_role(p, xcd, slot - (32 - GPX), call, lds);
    asm volatile("s_waitcnt vmcnt(0)" ::: "memory");
    __syncthreads();
    if (threadIdx.x == 0) {
        const unsigned t = __hip_atomic_fetch_add((unsigned*)(ws + OFF_DONE), 1u, __ATOMIC_RELAXED, __HIP_MEMORY_SCOPE_AGENT);
        if ((t + 1u) % (unsigned)GRID == 0u) {
            for (int l = 0; l < 27; ++l) st_agent(ws + OFF_QCTR + l * 128, 0u);
            st_agent(ws + OFF_SEQ, call);
            if (ld_agent(ws + OFF_TMO) == call) {
                const int t0 = tab8(p.forder[role_of(p.rank)], 0), qd = quad_of(p.rank);
                const size_t off = (size_t)((qd >> 1) * 4096 + (t0 >> 4) * 256) * 16384 + (size_t)((qd & 1) * 4096 + (t0 & 15) * 256) * 2;
                st_agent(p.out[p.rank] + off, 0x7fc07fc0u);
            }
        }
    }
}

}

extern "C" void kernel_launch(int rank, int nranks, ncclComm_t comm,
                              hipStream_t stream,
                              void* const* d_in, const int* in_sizes, int n_in,
                              void* d_out, int out_size,
                              const int* in_dtypes, int out_dtype,
                              void* const* d_out_all,
                              void* const* d_ws_all, size_t ws_bytes) {
    if (nranks != NR || (unsigned)rank >= (unsigned)NR || ws_bytes < WS_NEED || n_in < 2) {
        fprintf(stderr, "unsupported configuration\n");
        return;
    }
    static bool attr_set[NR] = {};
    if (!attr_set[rank]) {
        HIP_CHECK(hipFuncSetAttribute((const void*)fused_gemm_allreduce, hipFuncAttributeMaxDynamicSharedMemorySize, LDS_BYTES));
        attr_set[rank] = true;
    }
    Params p;
    p.x = (const char*)d_in[0];
    p.w = (const char*)d_in[1];
    for (int r = 0; r < NR; ++r) { p.out[r] = (char*)d_out_all[r]; p.ws[r] = (char*)d_ws_all[r]; }
    p.rank = rank;
    {
        static const unsigned char xl[NR][8] = {
            {6, 255, 2, 3, 1, 5, 4, 7}, {7, 255, 4, 5, 0, 3, 2, 6}, {3, 255, 0, 6, 4, 5, 1, 7}, {2, 255, 0, 6, 1, 5, 4, 7},
            {5, 255, 1, 7, 0, 2, 3, 6}, {4, 255, 1, 7, 2, 3, 0, 6}, {0, 255, 2, 3, 4, 7, 1, 5}, {1, 255, 4, 5, 3, 6, 0, 2}};
        for (int i = 0; i < 8; ++i) ((unsigned char*)p.xlink)[i] = xl[rank][i];
    }
    {
        int nf[2] = {0, 0}, nx[8] = {0, 0, 0, 0, 0, 0, 0, 0}, nt[8] = {0, 0, 0, 0, 0, 0, 0, 0};
        for (int g = 0; g < 8; ++g)
            for (int i = 0; i < XQN; ++i) { ((unsigned char*)p.xq[g])[i] = 0; ((unsigned char*)p.xm[g])[i] = 255; }
        for (int e = 0; e < 32; ++e)
            for (int t = 0; t < 256; ++t) {
                const int pm = t >> 4, pn = t & 15, a = 2 * pm, bb = 2 * pn + 1;
                if ((a > bb ? a : bb) != e) continue;
                const int h = (pm + pn) & 1, g = (pm & 3) * 2 + ((pn >> 1) & 1);
                ((unsigned char*)p.forder[h])[nf[h]] = (unsigned char)t;
                ((unsigned char*)p.pos)[t] = (unsigned char)nf[h]++;
                const bool split = h != role_of(rank) && nf[h] <= NHALF && nx[g] + 2 + (32 - nt[g] - 1) <= XQN;
                for (int m = split ? 1 : 0; m <= (split ? 2 : 0); ++m) {
                    ((unsigned char*)p.xq[g])[nx[g]] = (unsigned char)t;
                    ((unsigned char*)p.xm[g])[nx[g]++] = (unsigned char)m;
                }
                ++nt[g];
            }
    }
    hipLaunchKernelGGL(fused_gemm_allreduce, dim3(GRID), dim3(512), LDS_BYTES, stream, p);
    HIP_CHECK(hipGetLastError());
}
```

```cpp
#include <hip/hip_runtime.h>
#include <rccl/rccl.h>
#include <cstdio>
#include <cstdint>

#define HIP_CHECK(expr) do { hipError_t _e = (expr); if (_e != hipSuccess) { \
    fprintf(stderr, "HIP error %s at %s:%d\n", hipGetErrorString(_e), __FILE__, __LINE__); \
    return; } } while (0)

namespace {

#define LAS __attribute__((address_space(3)))
typedef short bf16x8 __attribute__((ext_vector_type(8)));
typedef float f32x4 __attribute__((ext_vector_type(4)));
typedef unsigned u32x4 __attribute__((ext_vector_type(4)));
typedef unsigned long long u64;

constexpr int NR = 8;
#define PM_ 7
constexpr int PM = PM_;
constexpr int KS = 1024;
constexpr int NPOS = 128;
#define NCW 14
#define NCP 12
#define EPUSH 8
#define NHALF 50
#define STG_SLOT 4
#define PGATE (N_PUSH - 48)
#define GSL 32
#define PSL_S 24
#define PSL_L 120
#define CSL 24
#define GPX 16
constexpr int GRID = 256;
constexpr int N_IN = 32 * 32;
constexpr int N_FIN = NPOS * 16;
constexpr int N_PUSH = EPUSH * 32;
constexpr int N_PIN = N_IN - N_PUSH;
constexpr int N_STG = 2 * 2 * (16 - EPUSH / 2) * 64;
static_assert(EPUSH % 2 == 0 && EPUSH >= 2 && EPUSH <= 32, "EPUSH");
constexpr int NSPARE = 32 - GPX - (NCW > NCP ? NCW : NCP);
constexpr int NSTG_WG = (NSPARE + GPX - STG_SLOT) * 8;
static_assert(NSTG_WG > 0, "stagers");
constexpr int XQN = 48;
constexpr int J_NONE = -N_IN - 1;
constexpr int SEGSH = 7;
constexpr int NSEG = N_FIN >> SEGSH;
static_assert(NSEG <= 64, "config");
static_assert(NCW <= 32 - GPX && NCP <= 32 - GPX && GPX >= 16 && GPX <= 32, "too many workgroups");

constexpr unsigned OFF_SEQ = 0, OFF_DONE = 128, OFF_TMO = 256;
constexpr unsigned OFF_INF = 4096;
constexpr unsigned OFF_OWNW = OFF_INF + 6 * 32 * 32 * 4;
constexpr unsigned OFF_RT = OFF_OWNW + 16 * 64 * 4;
constexpr unsigned OFF_RS = OFF_RT + 8 * 512;
constexpr unsigned OFF_PFLAG = OFF_RS + 512;
constexpr unsigned OFF_HAVE = OFF_PFLAG + 128 * 16 * 4;
constexpr unsigned OFF_SR = OFF_HAVE + 6 * 64 * 4;
constexpr unsigned OFF_HCNT = OFF_SR + 128;
constexpr unsigned OFF_QCTR = OFF_HCNT + 512;
constexpr unsigned OFF_DBG = 128 * 1024;
constexpr unsigned OFF_XS = 1u << 20;
constexpr unsigned OFF_WTS = OFF_XS + (24u << 20);
constexpr unsigned OFF_PS = OFF_WTS + (32u << 20);
constexpr unsigned OFF_PR = OFF_PS + (16u << 20);
constexpr unsigned OFF_XO = OFF_PR + (16u << 20);
constexpr unsigned OFF_WO = OFF_XO + (16u << 20);
constexpr unsigned WS_NEED = OFF_WO + (16u << 20);
static_assert(OFF_QCTR + 27 * 128 <= OFF_DBG && WS_NEED <= (128u << 20), "layout");
constexpr unsigned long long DEADLINE = 150000000ull;

constexpr int BM = 256, BK = 64, HALF = 128, HTB = HALF * BK * 2, STAGE_BYTES = 8 * HTB;
constexpr int LDS_BYTES = STAGE_BYTES;

struct Params {
    const char* x;
    const char* w;
    char* out[NR];
    char* ws[NR];
    int rank;
    unsigned forder[2][NPOS / 4];
    unsigned pos[256 / 4];
    unsigned xq[8][XQN / 4];
    unsigned xm[8][XQN / 4];
    unsigned xlink[2];
};
typedef const __attribute__((address_space(4))) Params& KP;

__host__ __device__ __forceinline__ int role_of(int r) { return (r >> 2) & 1; }
__host__ __device__ __forceinline__ int quad_of(int r) { return role_of(r) ? (r ^ PM) : r; }
__host__ __device__ __forceinline__ int uidx(int m) { return m < PM ? m - 1 : m - 2; }

__device__ __forceinline__ int lds_byte(int r, int c) { const int st = (r >> 4) * 2 + (c >> 5), rr = r & 15, cc = c & 31, ob = rr * 64 + cc * 2; return st * 1024 + (ob ^ (((ob >> 9) & 1) << 5)); }
__device__ __forceinline__ void stage_rc(int b, int& R, int& C) { const int st = b / 1024, sb = b % 1024, swz = sb ^ (((sb >> 9) & 1) << 5); R = (st >> 1) * 16 + swz / 64; C = (st & 1) * 32 + (swz % 64) / 2; }
__device__ __forceinline__ int perm32(int rho) { const int n = rho >> 4, i = rho & 15; return 8 * (i >> 2) + 4 * n + (i & 3); }

__device__ __forceinline__ unsigned ld_sys(const char* p) { return __hip_atomic_load((const unsigned*)p, __ATOMIC_RELAXED, __HIP_MEMORY_SCOPE_SYSTEM); }
__device__ __forceinline__ unsigned ld_agent(const char* p) { return __hip_atomic_load((const unsigned*)p, __ATOMIC_RELAXED, __HIP_MEMORY_SCOPE_AGENT); }
__device__ __forceinline__ void st_sys(char* p, unsigned v) { __hip_atomic_store((unsigned*)p, v, __ATOMIC_RELAXED, __HIP_MEMORY_SCOPE_SYSTEM); }
__device__ __forceinline__ void st_agent(char* p, unsigned v) { __hip_atomic_store((unsigned*)p, v, __ATOMIC_RELAXED, __HIP_MEMORY_SCOPE_AGENT); }
__device__ __forceinline__ unsigned cvt_pk_bf16(float lo, float hi) { unsigned r; asm("v_cvt_pk_bf16_f32 %0, %1, %2" : "=v"(r) : "v"(lo), "v"(hi)); return r; }
__device__ __forceinline__ float bf_lo(unsigned u) { return __uint_as_float(u << 16); }
__device__ __forceinline__ float bf_hi(unsigned u) { return __uint_as_float(u & 0xffff0000u); }
__device__ __forceinline__ float silu(float y) {
    const float t = fminf(fmaxf(y, -60.0f), 60.0f);
    const float e = __builtin_amdgcn_exp2f(t * -1.44269504088896f);
    return y * __builtin_amdgcn_rcpf(1.0f + e);
}
__device__ __forceinline__ __amdgpu_buffer_rsrc_t mk_rsrc(const void* p, unsigned bytes) {
    return __builtin_amdgcn_make_buffer_rsrc((void*)p, 0, bytes, 0x00020000);
}
template <class T> __device__ __forceinline__ int tab8(const T& t, int i) { return (t[i >> 2] >> (8 * (i & 3))) & 255; }
#define WAIT_V(n) asm volatile("s_waitcnt vmcnt(" #n ")" ::: "memory")
#define WAIT_L(n) asm volatile("s_waitcnt lgkmcnt(" #n ")" ::: "memory")

#define STAMP(idx) do { } while (0)

__device__ __forceinline__ bool give_up(char* ws, unsigned call, unsigned long long t0) {
    const unsigned tm = __builtin_amdgcn_readfirstlane(ld_agent(ws + OFF_TMO));
    if (tm == call || __builtin_amdgcn_s_memrealtime() - t0 > DEADLINE) {
        if ((threadIdx.x & 63) == 0) st_agent(ws + OFF_TMO, call);
        return true;
    }
    return false;
}

__device__ __forceinline__ u64 uni64(const void* p) {
    const u64 a = (u64)p;
    return ((u64)(unsigned)__builtin_amdgcn_readfirstlane((unsigned)(a >> 32)) << 32) | (unsigned)__builtin_amdgcn_readfirstlane((unsigned)a);
}
__device__ __forceinline__ u32x4 mk_srd(u64 a) {
    u32x4 r;
    r.x = (unsigned)a;
    r.y = (unsigned)(a >> 32) & 0xffffu;
    r.z = 128u << 20;
    r.w = 0x00020000u;
    return r;
}
#define CE_CLOB "memory", "scc", "vcc", "v156", "v157", "v158", "v159", "v160", "v161", "v162", "v163", "v164", "v165", "v166", "v167", "v168", "v169", "v170", "v171", "v172", "v173", "v174", "v175", "v176", "v177", "v178", "v179", "v180", "v181", "v182", "v183", "v184", "v185", "v186", "v187", "v188", "v189", "v190", "v191", "v192", "v193", "v194", "v195", "v196", "v197", "v198", "v199", "v200", "v201", "v202", "v203", "v204", "v205", "v206", "v207", "v208", "v209", "v210", "v211", "v212", "v213", "v214", "v215", "v216", "v217", "v218", "v219", "v220", "v221", "v222", "v223", "v224", "v225", "v226", "v227", "v228", "v229", "v230", "v231", "v232", "v233", "v234", "v235", "v236", "v237", "v238", "v239", "v240", "v241", "v242", "v243", "v244", "v245", "v246", "v247", "v248", "v249", "v250", "v251", "v252", "v253", "v254", "v255"
#define LDF "sc0 sc1"
template <int R> __device__ __forceinline__ void ld8(unsigned voff, const u32x4& srd, unsigned soff, unsigned step) {
    unsigned t;
    asm volatile(
        "s_nop 4\n\t"
        "buffer_load_dwordx4 v[%c5+0:%c5+3], %1, %2, %3 offen " LDF "\n\t"
        "s_add_u32 %0, %3, %4\n\t"
        "buffer_load_dwordx4 v[%c5+4:%c5+7], %1, %2, %0 offen " LDF "\n\t"
        "s_add_u32 %0, %0, %4\n\t"
        "buffer_load_dwordx4 v[%c5+8:%c5+11], %1, %2, %0 offen " LDF "\n\t"
        "s_add_u32 %0, %0, %4\n\t"
        "buffer_load_dwordx4 v[%c5+12:%c5+15], %1, %2, %0 offen " LDF "\n\t"
        "s_add_u32 %0, %0, %4\n\t"
        "buffer_load_dwordx4 v[%c5+16:%c5+19], %1, %2, %0 offen " LDF "\n\t"
        "s_add_u32 %0, %0, %4\n\t"
        "buffer_load_dwordx4 v[%c5+20:%c5+23], %1, %2, %0 offen " LDF "\n\t"
        "s_add_u32 %0, %0, %4\n\t"
        "buffer_load_dwordx4 v[%c5+24:%c5+27], %1, %2, %0 offen " LDF "\n\t"
        "s_add_u32 %0, %0, %4\n\t"
        "buffer_load_dwordx4 v[%c5+28:%c5+31], %1, %2, %0 offen " LDF "\n\t"
        "s_nop 0"
        : "=&s"(t)
        : "v"(voff), "s"(srd), "s"(soff), "s"(step), "i"(R)
        : CE_CLOB);
}
#define SWIN 4
#define SWN(j) ((j) < SWIN ? SWIN + 8 : SWIN - 1)
template <int R> __device__ __forceinline__ void st8(unsigned voff, const u32x4& srd, unsigned soff, unsigned step) {
    unsigned t;
    asm volatile(
        "s_nop 4\n\t"
        "s_waitcnt vmcnt(%c6)\n\t"
        "buffer_store_dwordx4 v[%c5+0:%c5+3], %1, %2, %3 offen sc0 sc1\n\t"
        "s_add_u32 %0, %3, %4\n\t"
        "s_waitcnt vmcnt(%c7)\n\t"
        "buffer_store_dwordx4 v[%c5+4:%c5+7], %1, %2, %0 offen sc0 sc1\n\t"
        "s_add_u32 %0, %0, %4\n\t"
        "s_waitcnt vmcnt(%c8)\n\t"
        "buffer_store_dwordx4 v[%c5+8:%c5+11], %1, %2, %0 offen sc0 sc1\n\t"
        "s_add_u32 %0, %0, %4\n\t"
        "s_waitcnt vmcnt(%c9)\n\t"
        "buffer_store_dwordx4 v[%c5+12:%c5+15], %1, %2, %0 offen sc0 sc1\n\t"
        "s_add_u32 %0, %0, %4\n\t"
        "s_waitcnt vmcnt(%c10)\n\t"
        "buffer_store_dwordx4 v[%c5+16:%c5+19], %1, %2, %0 offen sc0 sc1\n\t"
        "s_add_u32 %0, %0, %4\n\t"
        "s_waitcnt vmcnt(%c11)\n\t"
        "buffer_store_dwordx4 v[%c5+20:%c5+23], %1, %2, %0 offen sc0 sc1\n\t"
        "s_add_u32 %0, %0, %4\n\t"
        "s_waitcnt vmcnt(%c12)\n\t"
        "buffer_store_dwordx4 v[%c5+24:%c5+27], %1, %2, %0 offen sc0 sc1\n\t"
        "s_add_u32 %0, %0, %4\n\t"
        "s_waitcnt vmcnt(%c13)\n\t"
        "buffer_store_dwordx4 v[%c5+28:%c5+31], %1, %2, %0 offen sc0 sc1\n\t"
        "s_nop 1"
        : "=&s"(t)
        : "v"(voff), "s"(srd), "s"(soff), "s"(step), "i"(R), "i"(SWN(0)), "i"(SWN(1)), "i"(SWN(2)), "i"(SWN(3)), "i"(SWN(4)), "i"(SWN(5)), "i"(SWN(6)), "i"(SWN(7))
        : CE_CLOB);
}
template <int R> __device__ __forceinline__ void tr8() {
    asm volatile(
        "v_perm_b32 v224, v[%c2+4], v[%c2+0], %0\n\t"
        "v_perm_b32 v225, v[%c2+12], v[%c2+8], %0\n\t"
        "v_perm_b32 v226, v[%c2+20], v[%c2+16], %0\n\t"
        "v_perm_b32 v227, v[%c2+28], v[%c2+24], %0\n\t"
        "v_perm_b32 v228, v[%c2+4], v[%c2+0], %1\n\t"
        "v_perm_b32 v229, v[%c2+12], v[%c2+8], %1\n\t"
        "v_perm_b32 v230, v[%c2+20], v[%c2+16], %1\n\t"
        "v_perm_b32 v231, v[%c2+28], v[%c2+24], %1\n\t"
        "v_perm_b32 v232, v[%c2+5], v[%c2+1], %0\n\t"
        "v_perm_b32 v233, v[%c2+13], v[%c2+9], %0\n\t"
        "v_perm_b32 v234, v[%c2+21], v[%c2+17], %0\n\t"
        "v_perm_b32 v235, v[%c2+29], v[%c2+25], %0\n\t"
        "v_perm_b32 v236, v[%c2+5], v[%c2+1], %1\n\t"
        "v_perm_b32 v237, v[%c2+13], v[%c2+9], %1\n\t"
        "v_perm_b32 v238, v[%c2+21], v[%c2+17], %1\n\t"
        "v_perm_b32 v239, v[%c2+29], v[%c2+25], %1\n\t"
        "v_perm_b32 v240, v[%c2+6], v[%c2+2], %0\n\t"
        "v_perm_b32 v241, v[%c2+14], v[%c2+10], %0\n\t"
        "v_perm_b32 v242, v[%c2+22], v[%c2+18], %0\n\t"
        "v_perm_b32 v243, v[%c2+30], v[%c2+26], %0\n\t"
        "v_perm_b32 v244, v[%c2+6], v[%c2+2], %1\n\t"
        "v_perm_b32 v245, v[%c2+14], v[%c2+10], %1\n\t"
        "v_perm_b32 v246, v[%c2+22], v[%c2+18], %1\n\t"
        "v_perm_b32 v247, v[%c2+30], v[%c2+26], %1\n\t"
        "v_perm_b32 v248, v[%c2+7], v[%c2+3], %0\n\t"
        "v_perm_b32 v249, v[%c2+15], v[%c2+11], %0\n\t"
        "v_perm_b32 v250, v[%c2+23], v[%c2+19], %0\n\t"
        "v_perm_b32 v251, v[%c2+31], v[%c2+27], %0\n\t"
        "v_perm_b32 v252, v[%c2+7], v[%c2+3], %1\n\t"
        "v_perm_b32 v253, v[%c2+15], v[%c2+11], %1\n\t"
        "v_perm_b32 v254, v[%c2+23], v[%c2+19], %1\n\t"
        "v_perm_b32 v255, v[%c2+31], v[%c2+27], %1\n\t"
        "s_nop 1"
        :: "s"(0x05040100u), "s"(0x07060302u), "i"(R)
        : CE_CLOB);
}
template <int R> __device__ __forceinline__ void ld4(unsigned voff, const u32x4& srd, unsigned soff, unsigned step) {
    asm volatile(
        "s_nop 4\n\t"
        "buffer_load_dwordx4 v[%c4+0:%c4+3], %1, %2, %0 offen sc0 sc1\n\t"
        "s_add_u32 %0, %0, %3\n\t"
        "buffer_load_dwordx4 v[%c4+4:%c4+7], %1, %2, %0 offen sc0 sc1\n\t"
        "s_add_u32 %0, %0, %3\n\t"
        "buffer_load_dwordx4 v[%c4+8:%c4+11], %1, %2, %0 offen sc0 sc1\n\t"
        "s_add_u32 %0, %0, %3\n\t"
        "buffer_load_dwordx4 v[%c4+12:%c4+15], %1, %2, %0 offen sc0 sc1\n\t"
        "s_nop 0"
        : "+s"(soff)
        : "v"(voff), "s"(srd), "s"(step), "i"(R)
        : CE_CLOB);
}
template <int X, int Y> __device__ __forceinline__ void xfer(unsigned dst, unsigned src4, unsigned nsrc, unsigned voff, const u32x4& dsrd, unsigned step,
                                                         const u32x4& ssrd, unsigned nvoff, const u32x4& nsrd, unsigned nstep) {
    asm volatile(
        "s_nop 4\n\t"
        "s_waitcnt vmcnt(6)\n\t"
        "buffer_store_dwordx4 v[%c12+0:%c12+3], %3, %4, %0 offen sc0 sc1\n\t"
        "s_add_u32 %0, %0, %5\n\t"
        "buffer_load_dwordx4 v[%c12+16:%c12+19], %3, %6, %1 offen sc0 sc1\n\t"
        "s_add_u32 %1, %1, %5\n\t"
        "s_waitcnt vmcnt(6)\n\t"
        "buffer_store_dwordx4 v[%c12+4:%c12+7], %3, %4, %0 offen sc0 sc1\n\t"
        "s_add_u32 %0, %0, %5\n\t"
        "buffer_load_dwordx4 v[%c12+20:%c12+23], %3, %6, %1 offen sc0 sc1\n\t"
        "s_add_u32 %1, %1, %5\n\t"
        "s_waitcnt vmcnt(6)\n\t"
        "buffer_store_dwordx4 v[%c12+8:%c12+11], %3, %4, %0 offen sc0 sc1\n\t"
        "s_add_u32 %0, %0, %5\n\t"
        "buffer_load_dwordx4 v[%c12+24:%c12+27], %3, %6, %1 offen sc0 sc1\n\t"
        "s_add_u32 %1, %1, %5\n\t"
        "s_waitcnt vmcnt(6)\n\t"
        "buffer_store_dwordx4 v[%c12+12:%c12+15], %3, %4, %0 offen sc0 sc1\n\t"
        "s_add_u32 %0, %0, %5\n\t"
        "buffer_load_dwordx4 v[%c12+28:%c12+31], %3, %6, %1 offen sc0 sc1\n\t"
        "s_waitcnt vmcnt(6)\n\t"
        "buffer_store_dwordx4 v[%c12+16:%c12+19], %3, %4, %0 offen sc0 sc1\n\t"
        "s_add_u32 %0, %0, %5\n\t"
        "buffer_load_dwordx4 v[%c13+0:%c13+3], %7, %8, %2 offen sc0 sc1\n\t"
        "s_add_u32 %2, %2, %9\n\t"
        "s_waitcnt vmcnt(6)\n\t"
        "buffer_store_dwordx4 v[%c12+20:%c12+23], %3, %4, %0 offen sc0 sc1\n\t"
        "s_add_u32 %0, %0, %5\n\t"
        "buffer_load_dwordx4 v[%c13+4:%c13+7], %7, %8, %2 offen sc0 sc1\n\t"
        "s_add_u32 %2, %2, %9\n\t"
        "s_waitcnt vmcnt(6)\n\t"
        "buffer_store_dwordx4 v[%c12+24:%c12+27], %3, %4, %0 offen sc0 sc1\n\t"
        "s_add_u32 %0, %0, %5\n\t"
        "buffer_load_dwordx4 v[%c13+8:%c13+11], %7, %8, %2 offen sc0 sc1\n\t"
        "s_add_u32 %2, %2, %9\n\t"
        "s_waitcnt vmcnt(6)\n\t"
        "buffer_store_dwordx4 v[%c12+28:%c12+31], %3, %4, %0 offen sc0 sc1\n\t"
        "buffer_load_dwordx4 v[%c13+12:%c13+15], %7, %8, %2 offen sc0 sc1\n\t"
        "s_nop 1"
        : "+s"(dst), "+s"(src4), "+s"(nsrc)
        : "v"(voff), "s"(dsrd), "s"(step), "s"(ssrd), "v"(nvoff), "s"(nsrd), "s"(nstep), "i"(0), "i"(0), "i"(X), "i"(Y)
        : CE_CLOB);
}
__device__ __forceinline__ void ticket_issue(u64 ctr) {
    u64 sv;
    asm volatile(
        "s_mov_b64 %0, exec\n\t"
        "s_mov_b64 exec, 1\n\t"
        "v_mov_b32 v158, 0\n\t"
        "v_mov_b32 v159, 1\n\t"
        "s_nop 4\n\t"
        "global_atomic_add v159, v158, v159, %1 sc0\n\t"
        "s_mov_b64 exec, %0"
        : "=&s"(sv) : "s"(ctr) : CE_CLOB);
}
__device__ __forceinline__ int ticket_read() {
    int q;
    asm volatile("s_nop 1\n\tv_readfirstlane_b32 %0, v159\n\ts_nop 4" : "=s"(q) :: CE_CLOB);
    return q;
}
__device__ __forceinline__ void tflags_issue(u64 base, unsigned voff) {
    asm volatile(
        "s_nop 4\n\t"
        "global_load_dword v157, %0, %1 sc0 sc1\n\t"
        "global_load_dword v158, %0, %1 offset:256 sc0 sc1"
        :: "v"(voff), "s"(base) : CE_CLOB);
}
__device__ __forceinline__ int tflags_read(unsigned call) {
    u64 m0, m1;
    asm volatile("s_nop 1\n\tv_cmp_le_u32 %0, %2, v157\n\tv_cmp_le_u32 %1, %2, v158\n\ts_nop 4" : "=&s"(m0), "=&s"(m1) : "s"(call) : CE_CLOB);
    return (~m0 != 0ull) ? __builtin_ctzll(~m0) : 64 + ((~m1 != 0ull) ? __builtin_ctzll(~m1) : 64);
}
__device__ __forceinline__ void fcnt_issue(u64 base, unsigned voff) {
    asm volatile("s_nop 4\n\tglobal_load_dword v156, %0, %1 sc0 sc1" :: "v"(voff), "s"(base) : CE_CLOB);
}
__device__ __forceinline__ int fcnt_read(unsigned need) {
    u64 m0;
    asm volatile("s_nop 1\n\tv_cmp_le_u32 %0, %1, v156\n\ts_nop 4" : "=&s"(m0) : "s"(need) : CE_CLOB);
    return (~m0 != 0ull) ? __builtin_ctzll(~m0) : 64;
}
#define CE_WAIT(n) asm volatile("s_waitcnt vmcnt(" #n ")" ::: CE_CLOB)
constexpr int RA = 160, RB = 192, RT = 224;

__device__ __forceinline__ unsigned selv(bool c0, bool c1, unsigned a, unsigned b, unsigned c) {
    asm("" : "+v"(a)); asm("" : "+v"(b)); asm("" : "+v"(c));
    return c0 ? a : (c1 ? b : c);
}
__device__ __forceinline__ u64 sels(bool c, u64 a, u64 b) {
    asm("" : "+s"(a)); asm("" : "+s"(b));
    return uni64((const void*)(c ? a : b));
}

struct Piece { int type; unsigned src, dst, flag; };

struct Comm {
    KP p;
    int rank, peer, lane;
    unsigned call;
    bool partner;
    u64 rx, rw, ro_me, rws_me, rws_peer, ro_peer;
    unsigned lin_off, w_src_off, w_dst_off, o_off;
    char* wsp;
    char* wsh;
    char* wsm;

    __device__ __forceinline__ Comm(KP p_, int peer_, int lane_, unsigned call_) : p(p_), rank(p_.rank), peer(peer_), lane(lane_), call(call_) {
        partner = (rank ^ peer) == PM;
        rx = uni64(p.x);
        rw = uni64(p.w);
        ro_me = uni64(p.out[rank]);
        wsm = p.ws[rank];
        rws_me = uni64(wsm);
        wsp = p.ws[peer];
        wsh = p.ws[peer ^ PM];
        rws_peer = uni64(wsp);
        ro_peer = uni64(p.out[peer]);
        lin_off = lane * 16;
        w_src_off = (lane >> 3) * 8 * 16384 + (lane & 7) * 16;
        w_dst_off = (lane & 7) * 8 * 2048 + (lane >> 3) * 16;
        o_off = (lane >> 5) * 16384 + (lane & 31) * 16;
    }
    __device__ __forceinline__ unsigned out_off(int a, int o, int po) const {
        const int t = tab8(p.forder[role_of(a)], o), pm = t >> 4, pn = t & 15, qd = quad_of(a);
        return (unsigned)((qd >> 1) * 4096 + pm * 256 + po * 16) * 16384u + (unsigned)((qd & 1) * 4096 + pn * 256) * 2u;
    }
    __device__ __forceinline__ Piece desc_in(int q) const {
        Piece d;
        q = __builtin_amdgcn_readfirstlane(q);
        const int u = uidx(rank ^ peer);
        const int e = q >> 5, pc = q & 31, panel = e >> 1, hd = role_of(peer), qd = quad_of(peer);
        d.flag = OFF_INF + (u * 1024 + q) * 4;
        if (!(e & 1)) {
            d.type = 0;
            d.src = (unsigned)((qd >> 1) * 4096 + panel * 256 + pc * 8) * 2048u + hd * 1024u;
            d.dst = OFF_XS + (u >> 1) * (8u << 20) + (unsigned)(panel * 256 + pc * 8) * 2048u + (u & 1) * 1024u;
        } else {
            const int kb = pc >> 2, nb = pc & 3;
            d.type = 1;
            d.src = (unsigned)((hd * 512 + kb * 64) * 8192 + (qd & 1) * 4096 + panel * 256 + nb * 64) * 2u;
            d.dst = OFF_WTS + (u >> 1) * (8u << 20) + (unsigned)((panel * 256 + nb * 64) * 1024 + (u & 1) * 512 + kb * 64) * 2u;
        }
        d.type = __builtin_amdgcn_readfirstlane(d.type);
        d.src = __builtin_amdgcn_readfirstlane(d.src);
        d.dst = __builtin_amdgcn_readfirstlane(d.dst);
        d.flag = __builtin_amdgcn_readfirstlane(d.flag);
        return d;
    }
    template <int R> __device__ __forceinline__ void load(const Piece& d) const {
        ld8<R>(selv(d.type == 1, false, w_src_off, lin_off, lin_off), mk_srd(sels(d.type == 0, rx, rw)), d.src, d.type == 0 ? 2048u : 16384u);
    }
    template <int R> __device__ __forceinline__ void store(const Piece& d) const {
        if (d.type == 1) {
            tr8<R>();
            st8<RT>(w_dst_off, mk_srd(rws_peer), d.dst, 2048u);
        } else {
            st8<R>(lin_off, mk_srd(rws_peer), d.dst, 2048u);
        }
    }
    __device__ __forceinline__ void flag_in(int q) const {
        if (q >= 0 && lane == 0) st_sys(wsp + OFF_INF + (uidx(rank ^ peer) * 1024 + q) * 4, call);
    }
    __device__ __forceinline__ Piece desc_pull(int j) const {
        Piece d;
        j = __builtin_amdgcn_readfirstlane(j);
        if (j < -N_PIN) {
            d.type = 4; d.src = d.dst = OFF_PS; d.flag = 0;
        } else if (j < 0) {
            const int q = j + N_IN, e = q >> 5, pc = q & 31, panel = e >> 1, u = uidx(rank ^ peer), qd = quad_of(rank);
            const unsigned row = (unsigned)(panel * 256 + pc * 8) * 2048u;
            d.type = 5;
            d.src = ((e & 1) ? OFF_WO + (qd & 1) * (8u << 20) : OFF_XO + (qd >> 1) * (8u << 20)) + row + role_of(rank) * 1024u;
            d.dst = ((e & 1) ? OFF_WTS : OFF_XS) + (u >> 1) * (8u << 20) + row + (u & 1) * 1024u;
            d.flag = OFF_INF + (u * 1024 + q) * 4;
        } else if (j >= N_FIN) {
            const int qq = j - N_FIN;
            d.type = 2;
            d.src = d.dst = out_off(rank ^ PM, qq >> 4, qq & 15); d.flag = 0;
        } else if (partner) {
            d.type = 3;
            d.src = OFF_PS + j * 8192u; d.dst = OFF_PR + j * 8192u; d.flag = OFF_PFLAG + j * 4;
        } else {
            d.type = 2;
            d.src = d.dst = out_off(peer, j >> 4, j & 15);
            d.flag = (OFF_HAVE + (uidx(rank ^ peer ^ PM) * 64 + (j >> SEGSH)) * 4) | 1u;
        }
        d.type = __builtin_amdgcn_readfirstlane(d.type);
        d.src = __builtin_amdgcn_readfirstlane(d.src);
        d.dst = __builtin_amdgcn_readfirstlane(d.dst);
        d.flag = __builtin_amdgcn_readfirstlane(d.flag);
        return d;
    }
    __device__ __forceinline__ unsigned pvoff(const Piece& d) const { return selv(d.type == 2, false, o_off, lin_off, lin_off); }
    __device__ __forceinline__ unsigned pstep(const Piece& d) const { return d.type == 2 ? 32768u : (d.type == 5 ? 2048u : 1024u); }
    __device__ __forceinline__ u32x4 psrc(const Piece& d) const { return mk_srd(sels(d.type == 2, ro_peer, sels(d.type != 4, rws_peer, rws_me))); }
    __device__ __forceinline__ u32x4 pdst(const Piece& d) const { return mk_srd(sels(d.type == 2, ro_me, rws_me)); }
    template <int R> __device__ __forceinline__ void pull_first(const Piece& d) const { ld4<R>(pvoff(d), psrc(d), d.src, pstep(d)); }
    template <int X, int Y> __device__ __forceinline__ void pull_step(const Piece& c, const Piece& n) const {
        xfer<X, Y>(c.dst, c.src + 4 * pstep(c), n.src, pvoff(c), pdst(c), pstep(c), psrc(c), pvoff(n), psrc(n), pstep(n));
    }
    unsigned seg_off = 0, seg_cnt = 0;
    __device__ __forceinline__ void flush() {
        if (seg_cnt != 0 && lane == 0) __hip_atomic_fetch_add((unsigned*)(wsh + seg_off), seg_cnt, __ATOMIC_RELAXED, __HIP_MEMORY_SCOPE_SYSTEM);
        seg_cnt = 0;
    }
    __device__ __forceinline__ void flag_pull(unsigned off) {
        if (off & 1u) {
            if ((off & ~1u) != seg_off) { flush(); seg_off = off & ~1u; }
            ++seg_cnt;
        } else if (off != 0) { if (lane == 0) st_sys(wsm + off, call); }
        else flush();
    }
};

__device__ __forceinline__ void comm_role(KP p, int peer, int slot, int cidx) {
    const int rank = p.rank;
    char* ws = p.ws[rank];
    const int lane = threadIdx.x & 63, wid = __builtin_amdgcn_readfirstlane(threadIdx.x >> 6);
    if ((unsigned)peer >= (unsigned)NR || peer == rank) return;
    if (wid >= 2 || (wid == 0 && ((rank ^ peer) == PM || slot >= NCW)) || (wid == 1 && slot >= NCP)) return;
    const u64 ctr_own = uni64(ws + OFF_QCTR + (wid * 17 + peer) * 128);
    ticket_issue(ctr_own);
    const unsigned call = __builtin_amdgcn_readfirstlane(ld_agent(ws + OFF_SEQ)) + 1u;
    Comm c(p, peer, lane, call);
    STAMP(4096 + (cidx * 8 + wid) * 4 + 0);
    const bool partner = c.partner;
    CE_WAIT(0);
    int qn = ticket_read();

    if (wid == 0) {
        if (qn >= N_PUSH) return;
        int qc = qn, qp = -1;
        ticket_issue(ctr_own);
        c.load<RA>(c.desc_in(qc));
        CE_WAIT(8);
        qn = ticket_read();
        bool first = true;
#define CE_STEP(X, Y) { \
            if (qn >= N_PUSH) { CE_WAIT(0); c.flag_in(qp); c.store<X>(c.desc_in(qc)); CE_WAIT(0); c.flag_in(qc); break; } \
            ticket_issue(ctr_own); \
            c.load<Y>(c.desc_in(qn)); \
            if (first) CE_WAIT(9); else CE_WAIT(17); \
            first = false; \
            c.store<X>(c.desc_in(qc)); \
            CE_WAIT(16); \
            c.flag_in(qp); \
            qp = qc; qc = qn; \
            qn = ticket_read(); }
        for (;;) {
            CE_STEP(RA, RB)
            CE_STEP(RB, RA)
        }
#undef CE_STEP
        STAMP(4096 + (cidx * 8 + wid) * 4 + 1);
        return;
    }

    const u64 ctr_q = uni64(ws + OFF_QCTR + 16 * 128);
    const u64 rtf = uni64(ws + OFF_RT + peer * 512), rsf = uni64(ws + OFF_RS);
    const u64 hav = uni64(ws + OFF_HAVE + (partner ? 0 : uidx(rank ^ peer)) * 256);
    const unsigned tvoff = lane * 4;
    const unsigned long long t0 = __builtin_amdgcn_s_memrealtime();
    int qoff = partner ? 0 : -N_PIN;
    int qready = 0, qlim = N_FIN;
    bool inq = false;
    u64 ctr = ctr_own;
    if (!partner) {
        const char* gate = lane < 32 ? ws + OFF_INF + (uidx(rank ^ peer) * 1024 + PGATE + lane) * 4 : ws + OFF_SR + peer * 4;
        for (;;) {
            if (__all(ld_sys(gate) >= call)) break;
            if (give_up(ws, call, t0)) break;
            __builtin_amdgcn_s_sleep(GSL);
        }
    }
    STAMP(4096 + (cidx * 8 + wid) * 4 + 1);
    qn += qoff;
#define SNAP_ISSUE() do { tflags_issue(sels(partner && !inq, rsf, rtf), tvoff); if (inq) fcnt_issue(hav, tvoff); } while (0)
#define SNAP_APPLY() do { const int n_ = tflags_read(call); \
        int r_; \
        if (!inq) r_ = 16 * n_; \
        else r_ = N_FIN + (partner ? 16 * n_ : (min(fcnt_read(call << SEGSH), NSEG) << SEGSH)); \
        qready = __builtin_amdgcn_readfirstlane(max(qready, r_)); } while (0)
#define TO_QUEUE() do { STAMP(4096 + (cidx * 8 + wid) * 4 + 3); inq = true; qoff = N_FIN; qlim = 2 * N_FIN; ctr = ctr_q; qready = N_FIN; } while (0)
    for (;;) {
        if (qn >= qlim) {
            if (inq) break;
            TO_QUEUE();
            ticket_issue(ctr_q);
            SNAP_ISSUE();
            CE_WAIT(0);
            qn = qoff + ticket_read();
            SNAP_APPLY();
            continue;
        }
        while (qn >= qready) {
            SNAP_ISSUE();
            CE_WAIT(0);
            const int before = qready;
            SNAP_APPLY();
            if (qready != before) continue;
            if (give_up(ws, call, t0)) { qready = qlim; break; }
            __builtin_amdgcn_s_sleep(CSL);
        }
        qready = __builtin_amdgcn_readfirstlane(qready);
        ctr = uni64((const void*)ctr);
        int qc = qn, qp = J_NONE;
        ticket_issue(ctr);
        c.pull_first<RA>(c.desc_pull(qc));
        CE_WAIT(0);
        qn = qoff + ticket_read();
#define CE_STEP(X, Y) { \
            if (!inq && qn >= qlim) {         \
                TO_QUEUE(); ticket_issue(ctr_q); SNAP_ISSUE(); CE_WAIT(0); qn = qoff + ticket_read(); SNAP_APPLY(); \
                ctr = uni64((const void*)ctr); } \
            if (!(qn < qlim && qn < qready)) { \
                c.pull_step<X, Y>(c.desc_pull(qc), c.desc_pull(J_NONE)); \
                CE_WAIT(0); c.flag_pull(c.desc_pull(qp).flag); c.flag_pull(c.desc_pull(qc).flag); c.flush(); \
                break; } \
            const bool snap = qready < qlim && qn + (inq ? 16 : 3) * NCP >= qready; \
            ticket_issue(ctr); \
            if (snap) SNAP_ISSUE(); \
            c.pull_step<X, Y>(c.desc_pull(qc), c.desc_pull(qn)); \
            c.flag_pull(c.desc_pull(qp).flag); \
            qp = qc; qc = qn; \
            qn = qoff + ticket_read(); \
            if (qc < 0 && qn >= 0) STAMP(4096 + (cidx * 8 + 2) * 4); \
            if (snap) SNAP_APPLY(); }
        for (;;) {
            CE_STEP(RA, RB)
            CE_STEP(RB, RA)
        }
#undef CE_STEP
    }
    STAMP(4096 + (cidx * 8 + wid) * 4 + 2);
}

__device__ __forceinline__ void stage_role(KP p, int sidx) {
    const int rank = p.rank;
    char* ws = p.ws[rank];
    const int lane = threadIdx.x & 63, wid = __builtin_amdgcn_readfirstlane(threadIdx.x >> 6);
    const u32x4 sx = mk_srd(uni64(p.x)), sw = mk_srd(uni64(p.w)), sd = mk_srd(uni64(ws));
    const unsigned w_src_off = (lane >> 3) * 8 * 16384 + (lane & 7) * 16, w_dst_off = (lane & 7) * 8 * 2048 + (lane >> 3) * 16;
    for (int t = sidx * 8 + wid; t < N_STG / 2; t += NSTG_WG * 8) {
        const int pc = t & 63, hf = (t >> 6) & 1, panel = EPUSH / 2 + (t >> 7), kb = pc >> 2, nb = pc & 3;
        const unsigned xo = (unsigned)(hf * 4096 + panel * 256 + pc * 4) * 2048u;
        const int n0 = hf * 4096 + panel * 256 + nb * 64;
        ld8<RA>(lane * 16, sx, xo, 1024u);
        ld8<RB>(w_src_off, sw, (unsigned)(kb * 64 * 8192 + n0) * 2u, 16384u);
        CE_WAIT(8);
        st8<RA>(lane * 16, sd, OFF_XO + xo, 1024u);
        CE_WAIT(8);
        tr8<RB>();
        st8<RT>(w_dst_off, sd, OFF_WO + (unsigned)(n0 * 1024 + kb * 64) * 2u, 2048u);
    }
    CE_WAIT(0);
    __syncthreads();
    if (wid == 0) {
        unsigned tot = 0;
        if (lane == 0) tot = __hip_atomic_fetch_add((unsigned*)(ws + OFF_QCTR + 26 * 128), 1u, __ATOMIC_RELAXED, __HIP_MEMORY_SCOPE_AGENT) + 1u;
        if (__builtin_amdgcn_readfirstlane(tot) == (unsigned)NSTG_WG) {
            const unsigned call = ld_agent(ws + OFF_SEQ) + 1u;
            if (lane < NR && lane != rank) st_sys(p.ws[lane] + OFF_SR + rank * 4, call);
            STAMP(4094);
        }
    }
}

__device__ __forceinline__ void gemm_role(KP p, int xcd, int slot, unsigned call, LAS unsigned char* lds) {
    const int rank = p.rank;
    char* ws = p.ws[rank];
    const int tid = threadIdx.x, wid = __builtin_amdgcn_readfirstlane(tid >> 6), lane = tid & 63, wr = wid >> 2, wc = wid & 3, fr = lane & 15, fq = lane >> 4;
    const int qd = quad_of(rank), role = role_of(rank);

    if (slot < 16) {
        const int id = (slot * 8 + xcd) * 8 + wid, panel = id >> 6, pc = id & 63, kb = pc >> 2, nb = pc & 3;
        const unsigned w_src_off = (lane >> 3) * 8 * 16384 + (lane & 7) * 16, w_dst_off = (lane & 7) * 8 * 2048 + (lane >> 3) * 16;
        ld8<RA>(w_src_off, mk_srd(uni64(p.w)), (unsigned)(kb * 64 * 8192 + (qd & 1) * 4096 + panel * 256 + nb * 64) * 2u, 16384u);
        CE_WAIT(0);
        tr8<RA>();
        st8<RT>(w_dst_off, mk_srd(uni64(ws)), OFF_WTS + 3 * (8u << 20) + (unsigned)((panel * 256 + nb * 64) * 1024 + kb * 64) * 2u, 2048u);
        CE_WAIT(0);
        if (lane == 0) st_sys(ws + OFF_OWNW + id * 4, call);
    }

    if (slot >= STG_SLOT) stage_role(p, (NSPARE + slot - STG_SLOT) * 8 + xcd);
    unsigned voffA[2], voffB[2];
#pragma unroll
    for (int i = 0; i < 2; ++i) { int R, C; stage_rc(tid * 16 + i * 8192, R, C); const int Rb = (R & ~31) + perm32(R & 31);
        voffA[i] = (unsigned)(R * KS + C) * 2u; voffB[i] = (unsigned)(Rb * KS + C) * 2u; }
    constexpr size_t kstep = (size_t)(BK * 2);
    constexpr size_t hstep = (size_t)HALF * KS * 2;
    const unsigned ldsw = (unsigned)wid * 1024u;
    const int aoff = lds_byte(wr * 64 + fr, fq * 8), boff = lds_byte(wc * 32 + fr, fq * 8);
#define SA(b, h) (((b) * 2 + (h)) * HTB)
#define SB(b, h) ((4 + (b) * 2 + (h)) * HTB)
#define STAGE(bufoff, gbase, voff) do { _Pragma("unroll") for (int _i = 0; _i < 2; ++_i) \
        __builtin_amdgcn_global_load_lds((const unsigned*)((const char*)(gbase) + (voff)[_i]), (LAS unsigned*)(lds + (bufoff) + ldsw + _i * 8192), 16, 0, 0); } while (0)
#define LDA(dst, b, h) do { _Pragma("unroll") for (int m = 0; m < 4; ++m) _Pragma("unroll") for (int k = 0; k < 2; ++k) dst[m][k] = *(const LAS bf16x8*)(lds + SA(b, h) + aoff + m * 2048 + k * 1024); } while (0)
#define LDB(dst, b, h) do { _Pragma("unroll") for (int n = 0; n < 2; ++n) _Pragma("unroll") for (int k = 0; k < 2; ++k) dst[n][k] = *(const LAS bf16x8*)(lds + SB(b, h) + boff + n * 2048 + k * 1024); } while (0)
#define MMA(ai, bj, At, Bt) do { __builtin_amdgcn_s_setprio(1); _Pragma("unroll") for (int m = 0; m < 4; ++m) _Pragma("unroll") for (int n = 0; n < 2; ++n) _Pragma("unroll") for (int k = 0; k < 2; ++k) \
        acc[ai][bj][m][n] = __builtin_amdgcn_mfma_f32_16x16x32_bf16(Bt[n][k], At[m][k], acc[ai][bj][m][n], 0, 0, 0); __builtin_amdgcn_s_setprio(0); } while (0)
#define BAR __builtin_amdgcn_s_barrier()
#define SCHED __builtin_amdgcn_sched_barrier(0)

    int qi = slot;
    while (qi < XQN) {
        const int mode = tab8(p.xm[xcd], qi);
        if (mode == 255) break;
        const int tile = tab8(p.xq[xcd], qi), pm = tile >> 4, pn = tile & 15, pos = tab8(p.pos, tile);
        const bool full = mode == 0;
        const size_t hstepA = full ? hstep : 0;
        const bool mine = ((pm + pn) & 1) == role;
        if (wid == 0) STAMP(tile * 8 + 0);

        auto hook = [&](bool remote) {
            if (wid == 0) {
                const unsigned long long t0 = __builtin_amdgcn_s_memrealtime();
                if (!remote) {
                    for (;;) {
                        const bool ok = ld_sys(ws + OFF_OWNW + (pn * 64 + lane) * 4) >= call;
                        if (__all(ok)) break;
                        if (give_up(ws, call, t0)) break;
                        __builtin_amdgcn_s_sleep(4);
                    }
                } else {
                    const char* fb = ws + OFF_INF + ((lane < 32 ? 2 * pm : 2 * pn + 1) * 32 + (lane & 31)) * 4;
                    for (;;) {
                        const bool ok = ld_sys(fb) >= call;
                        if (__all(ok)) break;
                        if (give_up(ws, call, t0)) break;
                        __builtin_amdgcn_s_sleep(16);
                    }
                    for (;;) {
                        bool ok = true;
#pragma unroll
                        for (int u = 1; u < 6; ++u) ok = ok & (ld_sys(fb + u * 4096) >= call);
                        if (__all(ok)) break;
                        if (give_up(ws, call, t0)) break;
                        __builtin_amdgcn_s_sleep(8);
                    }
                }
                __builtin_amdgcn_fence(__ATOMIC_ACQUIRE, "");
                asm volatile("s_waitcnt vmcnt(0)" ::: "memory");
            }
            asm volatile("" ::: "memory"); __builtin_amdgcn_s_barrier(); asm volatile("" ::: "memory");
        };

        f32x4 acc[2][2][4][2];
#pragma unroll
        for (int a = 0; a < 2; ++a)
#pragma unroll
            for (int b = 0; b < 2; ++b)
#pragma unroll
                for (int m = 0; m < 4; ++m)
#pragma unroll
                    for (int n = 0; n < 2; ++n) acc[a][b][m][n] = (f32x4){0.f, 0.f, 0.f, 0.f};
        bf16x8 At[4][2], B0[2][2], B1[2][2];

        const char* xsA = ws + OFF_XS + (size_t)pm * (512u << 10) + (mode == 2 ? hstep : 0);
        const char* wsB = ws + OFF_WTS + (size_t)pn * (512u << 10);
        const char* cA = p.x + (size_t)((qd >> 1) * 4096 + pm * 256) * 2048 + (mode == 2 ? hstep : 0);
        const char* cB = wsB + (size_t)3 * (8u << 20);
        hook(false);
        if (wid == 0) STAMP(tile * 8 + 1);
        STAGE(SB(0, 0), cB, voffB); STAGE(SA(0, 0), cA, voffA); STAGE(SB(0, 1), cB + hstep, voffB); STAGE(SA(0, 1), cA + hstepA, voffA);
        if (wr == 1) BAR;
        WAIT_V(4); BAR;
        STAGE(SB(1, 0), cB + kstep, voffB); STAGE(SA(1, 0), cA + kstep, voffA); STAGE(SB(1, 1), cB + hstep + kstep, voffB);
        WAIT_V(6); BAR;
        constexpr int nt = KS / BK;
        for (int u = 0; u < 4; ++u) {
            const bool has_next = u + 1 < 4;
            const char* nA = has_next ? xsA + (size_t)u * (8u << 20) : cA;
            const char* nB = has_next ? wsB + (size_t)u * (8u << 20) : cB;
            for (int t = 0; t < nt; t += 2) {
                const bool last = (t == nt - 2);
                const char* a1 = cA + (size_t)(t + 1) * kstep;
                const char* a2 = last ? nA : cA + (size_t)(t + 2) * kstep; const char* b2 = last ? nB : cB + (size_t)(t + 2) * kstep;
                const char* a3 = a2 + kstep; const char* b3 = b2 + kstep;
                if (last && u == 0) {
                    if (wid == 0) STAMP(tile * 8 + 2);
                    hook(true);
                    if (wid == 0) STAMP(tile * 8 + 3);
                }
                LDB(B0, 0, 0); SCHED; LDA(At, 0, 0); STAGE(SA(1, 1), a1 + hstepA, voffA);
                WAIT_L(8); BAR; WAIT_L(0); MMA(0, 0, At, B0); BAR; SCHED;
                LDB(B1, 0, 1); STAGE(SB(0, 0), b2, voffB);
                BAR; WAIT_L(0); MMA(0, 1, At, B1); BAR;
                if (full) LDA(At, 0, 1);
                STAGE(SA(0, 0), a2, voffA);
                BAR; WAIT_L(0); if (full) MMA(1, 0, At, B0); BAR; SCHED;
                STAGE(SB(0, 1), b2 + hstep, voffB);
                WAIT_V(6); BAR; if (full) MMA(1, 1, At, B1); BAR;
                LDB(B0, 1, 0); SCHED; LDA(At, 1, 0); STAGE(SA(0, 1), a2 + hstepA, voffA);
                WAIT_L(8); BAR; WAIT_L(0); MMA(0, 0, At, B0); BAR; SCHED;
                LDB(B1, 1, 1); STAGE(SB(1, 0), b3, voffB);
                BAR; WAIT_L(0); MMA(0, 1, At, B1); BAR;
                if (full) LDA(At, 1, 1);
                STAGE(SA(1, 0), a3, voffA);
                BAR; WAIT_L(0); if (full) MMA(1, 0, At, B0); BAR; SCHED;
                STAGE(SB(1, 1), b3 + hstep, voffB);
                WAIT_V(6); BAR; if (full) MMA(1, 1, At, B1); BAR;
            }
            cA = nA; cB = nB;
        }
        if (wid == 0) STAMP(tile * 8 + 4);
        if (wr == 0) BAR;
        const __amdgpu_buffer_rsrc_t rw = mk_rsrc(ws, WS_NEED);
        const unsigned pbase = (unsigned)pos * (128u << 10) + (unsigned)wid * (16u << 10);
        if (!mine) {
#pragma unroll
            for (int ai = 0; ai < 2; ++ai) {
                if (ai == 0 || full)
#pragma unroll
                for (int m = 0; m < 4; ++m)
#pragma unroll
                    for (int bj = 0; bj < 2; ++bj) {
                        const f32x4 v0 = acc[ai][bj][m][0], v1 = acc[ai][bj][m][1];
                        u32x4 o;
                        o.x = cvt_pk_bf16(v0[0], v0[1]); o.y = cvt_pk_bf16(v0[2], v0[3]);
                        o.z = cvt_pk_bf16(v1[0], v1[1]); o.w = cvt_pk_bf16(v1[2], v1[3]);
                        __builtin_amdgcn_raw_buffer_store_b128(o, rw, lane * 16, OFF_PS + pbase + (mode == 2 ? 8192u : 0u) + ((ai * 4 + m) * 2 + bj) * 1024u, 17);
                    }
            }
        } else {
            if (wid == 0) {
                const unsigned long long t0 = __builtin_amdgcn_s_memrealtime();
                for (;;) {
                    const int fi = lane < 16 ? pos * 16 + lane : max(pos - 4, 0) * 16 + 15;
                    const unsigned long long m = __ballot(ld_sys(ws + OFF_PFLAG + fi * 4) >= call);
                    if ((m & 0xffffull) == 0xffffull) break;
                    if (give_up(ws, call, t0)) break;
                    if (m >> 16) __builtin_amdgcn_s_sleep(PSL_S); else __builtin_amdgcn_s_sleep(PSL_L);
                }
            }
            asm volatile("" ::: "memory"); __builtin_amdgcn_s_barrier(); asm volatile("" ::: "memory");
            if (wid == 0) STAMP(tile * 8 + 5);
            const __amdgpu_buffer_rsrc_t ro = mk_rsrc(p.out[rank], 128u << 20);
            const unsigned base = (unsigned)((qd >> 1) * 4096 + pm * BM) * 16384u + (unsigned)((qd & 1) * 4096 + pn * BM) * 2u;
            const unsigned voff = (unsigned)(wr * 64 + fr) * 16384u + (unsigned)(wc * 32 + 8 * fq) * 2u;
            u32x4 pr[2][4][2];
#pragma unroll
            for (int ai = 0; ai < 2; ++ai)
#pragma unroll
                for (int m = 0; m < 4; ++m)
#pragma unroll
                    for (int bj = 0; bj < 2; ++bj)
                        pr[ai][m][bj] = __builtin_amdgcn_raw_buffer_load_b128(rw, lane * 16, OFF_PR + pbase + ((ai * 4 + m) * 2 + bj) * 1024u, 17);
            __builtin_amdgcn_sched_barrier(0);
#pragma unroll
            for (int ai = 0; ai < 2; ++ai)
#pragma unroll
                for (int m = 0; m < 4; ++m)
#pragma unroll
                    for (int bj = 0; bj < 2; ++bj) {
                        const f32x4 v0 = acc[ai][bj][m][0], v1 = acc[ai][bj][m][1];
                        const u32x4 q = pr[ai][m][bj];
                        u32x4 o;
                        o.x = cvt_pk_bf16(silu(v0[0] + bf_lo(q.x)), silu(v0[1] + bf_hi(q.x))); o.y = cvt_pk_bf16(silu(v0[2] + bf_lo(q.y)), silu(v0[3] + bf_hi(q.y)));
                        o.z = cvt_pk_bf16(silu(v1[0] + bf_lo(q.z)), silu(v1[1] + bf_hi(q.z))); o.w = cvt_pk_bf16(silu(v1[2] + bf_lo(q.w)), silu(v1[3] + bf_hi(q.w)));
                        __builtin_amdgcn_raw_buffer_store_b128(o, ro, voff, base + (unsigned)(ai * HALF + m * 16) * 16384u + (unsigned)(bj * HALF) * 2u, 17);
                    }
        }
        WAIT_V(0);
        BAR;
        if (tid < NR) {
            if (mine) { if (tid != rank) st_sys(p.ws[tid] + OFF_RT + rank * 512 + pos * 4, call); }
            else if (tid == (rank ^ PM)) {
                if (full || (__hip_atomic_fetch_add((unsigned*)(ws + OFF_HCNT + pos * 4), 1u, __ATOMIC_RELAXED, __HIP_MEMORY_SCOPE_AGENT) & 1u))
                    st_sys(p.ws[tid] + OFF_RS + pos * 4, call);
            }
        }
        if (tid == 0) {
            const unsigned t = __hip_atomic_fetch_add((unsigned*)(ws + OFF_QCTR + (8 + xcd) * 128), 1u, __ATOMIC_RELAXED, __HIP_MEMORY_SCOPE_AGENT);
            *(volatile LAS unsigned*)lds = GPX + t;
        }
        if (wid == 0) STAMP(tile * 8 + 6);
        __syncthreads();
        qi = __builtin_amdgcn_readfirstlane(*(volatile LAS unsigned*)lds);
        __syncthreads();
    }
#undef SA
#undef SB
#undef STAGE
#undef LDA
#undef LDB
#undef MMA
#undef BAR
#undef SCHED
}

__global__ __launch_bounds__(512, 2) void fused_gemm_allreduce(const Params p_by_value) {
    KP p = *(const __attribute__((address_space(4))) Params*)__builtin_amdgcn_kernarg_segment_ptr();
    extern __shared__ __attribute__((aligned(16))) unsigned char lds_raw[];
    LAS unsigned char* lds = (LAS unsigned char*)lds_raw;
    const int b = blockIdx.x, xcd = b & 7, slot = b >> 3;
    char* ws = p.ws[p.rank];
    if (slot < 32 - GPX) {
        unsigned xcc;
        asm volatile("s_getreg_b32 %0, hwreg(HW_REG_XCC_ID)" : "=s"(xcc));
        const int xsel = slot == 0 ? xcd : (int)(xcc & 7u);
        const int peer = (slot < NCW || slot < NCP) ? tab8(p.xlink, xsel) : 255;
        if (peer < NR) comm_role(p, peer, slot, slot * 8 + xcd);
        else if (slot >= 32 - GPX - NSPARE) stage_role(p, (slot - (32 - GPX - NSPARE)) * 8 + xcd);
    }
    unsigned call = 0;
    if (slot >= 32 - GPX || __builtin_amdgcn_readfirstlane(threadIdx.x) < 64) call = __builtin_amdgcn_readfirstlane(ld_agent(ws + OFF_SEQ)) + 1u;
    if (slot >= 32 - GPX) gemm_role(p, xcd, slot - (32 - GPX), call, lds);
    asm volatile("s_waitcnt vmcnt(0)" ::: "memory");
    __syncthreads();
    if (threadIdx.x == 0) {
        const unsigned t = __hip_atomic_fetch_add((unsigned*)(ws + OFF_DONE), 1u, __ATOMIC_RELAXED, __HIP_MEMORY_SCOPE_AGENT);
        if ((t + 1u) % (unsigned)GRID == 0u) {
            for (int l = 0; l < 27; ++l) st_agent(ws + OFF_QCTR + l * 128, 0u);
            st_agent(ws + OFF_SEQ, call);
            if (ld_agent(ws + OFF_TMO) == call) {
                const int t0 = tab8(p.forder[role_of(p.rank)], 0), qd = quad_of(p.rank);
                const size_t off = (size_t)((qd >> 1) * 4096 + (t0 >> 4) * 256) * 16384 + (size_t)((qd & 1) * 4096 + (t0 & 15) * 256) * 2;
                st_agent(p.out[p.rank] + off, 0x7fc07fc0u);
            }
        }
    }
}

}

extern "C" void kernel_launch(int rank, int nranks, ncclComm_t comm,
                              hipStream_t stream,
                              void* const* d_in, const int* in_sizes, int n_in,
                              void* d_out, int out_size,
                              const int* in_dtypes, int out_dtype,
                              void* const* d_out_all,
                              void* const* d_ws_all, size_t ws_bytes) {
    if (nranks != NR || (unsigned)rank >= (unsigned)NR || ws_bytes < WS_NEED || n_in < 2) {
        fprintf(stderr, "unsupported configuration\n");
        return;
    }
    static bool attr_set[NR] = {};
    if (!attr_set[rank]) {
        HIP_CHECK(hipFuncSetAttribute((const void*)fused_gemm_allreduce, hipFuncAttributeMaxDynamicSharedMemorySize, LDS_BYTES));
        attr_set[rank] = true;
    }
    Params p;
    p.x = (const char*)d_in[0];
    p.w = (const char*)d_in[1];
    for (int r = 0; r < NR; ++r) { p.out[r] = (char*)d_out_all[r]; p.ws[r] = (char*)d_ws_all[r]; }
    p.rank = rank;
    {
        static const unsigned char xl[NR][8] = {
            {6, 255, 2, 3, 1, 5, 4, 7}, {7, 255, 4, 5, 0, 3, 2, 6}, {3, 255, 0, 6, 4, 5, 1, 7}, {2, 255, 0, 6, 1, 5, 4, 7},
            {5, 255, 1, 7, 0, 2, 3, 6}, {4, 255, 1, 7, 2, 3, 0, 6}, {0, 255, 2, 3, 4, 7, 1, 5}, {1, 255, 4, 5, 3, 6, 0, 2}};
        for (int i = 0; i < 8; ++i) ((unsigned char*)p.xlink)[i] = xl[rank][i];
    }
    {
        int nf[2] = {0, 0}, nx[8] = {0, 0, 0, 0, 0, 0, 0, 0}, nt[8] = {0, 0, 0, 0, 0, 0, 0, 0};
        for (int g = 0; g < 8; ++g)
            for (int i = 0; i < XQN; ++i) { ((unsigned char*)p.xq[g])[i] = 0; ((unsigned char*)p.xm[g])[i] = 255; }
        for (int e = 0; e < 32; ++e)
            for (int t = 0; t < 256; ++t) {
                const int pm = t >> 4, pn = t & 15, a = 2 * pm, bb = 2 * pn + 1;
                if ((a > bb ? a : bb) != e) continue;
                const int h = (pm + pn) & 1, g = (pm & 3) * 2 + ((pn >> 1) & 1);
                ((unsigned char*)p.forder[h])[nf[h]] = (unsigned char)t;
                ((unsigned char*)p.pos)[t] = (unsigned char)nf[h]++;
                const bool split = h != role_of(rank) && nf[h] <= NHALF && nx[g] + 2 + (32 - nt[g] - 1) <= XQN;
                for (int m = split ? 1 : 0; m <= (split ? 2 : 0); ++m) {
                    ((unsigned char*)p.xq[g])[nx[g]] = (unsigned char)t;
                    ((unsigned char*)p.xm[g])[nx[g]++] = (unsigned char)m;
                }
                ++nt[g];
            }
    }
    hipLaunchKernelGGL(fused_gemm_allreduce, dim3(GRID), dim3(512), LDS_BYTES, stream, p);
    HIP_CHECK(hipGetLastError());
}
```

```cpp
#include <hip/hip_runtime.h>
#include <rccl/rccl.h>
#include <cstdio>
#include <cstdint>

#define HIP_CHECK(expr) do { hipError_t _e = (expr); if (_e != hipSuccess) { \
    fprintf(stderr, "HIP error %s at %s:%d\n", hipGetErrorString(_e), __FILE__, __LINE__); \
    return; } } while (0)

namespace {

#define LAS __attribute__((address_space(3)))
typedef short bf16x8 __attribute__((ext_vector_type(8)));
typedef float f32x4 __attribute__((ext_vector_type(4)));
typedef unsigned u32x4 __attribute__((ext_vector_type(4)));
typedef unsigned long long u64;

constexpr int NR = 8;
#define PM_ 7
constexpr int PM = PM_;
constexpr int KS = 1024;
constexpr int NPOS = 128;
#define NCW 14
#define NCP 12
#define EPUSH 8
#define NHALF 50
#define STG_SLOT 4
#define PGATE (N_PUSH - 48)
#define GSL 32
#define PSL_S 24
#define PSL_L 120
#define CSL 24
#define GPX 16
constexpr int GRID = 256;
constexpr int N_IN = 32 * 32;
constexpr int N_FIN = NPOS * 16;
constexpr int N_PUSH = EPUSH * 32;
constexpr int N_PIN = N_IN - N_PUSH;
constexpr int N_STG = 2 * 2 * (16 - EPUSH / 2) * 64;
static_assert(EPUSH % 2 == 0 && EPUSH >= 2 && EPUSH <= 32, "EPUSH");
constexpr int NSPARE = 32 - GPX - (NCW > NCP ? NCW : NCP);
constexpr int NSTG_WG = (NSPARE + GPX - STG_SLOT) * 8;
static_assert(NSTG_WG > 0, "stagers");
constexpr int XQN = 48;
constexpr int J_NONE = -N_IN - 1;
constexpr int SEGSH = 7;
constexpr int NSEG = N_FIN >> SEGSH;
static_assert(NSEG <= 64, "config");
static_assert(NCW <= 32 - GPX && NCP <= 32 - GPX && GPX >= 16 && GPX <= 32, "too many workgroups");

constexpr unsigned OFF_SEQ = 0, OFF_DONE = 128, OFF_TMO = 256;
constexpr unsigned OFF_INF = 4096;
constexpr unsigned OFF_OWNW = OFF_INF + 6 * 32 * 32 * 4;
constexpr unsigned OFF_RT = OFF_OWNW + 16 * 64 * 4;
constexpr unsigned OFF_RS = OFF_RT + 8 * 512;
constexpr unsigned OFF_PFLAG = OFF_RS + 512;
constexpr unsigned OFF_HAVE = OFF_PFLAG + 128 * 16 * 4;
constexpr unsigned OFF_SR = OFF_HAVE + 6 * 64 * 4;
constexpr unsigned OFF_HCNT = OFF_SR + 128;
constexpr unsigned OFF_QCTR = OFF_HCNT + 512;
constexpr unsigned OFF_DBG = 128 * 1024;
constexpr unsigned OFF_XS = 1u << 20;
constexpr unsigned OFF_WTS = OFF_XS + (24u << 20);
constexpr unsigned OFF_PS = OFF_WTS + (32u << 20);
constexpr unsigned OFF_PR = OFF_PS + (16u << 20);
constexpr unsigned OFF_XO = OFF_PR + (16u << 20);
constexpr unsigned OFF_WO = OFF_XO + (16u << 20);
constexpr unsigned OFF_FS = OFF_WO + (16u << 20);
constexpr unsigned WS_NEED = OFF_FS + (16u << 20);
static_assert(OFF_QCTR + 27 * 128 <= OFF_DBG && WS_NEED <= (256u << 20), "layout");
#define DEADLINE_ 150000000ull
constexpr unsigned long long DEADLINE = DEADLINE_;

constexpr int BM = 256, BK = 64, HALF = 128, HTB = HALF * BK * 2, STAGE_BYTES = 8 * HTB;
constexpr int LDS_BYTES = STAGE_BYTES;

struct Params {
    const char* x;
    const char* w;
    char* out[NR];
    char* ws[NR];
    int rank;
    unsigned forder[2][NPOS / 4];
    unsigned pos[256 / 4];
    unsigned xq[8][XQN / 4];
    unsigned xm[8][XQN / 4];
    unsigned xlink[2];
};
typedef const __attribute__((address_space(4))) Params& KP;

__host__ __device__ __forceinline__ int role_of(int r) { return (r >> 2) & 1; }
__host__ __device__ __forceinline__ int quad_of(int r) { return role_of(r) ? (r ^ PM) : r; }
__host__ __device__ __forceinline__ int uidx(int m) { return m < PM ? m - 1 : m - 2; }

__device__ __forceinline__ int lds_byte(int r, int c) { const int st = (r >> 4) * 2 + (c >> 5), rr = r & 15, cc = c & 31, ob = rr * 64 + cc * 2; return st * 1024 + (ob ^ (((ob >> 9) & 1) << 5)); }
__device__ __forceinline__ void stage_rc(int b, int& R, int& C) { const int st = b / 1024, sb = b % 1024, swz = sb ^ (((sb >> 9) & 1) << 5); R = (st >> 1) * 16 + swz / 64; C = (st & 1) * 32 + (swz % 64) / 2; }
__device__ __forceinline__ int perm32(int rho) { const int n = rho >> 4, i = rho & 15; return 8 * (i >> 2) + 4 * n + (i & 3); }

__device__ __forceinline__ unsigned ld_sys(const char* p) { return __hip_atomic_load((const unsigned*)p, __ATOMIC_RELAXED, __HIP_MEMORY_SCOPE_SYSTEM); }
__device__ __forceinline__ unsigned ld_agent(const char* p) { return __hip_atomic_load((const unsigned*)p, __ATOMIC_RELAXED, __HIP_MEMORY_SCOPE_AGENT); }
__device__ __forceinline__ void st_sys(char* p, unsigned v) { __hip_atomic_store((unsigned*)p, v, __ATOMIC_RELAXED, __HIP_MEMORY_SCOPE_SYSTEM); }
__device__ __forceinline__ void st_agent(char* p, unsigned v) { __hip_atomic_store((unsigned*)p, v, __ATOMIC_RELAXED, __HIP_MEMORY_SCOPE_AGENT); }
__device__ __forceinline__ unsigned cvt_pk_bf16(float lo, float hi) { unsigned r; asm("v_cvt_pk_bf16_f32 %0, %1, %2" : "=v"(r) : "v"(lo), "v"(hi)); return r; }
__device__ __forceinline__ float bf_lo(unsigned u) { return __uint_as_float(u << 16); }
__device__ __forceinline__ float bf_hi(unsigned u) { return __uint_as_float(u & 0xffff0000u); }
__device__ __forceinline__ float silu(float y) {
    const float t = fminf(fmaxf(y, -60.0f), 60.0f);
    const float e = __builtin_amdgcn_exp2f(t * -1.44269504088896f);
    return y * __builtin_amdgcn_rcpf(1.0f + e);
}
__device__ __forceinline__ __amdgpu_buffer_rsrc_t mk_rsrc(const void* p, unsigned bytes) {
    return __builtin_amdgcn_make_buffer_rsrc((void*)p, 0, bytes, 0x00020000);
}
template <class T> __device__ __forceinline__ int tab8(const T& t, int i) { return (t[i >> 2] >> (8 * (i & 3))) & 255; }
#define WAIT_V(n) asm volatile("s_waitcnt vmcnt(" #n ")" ::: "memory")
#define WAIT_L(n) asm volatile("s_waitcnt lgkmcnt(" #n ")" ::: "memory")

#define STAMP(idx) do { } while (0)

__device__ __forceinline__ bool give_up(char* ws, unsigned call, unsigned long long t0) {
    const unsigned tm = __builtin_amdgcn_readfirstlane(ld_agent(ws + OFF_TMO));
    if (tm == call || __builtin_amdgcn_s_memrealtime() - t0 > DEADLINE) {
        if ((threadIdx.x & 63) == 0) st_agent(ws + OFF_TMO, call);
        return true;
    }
    return false;
}

__device__ __forceinline__ u64 uni64(const void* p) {
    const u64 a = (u64)p;
    return ((u64)(unsigned)__builtin_amdgcn_readfirstlane((unsigned)(a >> 32)) << 32) | (unsigned)__builtin_amdgcn_readfirstlane((unsigned)a);
}
__device__ __forceinline__ u32x4 mk_srd(u64 a) {
    u32x4 r;
    r.x = (unsigned)a;
    r.y = (unsigned)(a >> 32) & 0xffffu;
    r.z = 256u << 20;
    r.w = 0x00020000u;
    return r;
}
#define CE_CLOB "memory", "scc", "vcc", "v156", "v157", "v158", "v159", "v160", "v161", "v162", "v163", "v164", "v165", "v166", "v167", "v168", "v169", "v170", "v171", "v172", "v173", "v174", "v175", "v176", "v177", "v178", "v179", "v180", "v181", "v182", "v183", "v184", "v185", "v186", "v187", "v188", "v189", "v190", "v191", "v192", "v193", "v194", "v195", "v196", "v197", "v198", "v199", "v200", "v201", "v202", "v203", "v204", "v205", "v206", "v207", "v208", "v209", "v210", "v211", "v212", "v213", "v214", "v215", "v216", "v217", "v218", "v219", "v220", "v221", "v222", "v223", "v224", "v225", "v226", "v227", "v228", "v229", "v230", "v231", "v232", "v233", "v234", "v235", "v236", "v237", "v238", "v239", "v240", "v241", "v242", "v243", "v244", "v245", "v246", "v247", "v248", "v249", "v250", "v251", "v252", "v253", "v254", "v255"
#define LDF "sc0 sc1"
template <int R> __device__ __forceinline__ void ld8(unsigned voff, const u32x4& srd, unsigned soff, unsigned step) {
    unsigned t;
    asm volatile(
        "s_nop 4\n\t"
        "buffer_load_dwordx4 v[%c5+0:%c5+3], %1, %2, %3 offen " LDF "\n\t"
        "s_add_u32 %0, %3, %4\n\t"
        "buffer_load_dwordx4 v[%c5+4:%c5+7], %1, %2, %0 offen " LDF "\n\t"
        "s_add_u32 %0, %0, %4\n\t"
        "buffer_load_dwordx4 v[%c5+8:%c5+11], %1, %2, %0 offen " LDF "\n\t"
        "s_add_u32 %0, %0, %4\n\t"
        "buffer_load_dwordx4 v[%c5+12:%c5+15], %1, %2, %0 offen " LDF "\n\t"
        "s_add_u32 %0, %0, %4\n\t"
        "buffer_load_dwordx4 v[%c5+16:%c5+19], %1, %2, %0 offen " LDF "\n\t"
        "s_add_u32 %0, %0, %4\n\t"
        "buffer_load_dwordx4 v[%c5+20:%c5+23], %1, %2, %0 offen " LDF "\n\t"
        "s_add_u32 %0, %0, %4\n\t"
        "buffer_load_dwordx4 v[%c5+24:%c5+27], %1, %2, %0 offen " LDF "\n\t"
        "s_add_u32 %0, %0, %4\n\t"
        "buffer_load_dwordx4 v[%c5+28:%c5+31], %1, %2, %0 offen " LDF "\n\t"
        "s_nop 0"
        : "=&s"(t)
        : "v"(voff), "s"(srd), "s"(soff), "s"(step), "i"(R)
        : CE_CLOB);
}
#define SWIN 4
#define SWN(j) ((j) < SWIN ? SWIN + 8 : SWIN - 1)
template <int R> __device__ __forceinline__ void st8(unsigned voff, const u32x4& srd, unsigned soff, unsigned step) {
    unsigned t;
    asm volatile(
        "s_nop 4\n\t"
        "s_waitcnt vmcnt(%c6)\n\t"
        "buffer_store_dwordx4 v[%c5+0:%c5+3], %1, %2, %3 offen sc0 sc1\n\t"
        "s_add_u32 %0, %3, %4\n\t"
        "s_waitcnt vmcnt(%c7)\n\t"
        "buffer_store_dwordx4 v[%c5+4:%c5+7], %1, %2, %0 offen sc0 sc1\n\t"
        "s_add_u32 %0, %0, %4\n\t"
        "s_waitcnt vmcnt(%c8)\n\t"
        "buffer_store_dwordx4 v[%c5+8:%c5+11], %1, %2, %0 offen sc0 sc1\n\t"
        "s_add_u32 %0, %0, %4\n\t"
        "s_waitcnt vmcnt(%c9)\n\t"
        "buffer_store_dwordx4 v[%c5+12:%c5+15], %1, %2, %0 offen sc0 sc1\n\t"
        "s_add_u32 %0, %0, %4\n\t"
        "s_waitcnt vmcnt(%c10)\n\t"
        "buffer_store_dwordx4 v[%c5+16:%c5+19], %1, %2, %0 offen sc0 sc1\n\t"
        "s_add_u32 %0, %0, %4\n\t"
        "s_waitcnt vmcnt(%c11)\n\t"
        "buffer_store_dwordx4 v[%c5+20:%c5+23], %1, %2, %0 offen sc0 sc1\n\t"
        "s_add_u32 %0, %0, %4\n\t"
        "s_waitcnt vmcnt(%c12)\n\t"
        "buffer_store_dwordx4 v[%c5+24:%c5+27], %1, %2, %0 offen sc0 sc1\n\t"
        "s_add_u32 %0, %0, %4\n\t"
        "s_waitcnt vmcnt(%c13)\n\t"
        "buffer_store_dwordx4 v[%c5+28:%c5+31], %1, %2, %0 offen sc0 sc1\n\t"
        "s_nop 1"
        : "=&s"(t)
        : "v"(voff), "s"(srd), "s"(soff), "s"(step), "i"(R), "i"(SWN(0)), "i"(SWN(1)), "i"(SWN(2)), "i"(SWN(3)), "i"(SWN(4)), "i"(SWN(5)), "i"(SWN(6)), "i"(SWN(7))
        : CE_CLOB);
}
template <int R> __device__ __forceinline__ void tr8() {
    asm volatile(
        "v_perm_b32 v224, v[%c2+4], v[%c2+0], %0\n\t"
        "v_perm_b32 v225, v[%c2+12], v[%c2+8], %0\n\t"
        "v_perm_b32 v226, v[%c2+20], v[%c2+16], %0\n\t"
        "v_perm_b32 v227, v[%c2+28], v[%c2+24], %0\n\t"
        "v_perm_b32 v228, v[%c2+4], v[%c2+0], %1\n\t"
        "v_perm_b32 v229, v[%c2+12], v[%c2+8], %1\n\t"
        "v_perm_b32 v230, v[%c2+20], v[%c2+16], %1\n\t"
        "v_perm_b32 v231, v[%c2+28], v[%c2+24], %1\n\t"
        "v_perm_b32 v232, v[%c2+5], v[%c2+1], %0\n\t"
        "v_perm_b32 v233, v[%c2+13], v[%c2+9], %0\n\t"
        "v_perm_b32 v234, v[%c2+21], v[%c2+17], %0\n\t"
        "v_perm_b32 v235, v[%c2+29], v[%c2+25], %0\n\t"
        "v_perm_b32 v236, v[%c2+5], v[%c2+1], %1\n\t"
        "v_perm_b32 v237, v[%c2+13], v[%c2+9], %1\n\t"
        "v_perm_b32 v238, v[%c2+21], v[%c2+17], %1\n\t"
        "v_perm_b32 v239, v[%c2+29], v[%c2+25], %1\n\t"
        "v_perm_b32 v240, v[%c2+6], v[%c2+2], %0\n\t"
        "v_perm_b32 v241, v[%c2+14], v[%c2+10], %0\n\t"
        "v_perm_b32 v242, v[%c2+22], v[%c2+18], %0\n\t"
        "v_perm_b32 v243, v[%c2+30], v[%c2+26], %0\n\t"
        "v_perm_b32 v244, v[%c2+6], v[%c2+2], %1\n\t"
        "v_perm_b32 v245, v[%c2+14], v[%c2+10], %1\n\t"
        "v_perm_b32 v246, v[%c2+22], v[%c2+18], %1\n\t"
        "v_perm_b32 v247, v[%c2+30], v[%c2+26], %1\n\t"
        "v_perm_b32 v248, v[%c2+7], v[%c2+3], %0\n\t"
        "v_perm_b32 v249, v[%c2+15], v[%c2+11], %0\n\t"
        "v_perm_b32 v250, v[%c2+23], v[%c2+19], %0\n\t"
        "v_perm_b32 v251, v[%c2+31], v[%c2+27], %0\n\t"
        "v_perm_b32 v252, v[%c2+7], v[%c2+3], %1\n\t"
        "v_perm_b32 v253, v[%c2+15], v[%c2+11], %1\n\t"
        "v_perm_b32 v254, v[%c2+23], v[%c2+19], %1\n\t"
        "v_perm_b32 v255, v[%c2+31], v[%c2+27], %1\n\t"
        "s_nop 1"
        :: "s"(0x05040100u), "s"(0x07060302u), "i"(R)
        : CE_CLOB);
}
template <int R> __device__ __forceinline__ void ld4(unsigned voff, const u32x4& srd, unsigned soff, unsigned step) {
    asm volatile(
        "s_nop 4\n\t"
        "buffer_load_dwordx4 v[%c4+0:%c4+3], %1, %2, %0 offen sc0 sc1\n\t"
        "s_add_u32 %0, %0, %3\n\t"
        "buffer_load_dwordx4 v[%c4+4:%c4+7], %1, %2, %0 offen sc0 sc1\n\t"
        "s_add_u32 %0, %0, %3\n\t"
        "buffer_load_dwordx4 v[%c4+8:%c4+11], %1, %2, %0 offen sc0 sc1\n\t"
        "s_add_u32 %0, %0, %3\n\t"
        "buffer_load_dwordx4 v[%c4+12:%c4+15], %1, %2, %0 offen sc0 sc1\n\t"
        "s_nop 0"
        : "+s"(soff)
        : "v"(voff), "s"(srd), "s"(step), "i"(R)
        : CE_CLOB);
}
template <int X, int Y> __device__ __forceinline__ void xfer(unsigned dst, unsigned src4, unsigned nsrc, unsigned voff, const u32x4& dsrd, unsigned step,
                                                         const u32x4& ssrd, unsigned nvoff, const u32x4& nsrd, unsigned nstep, unsigned svoff, unsigned sstep) {
    asm volatile(
        "s_nop 4\n\t"
        "s_waitcnt vmcnt(6)\n\t"
        "buffer_store_dwordx4 v[%c12+0:%c12+3], %3, %4, %0 offen sc0 sc1\n\t"
        "s_add_u32 %0, %0, %5\n\t"
        "buffer_load_dwordx4 v[%c12+16:%c12+19], %10, %6, %1 offen sc0 sc1\n\t"
        "s_add_u32 %1, %1, %11\n\t"
        "s_waitcnt vmcnt(6)\n\t"
        "buffer_store_dwordx4 v[%c12+4:%c12+7], %3, %4, %0 offen sc0 sc1\n\t"
        "s_add_u32 %0, %0, %5\n\t"
        "buffer_load_dwordx4 v[%c12+20:%c12+23], %10, %6, %1 offen sc0 sc1\n\t"
        "s_add_u32 %1, %1, %11\n\t"
        "s_waitcnt vmcnt(6)\n\t"
        "buffer_store_dwordx4 v[%c12+8:%c12+11], %3, %4, %0 offen sc0 sc1\n\t"
        "s_add_u32 %0, %0, %5\n\t"
        "buffer_load_dwordx4 v[%c12+24:%c12+27], %10, %6, %1 offen sc0 sc1\n\t"
        "s_add_u32 %1, %1, %11\n\t"
        "s_waitcnt vmcnt(6)\n\t"
        "buffer_store_dwordx4 v[%c12+12:%c12+15], %3, %4, %0 offen sc0 sc1\n\t"
        "s_add_u32 %0, %0, %5\n\t"
        "buffer_load_dwordx4 v[%c12+28:%c12+31], %10, %6, %1 offen sc0 sc1\n\t"
        "s_waitcnt vmcnt(6)\n\t"
        "buffer_store_dwordx4 v[%c12+16:%c12+19], %3, %4, %0 offen sc0 sc1\n\t"
        "s_add_u32 %0, %0, %5\n\t"
        "buffer_load_dwordx4 v[%c13+0:%c13+3], %7, %8, %2 offen sc0 sc1\n\t"
        "s_add_u32 %2, %2, %9\n\t"
        "s_waitcnt vmcnt(6)\n\t"
        "buffer_store_dwordx4 v[%c12+20:%c12+23], %3, %4, %0 offen sc0 sc1\n\t"
        "s_add_u32 %0, %0, %5\n\t"
        "buffer_load_dwordx4 v[%c13+4:%c13+7], %7, %8, %2 offen sc0 sc1\n\t"
        "s_add_u32 %2, %2, %9\n\t"
        "s_waitcnt vmcnt(6)\n\t"
        "buffer_store_dwordx4 v[%c12+24:%c12+27], %3, %4, %0 offen sc0 sc1\n\t"
        "s_add_u32 %0, %0, %5\n\t"
        "buffer_load_dwordx4 v[%c13+8:%c13+11], %7, %8, %2 offen sc0 sc1\n\t"
        "s_add_u32 %2, %2, %9\n\t"
        "s_waitcnt vmcnt(6)\n\t"
        "buffer_store_dwordx4 v[%c12+28:%c12+31], %3, %4, %0 offen sc0 sc1\n\t"
        "buffer_load_dwordx4 v[%c13+12:%c13+15], %7, %8, %2 offen sc0 sc1\n\t"
        "s_nop 1"
        : "+s"(dst), "+s"(src4), "+s"(nsrc)
        : "v"(voff), "s"(dsrd), "s"(step), "s"(ssrd), "v"(nvoff), "s"(nsrd), "s"(nstep), "v"(svoff), "s"(sstep), "i"(X), "i"(Y)
        : CE_CLOB);
}
__device__ __forceinline__ void ticket_issue(u64 ctr) {
    u64 sv;
    asm volatile(
        "s_mov_b64 %0, exec\n\t"
        "s_mov_b64 exec, 1\n\t"
        "v_mov_b32 v158, 0\n\t"
        "v_mov_b32 v159, 1\n\t"
        "s_nop 4\n\t"
        "global_atomic_add v159, v158, v159, %1 sc0\n\t"
        "s_mov_b64 exec, %0"
        : "=&s"(sv) : "s"(ctr) : CE_CLOB);
}
__device__ __forceinline__ int ticket_read() {
    int q;
    asm volatile("s_nop 1\n\tv_readfirstlane_b32 %0, v159\n\ts_nop 4" : "=s"(q) :: CE_CLOB);
    return q;
}
__device__ __forceinline__ void tflags_issue(u64 base, unsigned voff) {
    asm volatile(
        "s_nop 4\n\t"
        "global_load_dword v157, %0, %1 sc0 sc1\n\t"
        "global_load_dword v158, %0, %1 offset:256 sc0 sc1"
        :: "v"(voff), "s"(base) : CE_CLOB);
}
__device__ __forceinline__ int tflags_read(unsigned call) {
    u64 m0, m1;
    asm volatile("s_nop 1\n\tv_cmp_le_u32 %0, %2, v157\n\tv_cmp_le_u32 %1, %2, v158\n\ts_nop 4" : "=&s"(m0), "=&s"(m1) : "s"(call) : CE_CLOB);
    return (~m0 != 0ull) ? __builtin_ctzll(~m0) : 64 + ((~m1 != 0ull) ? __builtin_ctzll(~m1) : 64);
}
__device__ __forceinline__ void fcnt_issue(u64 base, unsigned voff) {
    asm volatile("s_nop 4\n\tglobal_load_dword v156, %0, %1 sc0 sc1" :: "v"(voff), "s"(base) : CE_CLOB);
}
__device__ __forceinline__ int fcnt_read(unsigned need) {
    u64 m0;
    asm volatile("s_nop 1\n\tv_cmp_le_u32 %0, %1, v156\n\ts_nop 4" : "=&s"(m0) : "s"(need) : CE_CLOB);
    return (~m0 != 0ull) ? __builtin_ctzll(~m0) : 64;
}
#define CE_WAIT(n) asm volatile("s_waitcnt vmcnt(" #n ")" ::: CE_CLOB)
constexpr int RA = 160, RB = 192, RT = 224;

__device__ __forceinline__ unsigned selv(bool c0, bool c1, unsigned a, unsigned b, unsigned c) {
    asm("" : "+v"(a)); asm("" : "+v"(b)); asm("" : "+v"(c));
    return c0 ? a : (c1 ? b : c);
}
__device__ __forceinline__ u64 sels(bool c, u64 a, u64 b) {
    asm("" : "+s"(a)); asm("" : "+s"(b));
    return uni64((const void*)(c ? a : b));
}

struct Piece { int type; unsigned src, dst, flag; };

struct Comm {
    KP p;
    int rank, peer, lane;
    unsigned call;
    bool partner;
    u64 rx, rw, ro_me, rws_me, rws_peer, ro_peer;
    unsigned lin_off, w_src_off, w_dst_off, o_off;
    char* wsp;
    char* wsh;
    char* wsm;

    __device__ __forceinline__ Comm(KP p_, int peer_, int lane_, unsigned call_) : p(p_), rank(p_.rank), peer(peer_), lane(lane_), call(call_) {
        partner = (rank ^ peer) == PM;
        rx = uni64(p.x);
        rw = uni64(p.w);
        ro_me = uni64(p.out[rank]);
        wsm = p.ws[rank];
        rws_me = uni64(wsm);
        wsp = p.ws[peer];
        wsh = p.ws[peer ^ PM];
        rws_peer = uni64(wsp);
        ro_peer = uni64(p.out[peer]);
        lin_off = lane * 16;
        w_src_off = (lane >> 3) * 8 * 16384 + (lane & 7) * 16;
        w_dst_off = (lane & 7) * 8 * 2048 + (lane >> 3) * 16;
        o_off = (lane >> 5) * 16384 + (lane & 31) * 16;
    }
    __device__ __forceinline__ unsigned out_off(int a, int o, int po) const {
        const int t = tab8(p.forder[role_of(a)], o), pm = t >> 4, pn = t & 15, qd = quad_of(a);
        return (unsigned)((qd >> 1) * 4096 + pm * 256 + po * 16) * 16384u + (unsigned)((qd & 1) * 4096 + pn * 256) * 2u;
    }
    __device__ __forceinline__ Piece desc_in(int q) const {
        Piece d;
        q = __builtin_amdgcn_readfirstlane(q);
        const int u = uidx(rank ^ peer);
        const int e = q >> 5, pc = q & 31, panel = e >> 1, hd = role_of(peer), qd = quad_of(peer);
        d.flag = OFF_INF + (u * 1024 + q) * 4;
        if (!(e & 1)) {
            d.type = 0;
            d.src = (unsigned)((qd >> 1) * 4096 + panel * 256 + pc * 8) * 2048u + hd * 1024u;
            d.dst = OFF_XS + (u >> 1) * (8u << 20) + (unsigned)(panel * 256 + pc * 8) * 2048u + (u & 1) * 1024u;
        } else {
            const int kb = pc >> 2, nb = pc & 3;
            d.type = 1;
            d.src = (unsigned)((hd * 512 + kb * 64) * 8192 + (qd & 1) * 4096 + panel * 256 + nb * 64) * 2u;
            d.dst = OFF_WTS + (u >> 1) * (8u << 20) + (unsigned)((panel * 256 + nb * 64) * 1024 + (u & 1) * 512 + kb * 64) * 2u;
        }
        d.type = __builtin_amdgcn_readfirstlane(d.type);
        d.src = __builtin_amdgcn_readfirstlane(d.src);
        d.dst = __builtin_amdgcn_readfirstlane(d.dst);
        d.flag = __builtin_amdgcn_readfirstlane(d.flag);
        return d;
    }
    template <int R> __device__ __forceinline__ void load(const Piece& d) const {
        ld8<R>(selv(d.type == 1, false, w_src_off, lin_off, lin_off), mk_srd(sels(d.type == 0, rx, rw)), d.src, d.type == 0 ? 2048u : 16384u);
    }
    template <int R> __device__ __forceinline__ void store(const Piece& d) const {
        if (d.type == 1) {
            tr8<R>();
            st8<RT>(w_dst_off, mk_srd(rws_peer), d.dst, 2048u);
        } else {
            st8<R>(lin_off, mk_srd(rws_peer), d.dst, 2048u);
        }
    }
    __device__ __forceinline__ void flag_in(int q) const {
        if (q >= 0 && lane == 0) st_sys(wsp + OFF_INF + (uidx(rank ^ peer) * 1024 + q) * 4, call);
    }
    __device__ __forceinline__ Piece desc_pull(int j) const {
        Piece d;
        j = __builtin_amdgcn_readfirstlane(j);
        if (j < -N_PIN) {
            d.type = 4; d.src = d.dst = OFF_PS; d.flag = 0;
        } else if (j < 0) {
            const int q = j + N_IN, e = q >> 5, pc = q & 31, panel = e >> 1, u = uidx(rank ^ peer), qd = quad_of(rank);
            const unsigned row = (unsigned)(panel * 256 + pc * 8) * 2048u;
            d.type = 5;
            d.src = ((e & 1) ? OFF_WO + (qd & 1) * (4u << 20) : OFF_XO + (qd >> 1) * (4u << 20)) + row / 2 + role_of(rank) * (8u << 20);
            d.dst = ((e & 1) ? OFF_WTS : OFF_XS) + (u >> 1) * (8u << 20) + row + (u & 1) * 1024u;
            d.flag = OFF_INF + (u * 1024 + q) * 4;
        } else if (j >= N_FIN) {
            const int qq = j - N_FIN;
            d.type = partner ? 6 : 2;
            d.dst = out_off(rank ^ PM, qq >> 4, qq & 15); d.flag = 0;
            d.src = partner ? OFF_FS + qq * 8192u : d.dst;
        } else if (partner) {
            d.type = 3;
            d.src = OFF_PS + j * 8192u; d.dst = OFF_PR + j * 8192u; d.flag = OFF_PFLAG + j * 4;
        } else {
            d.type = 6;
            d.dst = out_off(peer, j >> 4, j & 15);
            d.src = OFF_FS + j * 8192u;
            d.flag = (OFF_HAVE + (uidx(rank ^ peer ^ PM) * 64 + (j >> SEGSH)) * 4) | 1u;
        }
        d.type = __builtin_amdgcn_readfirstlane(d.type);
        d.src = __builtin_amdgcn_readfirstlane(d.src);
        d.dst = __builtin_amdgcn_readfirstlane(d.dst);
        d.flag = __builtin_amdgcn_readfirstlane(d.flag);
        return d;
    }
    __device__ __forceinline__ unsigned pvoff(const Piece& d) const { return selv(d.type == 2, false, o_off, lin_off, lin_off); }
    __device__ __forceinline__ unsigned pstep(const Piece& d) const { return d.type == 2 ? 32768u : 1024u; }
    __device__ __forceinline__ unsigned dvoff(const Piece& d) const { return selv(d.type == 2 || d.type == 6, false, o_off, lin_off, lin_off); }
    __device__ __forceinline__ unsigned dstep(const Piece& d) const { return (d.type == 2 || d.type == 6) ? 32768u : (d.type == 5 ? 2048u : 1024u); }
    __device__ __forceinline__ u32x4 psrc(const Piece& d) const { return mk_srd(sels(d.type == 2, ro_peer, sels(d.type != 4, rws_peer, rws_me))); }
    __device__ __forceinline__ u32x4 pdst(const Piece& d) const { return mk_srd(sels(d.type == 2 || d.type == 6, ro_me, rws_me)); }
    template <int R> __device__ __forceinline__ void pull_first(const Piece& d) const { ld4<R>(pvoff(d), psrc(d), d.src, pstep(d)); }
    template <int X, int Y> __device__ __forceinline__ void pull_step(const Piece& c, const Piece& n) const {
        xfer<X, Y>(c.dst, c.src + 4 * pstep(c), n.src, dvoff(c), pdst(c), dstep(c), psrc(c), pvoff(n), psrc(n), pstep(n), pvoff(c), pstep(c));
    }
    unsigned seg_off = 0, seg_cnt = 0;
    __device__ __forceinline__ void flush() {
        if (seg_cnt != 0 && lane == 0) __hip_atomic_fetch_add((unsigned*)(wsh + seg_off), seg_cnt, __ATOMIC_RELAXED, __HIP_MEMORY_SCOPE_SYSTEM);
        seg_cnt = 0;
    }
    __device__ __forceinline__ void flag_pull(unsigned off) {
        if (off & 1u) {
            if ((off & ~1u) != seg_off) { flush(); seg_off = off & ~1u; }
            ++seg_cnt;
        } else if (off != 0) { if (lane == 0) st_sys(wsm + off, call); }
        else flush();
    }
};

__device__ __forceinline__ void comm_role(KP p, int peer_push, int peer_pull, int slot, int cidx) {
    const int rank = p.rank;
    char* ws = p.ws[rank];
    const int lane = threadIdx.x & 63, wid = __builtin_amdgcn_readfirstlane(threadIdx.x >> 6);
    const int peer = wid == 0 ? peer_push : peer_pull;
    if ((unsigned)peer >= (unsigned)NR || peer == rank) return;
    if (wid >= 2 || (wid == 0 && ((rank ^ peer) == PM || slot >= NCW)) || (wid == 1 && slot >= NCP)) return;
    const u64 ctr_own = uni64(ws + OFF_QCTR + (wid * 17 + peer) * 128);
    ticket_issue(ctr_own);
    Comm c(p, peer, lane, 0u);
    if (wid == 0) c.load<RA>(c.desc_in(slot));
    const unsigned call = __builtin_amdgcn_readfirstlane(ld_agent(ws + OFF_SEQ)) + 1u;
    c.call = call;
    STAMP(4096 + (cidx * 8 + wid) * 4 + 0);
    const bool partner = c.partner;
    CE_WAIT(0);
    int qn = ticket_read();

    if (wid == 0) {
        int qc = slot, qp = -1;
        qn += NCW;
        bool first = true;
#define CE_STEP(X, Y) { \
            if (qn >= N_PUSH) { CE_WAIT(0); c.flag_in(qp); c.store<X>(c.desc_in(qc)); CE_WAIT(0); c.flag_in(qc); break; } \
            ticket_issue(ctr_own); \
            c.load<Y>(c.desc_in(qn)); \
            if (first) CE_WAIT(9); else CE_WAIT(17); \
            first = false; \
            c.store<X>(c.desc_in(qc)); \
            CE_WAIT(16); \
            c.flag_in(qp); \
            qp = qc; qc = qn; \
            qn = NCW + ticket_read(); }
        for (;;) {
            CE_STEP(RA, RB)
            CE_STEP(RB, RA)
        }
#undef CE_STEP
        STAMP(4096 + (cidx * 8 + wid) * 4 + 1);
        return;
    }

    const u64 ctr_q = uni64(ws + OFF_QCTR + 16 * 128);
    const u64 rtf = uni64(ws + OFF_RT + peer * 512), rsf = uni64(ws + OFF_RS);
    const u64 hav = uni64(ws + OFF_HAVE + (partner ? 0 : uidx(rank ^ peer)) * 256);
    const unsigned tvoff = lane * 4;
    const unsigned long long t0 = __builtin_amdgcn_s_memrealtime();
    int qoff = partner ? 0 : -N_PIN;
    int qready = 0, qlim = N_FIN;
    bool inq = false;
    u64 ctr = ctr_own;
    if (!partner) {
        const char* gate = lane < 32 ? ws + OFF_INF + (uidx(rank ^ peer) * 1024 + PGATE + lane) * 4 : ws + OFF_SR + peer * 4;
        for (;;) {
            if (__all(ld_sys(gate) >= call)) break;
            if (give_up(ws, call, t0)) break;
            __builtin_amdgcn_s_sleep(GSL);
        }
    }
    STAMP(4096 + (cidx * 8 + wid) * 4 + 1);
    qn += qoff;
#define SNAP_ISSUE() do { tflags_issue(sels(partner && !inq, rsf, rtf), tvoff); if (inq) fcnt_issue(hav, tvoff); } while (0)
#define SNAP_APPLY() do { const int n_ = tflags_read(call); \
        int r_; \
        if (!inq) r_ = 16 * n_; \
        else r_ = N_FIN + (partner ? 16 * n_ : (min(fcnt_read(call << SEGSH), NSEG) << SEGSH)); \
        qready = __builtin_amdgcn_readfirstlane(max(qready, r_)); } while (0)
#define TO_QUEUE() do { STAMP(4096 + (cidx * 8 + wid) * 4 + 3); inq = true; qoff = N_FIN; qlim = 2 * N_FIN; ctr = ctr_q; qready = N_FIN; } while (0)
    for (;;) {
        if (qn >= qlim) {
            if (inq) break;
            TO_QUEUE();
            ticket_issue(ctr_q);
            SNAP_ISSUE();
            CE_WAIT(0);
            qn = qoff + ticket_read();
            SNAP_APPLY();
            continue;
        }
        while (qn >= qready) {
            SNAP_ISSUE();
            CE_WAIT(0);
            const int before = qready;
            SNAP_APPLY();
            if (qready != before) continue;
            if (give_up(ws, call, t0)) { qready = qlim; break; }
            __builtin_amdgcn_s_sleep(CSL);
        }
        qready = __builtin_amdgcn_readfirstlane(qready);
        ctr = uni64((const void*)ctr);
        int qc = qn, qp = J_NONE;
        ticket_issue(ctr);
        c.pull_first<RA>(c.desc_pull(qc));
        CE_WAIT(0);
        qn = qoff + ticket_read();
#define CE_STEP(X, Y) { \
            if (!inq && qn >= qlim) {         \
                TO_QUEUE(); ticket_issue(ctr_q); SNAP_ISSUE(); CE_WAIT(0); qn = qoff + ticket_read(); SNAP_APPLY(); \
                ctr = uni64((const void*)ctr); } \
            if (!(qn < qlim && qn < qready)) { \
                c.pull_step<X, Y>(c.desc_pull(qc), c.desc_pull(J_NONE)); \
                CE_WAIT(0); c.flag_pull(c.desc_pull(qp).flag); c.flag_pull(c.desc_pull(qc).flag); c.flush(); \
                break; } \
            const bool snap = qready < qlim && qn + (inq ? 16 : 3) * NCP >= qready; \
            ticket_issue(ctr); \
            if (snap) SNAP_ISSUE(); \
            c.pull_step<X, Y>(c.desc_pull(qc), c.desc_pull(qn)); \
            c.flag_pull(c.desc_pull(qp).flag); \
            qp = qc; qc = qn; \
            qn = qoff + ticket_read(); \
            if (qc < 0 && qn >= 0) STAMP(4096 + (cidx * 8 + 2) * 4); \
            if (snap) SNAP_APPLY(); }
        for (;;) {
            CE_STEP(RA, RB)
            CE_STEP(RB, RA)
        }
#undef CE_STEP
    }
    STAMP(4096 + (cidx * 8 + wid) * 4 + 2);
}

__device__ __forceinline__ void stage_role(KP p, int sidx) {
    const int rank = p.rank;
    char* ws = p.ws[rank];
    const int lane = threadIdx.x & 63, wid = __builtin_amdgcn_readfirstlane(threadIdx.x >> 6);
    const u32x4 sx = mk_srd(uni64(p.x)), sw = mk_srd(uni64(p.w)), sd = mk_srd(uni64(ws));
    const unsigned w_src_off = (lane >> 3) * 8 * 16384 + (lane & 7) * 16, w_dst_off = (lane & 7) * 8 * 1024 + (lane >> 3) * 16;
    for (int t = sidx * 8 + wid; t < N_STG / 2; t += NSTG_WG * 8) {
        const int pc = t & 63, hf = (t >> 6) & 1, panel = EPUSH / 2 + (t >> 7), kb = pc >> 2, nb = pc & 3;
        const unsigned r0 = (unsigned)(hf * 4096 + panel * 256 + (pc >> 1) * 8), kh = pc & 1;
        const int n0 = hf * 4096 + panel * 256 + nb * 64;
        ld8<RA>(lane * 16, sx, r0 * 2048u + kh * 1024u, 2048u);
        ld8<RB>(w_src_off, sw, (unsigned)(kb * 64 * 8192 + n0) * 2u, 16384u);
        CE_WAIT(8);
        st8<RA>(lane * 16, sd, OFF_XO + kh * (8u << 20) + r0 * 1024u, 1024u);
        CE_WAIT(8);
        tr8<RB>();
        st8<RT>(w_dst_off, sd, OFF_WO + (unsigned)(kb >> 3) * (8u << 20) + (unsigned)(n0 * 512 + (kb & 7) * 64) * 2u, 1024u);
    }
    CE_WAIT(0);
    __syncthreads();
    if (wid == 0) {
        unsigned tot = 0;
        if (lane == 0) tot = __hip_atomic_fetch_add((unsigned*)(ws + OFF_QCTR + 26 * 128), 1u, __ATOMIC_RELAXED, __HIP_MEMORY_SCOPE_AGENT) + 1u;
        if (__builtin_amdgcn_readfirstlane(tot) == (unsigned)NSTG_WG) {
            const unsigned call = ld_agent(ws + OFF_SEQ) + 1u;
            if (lane < NR && lane != rank) st_sys(p.ws[lane] + OFF_SR + rank * 4, call);
            STAMP(4094);
        }
    }
}

__device__ __forceinline__ void gemm_role(KP p, int xcd, int slot, unsigned call, LAS unsigned char* lds) {
    const int rank = p.rank;
    char* ws = p.ws[rank];
    const int tid = threadIdx.x, wid = __builtin_amdgcn_readfirstlane(tid >> 6), lane = tid & 63, wr = wid >> 2, wc = wid & 3, fr = lane & 15, fq = lane >> 4;
    const int qd = quad_of(rank), role = role_of(rank);

    if (slot < 16) {
        const int id = (slot * 8 + xcd) * 8 + wid, panel = id >> 6, pc = id & 63, kb = pc >> 2, nb = pc & 3;
        const unsigned w_src_off = (lane >> 3) * 8 * 16384 + (lane & 7) * 16, w_dst_off = (lane & 7) * 8 * 2048 + (lane >> 3) * 16;
        ld8<RA>(w_src_off, mk_srd(uni64(p.w)), (unsigned)(kb * 64 * 8192 + (qd & 1) * 4096 + panel * 256 + nb * 64) * 2u, 16384u);
        CE_WAIT(0);
        tr8<RA>();
        st8<RT>(w_dst_off, mk_srd(uni64(ws)), OFF_WTS + 3 * (8u << 20) + (unsigned)((panel * 256 + nb * 64) * 1024 + kb * 64) * 2u, 2048u);
        CE_WAIT(0);
        if (lane == 0) st_sys(ws + OFF_OWNW + id * 4, call);
    }

    if (slot >= STG_SLOT) stage_role(p, (NSPARE + slot - STG_SLOT) * 8 + xcd);
    unsigned voffA[2], voffB[2];
#pragma unroll
    for (int i = 0; i < 2; ++i) { int R, C; stage_rc(tid * 16 + i * 8192, R, C); const int Rb = (R & ~31) + perm32(R & 31);
        voffA[i] = (unsigned)(R * KS + C) * 2u; voffB[i] = (unsigned)(Rb * KS + C) * 2u; }
    constexpr size_t kstep = (size_t)(BK * 2);
    constexpr size_t hstep = (size_t)HALF * KS * 2;
    const unsigned ldsw = (unsigned)wid * 1024u;
    const int aoff = lds_byte(wr * 64 + fr, fq * 8), boff = lds_byte(wc * 32 + fr, fq * 8);
#define SA(b, h) (((b) * 2 + (h)) * HTB)
#define SB(b, h) ((4 + (b) * 2 + (h)) * HTB)
#define STAGE(bufoff, gbase, voff) do { _Pragma("unroll") for (int _i = 0; _i < 2; ++_i) \
        __builtin_amdgcn_global_load_lds((const unsigned*)((const char*)(gbase) + (voff)[_i]), (LAS unsigned*)(lds + (bufoff) + ldsw + _i * 8192), 16, 0, 0); } while (0)
#define LDA(dst, b, h) do { _Pragma("unroll") for (int m = 0; m < 4; ++m) _Pragma("unroll") for (int k = 0; k < 2; ++k) dst[m][k] = *(const LAS bf16x8*)(lds + SA(b, h) + aoff + m * 2048 + k * 1024); } while (0)
#define LDB(dst, b, h) do { _Pragma("unroll") for (int n = 0; n < 2; ++n) _Pragma("unroll") for (int k = 0; k < 2; ++k) dst[n][k] = *(const LAS bf16x8*)(lds + SB(b, h) + boff + n * 2048 + k * 1024); } while (0)
#define MMA(ai, bj, At, Bt) do { __builtin_amdgcn_s_setprio(1); _Pragma("unroll") for (int m = 0; m < 4; ++m) _Pragma("unroll") for (int n = 0; n < 2; ++n) _Pragma("unroll") for (int k = 0; k < 2; ++k) \
        acc[ai][bj][m][n] = __builtin_amdgcn_mfma_f32_16x16x32_bf16(Bt[n][k], At[m][k], acc[ai][bj][m][n], 0, 0, 0); __builtin_amdgcn_s_setprio(0); } while (0)
#define BAR __builtin_amdgcn_s_barrier()
#define SCHED __builtin_amdgcn_sched_barrier(0)

    int qi = slot;
    while (qi < XQN) {
        const int mode = tab8(p.xm[xcd], qi);
        if (mode == 255) break;
        const int tile = tab8(p.xq[xcd], qi), pm = tile >> 4, pn = tile & 15, pos = tab8(p.pos, tile);
        const bool full = mode == 0;
        const size_t hstepA = full ? hstep : 0;
        const bool mine = ((pm + pn) & 1) == role;
        if (wid == 0) STAMP(tile * 8 + 0);

        auto hook = [&](bool remote) {
            if (wid == 0) {
                const unsigned long long t0 = __builtin_amdgcn_s_memrealtime();
                if (!remote) {
                    for (;;) {
                        const bool ok = ld_sys(ws + OFF_OWNW + (pn * 64 + lane) * 4) >= call;
                        if (__all(ok)) break;
                        if (give_up(ws, call, t0)) break;
                        __builtin_amdgcn_s_sleep(4);
                    }
                } else {
                    const char* fb = ws + OFF_INF + ((lane < 32 ? 2 * pm : 2 * pn + 1) * 32 + (lane & 31)) * 4;
                    for (;;) {
                        const bool ok = ld_sys(fb) >= call;
                        if (__all(ok)) break;
                        if (give_up(ws, call, t0)) break;
                        __builtin_amdgcn_s_sleep(16);
                    }
                    for (;;) {
                        bool ok = true;
#pragma unroll
                        for (int u = 1; u < 6; ++u) ok = ok & (ld_sys(fb + u * 4096) >= call);
                        if (__all(ok)) break;
                        if (give_up(ws, call, t0)) break;
                        __builtin_amdgcn_s_sleep(8);
                    }
                }
                __builtin_amdgcn_fence(__ATOMIC_ACQUIRE, "");
                asm volatile("s_waitcnt vmcnt(0)" ::: "memory");
            }
            asm volatile("" ::: "memory"); __builtin_amdgcn_s_barrier(); asm volatile("" ::: "memory");
        };

        f32x4 acc[2][2][4][2];
#pragma unroll
        for (int a = 0; a < 2; ++a)
#pragma unroll
            for (int b = 0; b < 2; ++b)
#pragma unroll
                for (int m = 0; m < 4; ++m)
#pragma unroll
                    for (int n = 0; n < 2; ++n) acc[a][b][m][n] = (f32x4){0.f, 0.f, 0.f, 0.f};
        bf16x8 At[4][2], B0[2][2], B1[2][2];

        const char* xsA = ws + OFF_XS + (size_t)pm * (512u << 10) + (mode == 2 ? hstep : 0);
        const char* wsB = ws + OFF_WTS + (size_t)pn * (512u << 10);
        const char* cA = p.x + (size_t)((qd >> 1) * 4096 + pm * 256) * 2048 + (mode == 2 ? hstep : 0);
        const char* cB = wsB + (size_t)3 * (8u << 20);
        hook(false);
        if (wid == 0) STAMP(tile * 8 + 1);
        STAGE(SB(0, 0), cB, voffB); STAGE(SA(0, 0), cA, voffA); STAGE(SB(0, 1), cB + hstep, voffB); STAGE(SA(0, 1), cA + hstepA, voffA);
        if (wr == 1) BAR;
        WAIT_V(4); BAR;
        STAGE(SB(1, 0), cB + kstep, voffB); STAGE(SA(1, 0), cA + kstep, voffA); STAGE(SB(1, 1), cB + hstep + kstep, voffB);
        WAIT_V(6); BAR;
        constexpr int nt = KS / BK;
        for (int u = 0; u < 4; ++u) {
            const bool has_next = u + 1 < 4;
            const char* nA = has_next ? xsA + (size_t)u * (8u << 20) : cA;
            const char* nB = has_next ? wsB + (size_t)u * (8u << 20) : cB;
            for (int t = 0; t < nt; t += 2) {
                const bool last = (t == nt - 2);
                const char* a1 = cA + (size_t)(t + 1) * kstep;
                const char* a2 = last ? nA : cA + (size_t)(t + 2) * kstep; const char* b2 = last ? nB : cB + (size_t)(t + 2) * kstep;
                const char* a3 = a2 + kstep; const char* b3 = b2 + kstep;
                if (last && u == 0) {
                    if (wid == 0) STAMP(tile * 8 + 2);
                    hook(true);
                    if (wid == 0) STAMP(tile * 8 + 3);
                }
                LDB(B0, 0, 0); SCHED; LDA(At, 0, 0); STAGE(SA(1, 1), a1 + hstepA, voffA);
                WAIT_L(8); BAR; WAIT_L(0); MMA(0, 0, At, B0); BAR; SCHED;
                LDB(B1, 0, 1); STAGE(SB(0, 0), b2, voffB);
                BAR; WAIT_L(0); MMA(0, 1, At, B1); BAR;
                if (full) LDA(At, 0, 1);
                STAGE(SA(0, 0), a2, voffA);
                BAR; WAIT_L(0); if (full) MMA(1, 0, At, B0); BAR; SCHED;
                STAGE(SB(0, 1), b2 + hstep, voffB);
                WAIT_V(6); BAR; if (full) MMA(1, 1, At, B1); BAR;
                LDB(B0, 1, 0); SCHED; LDA(At, 1, 0); STAGE(SA(0, 1), a2 + hstepA, voffA);
                WAIT_L(8); BAR; WAIT_L(0); MMA(0, 0, At, B0); BAR; SCHED;
                LDB(B1, 1, 1); STAGE(SB(1, 0), b3, voffB);
                BAR; WAIT_L(0); MMA(0, 1, At, B1); BAR;
                if (full) LDA(At, 1, 1);
                STAGE(SA(1, 0), a3, voffA);
                BAR; WAIT_L(0); if (full) MMA(1, 0, At, B0); BAR; SCHED;
                STAGE(SB(1, 1), b3 + hstep, voffB);
                WAIT_V(6); BAR; if (full) MMA(1, 1, At, B1); BAR;
            }
            cA = nA; cB = nB;
        }
        if (wid == 0) STAMP(tile * 8 + 4);
        if (wr == 0) BAR;
        const __amdgpu_buffer_rsrc_t rw = mk_rsrc(ws, WS_NEED);
        const unsigned pbase = (unsigned)pos * (128u << 10) + (unsigned)wid * (16u << 10);
        if (!mine) {
#pragma unroll
            for (int ai = 0; ai < 2; ++ai) {
                if (ai == 0 || full)
#pragma unroll
                for (int m = 0; m < 4; ++m)
#pragma unroll
                    for (int bj = 0; bj < 2; ++bj) {
                        const f32x4 v0 = acc[ai][bj][m][0], v1 = acc[ai][bj][m][1];
                        u32x4 o;
                        o.x = cvt_pk_bf16(v0[0], v0[1]); o.y = cvt_pk_bf16(v0[2], v0[3]);
                        o.z = cvt_pk_bf16(v1[0], v1[1]); o.w = cvt_pk_bf16(v1[2], v1[3]);
                        __builtin_amdgcn_raw_buffer_store_b128(o, rw, lane * 16, OFF_PS + pbase + (mode == 2 ? 8192u : 0u) + ((ai * 4 + m) * 2 + bj) * 1024u, 17);
                    }
            }
        } else {
            if (wid == 0) {
                const unsigned long long t0 = __builtin_amdgcn_s_memrealtime();
                for (;;) {
                    const int fi = lane < 16 ? pos * 16 + lane : max(pos - 4, 0) * 16 + 15;
                    const unsigned long long m = __ballot(ld_sys(ws + OFF_PFLAG + fi * 4) >= call);
                    if ((m & 0xffffull) == 0xffffull) break;
                    if (give_up(ws, call, t0)) break;
                    if (m >> 16) __builtin_amdgcn_s_sleep(PSL_S); else __builtin_amdgcn_s_sleep(PSL_L);
                }
            }
            asm volatile("" ::: "memory"); __builtin_amdgcn_s_barrier(); asm volatile("" ::: "memory");
            if (wid == 0) STAMP(tile * 8 + 5);
            const __amdgpu_buffer_rsrc_t ro = mk_rsrc(p.out[rank], 128u << 20);
            const unsigned base = (unsigned)((qd >> 1) * 4096 + pm * BM) * 16384u + (unsigned)((qd & 1) * 4096 + pn * BM) * 2u;
            const unsigned voff = (unsigned)(wr * 64 + fr) * 16384u + (unsigned)(wc * 32 + 8 * fq) * 2u;
            const unsigned voff2 = (unsigned)(wr * 64 + fr) * 512u + (unsigned)(wc * 32 + 8 * fq) * 2u;
            u32x4 pr[2][4][2];
#pragma unroll
            for (int ai = 0; ai < 2; ++ai)
#pragma unroll
                for (int m = 0; m < 4; ++m)
#pragma unroll
                    for (int bj = 0; bj < 2; ++bj)
                        pr[ai][m][bj] = __builtin_amdgcn_raw_buffer_load_b128(rw, lane * 16, OFF_PR + pbase + ((ai * 4 + m) * 2 + bj) * 1024u, 17);
            __builtin_amdgcn_sched_barrier(0);
#pragma unroll
            for (int ai = 0; ai < 2; ++ai)
#pragma unroll
                for (int m = 0; m < 4; ++m)
#pragma unroll
                    for (int bj = 0; bj < 2; ++bj) {
                        const f32x4 v0 = acc[ai][bj][m][0], v1 = acc[ai][bj][m][1];
                        const u32x4 q = pr[ai][m][bj];
                        u32x4 o;
                        o.x = cvt_pk_bf16(silu(v0[0] + bf_lo(q.x)), silu(v0[1] + bf_hi(q.x))); o.y = cvt_pk_bf16(silu(v0[2] + bf_lo(q.y)), silu(v0[3] + bf_hi(q.y)));
                        o.z = cvt_pk_bf16(silu(v1[0] + bf_lo(q.z)), silu(v1[1] + bf_hi(q.z))); o.w = cvt_pk_bf16(silu(v1[2] + bf_lo(q.w)), silu(v1[3] + bf_hi(q.w)));
                        __builtin_amdgcn_raw_buffer_store_b128(o, rw, voff2, OFF_FS + (unsigned)pos * (128u << 10) + (unsigned)(ai * HALF + m * 16) * 512u + (unsigned)(bj * HALF) * 2u, 17);
                        __builtin_amdgcn_raw_buffer_store_b128(o, ro, voff, base + (unsigned)(ai * HALF + m * 16) * 16384u + (unsigned)(bj * HALF) * 2u, 16);
                    }
        }
        WAIT_V(0);
        BAR;
        if (tid < NR) {
            if (mine) { if (tid != rank) st_sys(p.ws[tid] + OFF_RT + rank * 512 + pos * 4, call); }
            else if (tid == (rank ^ PM)) {
                if (full || (__hip_atomic_fetch_add((unsigned*)(ws + OFF_HCNT + pos * 4), 1u, __ATOMIC_RELAXED, __HIP_MEMORY_SCOPE_AGENT) & 1u))
                    st_sys(p.ws[tid] + OFF_RS + pos * 4, call);
            }
        }
        if (tid == 0) {
            const unsigned t = __hip_atomic_fetch_add((unsigned*)(ws + OFF_QCTR + (8 + xcd) * 128), 1u, __ATOMIC_RELAXED, __HIP_MEMORY_SCOPE_AGENT);
            *(volatile LAS unsigned*)lds = GPX + t;
        }
        if (wid == 0) STAMP(tile * 8 + 6);
        __syncthreads();
        qi = __builtin_amdgcn_readfirstlane(*(volatile LAS unsigned*)lds);
        __syncthreads();
    }
#undef SA
#undef SB
#undef STAGE
#undef LDA
#undef LDB
#undef MMA
#undef BAR
#undef SCHED
}

__global__ __launch_bounds__(512, 2) void fused_gemm_allreduce(const Params p_by_value) {
    KP p = *(const __attribute__((address_space(4))) Params*)__builtin_amdgcn_kernarg_segment_ptr();
    extern __shared__ __attribute__((aligned(16))) unsigned char lds_raw[];
    LAS unsigned char* lds = (LAS unsigned char*)lds_raw;
    const int b = blockIdx.x, xcd = b & 7, slot = b >> 3;
    char* ws = p.ws[p.rank];
    if (slot < 32 - GPX) {
        unsigned xcc;
        asm volatile("s_getreg_b32 %0, hwreg(HW_REG_XCC_ID)" : "=s"(xcc));
        const int xsel = slot == 0 ? xcd : (int)(xcc & 7u);
        if (slot < NCW || slot < NCP) comm_role(p, tab8(p.xlink, xcd), tab8(p.xlink, xsel), slot, slot * 8 + xcd);
        else stage_role(p, (slot - (32 - GPX - NSPARE)) * 8 + xcd);
    }
    unsigned call = 0;
    if (slot >= 32 - GPX || __builtin_amdgcn_readfirstlane(threadIdx.x) < 64) call = __builtin_amdgcn_readfirstlane(ld_agent(ws + OFF_SEQ)) + 1u;
    if (slot >= 32 - GPX) gemm_role(p, xcd, slot - (32 - GPX), call, lds);
    asm volatile("s_waitcnt vmcnt(0)" ::: "memory");
    __syncthreads();
    if (threadIdx.x == 0) {
        const unsigned t = __hip_atomic_fetch_add((unsigned*)(ws + OFF_DONE), 1u, __ATOMIC_RELAXED, __HIP_MEMORY_SCOPE_AGENT);
        if ((t + 1u) % (unsigned)GRID == 0u) {
            for (int l = 0; l < 27; ++l) st_agent(ws + OFF_QCTR + l * 128, 0u);
            st_agent(ws + OFF_SEQ, call);
            if (ld_agent(ws + OFF_TMO) == call) {
                const int t0 = tab8(p.forder[role_of(p.rank)], 0), qd = quad_of(p.rank);
                const size_t off = (size_t)((qd >> 1) * 4096 + (t0 >> 4) * 256) * 16384 + (size_t)((qd & 1) * 4096 + (t0 & 15) * 256) * 2;
                st_agent(p.out[p.rank] + off, 0x7fc07fc0u);
            }
        }
    }
}

}

extern "C" void kernel_launch(int rank, int nranks, ncclComm_t comm,
                              hipStream_t stream,
                              void* const* d_in, const int* in_sizes, int n_in,
                              void* d_out, int out_size,
                              const int* in_dtypes, int out_dtype,
                              void* const* d_out_all,
                              void* const* d_ws_all, size_t ws_bytes) {
    if (nranks != NR || (unsigned)rank >= (unsigned)NR || ws_bytes < WS_NEED || n_in < 2) {
        fprintf(stderr, "unsupported configuration\n");
        return;
    }
    static bool attr_set[NR] = {};
    if (!attr_set[rank]) {
        HIP_CHECK(hipFuncSetAttribute((const void*)fused_gemm_allreduce, hipFuncAttributeMaxDynamicSharedMemorySize, LDS_BYTES));
        attr_set[rank] = true;
    }
    Params p;
    p.x = (const char*)d_in[0];
    p.w = (const char*)d_in[1];
    for (int r = 0; r < NR; ++r) { p.out[r] = (char*)d_out_all[r]; p.ws[r] = (char*)d_ws_all[r]; }
    p.rank = rank;
    {
        static const unsigned char xl[NR][8] = {
            {6, 255, 2, 3, 1, 5, 4, 7}, {7, 255, 4, 5, 0, 3, 2, 6}, {3, 255, 0, 6, 4, 5, 1, 7}, {2, 255, 0, 6, 1, 5, 4, 7},
            {5, 255, 1, 7, 0, 2, 3, 6}, {4, 255, 1, 7, 2, 3, 0, 6}, {0, 255, 2, 3, 4, 7, 1, 5}, {1, 255, 4, 5, 3, 6, 0, 2}};
        for (int i = 0; i < 8; ++i) ((unsigned char*)p.xlink)[i] = xl[rank][i];
    }
    {
        int nf[2] = {0, 0}, nx[8] = {0, 0, 0, 0, 0, 0, 0, 0}, nt[8] = {0, 0, 0, 0, 0, 0, 0, 0};
        for (int g = 0; g < 8; ++g)
            for (int i = 0; i < XQN; ++i) { ((unsigned char*)p.xq[g])[i] = 0; ((unsigned char*)p.xm[g])[i] = 255; }
        for (int e = 0; e < 32; ++e)
            for (int t = 0; t < 256; ++t) {
                const int pm = t >> 4, pn = t & 15, a = 2 * pm, bb = 2 * pn + 1;
                if ((a > bb ? a : bb) != e) continue;
                const int h = (pm + pn) & 1, g = (pm & 3) * 2 + ((pn >> 1) & 1);
                ((unsigned char*)p.forder[h])[nf[h]] = (unsigned char)t;
                ((unsigned char*)p.pos)[t] = (unsigned char)nf[h]++;
                const bool split = h != role_of(rank) && nf[h] <= NHALF && nx[g] + 2 + (32 - nt[g] - 1) <= XQN;
                for (int m = split ? 1 : 0; m <= (split ? 2 : 0); ++m) {
                    ((unsigned char*)p.xq[g])[nx[g]] = (unsigned char)t;
                    ((unsigned char*)p.xm[g])[nx[g]++] = (unsigned char)m;
                }
                ++nt[g];
            }
    }
    hipLaunchKernelGGL(fused_gemm_allreduce, dim3(GRID), dim3(512), LDS_BYTES, stream, p);
    HIP_CHECK(hipGetLastError());
}
```

```cpp
#include <hip/hip_runtime.h>
#include <rccl/rccl.h>
#include <cstdio>
#include <cstdint>

#define HIP_CHECK(expr) do { hipError_t _e = (expr); if (_e != hipSuccess) { \
    fprintf(stderr, "HIP error %s at %s:%d\n", hipGetErrorString(_e), __FILE__, __LINE__); \
    return; } } while (0)

namespace {

#define LAS __attribute__((address_space(3)))
typedef short bf16x8 __attribute__((ext_vector_type(8)));
typedef float f32x4 __attribute__((ext_vector_type(4)));
typedef unsigned u32x4 __attribute__((ext_vector_type(4)));
typedef unsigned long long u64;

constexpr int NR = 8;
#define PM_ 7
constexpr int PM = PM_;
constexpr int KS = 1024;
constexpr int NPOS = 128;
#define NCW 14
#define NCP 12
#define EPUSH 8
#define NHALF 64
#define FDELAY 2
#define GSLACK 6
#define STG_SLOT 4
#define PGATE (N_PUSH - 48)
#define GSL 32
#define PSL_S 24
#define PSL_L 120
#define CSL 24
#define GPX 18
constexpr int GRID = 256;
constexpr int N_IN = 32 * 32;
constexpr int N_FIN = NPOS * 16;
constexpr int N_PUSH = EPUSH * 32;
constexpr int N_PIN = N_IN - N_PUSH;
constexpr int N_STG = 2 * 2 * (16 - EPUSH / 2) * 64;
static_assert(EPUSH % 2 == 0 && EPUSH >= 2 && EPUSH <= 32, "EPUSH");
constexpr int NSPARE = 32 - GPX - (NCW > NCP ? NCW : NCP);
constexpr int NSTG_WG = (NSPARE + GPX - STG_SLOT) * 8;
static_assert(NSTG_WG > 0, "stagers");
constexpr int XQN = 64;
static_assert(256 + 3 * NHALF + 8 * 4 <= 8 * XQN, "job lists");
constexpr int J_NONE = -N_IN - 1;
constexpr int SEGSH = 7;
constexpr int NSEG = N_FIN >> SEGSH;
static_assert(NSEG <= 64, "config");
static_assert(NCW <= 32 - GPX && NCP <= 32 - GPX && GPX >= 16 && GPX <= 32, "too many workgroups");

constexpr unsigned OFF_SEQ = 0, OFF_DONE = 128, OFF_TMO = 256;
constexpr unsigned OFF_INF = 4096;
constexpr unsigned OFF_OWNW = OFF_INF + 6 * 32 * 32 * 4;
constexpr unsigned OFF_RT = OFF_OWNW + 16 * 64 * 4;
constexpr unsigned OFF_RS = OFF_RT + 8 * 512;
constexpr unsigned OFF_PFLAG = OFF_RS + 512;
constexpr unsigned OFF_HAVE = OFF_PFLAG + 128 * 16 * 4;
constexpr unsigned OFF_SR = OFF_HAVE + 6 * 64 * 4;
constexpr unsigned OFF_HCNT = OFF_SR + 128;
constexpr unsigned OFF_QCTR = OFF_HCNT + 512;
constexpr unsigned OFF_DBG = 128 * 1024;
constexpr unsigned OFF_XS = 1u << 20;
constexpr unsigned OFF_WTS = OFF_XS + (24u << 20);
constexpr unsigned OFF_PS = OFF_WTS + (32u << 20);
constexpr unsigned OFF_PR = OFF_PS + (16u << 20);
constexpr unsigned OFF_XO = OFF_PR + (16u << 20);
constexpr unsigned OFF_WO = OFF_XO + (16u << 20);
constexpr unsigned OFF_FS = OFF_WO + (16u << 20);
constexpr unsigned WS_NEED = OFF_FS + (16u << 20);
static_assert(OFF_QCTR + 27 * 128 <= OFF_DBG && WS_NEED <= (256u << 20), "layout");
#define DEADLINE_ 150000000ull
constexpr unsigned long long DEADLINE = DEADLINE_;

constexpr int BM = 256, BK = 64, HALF = 128, HTB = HALF * BK * 2, STAGE_BYTES = 8 * HTB;
constexpr int LDS_BYTES = STAGE_BYTES;

struct Params {
    const char* x;
    const char* w;
    char* out[NR];
    char* ws[NR];
    int rank;
    unsigned forder[2][NPOS / 4];
    unsigned pos[256 / 4];
    unsigned xq[8][XQN / 4];
    unsigned xm[8][XQN / 4];
    unsigned xlink[2];
};
typedef const __attribute__((address_space(4))) Params& KP;

__host__ __device__ __forceinline__ int role_of(int r) { return (r >> 2) & 1; }
__host__ __device__ __forceinline__ int quad_of(int r) { return role_of(r) ? (r ^ PM) : r; }
__host__ __device__ __forceinline__ int uidx(int m) { return m < PM ? m - 1 : m - 2; }

__device__ __forceinline__ int lds_byte(int r, int c) { const int st = (r >> 4) * 2 + (c >> 5), rr = r & 15, cc = c & 31, ob = rr * 64 + cc * 2; return st * 1024 + (ob ^ (((ob >> 9) & 1) << 5)); }
__device__ __forceinline__ void stage_rc(int b, int& R, int& C) { const int st = b / 1024, sb = b % 1024, swz = sb ^ (((sb >> 9) & 1) << 5); R = (st >> 1) * 16 + swz / 64; C = (st & 1) * 32 + (swz % 64) / 2; }
__device__ __forceinline__ int perm32(int rho) { const int n = rho >> 4, i = rho & 15; return 8 * (i >> 2) + 4 * n + (i & 3); }

__device__ __forceinline__ unsigned ld_sys(const char* p) { return __hip_atomic_load((const unsigned*)p, __ATOMIC_RELAXED, __HIP_MEMORY_SCOPE_SYSTEM); }
__device__ __forceinline__ unsigned ld_agent(const char* p) { return __hip_atomic_load((const unsigned*)p, __ATOMIC_RELAXED, __HIP_MEMORY_SCOPE_AGENT); }
__device__ __forceinline__ void st_sys(char* p, unsigned v) { __hip_atomic_store((unsigned*)p, v, __ATOMIC_RELAXED, __HIP_MEMORY_SCOPE_SYSTEM); }
__device__ __forceinline__ void st_agent(char* p, unsigned v) { __hip_atomic_store((unsigned*)p, v, __ATOMIC_RELAXED, __HIP_MEMORY_SCOPE_AGENT); }
__device__ __forceinline__ unsigned cvt_pk_bf16(float lo, float hi) { unsigned r; asm("v_cvt_pk_bf16_f32 %0, %1, %2" : "=v"(r) : "v"(lo), "v"(hi)); return r; }
__device__ __forceinline__ float bf_lo(unsigned u) { return __uint_as_float(u << 16); }
__device__ __forceinline__ float bf_hi(unsigned u) { return __uint_as_float(u & 0xffff0000u); }
__device__ __forceinline__ float silu(float y) {
    const float t = fminf(fmaxf(y, -60.0f), 60.0f);
    const float e = __builtin_amdgcn_exp2f(t * -1.44269504088896f);
    return y * __builtin_amdgcn_rcpf(1.0f + e);
}
__device__ __forceinline__ __amdgpu_buffer_rsrc_t mk_rsrc(const void* p, unsigned bytes) {
    return __builtin_amdgcn_make_buffer_rsrc((void*)p, 0, bytes, 0x00020000);
}
template <class T> __device__ __forceinline__ int tab8(const T& t, int i) { return (t[i >> 2] >> (8 * (i & 3))) & 255; }
#define WAIT_V(n) asm volatile("s_waitcnt vmcnt(" #n ")" ::: "memory")
#define WAIT_L(n) asm volatile("s_waitcnt lgkmcnt(" #n ")" ::: "memory")

#define STAMP(idx) do { } while (0)

__device__ __forceinline__ bool give_up(char* ws, unsigned call, unsigned long long t0) {
    const unsigned tm = __builtin_amdgcn_readfirstlane(ld_agent(ws + OFF_TMO));
    if (tm == call || __builtin_amdgcn_s_memrealtime() - t0 > DEADLINE) {
        if ((threadIdx.x & 63) == 0) st_agent(ws + OFF_TMO, call);
        return true;
    }
    return false;
}

__device__ __forceinline__ u64 uni64(const void* p) {
    const u64 a = (u64)p;
    return ((u64)(unsigned)__builtin_amdgcn_readfirstlane((unsigned)(a >> 32)) << 32) | (unsigned)__builtin_amdgcn_readfirstlane((unsigned)a);
}
__device__ __forceinline__ u32x4 mk_srd(u64 a) {
    u32x4 r;
    r.x = (unsigned)a;
    r.y = (unsigned)(a >> 32) & 0xffffu;
    r.z = 256u << 20;
    r.w = 0x00020000u;
    return r;
}
#define CE_CLOB "memory", "scc", "vcc", "v156", "v157", "v158", "v159", "v160", "v161", "v162", "v163", "v164", "v165", "v166", "v167", "v168", "v169", "v170", "v171", "v172", "v173", "v174", "v175", "v176", "v177", "v178", "v179", "v180", "v181", "v182", "v183", "v184", "v185", "v186", "v187", "v188", "v189", "v190", "v191", "v192", "v193", "v194", "v195", "v196", "v197", "v198", "v199", "v200", "v201", "v202", "v203", "v204", "v205", "v206", "v207", "v208", "v209", "v210", "v211", "v212", "v213", "v214", "v215", "v216", "v217", "v218", "v219", "v220", "v221", "v222", "v223", "v224", "v225", "v226", "v227", "v228", "v229", "v230", "v231", "v232", "v233", "v234", "v235", "v236", "v237", "v238", "v239", "v240", "v241", "v242", "v243", "v244", "v245", "v246", "v247", "v248", "v249", "v250", "v251", "v252", "v253", "v254", "v255"
#define LDF "sc0 sc1"
template <int R> __device__ __forceinline__ void ld8(unsigned voff, const u32x4& srd, unsigned soff, unsigned step) {
    unsigned t;
    asm volatile(
        "s_nop 4\n\t"
        "buffer_load_dwordx4 v[%c5+0:%c5+3], %1, %2, %3 offen " LDF "\n\t"
        "s_add_u32 %0, %3, %4\n\t"
        "buffer_load_dwordx4 v[%c5+4:%c5+7], %1, %2, %0 offen " LDF "\n\t"
        "s_add_u32 %0, %0, %4\n\t"
        "buffer_load_dwordx4 v[%c5+8:%c5+11], %1, %2, %0 offen " LDF "\n\t"
        "s_add_u32 %0, %0, %4\n\t"
        "buffer_load_dwordx4 v[%c5+12:%c5+15], %1, %2, %0 offen " LDF "\n\t"
        "s_add_u32 %0, %0, %4\n\t"
        "buffer_load_dwordx4 v[%c5+16:%c5+19], %1, %2, %0 offen " LDF "\n\t"
        "s_add_u32 %0, %0, %4\n\t"
        "buffer_load_dwordx4 v[%c5+20:%c5+23], %1, %2, %0 offen " LDF "\n\t"
        "s_add_u32 %0, %0, %4\n\t"
        "buffer_load_dwordx4 v[%c5+24:%c5+27], %1, %2, %0 offen " LDF "\n\t"
        "s_add_u32 %0, %0, %4\n\t"
        "buffer_load_dwordx4 v[%c5+28:%c5+31], %1, %2, %0 offen " LDF "\n\t"
        "s_nop 0"
        : "=&s"(t)
        : "v"(voff), "s"(srd), "s"(soff), "s"(step), "i"(R)
        : CE_CLOB);
}
#define SWIN 4
#define SWN(j) ((j) < SWIN ? SWIN + 8 : SWIN - 1)
template <int R> __device__ __forceinline__ void st8(unsigned voff, const u32x4& srd, unsigned soff, unsigned step) {
    unsigned t;
    asm volatile(
        "s_nop 4\n\t"
        "s_waitcnt vmcnt(%c6)\n\t"
        "buffer_store_dwordx4 v[%c5+0:%c5+3], %1, %2, %3 offen sc0 sc1\n\t"
        "s_add_u32 %0, %3, %4\n\t"
        "s_waitcnt vmcnt(%c7)\n\t"
        "buffer_store_dwordx4 v[%c5+4:%c5+7], %1, %2, %0 offen sc0 sc1\n\t"
        "s_add_u32 %0, %0, %4\n\t"
        "s_waitcnt vmcnt(%c8)\n\t"
        "buffer_store_dwordx4 v[%c5+8:%c5+11], %1, %2, %0 offen sc0 sc1\n\t"
        "s_add_u32 %0, %0, %4\n\t"
        "s_waitcnt vmcnt(%c9)\n\t"
        "buffer_store_dwordx4 v[%c5+12:%c5+15], %1, %2, %0 offen sc0 sc1\n\t"
        "s_add_u32 %0, %0, %4\n\t"
        "s_waitcnt vmcnt(%c10)\n\t"
        "buffer_store_dwordx4 v[%c5+16:%c5+19], %1, %2, %0 offen sc0 sc1\n\t"
        "s_add_u32 %0, %0, %4\n\t"
        "s_waitcnt vmcnt(%c11)\n\t"
        "buffer_store_dwordx4 v[%c5+20:%c5+23], %1, %2, %0 offen sc0 sc1\n\t"
        "s_add_u32 %0, %0, %4\n\t"
        "s_waitcnt vmcnt(%c12)\n\t"
        "buffer_store_dwordx4 v[%c5+24:%c5+27], %1, %2, %0 offen sc0 sc1\n\t"
        "s_add_u32 %0, %0, %4\n\t"
        "s_waitcnt vmcnt(%c13)\n\t"
        "buffer_store_dwordx4 v[%c5+28:%c5+31], %1, %2, %0 offen sc0 sc1\n\t"
        "s_nop 1"
        : "=&s"(t)
        : "v"(voff), "s"(srd), "s"(soff), "s"(step), "i"(R), "i"(SWN(0)), "i"(SWN(1)), "i"(SWN(2)), "i"(SWN(3)), "i"(SWN(4)), "i"(SWN(5)), "i"(SWN(6)), "i"(SWN(7))
        : CE_CLOB);
}
template <int R> __device__ __forceinline__ void tr8() {
    asm volatile(
        "v_perm_b32 v224, v[%c2+4], v[%c2+0], %0\n\t"
        "v_perm_b32 v225, v[%c2+12], v[%c2+8], %0\n\t"
        "v_perm_b32 v226, v[%c2+20], v[%c2+16], %0\n\t"
        "v_perm_b32 v227, v[%c2+28], v[%c2+24], %0\n\t"
        "v_perm_b32 v228, v[%c2+4], v[%c2+0], %1\n\t"
        "v_perm_b32 v229, v[%c2+12], v[%c2+8], %1\n\t"
        "v_perm_b32 v230, v[%c2+20], v[%c2+16], %1\n\t"
        "v_perm_b32 v231, v[%c2+28], v[%c2+24], %1\n\t"
        "v_perm_b32 v232, v[%c2+5], v[%c2+1], %0\n\t"
        "v_perm_b32 v233, v[%c2+13], v[%c2+9], %0\n\t"
        "v_perm_b32 v234, v[%c2+21], v[%c2+17], %0\n\t"
        "v_perm_b32 v235, v[%c2+29], v[%c2+25], %0\n\t"
        "v_perm_b32 v236, v[%c2+5], v[%c2+1], %1\n\t"
        "v_perm_b32 v237, v[%c2+13], v[%c2+9], %1\n\t"
        "v_perm_b32 v238, v[%c2+21], v[%c2+17], %1\n\t"
        "v_perm_b32 v239, v[%c2+29], v[%c2+25], %1\n\t"
        "v_perm_b32 v240, v[%c2+6], v[%c2+2], %0\n\t"
        "v_perm_b32 v241, v[%c2+14], v[%c2+10], %0\n\t"
        "v_perm_b32 v242, v[%c2+22], v[%c2+18], %0\n\t"
        "v_perm_b32 v243, v[%c2+30], v[%c2+26], %0\n\t"
        "v_perm_b32 v244, v[%c2+6], v[%c2+2], %1\n\t"
        "v_perm_b32 v245, v[%c2+14], v[%c2+10], %1\n\t"
        "v_perm_b32 v246, v[%c2+22], v[%c2+18], %1\n\t"
        "v_perm_b32 v247, v[%c2+30], v[%c2+26], %1\n\t"
        "v_perm_b32 v248, v[%c2+7], v[%c2+3], %0\n\t"
        "v_perm_b32 v249, v[%c2+15], v[%c2+11], %0\n\t"
        "v_perm_b32 v250, v[%c2+23], v[%c2+19], %0\n\t"
        "v_perm_b32 v251, v[%c2+31], v[%c2+27], %0\n\t"
        "v_perm_b32 v252, v[%c2+7], v[%c2+3], %1\n\t"
        "v_perm_b32 v253, v[%c2+15], v[%c2+11], %1\n\t"
        "v_perm_b32 v254, v[%c2+23], v[%c2+19], %1\n\t"
        "v_perm_b32 v255, v[%c2+31], v[%c2+27], %1\n\t"
        "s_nop 1"
        :: "s"(0x05040100u), "s"(0x07060302u), "i"(R)
        : CE_CLOB);
}
template <int R> __device__ __forceinline__ void ld4(unsigned voff, const u32x4& srd, unsigned soff, unsigned step) {
    asm volatile(
        "s_nop 4\n\t"
        "buffer_load_dwordx4 v[%c4+0:%c4+3], %1, %2, %0 offen sc0 sc1\n\t"
        "s_add_u32 %0, %0, %3\n\t"
        "buffer_load_dwordx4 v[%c4+4:%c4+7], %1, %2, %0 offen sc0 sc1\n\t"
        "s_add_u32 %0, %0, %3\n\t"
        "buffer_load_dwordx4 v[%c4+8:%c4+11], %1, %2, %0 offen sc0 sc1\n\t"
        "s_add_u32 %0, %0, %3\n\t"
        "buffer_load_dwordx4 v[%c4+12:%c4+15], %1, %2, %0 offen sc0 sc1\n\t"
        "s_nop 0"
        : "+s"(soff)
        : "v"(voff), "s"(srd), "s"(step), "i"(R)
        : CE_CLOB);
}
template <int X, int Y> __device__ __forceinline__ void xfer(unsigned dst, unsigned src4, unsigned nsrc, unsigned voff, const u32x4& dsrd, unsigned step,
                                                         const u32x4& ssrd, unsigned nvoff, const u32x4& nsrd, unsigned nstep, unsigned svoff, unsigned sstep) {
    asm volatile(
        "s_nop 4\n\t"
        "s_waitcnt vmcnt(6)\n\t"
        "buffer_store_dwordx4 v[%c12+0:%c12+3], %3, %4, %0 offen sc0 sc1\n\t"
        "s_add_u32 %0, %0, %5\n\t"
        "buffer_load_dwordx4 v[%c12+16:%c12+19], %10, %6, %1 offen sc0 sc1\n\t"
        "s_add_u32 %1, %1, %11\n\t"
        "s_waitcnt vmcnt(6)\n\t"
        "buffer_store_dwordx4 v[%c12+4:%c12+7], %3, %4, %0 offen sc0 sc1\n\t"
        "s_add_u32 %0, %0, %5\n\t"
        "buffer_load_dwordx4 v[%c12+20:%c12+23], %10, %6, %1 offen sc0 sc1\n\t"
        "s_add_u32 %1, %1, %11\n\t"
        "s_waitcnt vmcnt(6)\n\t"
        "buffer_store_dwordx4 v[%c12+8:%c12+11], %3, %4, %0 offen sc0 sc1\n\t"
        "s_add_u32 %0, %0, %5\n\t"
        "buffer_load_dwordx4 v[%c12+24:%c12+27], %10, %6, %1 offen sc0 sc1\n\t"
        "s_add_u32 %1, %1, %11\n\t"
        "s_waitcnt vmcnt(6)\n\t"
        "buffer_store_dwordx4 v[%c12+12:%c12+15], %3, %4, %0 offen sc0 sc1\n\t"
        "s_add_u32 %0, %0, %5\n\t"
        "buffer_load_dwordx4 v[%c12+28:%c12+31], %10, %6, %1 offen sc0 sc1\n\t"
        "s_waitcnt vmcnt(6)\n\t"
        "buffer_store_dwordx4 v[%c12+16:%c12+19], %3, %4, %0 offen sc0 sc1\n\t"
        "s_add_u32 %0, %0, %5\n\t"
        "buffer_load_dwordx4 v[%c13+0:%c13+3], %7, %8, %2 offen sc0 sc1\n\t"
        "s_add_u32 %2, %2, %9\n\t"
        "s_waitcnt vmcnt(6)\n\t"
        "buffer_store_dwordx4 v[%c12+20:%c12+23], %3, %4, %0 offen sc0 sc1\n\t"
        "s_add_u32 %0, %0, %5\n\t"
        "buffer_load_dwordx4 v[%c13+4:%c13+7], %7, %8, %2 offen sc0 sc1\n\t"
        "s_add_u32 %2, %2, %9\n\t"
        "s_waitcnt vmcnt(6)\n\t"
        "buffer_store_dwordx4 v[%c12+24:%c12+27], %3, %4, %0 offen sc0 sc1\n\t"
        "s_add_u32 %0, %0, %5\n\t"
        "buffer_load_dwordx4 v[%c13+8:%c13+11], %7, %8, %2 offen sc0 sc1\n\t"
        "s_add_u32 %2, %2, %9\n\t"
        "s_waitcnt vmcnt(6)\n\t"
        "buffer_store_dwordx4 v[%c12+28:%c12+31], %3, %4, %0 offen sc0 sc1\n\t"
        "buffer_load_dwordx4 v[%c13+12:%c13+15], %7, %8, %2 offen sc0 sc1\n\t"
        "s_nop 1"
        : "+s"(dst), "+s"(src4), "+s"(nsrc)
        : "v"(voff), "s"(dsrd), "s"(step), "s"(ssrd), "v"(nvoff), "s"(nsrd), "s"(nstep), "v"(svoff), "s"(sstep), "i"(X), "i"(Y)
        : CE_CLOB);
}
__device__ __forceinline__ void ticket_issue(u64 ctr) {
    u64 sv;
    asm volatile(
        "s_mov_b64 %0, exec\n\t"
        "s_mov_b64 exec, 1\n\t"
        "v_mov_b32 v158, 0\n\t"
        "v_mov_b32 v159, 1\n\t"
        "s_nop 4\n\t"
        "global_atomic_add v159, v158, v159, %1 sc0\n\t"
        "s_mov_b64 exec, %0"
        : "=&s"(sv) : "s"(ctr) : CE_CLOB);
}
__device__ __forceinline__ int ticket_read() {
    int q;
    asm volatile("s_nop 1\n\tv_readfirstlane_b32 %0, v159\n\ts_nop 4" : "=s"(q) :: CE_CLOB);
    return q;
}
__device__ __forceinline__ void tflags_issue(u64 base, unsigned voff) {
    asm volatile(
        "s_nop 4\n\t"
        "global_load_dword v157, %0, %1 sc0 sc1\n\t"
        "global_load_dword v158, %0, %1 offset:256 sc0 sc1"
        :: "v"(voff), "s"(base) : CE_CLOB);
}
__device__ __forceinline__ int tflags_read(unsigned call) {
    u64 m0, m1;
    asm volatile("s_nop 1\n\tv_cmp_le_u32 %0, %2, v157\n\tv_cmp_le_u32 %1, %2, v158\n\ts_nop 4" : "=&s"(m0), "=&s"(m1) : "s"(call) : CE_CLOB);
    return (~m0 != 0ull) ? __builtin_ctzll(~m0) : 64 + ((~m1 != 0ull) ? __builtin_ctzll(~m1) : 64);
}
__device__ __forceinline__ void fcnt_issue(u64 base, unsigned voff) {
    asm volatile("s_nop 4\n\tglobal_load_dword v156, %0, %1 sc0 sc1" :: "v"(voff), "s"(base) : CE_CLOB);
}
__device__ __forceinline__ int fcnt_read(unsigned need) {
    u64 m0;
    asm volatile("s_nop 1\n\tv_cmp_le_u32 %0, %1, v156\n\ts_nop 4" : "=&s"(m0) : "s"(need) : CE_CLOB);
    return (~m0 != 0ull) ? __builtin_ctzll(~m0) : 64;
}
#define CE_WAIT(n) asm volatile("s_waitcnt vmcnt(" #n ")" ::: CE_CLOB)
constexpr int RA = 160, RB = 192, RT = 224;

__device__ __forceinline__ unsigned selv(bool c0, bool c1, unsigned a, unsigned b, unsigned c) {
    asm("" : "+v"(a)); asm("" : "+v"(b)); asm("" : "+v"(c));
    return c0 ? a : (c1 ? b : c);
}
__device__ __forceinline__ u64 sels(bool c, u64 a, u64 b) {
    asm("" : "+s"(a)); asm("" : "+s"(b));
    return uni64((const void*)(c ? a : b));
}

struct Piece { int type; unsigned src, dst, flag; };

struct Comm {
    KP p;
    int rank, peer, lane;
    unsigned call;
    bool partner;
    u64 rx, rw, ro_me, rws_me, rws_peer, ro_peer;
    unsigned lin_off, w_src_off, w_dst_off, o_off;
    char* wsp;
    char* wsh;
    char* wsm;

    __device__ __forceinline__ Comm(KP p_, int peer_, int lane_, unsigned call_) : p(p_), rank(p_.rank), peer(peer_), lane(lane_), call(call_) {
        partner = (rank ^ peer) == PM;
        rx = uni64(p.x);
        rw = uni64(p.w);
        ro_me = uni64(p.out[rank]);
        wsm = p.ws[rank];
        rws_me = uni64(wsm);
        wsp = p.ws[peer];
        wsh = p.ws[peer ^ PM];
        rws_peer = uni64(wsp);
        ro_peer = uni64(p.out[peer]);
        lin_off = lane * 16;
        w_src_off = (lane >> 3) * 8 * 16384 + (lane & 7) * 16;
        w_dst_off = (lane & 7) * 8 * 2048 + (lane >> 3) * 16;
        o_off = (lane >> 5) * 16384 + (lane & 31) * 16;
    }
    __device__ __forceinline__ unsigned out_off(int a, int o, int po) const {
        const int t = tab8(p.forder[role_of(a)], o), pm = t >> 4, pn = t & 15, qd = quad_of(a);
        return (unsigned)((qd >> 1) * 4096 + pm * 256 + po * 16) * 16384u + (unsigned)((qd & 1) * 4096 + pn * 256) * 2u;
    }
    __device__ __forceinline__ Piece desc_in(int q) const {
        Piece d;
        q = __builtin_amdgcn_readfirstlane(q);
        const int u = uidx(rank ^ peer);
        const int e = q >> 5, pc = q & 31, panel = e >> 1, hd = role_of(peer), qd = quad_of(peer);
        d.flag = OFF_INF + (u * 1024 + q) * 4;
        if (!(e & 1)) {
            d.type = 0;
            d.src = (unsigned)((qd >> 1) * 4096 + panel * 256 + pc * 8) * 2048u + hd * 1024u;
            d.dst = OFF_XS + (u >> 1) * (8u << 20) + (unsigned)(panel * 256 + pc * 8) * 2048u + (u & 1) * 1024u;
        } else {
            const int kb = pc >> 2, nb = pc & 3;
            d.type = 1;
            d.src = (unsigned)((hd * 512 + kb * 64) * 8192 + (qd & 1) * 4096 + panel * 256 + nb * 64) * 2u;
            d.dst = OFF_WTS + (u >> 1) * (8u << 20) + (unsigned)((panel * 256 + nb * 64) * 1024 + (u & 1) * 512 + kb * 64) * 2u;
        }
        d.type = __builtin_amdgcn_readfirstlane(d.type);
        d.src = __builtin_amdgcn_readfirstlane(d.src);
        d.dst = __builtin_amdgcn_readfirstlane(d.dst);
        d.flag = __builtin_amdgcn_readfirstlane(d.flag);
        return d;
    }
    template <int R> __device__ __forceinline__ void load(const Piece& d) const {
        ld8<R>(selv(d.type == 1, false, w_src_off, lin_off, lin_off), mk_srd(sels(d.type == 0, rx, rw)), d.src, d.type == 0 ? 2048u : 16384u);
    }
    template <int R> __device__ __forceinline__ void store(const Piece& d) const {
        if (d.type == 1) {
            tr8<R>();
            st8<RT>(w_dst_off, mk_srd(rws_peer), d.dst, 2048u);
        } else {
            st8<R>(lin_off, mk_srd(rws_peer), d.dst, 2048u);
        }
    }
    __device__ __forceinline__ void flag_in(int q) const {
        if (q >= 0 && lane == 0) st_sys(wsp + OFF_INF + (uidx(rank ^ peer) * 1024 + q) * 4, call);
    }
    __device__ __forceinline__ Piece desc_pull(int j) const {
        Piece d;
        j = __builtin_amdgcn_readfirstlane(j);
        if (j < -N_PIN) {
            d.type = 4; d.src = d.dst = OFF_PS; d.flag = 0;
        } else if (j < 0) {
            const int q = j + N_IN, e = q >> 5, pc = q & 31, panel = e >> 1, u = uidx(rank ^ peer), qd = quad_of(rank);
            const unsigned row = (unsigned)(panel * 256 + pc * 8) * 2048u;
            d.type = 5;
            d.src = ((e & 1) ? OFF_WO + (qd & 1) * (4u << 20) : OFF_XO + (qd >> 1) * (4u << 20)) + row / 2 + role_of(rank) * (8u << 20);
            d.dst = ((e & 1) ? OFF_WTS : OFF_XS) + (u >> 1) * (8u << 20) + row + (u & 1) * 1024u;
            d.flag = OFF_INF + (u * 1024 + q) * 4;
        } else if (j >= N_FIN) {
            const int qq = j - N_FIN;
            d.type = partner ? 6 : 2;
            d.dst = out_off(rank ^ PM, qq >> 4, qq & 15); d.flag = 0;
            d.src = partner ? OFF_FS + qq * 8192u : d.dst;
        } else if (partner) {
            d.type = 3;
            d.src = OFF_PS + j * 8192u; d.dst = OFF_PR + j * 8192u; d.flag = OFF_PFLAG + j * 4;
        } else {
            d.type = 6;
            d.dst = out_off(peer, j >> 4, j & 15);
            d.src = OFF_FS + j * 8192u;
            d.flag = (OFF_HAVE + (uidx(rank ^ peer ^ PM) * 64 + (j >> SEGSH)) * 4) | 1u;
        }
        d.type = __builtin_amdgcn_readfirstlane(d.type);
        d.src = __builtin_amdgcn_readfirstlane(d.src);
        d.dst = __builtin_amdgcn_readfirstlane(d.dst);
        d.flag = __builtin_amdgcn_readfirstlane(d.flag);
        return d;
    }
    __device__ __forceinline__ unsigned pvoff(const Piece& d) const { return selv(d.type == 2, false, o_off, lin_off, lin_off); }
    __device__ __forceinline__ unsigned pstep(const Piece& d) const { return d.type == 2 ? 32768u : 1024u; }
    __device__ __forceinline__ unsigned dvoff(const Piece& d) const { return selv(d.type == 2 || d.type == 6, false, o_off, lin_off, lin_off); }
    __device__ __forceinline__ unsigned dstep(const Piece& d) const { return (d.type == 2 || d.type == 6) ? 32768u : (d.type == 5 ? 2048u : 1024u); }
    __device__ __forceinline__ u32x4 psrc(const Piece& d) const { return mk_srd(sels(d.type == 2, ro_peer, sels(d.type != 4, rws_peer, rws_me))); }
    __device__ __forceinline__ u32x4 pdst(const Piece& d) const { return mk_srd(sels(d.type == 2 || d.type == 6, ro_me, rws_me)); }
    template <int R> __device__ __forceinline__ void pull_first(const Piece& d) const { ld4<R>(pvoff(d), psrc(d), d.src, pstep(d)); }
    template <int X, int Y> __device__ __forceinline__ void pull_step(const Piece& c, const Piece& n) const {
        xfer<X, Y>(c.dst, c.src + 4 * pstep(c), n.src, dvoff(c), pdst(c), dstep(c), psrc(c), pvoff(n), psrc(n), pstep(n), pvoff(c), pstep(c));
    }
    unsigned seg_off = 0, seg_cnt = 0;
    __device__ __forceinline__ void flush() {
        if (seg_cnt != 0 && lane == 0) __hip_atomic_fetch_add((unsigned*)(wsh + seg_off), seg_cnt, __ATOMIC_RELAXED, __HIP_MEMORY_SCOPE_SYSTEM);
        seg_cnt = 0;
    }
    __device__ __forceinline__ void flag_pull(unsigned off) {
        if (off & 1u) {
            if ((off & ~1u) != seg_off) { flush(); seg_off = off & ~1u; }
            ++seg_cnt;
        } else if (off != 0) { if (lane == 0) st_sys(wsm + off, call); }
        else flush();
    }
};

__device__ __forceinline__ void comm_role(KP p, int peer_push, int peer_pull, int slot, int cidx) {
    const int rank = p.rank;
    char* ws = p.ws[rank];
    const int lane = threadIdx.x & 63, wid = __builtin_amdgcn_readfirstlane(threadIdx.x >> 6);
    const int peer = wid == 0 ? peer_push : peer_pull;
    if ((unsigned)peer >= (unsigned)NR || peer == rank) return;
    if (wid >= 2 || (wid == 0 && ((rank ^ peer) == PM || slot >= NCW)) || (wid == 1 && slot >= NCP)) return;
    const u64 ctr_own = uni64(ws + OFF_QCTR + (wid * 17 + peer) * 128);
    ticket_issue(ctr_own);
    Comm c(p, peer, lane, 0u);
    if (wid == 0) c.load<RA>(c.desc_in(slot));
    const unsigned call = __builtin_amdgcn_readfirstlane(ld_agent(ws + OFF_SEQ)) + 1u;
    c.call = call;
    STAMP(4096 + (cidx * 8 + wid) * 4 + 0);
    const bool partner = c.partner;
    CE_WAIT(0);
    int qn = ticket_read();

    if (wid == 0) {
        int qc = slot, qp = -1;
        qn += NCW;
        bool first = true;
#define CE_STEP(X, Y) { \
            if (qn >= N_PUSH) { CE_WAIT(0); c.flag_in(qp); c.store<X>(c.desc_in(qc)); CE_WAIT(0); c.flag_in(qc); break; } \
            ticket_issue(ctr_own); \
            c.load<Y>(c.desc_in(qn)); \
            if (first) CE_WAIT(9); else CE_WAIT(17); \
            first = false; \
            c.store<X>(c.desc_in(qc)); \
            CE_WAIT(16); \
            c.flag_in(qp); \
            qp = qc; qc = qn; \
            qn = NCW + ticket_read(); }
        for (;;) {
            CE_STEP(RA, RB)
            CE_STEP(RB, RA)
        }
#undef CE_STEP
        STAMP(4096 + (cidx * 8 + wid) * 4 + 1);
        return;
    }

    const u64 ctr_q = uni64(ws + OFF_QCTR + 16 * 128);
    const u64 rtf = uni64(ws + OFF_RT + peer * 512), rsf = uni64(ws + OFF_RS);
    const u64 hav = uni64(ws + OFF_HAVE + (partner ? 0 : uidx(rank ^ peer)) * 256);
    const unsigned tvoff = lane * 4;
    const unsigned long long t0 = __builtin_amdgcn_s_memrealtime();
    int qoff = partner ? 0 : -N_PIN;
    int qready = 0, qlim = N_FIN;
    bool inq = false;
    u64 ctr = ctr_own;
    if (!partner) {
        const char* gate = lane < 32 ? ws + OFF_INF + (uidx(rank ^ peer) * 1024 + PGATE + lane) * 4 : ws + OFF_SR + peer * 4;
        for (;;) {
            if (__all(ld_sys(gate) >= call)) break;
            if (give_up(ws, call, t0)) break;
            __builtin_amdgcn_s_sleep(GSL);
        }
    }
    STAMP(4096 + (cidx * 8 + wid) * 4 + 1);
    qn += qoff;
#define SNAP_ISSUE() do { tflags_issue(sels(partner && !inq, rsf, rtf), tvoff); if (inq) fcnt_issue(hav, tvoff); } while (0)
#define SNAP_APPLY() do { const int n_ = tflags_read(call); \
        int r_; \
        if (!inq) r_ = 16 * n_; \
        else r_ = N_FIN + (partner ? 16 * n_ : (min(fcnt_read(call << SEGSH), NSEG) << SEGSH)); \
        qready = __builtin_amdgcn_readfirstlane(max(qready, r_)); } while (0)
#define TO_QUEUE() do { STAMP(4096 + (cidx * 8 + wid) * 4 + 3); inq = true; qoff = N_FIN; qlim = 2 * N_FIN; ctr = ctr_q; qready = N_FIN; } while (0)
    for (;;) {
        if (qn >= qlim) {
            if (inq) break;
            TO_QUEUE();
            ticket_issue(ctr_q);
            SNAP_ISSUE();
            CE_WAIT(0);
            qn = qoff + ticket_read();
            SNAP_APPLY();
            continue;
        }
        while (qn >= qready) {
            SNAP_ISSUE();
            CE_WAIT(0);
            const int before = qready;
            SNAP_APPLY();
            if (qready != before) continue;
            if (give_up(ws, call, t0)) { qready = qlim; break; }
            __builtin_amdgcn_s_sleep(CSL);
        }
        qready = __builtin_amdgcn_readfirstlane(qready);
        ctr = uni64((const void*)ctr);
        int qc = qn, qp = J_NONE;
        ticket_issue(ctr);
        c.pull_first<RA>(c.desc_pull(qc));
        CE_WAIT(0);
        qn = qoff + ticket_read();
#define CE_STEP(X, Y) { \
            if (!inq && qn >= qlim) {         \
                TO_QUEUE(); ticket_issue(ctr_q); SNAP_ISSUE(); CE_WAIT(0); qn = qoff + ticket_read(); SNAP_APPLY(); \
                ctr = uni64((const void*)ctr); } \
            if (!(qn < qlim && qn < qready)) { \
                c.pull_step<X, Y>(c.desc_pull(qc), c.desc_pull(J_NONE)); \
                CE_WAIT(0); c.flag_pull(c.desc_pull(qp).flag); c.flag_pull(c.desc_pull(qc).flag); c.flush(); \
                break; } \
            const bool snap = qready < qlim && qn + (inq ? 16 : 3) * NCP >= qready; \
            ticket_issue(ctr); \
            if (snap) SNAP_ISSUE(); \
            c.pull_step<X, Y>(c.desc_pull(qc), c.desc_pull(qn)); \
            c.flag_pull(c.desc_pull(qp).flag); \
            qp = qc; qc = qn; \
            qn = qoff + ticket_read(); \
            if (qc < 0 && qn >= 0) STAMP(4096 + (cidx * 8 + 2) * 4); \
            if (snap) SNAP_APPLY(); }
        for (;;) {
            CE_STEP(RA, RB)
            CE_STEP(RB, RA)
        }
#undef CE_STEP
    }
    STAMP(4096 + (cidx * 8 + wid) * 4 + 2);
}

__device__ __forceinline__ void stage_role(KP p, int sidx) {
    const int rank = p.rank;
    char* ws = p.ws[rank];
    const int lane = threadIdx.x & 63, wid = __builtin_amdgcn_readfirstlane(threadIdx.x >> 6);
    const u32x4 sx = mk_srd(uni64(p.x)), sw = mk_srd(uni64(p.w)), sd = mk_srd(uni64(ws));
    const unsigned w_src_off = (lane >> 3) * 8 * 16384 + (lane & 7) * 16, w_dst_off = (lane & 7) * 8 * 1024 + (lane >> 3) * 16;
    for (int t = sidx * 8 + wid; t < N_STG / 2; t += NSTG_WG * 8) {
        const int pc = t & 63, hf = (t >> 6) & 1, panel = EPUSH / 2 + (t >> 7), kb = pc >> 2, nb = pc & 3;
        const unsigned r0 = (unsigned)(hf * 4096 + panel * 256 + (pc >> 1) * 8), kh = pc & 1;
        const int n0 = hf * 4096 + panel * 256 + nb * 64;
        ld8<RA>(lane * 16, sx, r0 * 2048u + kh * 1024u, 2048u);
        ld8<RB>(w_src_off, sw, (unsigned)(kb * 64 * 8192 + n0) * 2u, 16384u);
        CE_WAIT(8);
        st8<RA>(lane * 16, sd, OFF_XO + kh * (8u << 20) + r0 * 1024u, 1024u);
        CE_WAIT(8);
        tr8<RB>();
        st8<RT>(w_dst_off, sd, OFF_WO + (unsigned)(kb >> 3) * (8u << 20) + (unsigned)(n0 * 512 + (kb & 7) * 64) * 2u, 1024u);
    }
    CE_WAIT(0);
    __syncthreads();
    if (wid == 0) {
        unsigned tot = 0;
        if (lane == 0) tot = __hip_atomic_fetch_add((unsigned*)(ws + OFF_QCTR + 26 * 128), 1u, __ATOMIC_RELAXED, __HIP_MEMORY_SCOPE_AGENT) + 1u;
        if (__builtin_amdgcn_readfirstlane(tot) == (unsigned)NSTG_WG) {
            const unsigned call = ld_agent(ws + OFF_SEQ) + 1u;
            if (lane < NR && lane != rank) st_sys(p.ws[lane] + OFF_SR + rank * 4, call);
            STAMP(4094);
        }
    }
}

__device__ __forceinline__ void gemm_role(KP p, int xcd, int slot, unsigned call, LAS unsigned char* lds) {
    const int rank = p.rank;
    char* ws = p.ws[rank];
    const int tid = threadIdx.x, wid = __builtin_amdgcn_readfirstlane(tid >> 6), lane = tid & 63, wr = wid >> 2, wc = wid & 3, fr = lane & 15, fq = lane >> 4;
    const int qd = quad_of(rank), role = role_of(rank);

    if (slot < 16) {
        const int id = (slot * 8 + xcd) * 8 + wid, panel = id >> 6, pc = id & 63, kb = pc >> 2, nb = pc & 3;
        const unsigned w_src_off = (lane >> 3) * 8 * 16384 + (lane & 7) * 16, w_dst_off = (lane & 7) * 8 * 2048 + (lane >> 3) * 16;
        ld8<RA>(w_src_off, mk_srd(uni64(p.w)), (unsigned)(kb * 64 * 8192 + (qd & 1) * 4096 + panel * 256 + nb * 64) * 2u, 16384u);
        CE_WAIT(0);
        tr8<RA>();
        st8<RT>(w_dst_off, mk_srd(uni64(ws)), OFF_WTS + 3 * (8u << 20) + (unsigned)((panel * 256 + nb * 64) * 1024 + kb * 64) * 2u, 2048u);
        CE_WAIT(0);
        if (lane == 0) st_sys(ws + OFF_OWNW + id * 4, call);
    }

    if (slot >= STG_SLOT) stage_role(p, (NSPARE + slot - STG_SLOT) * 8 + xcd);
    unsigned voffA[2], voffB[2];
#pragma unroll
    for (int i = 0; i < 2; ++i) { int R, C; stage_rc(tid * 16 + i * 8192, R, C); const int Rb = (R & ~31) + perm32(R & 31);
        voffA[i] = (unsigned)(R * KS + C) * 2u; voffB[i] = (unsigned)(Rb * KS + C) * 2u; }
    constexpr size_t kstep = (size_t)(BK * 2);
    constexpr size_t hstep = (size_t)HALF * KS * 2;
    const unsigned ldsw = (unsigned)wid * 1024u;
    const int aoff = lds_byte(wr * 64 + fr, fq * 8), boff = lds_byte(wc * 32 + fr, fq * 8);
#define SA(b, h) (((b) * 2 + (h)) * HTB)
#define SB(b, h) ((4 + (b) * 2 + (h)) * HTB)
#define STAGE(bufoff, gbase, voff) do { _Pragma("unroll") for (int _i = 0; _i < 2; ++_i) \
        __builtin_amdgcn_global_load_lds((const unsigned*)((const char*)(gbase) + (voff)[_i]), (LAS unsigned*)(lds + (bufoff) + ldsw + _i * 8192), 16, 0, 0); } while (0)
#define LDA(dst, b, h) do { _Pragma("unroll") for (int m = 0; m < 4; ++m) _Pragma("unroll") for (int k = 0; k < 2; ++k) dst[m][k] = *(const LAS bf16x8*)(lds + SA(b, h) + aoff + m * 2048 + k * 1024); } while (0)
#define LDB(dst, b, h) do { _Pragma("unroll") for (int n = 0; n < 2; ++n) _Pragma("unroll") for (int k = 0; k < 2; ++k) dst[n][k] = *(const LAS bf16x8*)(lds + SB(b, h) + boff + n * 2048 + k * 1024); } while (0)
#define MMA(ai, bj, At, Bt) do { __builtin_amdgcn_s_setprio(1); _Pragma("unroll") for (int m = 0; m < 4; ++m) _Pragma("unroll") for (int n = 0; n < 2; ++n) _Pragma("unroll") for (int k = 0; k < 2; ++k) \
        acc[ai][bj][m][n] = __builtin_amdgcn_mfma_f32_16x16x32_bf16(Bt[n][k], At[m][k], acc[ai][bj][m][n], 0, 0, 0); __builtin_amdgcn_s_setprio(0); } while (0)
#define BAR __builtin_amdgcn_s_barrier()
#define SCHED __builtin_amdgcn_sched_barrier(0)

    int qi = slot;
    while (qi < XQN) {
        const int mode = tab8(p.xm[xcd], qi);
        if (mode == 255) break;
        const int tile = tab8(p.xq[xcd], qi), pm = tile >> 4, pn = tile & 15, pos = tab8(p.pos, tile);
        const bool full = mode == 0;
        const int qa = (mode - 1) >> 1, qb = (mode - 1) & 1;
        const bool mine = ((pm + pn) & 1) == role;
        if (wid == 0) STAMP(tile * 8 + 0);

        auto hook = [&](bool remote) {
            if (wid == 0) {
                const unsigned long long t0 = __builtin_amdgcn_s_memrealtime();
                if (!remote) {
                    for (;;) {
                        const bool ok = ld_sys(ws + OFF_OWNW + (pn * 64 + lane) * 4) >= call;
                        if (__all(ok)) break;
                        if (give_up(ws, call, t0)) break;
                        __builtin_amdgcn_s_sleep(4);
                    }
                } else {
                    const char* fb = ws + OFF_INF + ((lane < 32 ? 2 * pm : 2 * pn + 1) * 32 + (lane & 31)) * 4;
                    for (;;) {
                        const bool ok = ld_sys(fb) >= call;
                        if (__all(ok)) break;
                        if (give_up(ws, call, t0)) break;
                        __builtin_amdgcn_s_sleep(16);
                    }
                    for (;;) {
                        bool ok = true;
#pragma unroll
                        for (int u = 1; u < 6; ++u) ok = ok & (ld_sys(fb + u * 4096) >= call);
                        if (__all(ok)) break;
                        if (give_up(ws, call, t0)) break;
                        __builtin_amdgcn_s_sleep(8);
                    }
                }
                __builtin_amdgcn_fence(__ATOMIC_ACQUIRE, "");
                asm volatile("s_waitcnt vmcnt(0)" ::: "memory");
            }
            asm volatile("" ::: "memory"); __builtin_amdgcn_s_barrier(); asm volatile("" ::: "memory");
        };

        f32x4 acc[2][2][4][2];
#pragma unroll
        for (int a = 0; a < 2; ++a)
#pragma unroll
            for (int b = 0; b < 2; ++b)
#pragma unroll
                for (int m = 0; m < 4; ++m)
#pragma unroll
                    for (int n = 0; n < 2; ++n) acc[a][b][m][n] = (f32x4){0.f, 0.f, 0.f, 0.f};
        bf16x8 At[4][2], B0[2][2], B1[2][2];

        const char* xsA = ws + OFF_XS + (size_t)pm * (512u << 10);
        const char* wsB = ws + OFF_WTS + (size_t)pn * (512u << 10);
        const char* cA = p.x + (size_t)((qd >> 1) * 4096 + pm * 256) * 2048;
        const char* cB = wsB + (size_t)3 * (8u << 20);
        hook(false);
        if (wid == 0) STAMP(tile * 8 + 1);
        if (!full) {
            const char* qA = cA + (size_t)qa * hstep; const char* qB = cB + (size_t)qb * hstep;
            const char* rA = xsA + (size_t)qa * hstep; const char* rB = wsB + (size_t)qb * hstep;
            auto issue = [&](int t, int sl) {
                t = min(t, 63);
                const int u = t >> 4;
                const char* a = (u == 0 ? qA : rA + (size_t)(u - 1) * (8u << 20)) + (size_t)(t & 15) * kstep;
                const char* b = (u == 0 ? qB : rB + (size_t)(u - 1) * (8u << 20)) + (size_t)(t & 15) * kstep;
                STAGE(sl * HTB, a, voffA); STAGE((4 + sl) * HTB, b, voffB);
            };
            issue(0, 0); issue(1, 1); issue(2, 2);
            for (int t0 = 0; t0 < 64; t0 += 4) {
#pragma unroll
                for (int sl = 0; sl < 4; ++sl) {
                    if (t0 + sl == 13) {
                        if (wid == 0) STAMP(tile * 8 + 2);
                        hook(true);
                        if (wid == 0) STAMP(tile * 8 + 3);
                    }
                    WAIT_V(8); BAR;
                    issue(t0 + sl + 3, (sl + 3) & 3);
#pragma unroll
                    for (int n = 0; n < 2; ++n)
#pragma unroll
                        for (int k = 0; k < 2; ++k) B0[n][k] = *(const LAS bf16x8*)(lds + (4 + sl) * HTB + boff + n * 2048 + k * 1024);
#pragma unroll
                    for (int m = 0; m < 4; ++m)
#pragma unroll
                        for (int k = 0; k < 2; ++k) At[m][k] = *(const LAS bf16x8*)(lds + sl * HTB + aoff + m * 2048 + k * 1024);
                    WAIT_L(0); SCHED;
                    MMA(0, 0, At, B0);
                    SCHED;
                }
            }
        } else {
        STAGE(SB(0, 0), cB, voffB); STAGE(SA(0, 0), cA, voffA); STAGE(SB(0, 1), cB + hstep, voffB); STAGE(SA(0, 1), cA + hstep, voffA);
        if (wr == 1) BAR;
        WAIT_V(4); BAR;
        STAGE(SB(1, 0), cB + kstep, voffB); STAGE(SA(1, 0), cA + kstep, voffA); STAGE(SB(1, 1), cB + hstep + kstep, voffB);
        WAIT_V(6); BAR;
        constexpr int nt = KS / BK;
        for (int u = 0; u < 4; ++u) {
            const bool has_next = u + 1 < 4;
            const char* nA = has_next ? xsA + (size_t)u * (8u << 20) : cA;
            const char* nB = has_next ? wsB + (size_t)u * (8u << 20) : cB;
            for (int t = 0; t < nt; t += 2) {
                const bool last = (t == nt - 2);
                const char* a1 = cA + (size_t)(t + 1) * kstep;
                const char* a2 = last ? nA : cA + (size_t)(t + 2) * kstep; const char* b2 = last ? nB : cB + (size_t)(t + 2) * kstep;
                const char* a3 = a2 + kstep; const char* b3 = b2 + kstep;
                if (last && u == 0) {
                    if (wid == 0) STAMP(tile * 8 + 2);
                    hook(true);
                    if (wid == 0) STAMP(tile * 8 + 3);
                }
                LDB(B0, 0, 0); SCHED; LDA(At, 0, 0); STAGE(SA(1, 1), a1 + hstep, voffA);
                WAIT_L(8); BAR; WAIT_L(0); MMA(0, 0, At, B0); BAR; SCHED;
                LDB(B1, 0, 1); STAGE(SB(0, 0), b2, voffB);
                BAR; WAIT_L(0); MMA(0, 1, At, B1); BAR;
                LDA(At, 0, 1); STAGE(SA(0, 0), a2, voffA);
                BAR; WAIT_L(0); MMA(1, 0, At, B0); BAR; SCHED;
                STAGE(SB(0, 1), b2 + hstep, voffB);
                WAIT_V(6); BAR; MMA(1, 1, At, B1); BAR;
                LDB(B0, 1, 0); SCHED; LDA(At, 1, 0); STAGE(SA(0, 1), a2 + hstep, voffA);
                WAIT_L(8); BAR; WAIT_L(0); MMA(0, 0, At, B0); BAR; SCHED;
                LDB(B1, 1, 1); STAGE(SB(1, 0), b3, voffB);
                BAR; WAIT_L(0); MMA(0, 1, At, B1); BAR;
                LDA(At, 1, 1); STAGE(SA(1, 0), a3, voffA);
                BAR; WAIT_L(0); MMA(1, 0, At, B0); BAR; SCHED;
                STAGE(SB(1, 1), b3 + hstep, voffB);
                WAIT_V(6); BAR; MMA(1, 1, At, B1); BAR;
            }
            cA = nA; cB = nB;
        }
        if (wr == 0) BAR;
        }
        if (wid == 0) STAMP(tile * 8 + 4);
        const __amdgpu_buffer_rsrc_t rw = mk_rsrc(ws, WS_NEED);
        const unsigned pbase = (unsigned)pos * (128u << 10) + (unsigned)wid * (16u << 10);
        if (!mine) {
#pragma unroll
            for (int ai = 0; ai < 2; ++ai)
#pragma unroll
                for (int m = 0; m < 4; ++m)
#pragma unroll
                    for (int bj = 0; bj < 2; ++bj)
                        if (full || (ai == 0 && bj == 0)) {
                            const f32x4 v0 = acc[ai][bj][m][0], v1 = acc[ai][bj][m][1];
                            u32x4 o;
                            o.x = cvt_pk_bf16(v0[0], v0[1]); o.y = cvt_pk_bf16(v0[2], v0[3]);
                            o.z = cvt_pk_bf16(v1[0], v1[1]); o.w = cvt_pk_bf16(v1[2], v1[3]);
                            __builtin_amdgcn_raw_buffer_store_b128(o, rw, lane * 16, OFF_PS + pbase + (full ? 0u : qa * 8192u + qb * 1024u) + ((ai * 4 + m) * 2 + bj) * 1024u, 17);
                        }
        } else {
            if (wid == 0) {
                const unsigned long long t0 = __builtin_amdgcn_s_memrealtime();
                for (;;) {
                    const int fi = lane < 16 ? pos * 16 + lane : max(pos - 4, 0) * 16 + 15;
                    const unsigned long long m = __ballot(ld_sys(ws + OFF_PFLAG + fi * 4) >= call);
                    if ((m & 0xffffull) == 0xffffull) break;
                    if (give_up(ws, call, t0)) break;
                    if (m >> 16) __builtin_amdgcn_s_sleep(PSL_S); else __builtin_amdgcn_s_sleep(PSL_L);
                }
            }
            asm volatile("" ::: "memory"); __builtin_amdgcn_s_barrier(); asm volatile("" ::: "memory");
            if (wid == 0) STAMP(tile * 8 + 5);
            const __amdgpu_buffer_rsrc_t ro = mk_rsrc(p.out[rank], 128u << 20);
            const unsigned base = (unsigned)((qd >> 1) * 4096 + pm * BM) * 16384u + (unsigned)((qd & 1) * 4096 + pn * BM) * 2u;
            const unsigned voff = (unsigned)(wr * 64 + fr) * 16384u + (unsigned)(wc * 32 + 8 * fq) * 2u;
            const unsigned voff2 = (unsigned)(wr * 64 + fr) * 512u + (unsigned)(wc * 32 + 8 * fq) * 2u;
            u32x4 pr[2][4][2];
#pragma unroll
            for (int ai = 0; ai < 2; ++ai)
#pragma unroll
                for (int m = 0; m < 4; ++m)
#pragma unroll
                    for (int bj = 0; bj < 2; ++bj)
                        pr[ai][m][bj] = __builtin_amdgcn_raw_buffer_load_b128(rw, lane * 16, OFF_PR + pbase + ((ai * 4 + m) * 2 + bj) * 1024u, 17);
            __builtin_amdgcn_sched_barrier(0);
#pragma unroll
            for (int ai = 0; ai < 2; ++ai)
#pragma unroll
                for (int m = 0; m < 4; ++m)
#pragma unroll
                    for (int bj = 0; bj < 2; ++bj) {
                        const f32x4 v0 = acc[ai][bj][m][0], v1 = acc[ai][bj][m][1];
                        const u32x4 q = pr[ai][m][bj];
                        u32x4 o;
                        o.x = cvt_pk_bf16(silu(v0[0] + bf_lo(q.x)), silu(v0[1] + bf_hi(q.x))); o.y = cvt_pk_bf16(silu(v0[2] + bf_lo(q.y)), silu(v0[3] + bf_hi(q.y)));
                        o.z = cvt_pk_bf16(silu(v1[0] + bf_lo(q.z)), silu(v1[1] + bf_hi(q.z))); o.w = cvt_pk_bf16(silu(v1[2] + bf_lo(q.w)), silu(v1[3] + bf_hi(q.w)));
                        __builtin_amdgcn_raw_buffer_store_b128(o, rw, voff2, OFF_FS + (unsigned)pos * (128u << 10) + (unsigned)(ai * HALF + m * 16) * 512u + (unsigned)(bj * HALF) * 2u, 17);
                        __builtin_amdgcn_raw_buffer_store_b128(o, ro, voff, base + (unsigned)(ai * HALF + m * 16) * 16384u + (unsigned)(bj * HALF) * 2u, 16);
                    }
        }
        WAIT_V(0);
        BAR;
        if (tid < NR) {
            if (mine) { if (tid != rank) st_sys(p.ws[tid] + OFF_RT + rank * 512 + pos * 4, call); }
            else if (tid == (rank ^ PM)) {
                if (full || (__hip_atomic_fetch_add((unsigned*)(ws + OFF_HCNT + pos * 4), 1u, __ATOMIC_RELAXED, __HIP_MEMORY_SCOPE_AGENT) & 3u) == 3u)
                    st_sys(p.ws[tid] + OFF_RS + pos * 4, call);
            }
        }
        if (tid == 0) {
            const unsigned t = __hip_atomic_fetch_add((unsigned*)(ws + OFF_QCTR + (8 + xcd) * 128), 1u, __ATOMIC_RELAXED, __HIP_MEMORY_SCOPE_AGENT);
            *(volatile LAS unsigned*)lds = GPX + t;
        }
        if (wid == 0) STAMP(tile * 8 + 6);
        __syncthreads();
        qi = __builtin_amdgcn_readfirstlane(*(volatile LAS unsigned*)lds);
        __syncthreads();
    }
#undef SA
#undef SB
#undef STAGE
#undef LDA
#undef LDB
#undef MMA
#undef BAR
#undef SCHED
}

__global__ __launch_bounds__(512, 2) void fused_gemm_allreduce(const Params p_by_value) {
    KP p = *(const __attribute__((address_space(4))) Params*)__builtin_amdgcn_kernarg_segment_ptr();
    extern __shared__ __attribute__((aligned(16))) unsigned char lds_raw[];
    LAS unsigned char* lds = (LAS unsigned char*)lds_raw;
    const int b = blockIdx.x, xcd = b & 7, slot = b >> 3;
    char* ws = p.ws[p.rank];
    if (slot < 32 - GPX) {
        unsigned xcc;
        asm volatile("s_getreg_b32 %0, hwreg(HW_REG_XCC_ID)" : "=s"(xcc));
        const int xsel = slot == 0 ? xcd : (int)(xcc & 7u);
        if (slot < NCW || slot < NCP) comm_role(p, tab8(p.xlink, xcd), tab8(p.xlink, xsel), slot, slot * 8 + xcd);
        else stage_role(p, (slot - (32 - GPX - NSPARE)) * 8 + xcd);
    }
    unsigned call = 0;
    if (slot >= 32 - GPX || __builtin_amdgcn_readfirstlane(threadIdx.x) < 64) call = __builtin_amdgcn_readfirstlane(ld_agent(ws + OFF_SEQ)) + 1u;
    if (slot >= 32 - GPX) gemm_role(p, xcd, slot - (32 - GPX), call, lds);
    asm volatile("s_waitcnt vmcnt(0)" ::: "memory");
    __syncthreads();
    if (threadIdx.x == 0) {
        const unsigned t = __hip_atomic_fetch_add((unsigned*)(ws + OFF_DONE), 1u, __ATOMIC_RELAXED, __HIP_MEMORY_SCOPE_AGENT);
        if ((t + 1u) % (unsigned)GRID == 0u) {
            for (int l = 0; l < 27; ++l) st_agent(ws + OFF_QCTR + l * 128, 0u);
            st_agent(ws + OFF_SEQ, call);
            if (ld_agent(ws + OFF_TMO) == call) {
                const int t0 = tab8(p.forder[role_of(p.rank)], 0), qd = quad_of(p.rank);
                const size_t off = (size_t)((qd >> 1) * 4096 + (t0 >> 4) * 256) * 16384 + (size_t)((qd & 1) * 4096 + (t0 & 15) * 256) * 2;
                st_agent(p.out[p.rank] + off, 0x7fc07fc0u);
            }
        }
    }
}

}

extern "C" void kernel_launch(int rank, int nranks, ncclComm_t comm,
                              hipStream_t stream,
                              void* const* d_in, const int* in_sizes, int n_in,
                              void* d_out, int out_size,
                              const int* in_dtypes, int out_dtype,
                              void* const* d_out_all,
                              void* const* d_ws_all, size_t ws_bytes) {
    if (nranks != NR || (unsigned)rank >= (unsigned)NR || ws_bytes < WS_NEED || n_in < 2) {
        fprintf(stderr, "unsupported configuration\n");
        return;
    }
    static bool attr_set[NR] = {};
    if (!attr_set[rank]) {
        HIP_CHECK(hipFuncSetAttribute((const void*)fused_gemm_allreduce, hipFuncAttributeMaxDynamicSharedMemorySize, LDS_BYTES));
        attr_set[rank] = true;
    }
    Params p;
    p.x = (const char*)d_in[0];
    p.w = (const char*)d_in[1];
    for (int r = 0; r < NR; ++r) { p.out[r] = (char*)d_out_all[r]; p.ws[r] = (char*)d_ws_all[r]; }
    p.rank = rank;
    {
        static const unsigned char xl[NR][8] = {
            {6, 255, 2, 3, 1, 5, 4, 7}, {7, 255, 4, 5, 0, 3, 2, 6}, {3, 255, 0, 6, 4, 5, 1, 7}, {2, 255, 0, 6, 1, 5, 4, 7},
            {5, 255, 1, 7, 0, 2, 3, 6}, {4, 255, 1, 7, 2, 3, 0, 6}, {0, 255, 2, 3, 4, 7, 1, 5}, {1, 255, 4, 5, 3, 6, 0, 2}};
        for (int i = 0; i < 8; ++i) ((unsigned char*)p.xlink)[i] = xl[rank][i];
    }
    {
        int nf[2] = {0, 0}, nx[8] = {0, 0, 0, 0, 0, 0, 0, 0}, load[8] = {0, 0, 0, 0, 0, 0, 0, 0};
        for (int g = 0; g < 8; ++g)
            for (int i = 0; i < XQN; ++i) { ((unsigned char*)p.xq[g])[i] = 0; ((unsigned char*)p.xm[g])[i] = 255; }
        for (int e = 0; e < 32; ++e)
            for (int t = 0; t < 256; ++t) {
                const int pm = t >> 4, pn = t & 15, a = 2 * pm, bb = 2 * pn + 1;
                if ((a > bb ? a : bb) != e) continue;
                const int h = (pm + pn) & 1;
                ((unsigned char*)p.forder[h])[nf[h]] = (unsigned char)t;
                ((unsigned char*)p.pos)[t] = (unsigned char)nf[h]++;
            }
        for (int e = 0; e < 32 + FDELAY; ++e)
            for (int fin = 0; fin < 2; ++fin)
                for (int t = 0; t < 256; ++t) {
                    const int pm = t >> 4, pn = t & 15, a = 2 * pm, bb = 2 * pn + 1;
                    const bool mine = ((pm + pn) & 1) == role_of(rank);
                    if (mine != (fin == 1) || (a > bb ? a : bb) + (mine ? FDELAY : 0) != e) continue;
                    const bool split = !mine && ((unsigned char*)p.pos)[t] < NHALF;
                    const int nj = split ? 4 : 1;
                    int g = (pm & 3) * 2 + ((pn >> 1) & 1), gm = -1;
                    for (int i = 0; i < 8; ++i)
                        if (nx[i] + nj <= XQN && (gm < 0 || load[i] < load[gm])) gm = i;
                    if (nx[g] + nj > XQN || load[g] > load[gm] + GSLACK) g = gm;
                    for (int m = split ? 1 : 0; m <= (split ? 4 : 0); ++m) {
                        ((unsigned char*)p.xq[g])[nx[g]] = (unsigned char)t;
                        ((unsigned char*)p.xm[g])[nx[g]++] = (unsigned char)m;
                    }
                    load[g] += split ? 5 : 3;
                }
    }
    hipLaunchKernelGGL(fused_gemm_allreduce, dim3(GRID), dim3(512), LDS_BYTES, stream, p);
    HIP_CHECK(hipGetLastError());
}
```

```cpp
#include <hip/hip_runtime.h>
#include <rccl/rccl.h>
#include <cstdio>
#include <cstdint>

#define HIP_CHECK(expr) do { hipError_t _e = (expr); if (_e != hipSuccess) { \
    fprintf(stderr, "HIP error %s at %s:%d\n", hipGetErrorString(_e), __FILE__, __LINE__); \
    return; } } while (0)

namespace {

#define LAS __attribute__((address_space(3)))
typedef short bf16x8 __attribute__((ext_vector_type(8)));
typedef float f32x4 __attribute__((ext_vector_type(4)));
typedef unsigned u32x4 __attribute__((ext_vector_type(4)));
typedef unsigned long long u64;

constexpr int NR = 8;
#define PM_ 7
constexpr int PM = PM_;
constexpr int KS = 1024;
constexpr int NPOS = 128;
#define NCW 14
#define NCP 12
#define EPUSH 6
#define NHALF 64
#define FDELAY 2
#define GSLACK 6
#define STG_SLOT 4
#define PGATE (N_PUSH - 48)
#define GSL 32
#define PSL_S 24
#define PSL_L 120
#define CSL 24
#define GPX 18
constexpr int GRID = 256;
constexpr int N_IN = 32 * 32;
constexpr int N_FIN = NPOS * 16;
constexpr int N_PUSH = EPUSH * 32;
constexpr int N_PIN = N_IN - N_PUSH;
constexpr int N_STG = 2 * 2 * (16 - EPUSH / 2) * 64;
static_assert(EPUSH % 2 == 0 && EPUSH >= 2 && EPUSH <= 32, "EPUSH");
constexpr int NSPARE = 32 - GPX - (NCW > NCP ? NCW : NCP);
constexpr int NSTG_WG = (NSPARE + GPX - STG_SLOT) * 8;
static_assert(NSTG_WG > 0, "stagers");
constexpr int XQN = 64;
static_assert(256 + 3 * NHALF + 8 * 4 <= 8 * XQN, "job lists");
constexpr int J_MS1 = (EPUSH / 2 + NSTG_WG * 8 / 128 >= 16) ? 0 : (EPUSH / 2 + NSTG_WG * 8 / 128) * 64 - N_IN;
constexpr int J_NONE = -N_IN - 1;
constexpr int SEGSH = 7;
constexpr int NSEG = N_FIN >> SEGSH;
static_assert(NSEG <= 64, "config");
static_assert(NCW <= 32 - GPX && NCP <= 32 - GPX && GPX >= 16 && GPX <= 32, "too many workgroups");

constexpr unsigned OFF_SEQ = 0, OFF_DONE = 128, OFF_TMO = 256;
constexpr unsigned OFF_INF = 4096;
constexpr unsigned OFF_OWNW = OFF_INF + 6 * 32 * 32 * 4;
constexpr unsigned OFF_RT = OFF_OWNW + 16 * 64 * 4;
constexpr unsigned OFF_RS = OFF_RT + 8 * 512;
constexpr unsigned OFF_PFLAG = OFF_RS + 512;
constexpr unsigned OFF_HAVE = OFF_PFLAG + 128 * 16 * 4;
constexpr unsigned OFF_SR = OFF_HAVE + 6 * 64 * 4;
constexpr unsigned OFF_HCNT = OFF_SR + 128;
constexpr unsigned OFF_QCTR = OFF_HCNT + 512;
constexpr unsigned OFF_DBG = 128 * 1024;
constexpr unsigned OFF_XS = 1u << 20;
constexpr unsigned OFF_WTS = OFF_XS + (24u << 20);
constexpr unsigned OFF_PS = OFF_WTS + (32u << 20);
constexpr unsigned OFF_PR = OFF_PS + (16u << 20);
constexpr unsigned OFF_XO = OFF_PR + (16u << 20);
constexpr unsigned OFF_WO = OFF_XO + (16u << 20);
constexpr unsigned OFF_FS = OFF_WO + (16u << 20);
constexpr unsigned WS_NEED = OFF_FS + (16u << 20);
static_assert(OFF_QCTR + 27 * 128 <= OFF_DBG && WS_NEED <= (256u << 20), "layout");
#define DEADLINE_ 150000000ull
constexpr unsigned long long DEADLINE = DEADLINE_;

constexpr int BM = 256, BK = 64, HALF = 128, HTB = HALF * BK * 2, STAGE_BYTES = 8 * HTB;
constexpr int LDS_BYTES = STAGE_BYTES;

struct Params {
    const char* x;
    const char* w;
    char* out[NR];
    char* ws[NR];
    int rank;
    unsigned forder[2][NPOS / 4];
    unsigned pos[256 / 4];
    unsigned xq[8][XQN / 4];
    unsigned xm[8][XQN / 4];
    unsigned xlink[2];
};
typedef const __attribute__((address_space(4))) Params& KP;

__host__ __device__ __forceinline__ int role_of(int r) { return (r >> 2) & 1; }
__host__ __device__ __forceinline__ int quad_of(int r) { return role_of(r) ? (r ^ PM) : r; }
__host__ __device__ __forceinline__ int uidx(int m) { return m < PM ? m - 1 : m - 2; }

__device__ __forceinline__ int lds_byte(int r, int c) { const int st = (r >> 4) * 2 + (c >> 5), rr = r & 15, cc = c & 31, ob = rr * 64 + cc * 2; return st * 1024 + (ob ^ (((ob >> 9) & 1) << 5)); }
__device__ __forceinline__ void stage_rc(int b, int& R, int& C) { const int st = b / 1024, sb = b % 1024, swz = sb ^ (((sb >> 9) & 1) << 5); R = (st >> 1) * 16 + swz / 64; C = (st & 1) * 32 + (swz % 64) / 2; }
__device__ __forceinline__ int perm32(int rho) { const int n = rho >> 4, i = rho & 15; return 8 * (i >> 2) + 4 * n + (i & 3); }

__device__ __forceinline__ unsigned ld_sys(const char* p) { return __hip_atomic_load((const unsigned*)p, __ATOMIC_RELAXED, __HIP_MEMORY_SCOPE_SYSTEM); }
__device__ __forceinline__ unsigned ld_agent(const char* p) { return __hip_atomic_load((const unsigned*)p, __ATOMIC_RELAXED, __HIP_MEMORY_SCOPE_AGENT); }
__device__ __forceinline__ void st_sys(char* p, unsigned v) { __hip_atomic_store((unsigned*)p, v, __ATOMIC_RELAXED, __HIP_MEMORY_SCOPE_SYSTEM); }
__device__ __forceinline__ void st_agent(char* p, unsigned v) { __hip_atomic_store((unsigned*)p, v, __ATOMIC_RELAXED, __HIP_MEMORY_SCOPE_AGENT); }
__device__ __forceinline__ unsigned cvt_pk_bf16(float lo, float hi) { unsigned r; asm("v_cvt_pk_bf16_f32 %0, %1, %2" : "=v"(r) : "v"(lo), "v"(hi)); return r; }
__device__ __forceinline__ float bf_lo(unsigned u) { return __uint_as_float(u << 16); }
__device__ __forceinline__ float bf_hi(unsigned u) { return __uint_as_float(u & 0xffff0000u); }
__device__ __forceinline__ float silu(float y) {
    const float t = fminf(fmaxf(y, -60.0f), 60.0f);
    const float e = __builtin_amdgcn_exp2f(t * -1.44269504088896f);
    return y * __builtin_amdgcn_rcpf(1.0f + e);
}
__device__ __forceinline__ __amdgpu_buffer_rsrc_t mk_rsrc(const void* p, unsigned bytes) {
    return __builtin_amdgcn_make_buffer_rsrc((void*)p, 0, bytes, 0x00020000);
}
template <class T> __device__ __forceinline__ int tab8(const T& t, int i) { return (t[i >> 2] >> (8 * (i & 3))) & 255; }
#define WAIT_V(n) asm volatile("s_waitcnt vmcnt(" #n ")" ::: "memory")
#define WAIT_L(n) asm volatile("s_waitcnt lgkmcnt(" #n ")" ::: "memory")

#define STAMP(idx) do { } while (0)

__device__ __forceinline__ bool give_up(char* ws, unsigned call, unsigned long long t0) {
    const unsigned tm = __builtin_amdgcn_readfirstlane(ld_agent(ws + OFF_TMO));
    if (tm == call || __builtin_amdgcn_s_memrealtime() - t0 > DEADLINE) {
        if ((threadIdx.x & 63) == 0) st_agent(ws + OFF_TMO, call);
        return true;
    }
    return false;
}

__device__ __forceinline__ u64 uni64(const void* p) {
    const u64 a = (u64)p;
    return ((u64)(unsigned)__builtin_amdgcn_readfirstlane((unsigned)(a >> 32)) << 32) | (unsigned)__builtin_amdgcn_readfirstlane((unsigned)a);
}
__device__ __forceinline__ u32x4 mk_srd(u64 a) {
    u32x4 r;
    r.x = (unsigned)a;
    r.y = (unsigned)(a >> 32) & 0xffffu;
    r.z = 256u << 20;
    r.w = 0x00020000u;
    return r;
}
#define CE_CLOB "memory", "scc", "vcc", "v156", "v157", "v158", "v159", "v160", "v161", "v162", "v163", "v164", "v165", "v166", "v167", "v168", "v169", "v170", "v171", "v172", "v173", "v174", "v175", "v176", "v177", "v178", "v179", "v180", "v181", "v182", "v183", "v184", "v185", "v186", "v187", "v188", "v189", "v190", "v191", "v192", "v193", "v194", "v195", "v196", "v197", "v198", "v199", "v200", "v201", "v202", "v203", "v204", "v205", "v206", "v207", "v208", "v209", "v210", "v211", "v212", "v213", "v214", "v215", "v216", "v217", "v218", "v219", "v220", "v221", "v222", "v223", "v224", "v225", "v226", "v227", "v228", "v229", "v230", "v231", "v232", "v233", "v234", "v235", "v236", "v237", "v238", "v239", "v240", "v241", "v242", "v243", "v244", "v245", "v246", "v247", "v248", "v249", "v250", "v251", "v252", "v253", "v254", "v255"
#define LDF "sc0 sc1"
template <int R> __device__ __forceinline__ void ld8(unsigned voff, const u32x4& srd, unsigned soff, unsigned step) {
    unsigned t;
    asm volatile(
        "s_nop 4\n\t"
        "buffer_load_dwordx4 v[%c5+0:%c5+3], %1, %2, %3 offen " LDF "\n\t"
        "s_add_u32 %0, %3, %4\n\t"
        "buffer_load_dwordx4 v[%c5+4:%c5+7], %1, %2, %0 offen " LDF "\n\t"
        "s_add_u32 %0, %0, %4\n\t"
        "buffer_load_dwordx4 v[%c5+8:%c5+11], %1, %2, %0 offen " LDF "\n\t"
        "s_add_u32 %0, %0, %4\n\t"
        "buffer_load_dwordx4 v[%c5+12:%c5+15], %1, %2, %0 offen " LDF "\n\t"
        "s_add_u32 %0, %0, %4\n\t"
        "buffer_load_dwordx4 v[%c5+16:%c5+19], %1, %2, %0 offen " LDF "\n\t"
        "s_add_u32 %0, %0, %4\n\t"
        "buffer_load_dwordx4 v[%c5+20:%c5+23], %1, %2, %0 offen " LDF "\n\t"
        "s_add_u32 %0, %0, %4\n\t"
        "buffer_load_dwordx4 v[%c5+24:%c5+27], %1, %2, %0 offen " LDF "\n\t"
        "s_add_u32 %0, %0, %4\n\t"
        "buffer_load_dwordx4 v[%c5+28:%c5+31], %1, %2, %0 offen " LDF "\n\t"
        "s_nop 0"
        : "=&s"(t)
        : "v"(voff), "s"(srd), "s"(soff), "s"(step), "i"(R)
        : CE_CLOB);
}
#define SWIN 4
#define SWN(j) ((j) < SWIN ? SWIN + 8 : SWIN - 1)
template <int R> __device__ __forceinline__ void st8(unsigned voff, const u32x4& srd, unsigned soff, unsigned step) {
    unsigned t;
    asm volatile(
        "s_nop 4\n\t"
        "s_waitcnt vmcnt(%c6)\n\t"
        "buffer_store_dwordx4 v[%c5+0:%c5+3], %1, %2, %3 offen sc0 sc1\n\t"
        "s_add_u32 %0, %3, %4\n\t"
        "s_waitcnt vmcnt(%c7)\n\t"
        "buffer_store_dwordx4 v[%c5+4:%c5+7], %1, %2, %0 offen sc0 sc1\n\t"
        "s_add_u32 %0, %0, %4\n\t"
        "s_waitcnt vmcnt(%c8)\n\t"
        "buffer_store_dwordx4 v[%c5+8:%c5+11], %1, %2, %0 offen sc0 sc1\n\t"
        "s_add_u32 %0, %0, %4\n\t"
        "s_waitcnt vmcnt(%c9)\n\t"
        "buffer_store_dwordx4 v[%c5+12:%c5+15], %1, %2, %0 offen sc0 sc1\n\t"
        "s_add_u32 %0, %0, %4\n\t"
        "s_waitcnt vmcnt(%c10)\n\t"
        "buffer_store_dwordx4 v[%c5+16:%c5+19], %1, %2, %0 offen sc0 sc1\n\t"
        "s_add_u32 %0, %0, %4\n\t"
        "s_waitcnt vmcnt(%c11)\n\t"
        "buffer_store_dwordx4 v[%c5+20:%c5+23], %1, %2, %0 offen sc0 sc1\n\t"
        "s_add_u32 %0, %0, %4\n\t"
        "s_waitcnt vmcnt(%c12)\n\t"
        "buffer_store_dwordx4 v[%c5+24:%c5+27], %1, %2, %0 offen sc0 sc1\n\t"
        "s_add_u32 %0, %0, %4\n\t"
        "s_waitcnt vmcnt(%c13)\n\t"
        "buffer_store_dwordx4 v[%c5+28:%c5+31], %1, %2, %0 offen sc0 sc1\n\t"
        "s_nop 1"
        : "=&s"(t)
        : "v"(voff), "s"(srd), "s"(soff), "s"(step), "i"(R), "i"(SWN(0)), "i"(SWN(1)), "i"(SWN(2)), "i"(SWN(3)), "i"(SWN(4)), "i"(SWN(5)), "i"(SWN(6)), "i"(SWN(7))
        : CE_CLOB);
}
template <int R> __device__ __forceinline__ void tr8() {
    asm volatile(
        "v_perm_b32 v224, v[%c2+4], v[%c2+0], %0\n\t"
        "v_perm_b32 v225, v[%c2+12], v[%c2+8], %0\n\t"
        "v_perm_b32 v226, v[%c2+20], v[%c2+16], %0\n\t"
        "v_perm_b32 v227, v[%c2+28], v[%c2+24], %0\n\t"
        "v_perm_b32 v228, v[%c2+4], v[%c2+0], %1\n\t"
        "v_perm_b32 v229, v[%c2+12], v[%c2+8], %1\n\t"
        "v_perm_b32 v230, v[%c2+20], v[%c2+16], %1\n\t"
        "v_perm_b32 v231, v[%c2+28], v[%c2+24], %1\n\t"
        "v_perm_b32 v232, v[%c2+5], v[%c2+1], %0\n\t"
        "v_perm_b32 v233, v[%c2+13], v[%c2+9], %0\n\t"
        "v_perm_b32 v234, v[%c2+21], v[%c2+17], %0\n\t"
        "v_perm_b32 v235, v[%c2+29], v[%c2+25], %0\n\t"
        "v_perm_b32 v236, v[%c2+5], v[%c2+1], %1\n\t"
        "v_perm_b32 v237, v[%c2+13], v[%c2+9], %1\n\t"
        "v_perm_b32 v238, v[%c2+21], v[%c2+17], %1\n\t"
        "v_perm_b32 v239, v[%c2+29], v[%c2+25], %1\n\t"
        "v_perm_b32 v240, v[%c2+6], v[%c2+2], %0\n\t"
        "v_perm_b32 v241, v[%c2+14], v[%c2+10], %0\n\t"
        "v_perm_b32 v242, v[%c2+22], v[%c2+18], %0\n\t"
        "v_perm_b32 v243, v[%c2+30], v[%c2+26], %0\n\t"
        "v_perm_b32 v244, v[%c2+6], v[%c2+2], %1\n\t"
        "v_perm_b32 v245, v[%c2+14], v[%c2+10], %1\n\t"
        "v_perm_b32 v246, v[%c2+22], v[%c2+18], %1\n\t"
        "v_perm_b32 v247, v[%c2+30], v[%c2+26], %1\n\t"
        "v_perm_b32 v248, v[%c2+7], v[%c2+3], %0\n\t"
        "v_perm_b32 v249, v[%c2+15], v[%c2+11], %0\n\t"
        "v_perm_b32 v250, v[%c2+23], v[%c2+19], %0\n\t"
        "v_perm_b32 v251, v[%c2+31], v[%c2+27], %0\n\t"
        "v_perm_b32 v252, v[%c2+7], v[%c2+3], %1\n\t"
        "v_perm_b32 v253, v[%c2+15], v[%c2+11], %1\n\t"
        "v_perm_b32 v254, v[%c2+23], v[%c2+19], %1\n\t"
        "v_perm_b32 v255, v[%c2+31], v[%c2+27], %1\n\t"
        "s_nop 1"
        :: "s"(0x05040100u), "s"(0x07060302u), "i"(R)
        : CE_CLOB);
}
template <int R> __device__ __forceinline__ void ld4(unsigned voff, const u32x4& srd, unsigned soff, unsigned step) {
    asm volatile(
        "s_nop 4\n\t"
        "buffer_load_dwordx4 v[%c4+0:%c4+3], %1, %2, %0 offen sc0 sc1\n\t"
        "s_add_u32 %0, %0, %3\n\t"
        "buffer_load_dwordx4 v[%c4+4:%c4+7], %1, %2, %0 offen sc0 sc1\n\t"
        "s_add_u32 %0, %0, %3\n\t"
        "buffer_load_dwordx4 v[%c4+8:%c4+11], %1, %2, %0 offen sc0 sc1\n\t"
        "s_add_u32 %0, %0, %3\n\t"
        "buffer_load_dwordx4 v[%c4+12:%c4+15], %1, %2, %0 offen sc0 sc1\n\t"
        "s_nop 0"
        : "+s"(soff)
        : "v"(voff), "s"(srd), "s"(step), "i"(R)
        : CE_CLOB);
}
template <int X, int Y> __device__ __forceinline__ void xfer(unsigned dst, unsigned src4, unsigned nsrc, unsigned voff, const u32x4& dsrd, unsigned step,
                                                         const u32x4& ssrd, unsigned nvoff, const u32x4& nsrd, unsigned nstep, unsigned svoff, unsigned sstep) {
    asm volatile(
        "s_nop 4\n\t"
        "s_waitcnt vmcnt(6)\n\t"
        "buffer_store_dwordx4 v[%c12+0:%c12+3], %3, %4, %0 offen sc0 sc1\n\t"
        "s_add_u32 %0, %0, %5\n\t"
        "buffer_load_dwordx4 v[%c12+16:%c12+19], %10, %6, %1 offen sc0 sc1\n\t"
        "s_add_u32 %1, %1, %11\n\t"
        "s_waitcnt vmcnt(6)\n\t"
        "buffer_store_dwordx4 v[%c12+4:%c12+7], %3, %4, %0 offen sc0 sc1\n\t"
        "s_add_u32 %0, %0, %5\n\t"
        "buffer_load_dwordx4 v[%c12+20:%c12+23], %10, %6, %1 offen sc0 sc1\n\t"
        "s_add_u32 %1, %1, %11\n\t"
        "s_waitcnt vmcnt(6)\n\t"
        "buffer_store_dwordx4 v[%c12+8:%c12+11], %3, %4, %0 offen sc0 sc1\n\t"
        "s_add_u32 %0, %0, %5\n\t"
        "buffer_load_dwordx4 v[%c12+24:%c12+27], %10, %6, %1 offen sc0 sc1\n\t"
        "s_add_u32 %1, %1, %11\n\t"
        "s_waitcnt vmcnt(6)\n\t"
        "buffer_store_dwordx4 v[%c12+12:%c12+15], %3, %4, %0 offen sc0 sc1\n\t"
        "s_add_u32 %0, %0, %5\n\t"
        "buffer_load_dwordx4 v[%c12+28:%c12+31], %10, %6, %1 offen sc0 sc1\n\t"
        "s_waitcnt vmcnt(6)\n\t"
        "buffer_store_dwordx4 v[%c12+16:%c12+19], %3, %4, %0 offen sc0 sc1\n\t"
        "s_add_u32 %0, %0, %5\n\t"
        "buffer_load_dwordx4 v[%c13+0:%c13+3], %7, %8, %2 offen sc0 sc1\n\t"
        "s_add_u32 %2, %2, %9\n\t"
        "s_waitcnt vmcnt(6)\n\t"
        "buffer_store_dwordx4 v[%c12+20:%c12+23], %3, %4, %0 offen sc0 sc1\n\t"
        "s_add_u32 %0, %0, %5\n\t"
        "buffer_load_dwordx4 v[%c13+4:%c13+7], %7, %8, %2 offen sc0 sc1\n\t"
        "s_add_u32 %2, %2, %9\n\t"
        "s_waitcnt vmcnt(6)\n\t"
        "buffer_store_dwordx4 v[%c12+24:%c12+27], %3, %4, %0 offen sc0 sc1\n\t"
        "s_add_u32 %0, %0, %5\n\t"
        "buffer_load_dwordx4 v[%c13+8:%c13+11], %7, %8, %2 offen sc0 sc1\n\t"
        "s_add_u32 %2, %2, %9\n\t"
        "s_waitcnt vmcnt(6)\n\t"
        "buffer_store_dwordx4 v[%c12+28:%c12+31], %3, %4, %0 offen sc0 sc1\n\t"
        "buffer_load_dwordx4 v[%c13+12:%c13+15], %7, %8, %2 offen sc0 sc1\n\t"
        "s_nop 1"
        : "+s"(dst), "+s"(src4), "+s"(nsrc)
        : "v"(voff), "s"(dsrd), "s"(step), "s"(ssrd), "v"(nvoff), "s"(nsrd), "s"(nstep), "v"(svoff), "s"(sstep), "i"(X), "i"(Y)
        : CE_CLOB);
}
__device__ __forceinline__ void ticket_issue(u64 ctr) {
    u64 sv;
    asm volatile(
        "s_mov_b64 %0, exec\n\t"
        "s_mov_b64 exec, 1\n\t"
        "v_mov_b32 v158, 0\n\t"
        "v_mov_b32 v159, 1\n\t"
        "s_nop 4\n\t"
        "global_atomic_add v159, v158, v159, %1 sc0\n\t"
        "s_mov_b64 exec, %0"
        : "=&s"(sv) : "s"(ctr) : CE_CLOB);
}
__device__ __forceinline__ int ticket_read() {
    int q;
    asm volatile("s_nop 1\n\tv_readfirstlane_b32 %0, v159\n\ts_nop 4" : "=s"(q) :: CE_CLOB);
    return q;
}
__device__ __forceinline__ void tflags_issue(u64 base, unsigned voff) {
    asm volatile(
        "s_nop 4\n\t"
        "global_load_dword v157, %0, %1 sc0 sc1\n\t"
        "global_load_dword v158, %0, %1 offset:256 sc0 sc1"
        :: "v"(voff), "s"(base) : CE_CLOB);
}
__device__ __forceinline__ int tflags_read(unsigned call) {
    u64 m0, m1;
    asm volatile("s_nop 1\n\tv_cmp_le_u32 %0, %2, v157\n\tv_cmp_le_u32 %1, %2, v158\n\ts_nop 4" : "=&s"(m0), "=&s"(m1) : "s"(call) : CE_CLOB);
    return (~m0 != 0ull) ? __builtin_ctzll(~m0) : 64 + ((~m1 != 0ull) ? __builtin_ctzll(~m1) : 64);
}
__device__ __forceinline__ void fcnt_issue(u64 base, unsigned voff) {
    asm volatile("s_nop 4\n\tglobal_load_dword v156, %0, %1 sc0 sc1" :: "v"(voff), "s"(base) : CE_CLOB);
}
__device__ __forceinline__ int fcnt_read(unsigned need) {
    u64 m0;
    asm volatile("s_nop 1\n\tv_cmp_le_u32 %0, %1, v156\n\ts_nop 4" : "=&s"(m0) : "s"(need) : CE_CLOB);
    return (~m0 != 0ull) ? __builtin_ctzll(~m0) : 64;
}
#define CE_WAIT(n) asm volatile("s_waitcnt vmcnt(" #n ")" ::: CE_CLOB)
constexpr int RA = 160, RB = 192, RT = 224;

__device__ __forceinline__ unsigned selv(bool c0, bool c1, unsigned a, unsigned b, unsigned c) {
    asm("" : "+v"(a)); asm("" : "+v"(b)); asm("" : "+v"(c));
    return c0 ? a : (c1 ? b : c);
}
__device__ __forceinline__ u64 sels(bool c, u64 a, u64 b) {
    asm("" : "+s"(a)); asm("" : "+s"(b));
    return uni64((const void*)(c ? a : b));
}

struct Piece { int type; unsigned src, dst, flag; };

struct Comm {
    KP p;
    int rank, peer, lane;
    unsigned call;
    bool partner;
    u64 rx, rw, ro_me, rws_me, rws_peer, ro_peer;
    unsigned lin_off, w_src_off, w_dst_off, o_off;
    char* wsp;
    char* wsh;
    char* wsm;

    __device__ __forceinline__ Comm(KP p_, int peer_, int lane_, unsigned call_) : p(p_), rank(p_.rank), peer(peer_), lane(lane_), call(call_) {
        partner = (rank ^ peer) == PM;
        rx = uni64(p.x);
        rw = uni64(p.w);
        ro_me = uni64(p.out[rank]);
        wsm = p.ws[rank];
        rws_me = uni64(wsm);
        wsp = p.ws[peer];
        wsh = p.ws[peer ^ PM];
        rws_peer = uni64(wsp);
        ro_peer = uni64(p.out[peer]);
        lin_off = lane * 16;
        w_src_off = (lane >> 3) * 8 * 16384 + (lane & 7) * 16;
        w_dst_off = (lane & 7) * 8 * 2048 + (lane >> 3) * 16;
        o_off = (lane >> 5) * 16384 + (lane & 31) * 16;
    }
    __device__ __forceinline__ unsigned out_off(int a, int o, int po) const {
        const int t = tab8(p.forder[role_of(a)], o), pm = t >> 4, pn = t & 15, qd = quad_of(a);
        return (unsigned)((qd >> 1) * 4096 + pm * 256 + po * 16) * 16384u + (unsigned)((qd & 1) * 4096 + pn * 256) * 2u;
    }
    __device__ __forceinline__ Piece desc_in(int q) const {
        Piece d;
        q = __builtin_amdgcn_readfirstlane(q);
        const int u = uidx(rank ^ peer);
        const int e = q >> 5, pc = q & 31, panel = e >> 1, hd = role_of(peer), qd = quad_of(peer);
        d.flag = OFF_INF + (u * 1024 + q) * 4;
        if (!(e & 1)) {
            d.type = 0;
            d.src = (unsigned)((qd >> 1) * 4096 + panel * 256 + pc * 8) * 2048u + hd * 1024u;
            d.dst = OFF_XS + (u >> 1) * (8u << 20) + (unsigned)(panel * 256 + pc * 8) * 2048u + (u & 1) * 1024u;
        } else {
            const int kb = pc >> 2, nb = pc & 3;
            d.type = 1;
            d.src = (unsigned)((hd * 512 + kb * 64) * 8192 + (qd & 1) * 4096 + panel * 256 + nb * 64) * 2u;
            d.dst = OFF_WTS + (u >> 1) * (8u << 20) + (unsigned)((panel * 256 + nb * 64) * 1024 + (u & 1) * 512 + kb * 64) * 2u;
        }
        d.type = __builtin_amdgcn_readfirstlane(d.type);
        d.src = __builtin_amdgcn_readfirstlane(d.src);
        d.dst = __builtin_amdgcn_readfirstlane(d.dst);
        d.flag = __builtin_amdgcn_readfirstlane(d.flag);
        return d;
    }
    template <int R> __device__ __forceinline__ void load(const Piece& d) const {
        ld8<R>(selv(d.type == 1, false, w_src_off, lin_off, lin_off), mk_srd(sels(d.type == 0, rx, rw)), d.src, d.type == 0 ? 2048u : 16384u);
    }
    template <int R> __device__ __forceinline__ void store(const Piece& d) const {
        if (d.type == 1) {
            tr8<R>();
            st8<RT>(w_dst_off, mk_srd(rws_peer), d.dst, 2048u);
        } else {
            st8<R>(lin_off, mk_srd(rws_peer), d.dst, 2048u);
        }
    }
    __device__ __forceinline__ void flag_in(int q) const {
        if (q >= 0 && lane == 0) st_sys(wsp + OFF_INF + (uidx(rank ^ peer) * 1024 + q) * 4, call);
    }
    __device__ __forceinline__ Piece desc_pull(int j) const {
        Piece d;
        j = __builtin_amdgcn_readfirstlane(j);
        if (j < -N_PIN) {
            d.type = 4; d.src = d.dst = OFF_PS; d.flag = 0;
        } else if (j < 0) {
            const int q = j + N_IN, e = q >> 5, pc = q & 31, panel = e >> 1, u = uidx(rank ^ peer), qd = quad_of(rank);
            const unsigned row = (unsigned)(panel * 256 + pc * 8) * 2048u;
            d.type = 5;
            d.src = ((e & 1) ? OFF_WO + (qd & 1) * (4u << 20) : OFF_XO + (qd >> 1) * (4u << 20)) + row / 2 + role_of(rank) * (8u << 20);
            d.dst = ((e & 1) ? OFF_WTS : OFF_XS) + (u >> 1) * (8u << 20) + row + (u & 1) * 1024u;
            d.flag = OFF_INF + (u * 1024 + q) * 4;
        } else if (j >= N_FIN) {
            const int qq = j - N_FIN;
            d.type = partner ? 6 : 2;
            d.dst = out_off(rank ^ PM, qq >> 4, qq & 15); d.flag = 0;
            d.src = partner ? OFF_FS + qq * 8192u : d.dst;
        } else if (partner) {
            d.type = 3;
            d.src = OFF_PS + j * 8192u; d.dst = OFF_PR + j * 8192u; d.flag = OFF_PFLAG + j * 4;
        } else {
            d.type = 6;
            d.dst = out_off(peer, j >> 4, j & 15);
            d.src = OFF_FS + j * 8192u;
            d.flag = (OFF_HAVE + (uidx(rank ^ peer ^ PM) * 64 + (j >> SEGSH)) * 4) | 1u;
        }
        d.type = __builtin_amdgcn_readfirstlane(d.type);
        d.src = __builtin_amdgcn_readfirstlane(d.src);
        d.dst = __builtin_amdgcn_readfirstlane(d.dst);
        d.flag = __builtin_amdgcn_readfirstlane(d.flag);
        return d;
    }
    __device__ __forceinline__ unsigned pvoff(const Piece& d) const { return selv(d.type == 2, false, o_off, lin_off, lin_off); }
    __device__ __forceinline__ unsigned pstep(const Piece& d) const { return d.type == 2 ? 32768u : 1024u; }
    __device__ __forceinline__ unsigned dvoff(const Piece& d) const { return selv(d.type == 2 || d.type == 6, false, o_off, lin_off, lin_off); }
    __device__ __forceinline__ unsigned dstep(const Piece& d) const { return (d.type == 2 || d.type == 6) ? 32768u : (d.type == 5 ? 2048u : 1024u); }
    __device__ __forceinline__ u32x4 psrc(const Piece& d) const { return mk_srd(sels(d.type == 2, ro_peer, sels(d.type != 4, rws_peer, rws_me))); }
    __device__ __forceinline__ u32x4 pdst(const Piece& d) const { return mk_srd(sels(d.type == 2 || d.type == 6, ro_me, rws_me)); }
    template <int R> __device__ __forceinline__ void pull_first(const Piece& d) const { ld4<R>(pvoff(d), psrc(d), d.src, pstep(d)); }
    template <int X, int Y> __device__ __forceinline__ void pull_step(const Piece& c, const Piece& n) const {
        xfer<X, Y>(c.dst, c.src + 4 * pstep(c), n.src, dvoff(c), pdst(c), dstep(c), psrc(c), pvoff(n), psrc(n), pstep(n), pvoff(c), pstep(c));
    }
    unsigned seg_off = 0, seg_cnt = 0;
    __device__ __forceinline__ void flush() {
        if (seg_cnt != 0 && lane == 0) __hip_atomic_fetch_add((unsigned*)(wsh + seg_off), seg_cnt, __ATOMIC_RELAXED, __HIP_MEMORY_SCOPE_SYSTEM);
        seg_cnt = 0;
    }
    __device__ __forceinline__ void flag_pull(unsigned off) {
        if (off & 1u) {
            if ((off & ~1u) != seg_off) { flush(); seg_off = off & ~1u; }
            ++seg_cnt;
        } else if (off != 0) { if (lane == 0) st_sys(wsm + off, call); }
        else flush();
    }
};

__device__ __forceinline__ void comm_role(KP p, int peer_push, int peer_pull, int slot, int cidx) {
    const int rank = p.rank;
    char* ws = p.ws[rank];
    const int lane = threadIdx.x & 63, wid = __builtin_amdgcn_readfirstlane(threadIdx.x >> 6);
    const int peer = wid == 0 ? peer_push : peer_pull;
    if ((unsigned)peer >= (unsigned)NR || peer == rank) return;
    if (wid >= 2 || (wid == 0 && ((rank ^ peer) == PM || slot >= NCW)) || (wid == 1 && slot >= NCP)) return;
    const u64 ctr_own = uni64(ws + OFF_QCTR + (wid * 17 + peer) * 128);
    ticket_issue(ctr_own);
    Comm c(p, peer, lane, 0u);
    if (wid == 0) c.load<RA>(c.desc_in(slot));
    const unsigned call = __builtin_amdgcn_readfirstlane(ld_agent(ws + OFF_SEQ)) + 1u;
    c.call = call;
    STAMP(4096 + (cidx * 8 + wid) * 4 + 0);
    const bool partner = c.partner;
    CE_WAIT(0);
    int qn = ticket_read();

    if (wid == 0) {
        int qc = slot, qp = -1;
        qn += NCW;
        bool first = true;
#define CE_STEP(X, Y) { \
            if (qn >= N_PUSH) { CE_WAIT(0); c.flag_in(qp); c.store<X>(c.desc_in(qc)); CE_WAIT(0); c.flag_in(qc); break; } \
            ticket_issue(ctr_own); \
            c.load<Y>(c.desc_in(qn)); \
            if (first) CE_WAIT(9); else CE_WAIT(17); \
            first = false; \
            c.store<X>(c.desc_in(qc)); \
            CE_WAIT(16); \
            c.flag_in(qp); \
            qp = qc; qc = qn; \
            qn = NCW + ticket_read(); }
        for (;;) {
            CE_STEP(RA, RB)
            CE_STEP(RB, RA)
        }
#undef CE_STEP
        STAMP(4096 + (cidx * 8 + wid) * 4 + 1);
        return;
    }

    const u64 ctr_q = uni64(ws + OFF_QCTR + 16 * 128);
    const u64 rtf = uni64(ws + OFF_RT + peer * 512), rsf = uni64(ws + OFF_RS);
    const u64 hav = uni64(ws + OFF_HAVE + (partner ? 0 : uidx(rank ^ peer)) * 256);
    const unsigned tvoff = lane * 4;
    const unsigned long long t0 = __builtin_amdgcn_s_memrealtime();
    int qoff = partner ? 0 : -N_PIN;
    int qready = partner ? 0 : J_MS1, qlim = N_FIN;
    const u64 sr2 = uni64(ws + OFF_SR + (8 + peer) * 4);
    bool inq = false;
    u64 ctr = ctr_own;
    if (!partner) {
        const char* gate = lane < 32 ? ws + OFF_INF + (uidx(rank ^ peer) * 1024 + PGATE + lane) * 4 : ws + OFF_SR + peer * 4;
        for (;;) {
            if (__all(ld_sys(gate) >= call)) break;
            if (give_up(ws, call, t0)) break;
            __builtin_amdgcn_s_sleep(GSL);
        }
    }
    STAMP(4096 + (cidx * 8 + wid) * 4 + 1);
    qn += qoff;
#define SNAP_ISSUE() do { tflags_issue(sels(partner && !inq, rsf, rtf), tvoff); if (inq || qready < 0) fcnt_issue(sels(inq, hav, sr2), tvoff); } while (0)
#define SNAP_APPLY() do { const int n_ = tflags_read(call); \
        int r_; \
        if (!inq) r_ = (qready < 0 && fcnt_read(call) == 0) ? qready : 16 * n_; \
        else r_ = N_FIN + (partner ? 16 * n_ : (min(fcnt_read(call << SEGSH), NSEG) << SEGSH)); \
        qready = __builtin_amdgcn_readfirstlane(max(qready, r_)); } while (0)
#define TO_QUEUE() do { STAMP(4096 + (cidx * 8 + wid) * 4 + 3); inq = true; qoff = N_FIN; qlim = 2 * N_FIN; ctr = ctr_q; qready = N_FIN; } while (0)
    for (;;) {
        if (qn >= qlim) {
            if (inq) break;
            TO_QUEUE();
            ticket_issue(ctr_q);
            SNAP_ISSUE();
            CE_WAIT(0);
            qn = qoff + ticket_read();
            SNAP_APPLY();
            continue;
        }
        while (qn >= qready) {
            SNAP_ISSUE();
            CE_WAIT(0);
            const int before = qready;
            SNAP_APPLY();
            if (qready != before) continue;
            if (give_up(ws, call, t0)) { qready = qlim; break; }
            __builtin_amdgcn_s_sleep(CSL);
        }
        qready = __builtin_amdgcn_readfirstlane(qready);
        ctr = uni64((const void*)ctr);
        int qc = qn, qp = J_NONE;
        ticket_issue(ctr);
        c.pull_first<RA>(c.desc_pull(qc));
        CE_WAIT(0);
        qn = qoff + ticket_read();
#define CE_STEP(X, Y) { \
            if (!inq && qn >= qlim) {         \
                TO_QUEUE(); ticket_issue(ctr_q); SNAP_ISSUE(); CE_WAIT(0); qn = qoff + ticket_read(); SNAP_APPLY(); \
                ctr = uni64((const void*)ctr); } \
            if (!(qn < qlim && qn < qready)) { \
                c.pull_step<X, Y>(c.desc_pull(qc), c.desc_pull(J_NONE)); \
                CE_WAIT(0); c.flag_pull(c.desc_pull(qp).flag); c.flag_pull(c.desc_pull(qc).flag); c.flush(); \
                break; } \
            const bool snap = qready < qlim && qn + (inq ? 16 : 3) * NCP >= qready; \
            ticket_issue(ctr); \
            if (snap) SNAP_ISSUE(); \
            c.pull_step<X, Y>(c.desc_pull(qc), c.desc_pull(qn)); \
            c.flag_pull(c.desc_pull(qp).flag); \
            qp = qc; qc = qn; \
            qn = qoff + ticket_read(); \
            if (qc < 0 && qn >= 0) STAMP(4096 + (cidx * 8 + 2) * 4); \
            if (snap) SNAP_APPLY(); }
        for (;;) {
            CE_STEP(RA, RB)
            CE_STEP(RB, RA)
        }
#undef CE_STEP
    }
    STAMP(4096 + (cidx * 8 + wid) * 4 + 2);
}

__device__ __forceinline__ void stage_role(KP p, int sidx) {
    const int rank = p.rank;
    char* ws = p.ws[rank];
    const int lane = threadIdx.x & 63, wid = __builtin_amdgcn_readfirstlane(threadIdx.x >> 6);
    const u32x4 sx = mk_srd(uni64(p.x)), sw = mk_srd(uni64(p.w)), sd = mk_srd(uni64(ws));
    const unsigned w_src_off = (lane >> 3) * 8 * 16384 + (lane & 7) * 16, w_dst_off = (lane & 7) * 8 * 1024 + (lane >> 3) * 16;
    for (int ms = 0; ms < 2; ++ms) {
        for (int t = sidx * 8 + wid + ms * NSTG_WG * 8; t < (ms == 0 ? min(N_STG / 2, NSTG_WG * 8) : N_STG / 2); t += NSTG_WG * 8) {
        const int pc = t & 63, hf = (t >> 6) & 1, panel = EPUSH / 2 + (t >> 7), kb = pc >> 2, nb = pc & 3;
        const unsigned r0 = (unsigned)(hf * 4096 + panel * 256 + (pc >> 1) * 8), kh = pc & 1;
        const int n0 = hf * 4096 + panel * 256 + nb * 64;
        ld8<RA>(lane * 16, sx, r0 * 2048u + kh * 1024u, 2048u);
        ld8<RB>(w_src_off, sw, (unsigned)(kb * 64 * 8192 + n0) * 2u, 16384u);
        CE_WAIT(8);
        st8<RA>(lane * 16, sd, OFF_XO + kh * (8u << 20) + r0 * 1024u, 1024u);
        CE_WAIT(8);
        tr8<RB>();
        st8<RT>(w_dst_off, sd, OFF_WO + (unsigned)(kb >> 3) * (8u << 20) + (unsigned)(n0 * 512 + (kb & 7) * 64) * 2u, 1024u);
        }
        CE_WAIT(0);
        __syncthreads();
        if (wid == 0) {
            unsigned tot = 0;
            if (lane == 0) tot = __hip_atomic_fetch_add((unsigned*)(ws + OFF_QCTR + (25 + ms) * 128), 1u, __ATOMIC_RELAXED, __HIP_MEMORY_SCOPE_AGENT) + 1u;
            if (__builtin_amdgcn_readfirstlane(tot) == (unsigned)NSTG_WG) {
                const unsigned call = ld_agent(ws + OFF_SEQ) + 1u;
                if (lane < NR && lane != rank) st_sys(p.ws[lane] + OFF_SR + (ms * 8 + rank) * 4, call);
                STAMP(4093 + ms);
            }
        }
    }
}

__device__ __forceinline__ void gemm_role(KP p, int xcd, int slot, unsigned call, LAS unsigned char* lds) {
    const int rank = p.rank;
    char* ws = p.ws[rank];
    const int tid = threadIdx.x, wid = __builtin_amdgcn_readfirstlane(tid >> 6), lane = tid & 63, wr = wid >> 2, wc = wid & 3, fr = lane & 15, fq = lane >> 4;
    const int qd = quad_of(rank), role = role_of(rank);

    if (slot < 16) {
        const int id = (slot * 8 + xcd) * 8 + wid, panel = id >> 6, pc = id & 63, kb = pc >> 2, nb = pc & 3;
        const unsigned w_src_off = (lane >> 3) * 8 * 16384 + (lane & 7) * 16, w_dst_off = (lane & 7) * 8 * 2048 + (lane >> 3) * 16;
        ld8<RA>(w_src_off, mk_srd(uni64(p.w)), (unsigned)(kb * 64 * 8192 + (qd & 1) * 4096 + panel * 256 + nb * 64) * 2u, 16384u);
        CE_WAIT(0);
        tr8<RA>();
        st8<RT>(w_dst_off, mk_srd(uni64(ws)), OFF_WTS + 3 * (8u << 20) + (unsigned)((panel * 256 + nb * 64) * 1024 + kb * 64) * 2u, 2048u);
        CE_WAIT(0);
        if (lane == 0) st_sys(ws + OFF_OWNW + id * 4, call);
    }

    if (slot >= STG_SLOT) stage_role(p, (NSPARE + slot - STG_SLOT) * 8 + xcd);
    unsigned voffA[2], voffB[2];
#pragma unroll
    for (int i = 0; i < 2; ++i) { int R, C; stage_rc(tid * 16 + i * 8192, R, C); const int Rb = (R & ~31) + perm32(R & 31);
        voffA[i] = (unsigned)(R * KS + C) * 2u; voffB[i] = (unsigned)(Rb * KS + C) * 2u; }
    constexpr size_t kstep = (size_t)(BK * 2);
    constexpr size_t hstep = (size_t)HALF * KS * 2;
    const unsigned ldsw = (unsigned)wid * 1024u;
    const int aoff = lds_byte(wr * 64 + fr, fq * 8), boff = lds_byte(wc * 32 + fr, fq * 8);
#define SA(b, h) (((b) * 2 + (h)) * HTB)
#define SB(b, h) ((4 + (b) * 2 + (h)) * HTB)
#define STAGE(bufoff, gbase, voff) do { _Pragma("unroll") for (int _i = 0; _i < 2; ++_i) \
        __builtin_amdgcn_global_load_lds((const unsigned*)((const char*)(gbase) + (voff)[_i]), (LAS unsigned*)(lds + (bufoff) + ldsw + _i * 8192), 16, 0, 0); } while (0)
#define LDA(dst, b, h) do { _Pragma("unroll") for (int m = 0; m < 4; ++m) _Pragma("unroll") for (int k = 0; k < 2; ++k) dst[m][k] = *(const LAS bf16x8*)(lds + SA(b, h) + aoff + m * 2048 + k * 1024); } while (0)
#define LDB(dst, b, h) do { _Pragma("unroll") for (int n = 0; n < 2; ++n) _Pragma("unroll") for (int k = 0; k < 2; ++k) dst[n][k] = *(const LAS bf16x8*)(lds + SB(b, h) + boff + n * 2048 + k * 1024); } while (0)
#define MMA(ai, bj, At, Bt) do { __builtin_amdgcn_s_setprio(1); _Pragma("unroll") for (int m = 0; m < 4; ++m) _Pragma("unroll") for (int n = 0; n < 2; ++n) _Pragma("unroll") for (int k = 0; k < 2; ++k) \
        acc[ai][bj][m][n] = __builtin_amdgcn_mfma_f32_16x16x32_bf16(Bt[n][k], At[m][k], acc[ai][bj][m][n], 0, 0, 0); __builtin_amdgcn_s_setprio(0); } while (0)
#define BAR __builtin_amdgcn_s_barrier()
#define SCHED __builtin_amdgcn_sched_barrier(0)

    int qi = slot;
    while (qi < XQN) {
        const int mode = tab8(p.xm[xcd], qi);
        if (mode == 255) break;
        const int tile = tab8(p.xq[xcd], qi), pm = tile >> 4, pn = tile & 15, pos = tab8(p.pos, tile);
        const bool full = mode == 0;
        const int qa = (mode - 1) >> 1, qb = (mode - 1) & 1;
        const bool mine = ((pm + pn) & 1) == role;
        if (wid == 0) STAMP(tile * 8 + 0);

        auto hook = [&](bool remote) {
            if (wid == 0) {
                const unsigned long long t0 = __builtin_amdgcn_s_memrealtime();
                if (!remote) {
                    for (;;) {
                        const bool ok = ld_sys(ws + OFF_OWNW + (pn * 64 + lane) * 4) >= call;
                        if (__all(ok)) break;
                        if (give_up(ws, call, t0)) break;
                        __builtin_amdgcn_s_sleep(4);
                    }
                } else {
                    const char* fb = ws + OFF_INF + ((lane < 32 ? 2 * pm : 2 * pn + 1) * 32 + (lane & 31)) * 4;
                    for (;;) {
                        const bool ok = ld_sys(fb) >= call;
                        if (__all(ok)) break;
                        if (give_up(ws, call, t0)) break;
                        __builtin_amdgcn_s_sleep(16);
                    }
                    for (;;) {
                        bool ok = true;
#pragma unroll
                        for (int u = 1; u < 6; ++u) ok = ok & (ld_sys(fb + u * 4096) >= call);
                        if (__all(ok)) break;
                        if (give_up(ws, call, t0)) break;
                        __builtin_amdgcn_s_sleep(8);
                    }
                }
                __builtin_amdgcn_fence(__ATOMIC_ACQUIRE, "");
                asm volatile("s_waitcnt vmcnt(0)" ::: "memory");
            }
            asm volatile("" ::: "memory"); __builtin_amdgcn_s_barrier(); asm volatile("" ::: "memory");
        };

        f32x4 acc[2][2][4][2];
#pragma unroll
        for (int a = 0; a < 2; ++a)
#pragma unroll
            for (int b = 0; b < 2; ++b)
#pragma unroll
                for (int m = 0; m < 4; ++m)
#pragma unroll
                    for (int n = 0; n < 2; ++n) acc[a][b][m][n] = (f32x4){0.f, 0.f, 0.f, 0.f};
        bf16x8 At[4][2], B0[2][2], B1[2][2];

        const char* xsA = ws + OFF_XS + (size_t)pm * (512u << 10);
        const char* wsB = ws + OFF_WTS + (size_t)pn * (512u << 10);
        const char* cA = p.x + (size_t)((qd >> 1) * 4096 + pm * 256) * 2048;
        const char* cB = wsB + (size_t)3 * (8u << 20);
        hook(false);
        if (wid == 0) STAMP(tile * 8 + 1);
        if (!full) {
            const char* qA = cA + (size_t)qa * hstep; const char* qB = cB + (size_t)qb * hstep;
            const char* rA = xsA + (size_t)qa * hstep; const char* rB = wsB + (size_t)qb * hstep;
            auto issue = [&](int t, int sl) {
                t = min(t, 63);
                const int u = t >> 4;
                const char* a = (u == 0 ? qA : rA + (size_t)(u - 1) * (8u << 20)) + (size_t)(t & 15) * kstep;
                const char* b = (u == 0 ? qB : rB + (size_t)(u - 1) * (8u << 20)) + (size_t)(t & 15) * kstep;
                STAGE(sl * HTB, a, voffA); STAGE((4 + sl) * HTB, b, voffB);
            };
            issue(0, 0); issue(1, 1); issue(2, 2);
            for (int t0 = 0; t0 < 64; t0 += 4) {
#pragma unroll
                for (int sl = 0; sl < 4; ++sl) {
                    if (t0 + sl == 13) {
                        if (wid == 0) STAMP(tile * 8 + 2);
                        hook(true);
                        if (wid == 0) STAMP(tile * 8 + 3);
                    }
                    WAIT_V(8); BAR;
                    issue(t0 + sl + 3, (sl + 3) & 3);
#pragma unroll
                    for (int n = 0; n < 2; ++n)
#pragma unroll
                        for (int k = 0; k < 2; ++k) B0[n][k] = *(const LAS bf16x8*)(lds + (4 + sl) * HTB + boff + n * 2048 + k * 1024);
#pragma unroll
                    for (int m = 0; m < 4; ++m)
#pragma unroll
                        for (int k = 0; k < 2; ++k) At[m][k] = *(const LAS bf16x8*)(lds + sl * HTB + aoff + m * 2048 + k * 1024);
                    WAIT_L(0); SCHED;
                    MMA(0, 0, At, B0);
                    SCHED;
                }
            }
        } else {
        STAGE(SB(0, 0), cB, voffB); STAGE(SA(0, 0), cA, voffA); STAGE(SB(0, 1), cB + hstep, voffB); STAGE(SA(0, 1), cA + hstep, voffA);
        if (wr == 1) BAR;
        WAIT_V(4); BAR;
        STAGE(SB(1, 0), cB + kstep, voffB); STAGE(SA(1, 0), cA + kstep, voffA); STAGE(SB(1, 1), cB + hstep + kstep, voffB);
        WAIT_V(6); BAR;
        constexpr int nt = KS / BK;
        for (int u = 0; u < 4; ++u) {
            const bool has_next = u + 1 < 4;
            const char* nA = has_next ? xsA + (size_t)u * (8u << 20) : cA;
            const char* nB = has_next ? wsB + (size_t)u * (8u << 20) : cB;
            for (int t = 0; t < nt; t += 2) {
                const bool last = (t == nt - 2);
                const char* a1 = cA + (size_t)(t + 1) * kstep;
                const char* a2 = last ? nA : cA + (size_t)(t + 2) * kstep; const char* b2 = last ? nB : cB + (size_t)(t + 2) * kstep;
                const char* a3 = a2 + kstep; const char* b3 = b2 + kstep;
                if (last && u == 0) {
                    if (wid == 0) STAMP(tile * 8 + 2);
                    hook(true);
                    if (wid == 0) STAMP(tile * 8 + 3);
                }
                LDB(B0, 0, 0); SCHED; LDA(At, 0, 0); STAGE(SA(1, 1), a1 + hstep, voffA);
                WAIT_L(8); BAR; WAIT_L(0); MMA(0, 0, At, B0); BAR; SCHED;
                LDB(B1, 0, 1); STAGE(SB(0, 0), b2, voffB);
                BAR; WAIT_L(0); MMA(0, 1, At, B1); BAR;
                LDA(At, 0, 1); STAGE(SA(0, 0), a2, voffA);
                BAR; WAIT_L(0); MMA(1, 0, At, B0); BAR; SCHED;
                STAGE(SB(0, 1), b2 + hstep, voffB);
                WAIT_V(6); BAR; MMA(1, 1, At, B1); BAR;
                LDB(B0, 1, 0); SCHED; LDA(At, 1, 0); STAGE(SA(0, 1), a2 + hstep, voffA);
                WAIT_L(8); BAR; WAIT_L(0); MMA(0, 0, At, B0); BAR; SCHED;
                LDB(B1, 1, 1); STAGE(SB(1, 0), b3, voffB);
                BAR; WAIT_L(0); MMA(0, 1, At, B1); BAR;
                LDA(At, 1, 1); STAGE(SA(1, 0), a3, voffA);
                BAR; WAIT_L(0); MMA(1, 0, At, B0); BAR; SCHED;
                STAGE(SB(1, 1), b3 + hstep, voffB);
                WAIT_V(6); BAR; MMA(1, 1, At, B1); BAR;
            }
            cA = nA; cB = nB;
        }
        if (wr == 0) BAR;
        }
        if (wid == 0) STAMP(tile * 8 + 4);
        const __amdgpu_buffer_rsrc_t rw = mk_rsrc(ws, WS_NEED);
        const unsigned pbase = (unsigned)pos * (128u << 10) + (unsigned)wid * (16u << 10);
        if (!mine) {
#pragma unroll
            for (int ai = 0; ai < 2; ++ai)
#pragma unroll
                for (int m = 0; m < 4; ++m)
#pragma unroll
                    for (int bj = 0; bj < 2; ++bj)
                        if (full || (ai == 0 && bj == 0)) {
                            const f32x4 v0 = acc[ai][bj][m][0], v1 = acc[ai][bj][m][1];
                            u32x4 o;
                            o.x = cvt_pk_bf16(v0[0], v0[1]); o.y = cvt_pk_bf16(v0[2], v0[3]);
                            o.z = cvt_pk_bf16(v1[0], v1[1]); o.w = cvt_pk_bf16(v1[2], v1[3]);
                            __builtin_amdgcn_raw_buffer_store_b128(o, rw, lane * 16, OFF_PS + pbase + (full ? 0u : qa * 8192u + qb * 1024u) + ((ai * 4 + m) * 2 + bj) * 1024u, 17);
                        }
        } else {
            if (wid == 0) {
                const unsigned long long t0 = __builtin_amdgcn_s_memrealtime();
                for (;;) {
                    const int fi = lane < 16 ? pos * 16 + lane : max(pos - 4, 0) * 16 + 15;
                    const unsigned long long m = __ballot(ld_sys(ws + OFF_PFLAG + fi * 4) >= call);
                    if ((m & 0xffffull) == 0xffffull) break;
                    if (give_up(ws, call, t0)) break;
                    if (m >> 16) __builtin_amdgcn_s_sleep(PSL_S); else __builtin_amdgcn_s_sleep(PSL_L);
                }
            }
            asm volatile("" ::: "memory"); __builtin_amdgcn_s_barrier(); asm volatile("" ::: "memory");
            if (wid == 0) STAMP(tile * 8 + 5);
            const __amdgpu_buffer_rsrc_t ro = mk_rsrc(p.out[rank], 128u << 20);
            const unsigned base = (unsigned)((qd >> 1) * 4096 + pm * BM) * 16384u + (unsigned)((qd & 1) * 4096 + pn * BM) * 2u;
            const unsigned voff = (unsigned)(wr * 64 + fr) * 16384u + (unsigned)(wc * 32 + 8 * fq) * 2u;
            const unsigned voff2 = (unsigned)(wr * 64 + fr) * 512u + (unsigned)(wc * 32 + 8 * fq) * 2u;
            u32x4 pr[2][4][2];
#pragma unroll
            for (int ai = 0; ai < 2; ++ai)
#pragma unroll
                for (int m = 0; m < 4; ++m)
#pragma unroll
                    for (int bj = 0; bj < 2; ++bj)
                        pr[ai][m][bj] = __builtin_amdgcn_raw_buffer_load_b128(rw, lane * 16, OFF_PR + pbase + ((ai * 4 + m) * 2 + bj) * 1024u, 17);
            __builtin_amdgcn_sched_barrier(0);
#pragma unroll
            for (int ai = 0; ai < 2; ++ai)
#pragma unroll
                for (int m = 0; m < 4; ++m)
#pragma unroll
                    for (int bj = 0; bj < 2; ++bj) {
                        const f32x4 v0 = acc[ai][bj][m][0], v1 = acc[ai][bj][m][1];
                        const u32x4 q = pr[ai][m][bj];
                        u32x4 o;
                        o.x = cvt_pk_bf16(silu(v0[0] + bf_lo(q.x)), silu(v0[1] + bf_hi(q.x))); o.y = cvt_pk_bf16(silu(v0[2] + bf_lo(q.y)), silu(v0[3] + bf_hi(q.y)));
                        o.z = cvt_pk_bf16(silu(v1[0] + bf_lo(q.z)), silu(v1[1] + bf_hi(q.z))); o.w = cvt_pk_bf16(silu(v1[2] + bf_lo(q.w)), silu(v1[3] + bf_hi(q.w)));
                        __builtin_amdgcn_raw_buffer_store_b128(o, rw, voff2, OFF_FS + (unsigned)pos * (128u << 10) + (unsigned)(ai * HALF + m * 16) * 512u + (unsigned)(bj * HALF) * 2u, 17);
                        __builtin_amdgcn_raw_buffer_store_b128(o, ro, voff, base + (unsigned)(ai * HALF + m * 16) * 16384u + (unsigned)(bj * HALF) * 2u, 16);
                    }
        }
        WAIT_V(0);
        BAR;
        if (tid < NR) {
            if (mine) { if (tid != rank) st_sys(p.ws[tid] + OFF_RT + rank * 512 + pos * 4, call); }
            else if (tid == (rank ^ PM)) {
                if (full || (__hip_atomic_fetch_add((unsigned*)(ws + OFF_HCNT + pos * 4), 1u, __ATOMIC_RELAXED, __HIP_MEMORY_SCOPE_AGENT) & 3u) == 3u)
                    st_sys(p.ws[tid] + OFF_RS + pos * 4, call);
            }
        }
        if (tid == 0) {
            const unsigned t = __hip_atomic_fetch_add((unsigned*)(ws + OFF_QCTR + (8 + xcd) * 128), 1u, __ATOMIC_RELAXED, __HIP_MEMORY_SCOPE_AGENT);
            *(volatile LAS unsigned*)lds = GPX + t;
        }
        if (wid == 0) STAMP(tile * 8 + 6);
        __syncthreads();
        qi = __builtin_amdgcn_readfirstlane(*(volatile LAS unsigned*)lds);
        __syncthreads();
    }
#undef SA
#undef SB
#undef STAGE
#undef LDA
#undef LDB
#undef MMA
#undef BAR
#undef SCHED
}

__global__ __launch_bounds__(512, 2) void fused_gemm_allreduce(const Params p_by_value) {
    KP p = *(const __attribute__((address_space(4))) Params*)__builtin_amdgcn_kernarg_segment_ptr();
    extern __shared__ __attribute__((aligned(16))) unsigned char lds_raw[];
    LAS unsigned char* lds = (LAS unsigned char*)lds_raw;
    const int b = blockIdx.x, xcd = b & 7, slot = b >> 3;
    char* ws = p.ws[p.rank];
    if (slot < 32 - GPX) {
        unsigned xcc;
        asm volatile("s_getreg_b32 %0, hwreg(HW_REG_XCC_ID)" : "=s"(xcc));
        const int xsel = slot == 0 ? xcd : (int)(xcc & 7u);
        if (slot < NCW || slot < NCP) comm_role(p, tab8(p.xlink, xcd), tab8(p.xlink, xsel), slot, slot * 8 + xcd);
        else stage_role(p, (slot - (32 - GPX - NSPARE)) * 8 + xcd);
    }
    unsigned call = 0;
    if (slot >= 32 - GPX || __builtin_amdgcn_readfirstlane(threadIdx.x) < 64) call = __builtin_amdgcn_readfirstlane(ld_agent(ws + OFF_SEQ)) + 1u;
    if (slot >= 32 - GPX) gemm_role(p, xcd, slot - (32 - GPX), call, lds);
    asm volatile("s_waitcnt vmcnt(0)" ::: "memory");
    __syncthreads();
    if (threadIdx.x == 0) {
        const unsigned t = __hip_atomic_fetch_add((unsigned*)(ws + OFF_DONE), 1u, __ATOMIC_RELAXED, __HIP_MEMORY_SCOPE_AGENT);
        if ((t + 1u) % (unsigned)GRID == 0u) {
            for (int l = 0; l < 27; ++l) st_agent(ws + OFF_QCTR + l * 128, 0u);
            st_agent(ws + OFF_SEQ, call);
            if (ld_agent(ws + OFF_TMO) == call) {
                const int t0 = tab8(p.forder[role_of(p.rank)], 0), qd = quad_of(p.rank);
                const size_t off = (size_t)((qd >> 1) * 4096 + (t0 >> 4) * 256) * 16384 + (size_t)((qd & 1) * 4096 + (t0 & 15) * 256) * 2;
                st_agent(p.out[p.rank] + off, 0x7fc07fc0u);
            }
        }
    }
}

}

extern "C" void kernel_launch(int rank, int nranks, ncclComm_t comm,
                              hipStream_t stream,
                              void* const* d_in, const int* in_sizes, int n_in,
                              void* d_out, int out_size,
                              const int* in_dtypes, int out_dtype,
                              void* const* d_out_all,
                              void* const* d_ws_all, size_t ws_bytes) {
    if (nranks != NR || (unsigned)rank >= (unsigned)NR || ws_bytes < WS_NEED || n_in < 2) {
        fprintf(stderr, "unsupported configuration\n");
        return;
    }
    static bool attr_set[NR] = {};
    if (!attr_set[rank]) {
        HIP_CHECK(hipFuncSetAttribute((const void*)fused_gemm_allreduce, hipFuncAttributeMaxDynamicSharedMemorySize, LDS_BYTES));
        attr_set[rank] = true;
    }
    Params p;
    p.x = (const char*)d_in[0];
    p.w = (const char*)d_in[1];
    for (int r = 0; r < NR; ++r) { p.out[r] = (char*)d_out_all[r]; p.ws[r] = (char*)d_ws_all[r]; }
    p.rank = rank;
    {
        static const unsigned char xl[NR][8] = {
            {6, 255, 2, 3, 1, 5, 4, 7}, {7, 255, 4, 5, 0, 3, 2, 6}, {3, 255, 0, 6, 4, 5, 1, 7}, {2, 255, 0, 6, 1, 5, 4, 7},
            {5, 255, 1, 7, 0, 2, 3, 6}, {4, 255, 1, 7, 2, 3, 0, 6}, {0, 255, 2, 3, 4, 7, 1, 5}, {1, 255, 4, 5, 3, 6, 0, 2}};
        for (int i = 0; i < 8; ++i) ((unsigned char*)p.xlink)[i] = xl[rank][i];
    }
    {
        int nf[2] = {0, 0}, nx[8] = {0, 0, 0, 0, 0, 0, 0, 0}, load[8] = {0, 0, 0, 0, 0, 0, 0, 0};
        for (int g = 0; g < 8; ++g)
            for (int i = 0; i < XQN; ++i) { ((unsigned char*)p.xq[g])[i] = 0; ((unsigned char*)p.xm[g])[i] = 255; }
        for (int e = 0; e < 32; ++e)
            for (int t = 0; t < 256; ++t) {
                const int pm = t >> 4, pn = t & 15, a = 2 * pm, bb = 2 * pn + 1;
                if ((a > bb ? a : bb) != e) continue;
                const int h = (pm + pn) & 1;
                ((unsigned char*)p.forder[h])[nf[h]] = (unsigned char)t;
                ((unsigned char*)p.pos)[t] = (unsigned char)nf[h]++;
            }
        for (int e = 0; e < 32 + FDELAY; ++e)
            for (int fin = 0; fin < 2; ++fin)
                for (int t = 0; t < 256; ++t) {
                    const int pm = t >> 4, pn = t & 15, a = 2 * pm, bb = 2 * pn + 1;
                    const bool mine = ((pm + pn) & 1) == role_of(rank);
                    if (mine != (fin == 1) || (a > bb ? a : bb) + (mine ? FDELAY : 0) != e) continue;
                    const bool split = !mine && ((unsigned char*)p.pos)[t] < NHALF;
                    const int nj = split ? 4 : 1;
                    int g = (pm & 3) * 2 + ((pn >> 1) & 1), gm = -1;
                    for (int i = 0; i < 8; ++i)
                        if (nx[i] + nj <= XQN && (gm < 0 || load[i] < load[gm])) gm = i;
                    if (nx[g] + nj > XQN || load[g] > load[gm] + GSLACK) g = gm;
                    for (int m = split ? 1 : 0; m <= (split ? 4 : 0); ++m) {
                        ((unsigned char*)p.xq[g])[nx[g]] = (unsigned char)t;
                        ((unsigned char*)p.xm[g])[nx[g]++] = (unsigned char)m;
                    }
                    load[g] += split ? 5 : 3;
                }
    }
    hipLaunchKernelGGL(fused_gemm_allreduce, dim3(GRID), dim3(512), LDS_BYTES, stream, p);
    HIP_CHECK(hipGetLastError());
}
```
